```python
import numpy as np

kernel_name = "gpu_gemm_ar_m4096_k4096_n32768_bf16_silu"


ABSMAX_RTOL = 0.008

COMPUTE_DTYPE = "bfloat16"
EPILOGUE = "silu"
IN_QUANT = None
OUT_QUANT = None
IN_DTYPES = {"x": "bfloat16", "w_mat": "bfloat16"}
WORLD_SIZE = 8

def _to_bf16(x):
    u = np.ascontiguousarray(x, dtype=np.float32).view(np.uint32)
    return ((u + 0x7FFF + ((u >> 16) & 1)) & 0xFFFF0000).view(np.float32)

def setup_inputs():

    x = _to_bf16(np.random.randn(4096, 4096).astype(np.float32))
    w_mat = _to_bf16((np.random.randn(4096, 32768) / np.sqrt(4096)).astype(np.float32))
    return {"x": x, "w_mat": w_mat}

def _epilogue(y):
    return (y / (1.0 + np.exp(-np.clip(y, -60.0, 60.0)))).astype(np.float32)

def reference(x, w_mat):
    y = (x @ w_mat).astype(np.float32)
    return _epilogue(y)
```

```cpp
#include <hip/hip_runtime.h>
#include <rccl/rccl.h>
#include <cstdio>

namespace {

typedef unsigned short u16;
typedef short bf16x8 __attribute__((ext_vector_type(8)));
typedef float f32x4 __attribute__((ext_vector_type(4)));
typedef unsigned u32x4 __attribute__((ext_vector_type(4)));
#define LAS __attribute__((address_space(3)))
#define GAS __attribute__((address_space(1)))

constexpr int NR = 8;
constexpr int MM = 4096, NN = 32768, KSH = 512;
#define A_OWN 8
#define NCOMM 16
constexpr int NCT = 128 - 7 * A_OWN;
constexpr int GRID = 256;
constexpr int WG_PER_GRP = (GRID - NCOMM) / 8;
static_assert(NCOMM % 8 == 0, "comm workgroups: a multiple of 8");

constexpr unsigned OFF_TICKET = 0, OFF_TMO = 128;
constexpr unsigned OFF_FX = 4096;
constexpr unsigned OFF_FW = 8192;
constexpr unsigned OFF_TD = 32768;
constexpr unsigned OFF_FO = 40960;
constexpr unsigned OFF_DATA = 1u << 20;
constexpr unsigned XA_BYTES = 32u << 20;
constexpr unsigned PANEL_BYTES = 2u << 20;
constexpr unsigned BUF_BYTES = XA_BYTES + (unsigned)NCT * PANEL_BYTES;
constexpr size_t WS_NEED = (size_t)OFF_DATA + 2 * (size_t)BUF_BYTES;
constexpr unsigned OUT_BYTES = 256u << 20;

constexpr int NW_ITEMS = NCT * 4;
constexpr int NO_ITEMS = A_OWN * 16 * 2;
constexpr int N_XW = 64 + NW_ITEMS;
constexpr int NITEMS = N_XW + NO_ITEMS;
constexpr int MAX_LOCAL_ITEMS = (NITEMS + NCOMM - 1) / NCOMM + 2;

constexpr unsigned long long DEADLINE_TICKS = 100000000ull;
constexpr int STAGE_BYTES = 131072;
constexpr int LDS_BYTES = STAGE_BYTES;

struct Params {
    const u16* x; const u16* w;
    char* ws[NR]; char* out[NR];
    int rank; unsigned ws_bytes;
};

__device__ __forceinline__ int rfl(int v) { return __builtin_amdgcn_readfirstlane(v); }

__device__ __forceinline__ __amdgpu_buffer_rsrc_t mk_rsrc(const void* p, unsigned bytes) {
    return __builtin_amdgcn_make_buffer_rsrc(const_cast<void*>(p), 0, (int)bytes, 0x00020000);
}

template <int SCOPE>
__device__ __forceinline__ bool wait_ge(const unsigned* p, bool active, unsigned target, unsigned* tmo, unsigned call) {
    const unsigned long long t0 = __builtin_amdgcn_s_memrealtime();
    for (;;) {
        unsigned v = target;
        if (active) v = __hip_atomic_load(p, __ATOMIC_RELAXED, SCOPE);
        if (__all(v >= target)) return true;
        const unsigned tv = (unsigned)rfl((int)__hip_atomic_load(tmo, __ATOMIC_RELAXED, __HIP_MEMORY_SCOPE_AGENT));
        if (tv == call || __builtin_amdgcn_s_memrealtime() - t0 > DEADLINE_TICKS) {
            __hip_atomic_store(tmo, call, __ATOMIC_RELAXED, __HIP_MEMORY_SCOPE_AGENT);
            return false;
        }
        __builtin_amdgcn_s_sleep(4);
    }
}

#define WAIT_V(n) asm volatile("s_waitcnt vmcnt(" #n ")" ::: "memory")
#define WAIT_L(n) asm volatile("s_waitcnt lgkmcnt(" #n ")" ::: "memory")
#define DRAIN_V() do { __builtin_amdgcn_sched_barrier(0); __builtin_amdgcn_s_waitcnt(0x0F70); WAIT_V(0); __builtin_amdgcn_sched_barrier(0); } while (0)

constexpr int AUX_SYS = 17;
constexpr int AUX_AGT = 16;

__device__ __forceinline__ void transpose8(const u32x4 (&in)[8], u32x4 (&o)[8]) {
#pragma unroll
    for (int j = 0; j < 8; ++j)
#pragma unroll
        for (int e = 0; e < 4; ++e)
            o[j][e] = __builtin_amdgcn_perm(in[2 * e + 1][j >> 1], in[2 * e][j >> 1], (j & 1) ? 0x07060302u : 0x05040100u);
}

__device__ __forceinline__ void comm_role(const Params& P, LAS unsigned* lcnt, unsigned call, int c) {
    const int tid = threadIdx.x, wid = rfl(tid >> 6), lane = tid & 63;
    const int rank = P.rank;
    const unsigned dbase = OFF_DATA + (call & 1u) * BUF_BYTES;
    unsigned* tmo = (unsigned*)(P.ws[rank] + OFF_TMO);
    if (tid < MAX_LOCAL_ITEMS) lcnt[tid] = 0u;
    __syncthreads();
    char* wsl = P.ws[0];
#pragma unroll
    for (int p = 1; p < NR; ++p) wsl = (lane == p) ? P.ws[p] : wsl;

    int prev_flag = -1, li = 0;
    bool bad = false;
    auto signal_prev = [&]() {
        if (prev_flag >= 0) {
            unsigned old = 0;
            if (lane == 0) old = __hip_atomic_fetch_add(lcnt + (li - 1), 1u, __ATOMIC_RELAXED, __HIP_MEMORY_SCOPE_WORKGROUP);
            old = (unsigned)rfl((int)old);
            if (old == 7u && lane < NR)
                __hip_atomic_store((unsigned*)(wsl + prev_flag), call, __ATOMIC_RELAXED, __HIP_MEMORY_SCOPE_SYSTEM);
        }
    };
    const int qi = 4 * (lane & 3) + ((lane & 15) >> 2);

    for (int n = c; n < NITEMS; n += NCOMM, ++li) {
        u32x4 v[8];
        if (n < 128 && !(n & 1)) {
            const int u = n >> 1, mbi = u >> 2, kb = u & 3, mb = (mbi & 3) * 4 + (mbi >> 2);
            const u16* src = P.x + (size_t)(mb * 256 + wid * 32 + (lane & 15)) * KSH + kb * 128 + (lane >> 4) * 8;
#pragma unroll
            for (int t = 0; t < 8; ++t) v[t] = *(const u32x4*)(src + (t >> 2) * 16 * KSH + (t & 3) * 32);
            DRAIN_V();
            signal_prev();
            const unsigned dst = dbase + (unsigned)(mb * 64 + rank * 8 + kb * 2) * 32768u + (wid >> 2) * 16384u + (wid & 3) * 4096u + lane * 16u;
#pragma unroll
            for (int q = 0; q < NR; ++q) {
                const int p = (rank + 1 + q + wid) & 7;
                const __amdgpu_buffer_rsrc_t r = mk_rsrc(P.ws[p], P.ws_bytes);
#pragma unroll
                for (int t = 0; t < 8; ++t)
                    __builtin_amdgcn_raw_buffer_store_b128(v[t], r, dst + (t >> 2) * 2048 + ((t >> 1) & 1) * 32768 + (t & 1) * 1024, 0, AUX_SYS);
            }
            prev_flag = OFF_FX + (mb * 32 + rank * 4 + kb) * 4;
        } else if (n < N_XW) {
            const int u = n < 128 ? (n >> 1) : (n - 64), j = u >> 2, kb = u & 3;
            const int ktl = wid >> 2, kh = (wid >> 1) & 1, wr = wid & 1;
            const u16* src0 = P.w + (size_t)(kb * 128 + ktl * 64 + kh * 32 + (lane >> 4) * 8) * NN + wr * 128 + qi * 8;
            const unsigned dst = dbase + XA_BYTES + (unsigned)(j * 64 + rank * 8 + kb * 2 + ktl) * 32768u + (wr * 8 + kh) * 1024u + lane * 16u;
            if (j >= A_OWN) {
                const u16* src = src0 + (size_t)(7 * A_OWN + j) * 256;
#pragma unroll
                for (int t = 0; t < 8; ++t) v[t] = *(const u32x4*)(src + (size_t)t * NN);
                DRAIN_V();
                signal_prev();
                u32x4 o[8];
                transpose8(v, o);
#pragma unroll
                for (int q = 0; q < NR; ++q) {
                    const int p = (rank + 1 + q + wid) & 7;
                    const __amdgpu_buffer_rsrc_t r = mk_rsrc(P.ws[p], P.ws_bytes);
#pragma unroll
                    for (int t = 0; t < 8; ++t)
                        __builtin_amdgcn_raw_buffer_store_b128(o[t], r, dst + (t >> 2) * 16384 + (t & 3) * 2048, 0, AUX_SYS);
                }
            } else {
#pragma unroll
                for (int hb = 0; hb < 2; ++hb) {
                    u32x4 vv[4][8];
#pragma unroll
                    for (int q = 0; q < 4; ++q) {
                        const int p = (rank + 1 + hb * 4 + q + wid) & 7;
                        const u16* src = src0 + (size_t)(p * A_OWN + j) * 256;
#pragma unroll
                        for (int t = 0; t < 8; ++t) vv[q][t] = *(const u32x4*)(src + (size_t)t * NN);
                    }
                    DRAIN_V();
                    if (hb == 0) signal_prev();
#pragma unroll
                    for (int q = 0; q < 4; ++q) {
                        const int p = (rank + 1 + hb * 4 + q + wid) & 7;
                        const __amdgpu_buffer_rsrc_t r = mk_rsrc(P.ws[p], P.ws_bytes);
                        u32x4 o[8];
                        transpose8(vv[q], o);
#pragma unroll
                        for (int t = 0; t < 8; ++t)
                            __builtin_amdgcn_raw_buffer_store_b128(o[t], r, dst + (t >> 2) * 16384 + (t & 3) * 2048, 0, AUX_SYS);
                    }
                }
            }
            prev_flag = OFF_FW + (j * 32 + rank * 4 + kb) * 4;
        } else {
            const int o = n - N_XW, ti = o >> 1, hh = o & 1, jj = ti >> 4, mb = ti & 15;
            DRAIN_V();
            signal_prev();
            prev_flag = -1;
            if (!wait_ge<__HIP_MEMORY_SCOPE_AGENT>((const unsigned*)(P.ws[rank] + OFF_TD) + ti, true, 8u * call, tmo, call)) bad = true;
            __builtin_amdgcn_fence(__ATOMIC_ACQUIRE, "agent");
            const unsigned off = (unsigned)(mb * 256 + hh * 128 + wid * 16 + (lane >> 5)) * 65536u + (unsigned)(rank * A_OWN + jj) * 512u + (lane & 31) * 16u;
            const __amdgpu_buffer_rsrc_t rs = mk_rsrc(P.out[rank], OUT_BYTES);
#pragma unroll
            for (int t = 0; t < 8; ++t) v[t] = __builtin_amdgcn_raw_buffer_load_b128(rs, off + t * 131072, 0, AUX_AGT);
            if (bad) {
#pragma unroll
                for (int t = 0; t < 8; ++t) v[t] = (u32x4){0x7fc07fc0u, 0x7fc07fc0u, 0x7fc07fc0u, 0x7fc07fc0u};
            }
#pragma unroll
            for (int q = 0; q < NR - 1; ++q) {
                int qq = q + wid; qq = qq >= 7 ? qq - 7 : qq; qq = qq >= 7 ? qq - 7 : qq;
                const int p = (rank + 1 + qq) & 7;
                const __amdgpu_buffer_rsrc_t r = mk_rsrc(P.out[p], OUT_BYTES);
#pragma unroll
                for (int t = 0; t < 8; ++t)
                    __builtin_amdgcn_raw_buffer_store_b128(v[t], r, off + t * 131072, 0, AUX_SYS);
            }
        }
    }
    DRAIN_V();
    signal_prev();
    unsigned old = 0;
    if (lane == 0) old = __hip_atomic_fetch_add(lcnt + (MAX_LOCAL_ITEMS - 1), 1u, __ATOMIC_RELAXED, __HIP_MEMORY_SCOPE_WORKGROUP);
    old = (unsigned)rfl((int)old);
    if (old == 7u) {
        if (lane < NR && lane != rank)
            __hip_atomic_store((unsigned*)(wsl + OFF_FO + c * 128 + rank * 4), call, __ATOMIC_RELAXED, __HIP_MEMORY_SCOPE_SYSTEM);
        const bool ok = wait_ge<__HIP_MEMORY_SCOPE_SYSTEM>((const unsigned*)(P.ws[rank] + OFF_FO + c * 128) + (lane & 7), lane < NR && lane != rank, call, tmo, call);
        __builtin_amdgcn_fence(__ATOMIC_ACQUIRE, "");
        if (!ok && lane == 0)
            __hip_atomic_store((unsigned*)(P.out[rank] + (size_t)rank * A_OWN * 512), 0x7fc07fc0u, __ATOMIC_RELAXED, __HIP_MEMORY_SCOPE_SYSTEM);
    }
}

struct Unit { int mb, j; };

__device__ __forceinline__ unsigned cvt_pk_bf16(float lo, float hi) {
    unsigned r; asm volatile("v_cvt_pk_bf16_f32 %0, %1, %2" : "=v"(r) : "v"(lo), "v"(hi)); return r;
}
__device__ __forceinline__ float silu(float y) {
    const float t = __builtin_amdgcn_fmed3f(y, -60.0f, 60.0f);
    const float e = __builtin_amdgcn_exp2f(t * -1.44269504088896341f);
    return y * __builtin_amdgcn_rcpf(1.0f + e);
}

__device__ __forceinline__ void gemm_role(const Params& P, LAS unsigned char* lds, unsigned call, int g, int lw) {
    const int tid = threadIdx.x, wid = rfl(tid >> 6), lane = tid & 63, wr = wid >> 2, wc = wid & 3, fr = lane & 15, fq = lane >> 4;
    const int rank = P.rank;
    char* const ws = P.ws[rank];
    unsigned* tmo = (unsigned*)(ws + OFF_TMO);
    const char* XA = ws + OFF_DATA + (size_t)(call & 1u) * BUF_BYTES;
    const char* WB = XA + XA_BYTES;
    constexpr int nt = 64, HTB = 16384;
    constexpr size_t kstep = 32768, hstep = 16384;
    const unsigned voff0 = tid * 16u, voff1 = tid * 16u + 8192u;
    const unsigned ldsw = (unsigned)wid * 1024u;
    const int aoff = wr * 8192 + lane * 16, boff = wc * 4096 + lane * 16;
    bool bad = false;

    auto next = [&](int i, Unit& u) -> bool {
        const int e = lw + i * WG_PER_GRP;
        u.j = 2 * (e >> 2) + (g >> 2); u.mb = 4 * (g & 3) + (e & 3);
        return u.j < NCT;
    };
    auto ready = [&](const Unit& u) {
        if (wid == 0) {
            const unsigned* f = lane < 32 ? (const unsigned*)(ws + OFF_FX) + u.mb * 32 + lane : (const unsigned*)(ws + OFF_FW) + u.j * 32 + (lane - 32);
            if (!wait_ge<__HIP_MEMORY_SCOPE_SYSTEM>(f, true, call, tmo, call)) bad = true;
            __builtin_amdgcn_fence(__ATOMIC_ACQUIRE, "");
            WAIT_V(0);
        }
        asm volatile("" ::: "memory"); __builtin_amdgcn_s_barrier(); asm volatile("" ::: "memory");
    };

#define SA(b, h) (((b) * 2 + (h)) * HTB)
#define SB(b, h) ((4 + (b) * 2 + (h)) * HTB)
#define STAGE(bufoff, gbase) do { \
        __builtin_amdgcn_global_load_lds((const GAS unsigned*)((const char*)(gbase) + voff0), (LAS unsigned*)(lds + (bufoff) + ldsw), 16, 0, 0); \
        __builtin_amdgcn_global_load_lds((const GAS unsigned*)((const char*)(gbase) + voff1), (LAS unsigned*)(lds + (bufoff) + ldsw + 8192), 16, 0, 0); } while (0)
#define LDA(dst, b, h) do { _Pragma("unroll") for (int m = 0; m < 4; ++m) _Pragma("unroll") for (int k = 0; k < 2; ++k) dst[m][k] = *(const LAS bf16x8*)(lds + SA(b, h) + aoff + m * 2048 + k * 1024); } while (0)
#define LDB(dst, b, h) do { _Pragma("unroll") for (int n = 0; n < 2; ++n) _Pragma("unroll") for (int k = 0; k < 2; ++k) dst[n][k] = *(const LAS bf16x8*)(lds + SB(b, h) + boff + n * 2048 + k * 1024); } while (0)
#define MMA(ai, bj, At, Bt) do { __builtin_amdgcn_s_setprio(1); _Pragma("unroll") for (int m = 0; m < 4; ++m) _Pragma("unroll") for (int n = 0; n < 2; ++n) _Pragma("unroll") for (int k = 0; k < 2; ++k) \
        acc[ai][bj][m][n] = __builtin_amdgcn_mfma_f32_16x16x32_bf16(At[m][k], Bt[n][k], acc[ai][bj][m][n], 0, 0, 0); __builtin_amdgcn_s_setprio(0); } while (0)
#define BAR __builtin_amdgcn_s_barrier()
#define SCHED __builtin_amdgcn_sched_barrier(0)

    Unit cur, nxt; int ui = 0;
    if (!next(0, cur)) return;
    f32x4 acc[2][2][4][2];
#pragma unroll
    for (int a = 0; a < 2; ++a)
#pragma unroll
        for (int b = 0; b < 2; ++b)
#pragma unroll
            for (int m = 0; m < 4; ++m)
#pragma unroll
                for (int n = 0; n < 2; ++n) acc[a][b][m][n] = (f32x4){0.f, 0.f, 0.f, 0.f};
    bf16x8 At[4][2], B0[2][2], B1[2][2];
    const char* cA = WB + (size_t)cur.j * PANEL_BYTES; const char* cB = XA + (size_t)cur.mb * PANEL_BYTES;
    ready(cur);
    STAGE(SB(0, 0), cB); STAGE(SA(0, 0), cA); STAGE(SB(0, 1), cB + hstep); STAGE(SA(0, 1), cA + hstep);
    if (wr == 1) BAR;
    WAIT_V(4); BAR;
    STAGE(SB(1, 0), cB + kstep); STAGE(SA(1, 0), cA + kstep); STAGE(SB(1, 1), cB + hstep + kstep);
    WAIT_V(6); BAR;
    const __amdgpu_buffer_rsrc_t ro = mk_rsrc(P.out[rank], OUT_BYTES);
    for (;;) {
        const bool has_next = next(ui + 1, nxt);
        const char* nA = has_next ? WB + (size_t)nxt.j * PANEL_BYTES : cA; const char* nB = has_next ? XA + (size_t)nxt.mb * PANEL_BYTES : cB;
        for (int t = 0; t < nt; t += 2) {
            const bool last = (t == nt - 2);
            const char* a1 = cA + (size_t)(t + 1) * kstep;
            const char* a2 = last ? nA : cA + (size_t)(t + 2) * kstep; const char* b2 = last ? nB : cB + (size_t)(t + 2) * kstep;
            const char* a3 = a2 + kstep; const char* b3 = b2 + kstep;
            if (last && has_next) ready(nxt);
            LDB(B0, 0, 0); SCHED; LDA(At, 0, 0); STAGE(SA(1, 1), a1 + hstep);
            WAIT_L(8); BAR; WAIT_L(0); MMA(0, 0, At, B0); BAR; SCHED;
            LDB(B1, 0, 1); STAGE(SB(0, 0), b2);
            BAR; WAIT_L(0); MMA(0, 1, At, B1); BAR;
            LDA(At, 0, 1); STAGE(SA(0, 0), a2);
            BAR; WAIT_L(0); MMA(1, 0, At, B0); BAR; SCHED;
            STAGE(SB(0, 1), b2 + hstep);
            WAIT_V(6); BAR; MMA(1, 1, At, B1); BAR;
            LDB(B0, 1, 0); SCHED; LDA(At, 1, 0); STAGE(SA(0, 1), a2 + hstep);
            WAIT_L(8); BAR; WAIT_L(0); MMA(0, 0, At, B0); BAR; SCHED;
            LDB(B1, 1, 1); STAGE(SB(1, 0), b3);
            BAR; WAIT_L(0); MMA(0, 1, At, B1); BAR;
            LDA(At, 1, 1); STAGE(SA(1, 0), a3);
            BAR; WAIT_L(0); MMA(1, 0, At, B0); BAR; SCHED;
            STAGE(SB(1, 1), b3 + hstep);
            WAIT_V(6); BAR; MMA(1, 1, At, B1); BAR;
        }
        {
            const bool own = cur.j < A_OWN;
            const int ct = own ? rank * A_OWN + cur.j : 7 * A_OWN + cur.j;
            const unsigned obase = (unsigned)(cur.mb * 256 + wc * 32 + fr) * 65536u + (unsigned)(ct * 256 + wr * 128 + fq * 8) * 2u;
#pragma unroll
            for (int bj = 0; bj < 2; ++bj)
#pragma unroll
                for (int n = 0; n < 2; ++n)
#pragma unroll
                    for (int r = 0; r < 4; ++r) {
                        u32x4 o;
                        o.x = cvt_pk_bf16(silu(acc[0][bj][0][n][r]), silu(acc[0][bj][1][n][r]));
                        o.y = cvt_pk_bf16(silu(acc[0][bj][2][n][r]), silu(acc[0][bj][3][n][r]));
                        o.z = cvt_pk_bf16(silu(acc[1][bj][0][n][r]), silu(acc[1][bj][1][n][r]));
                        o.w = cvt_pk_bf16(silu(acc[1][bj][2][n][r]), silu(acc[1][bj][3][n][r]));
                        if (bad) o = (u32x4){0x7fc07fc0u, 0x7fc07fc0u, 0x7fc07fc0u, 0x7fc07fc0u};
                        __builtin_amdgcn_raw_buffer_store_b128(o, ro, obase + r * 64u + (bj * 128 + n * 16) * 65536u, 0, AUX_AGT);
                    }
            if (own) {
                WAIT_V(0);
                if (lane == 0) __hip_atomic_fetch_add((unsigned*)(ws + OFF_TD) + cur.j * 16 + cur.mb, 1u, __ATOMIC_RELAXED, __HIP_MEMORY_SCOPE_AGENT);
            }
        }
        if (!has_next) break;
#pragma unroll
        for (int a = 0; a < 2; ++a)
#pragma unroll
            for (int b = 0; b < 2; ++b)
#pragma unroll
                for (int m = 0; m < 4; ++m)
#pragma unroll
                    for (int n = 0; n < 2; ++n) acc[a][b][m][n] = (f32x4){0.f, 0.f, 0.f, 0.f};
        cur = nxt; cA = nA; cB = nB; ++ui;
    }
    WAIT_V(0);
    if (wr == 0) BAR;
}

__global__ __launch_bounds__(512, 2) void fused_kernel(Params P) {
    __shared__ __attribute__((aligned(16))) unsigned char smem[LDS_BYTES];
    LAS unsigned char* lds = (LAS unsigned char*)smem;
    unsigned tk = 0;
    if (threadIdx.x == 0) {
        tk = __hip_atomic_fetch_add((unsigned*)(P.ws[P.rank] + OFF_TICKET), 1u, __ATOMIC_RELAXED, __HIP_MEMORY_SCOPE_AGENT);
        *(LAS unsigned*)(lds + 1024) = tk;
    }
    __syncthreads();
    const unsigned call = (unsigned)rfl((int)*(LAS unsigned*)(lds + 1024)) / GRID + 1u;
    __syncthreads();
    const int b = blockIdx.x;
    if (b < NCOMM) comm_role(P, (LAS unsigned*)lds, call, b);
    else gemm_role(P, lds, call, b & 7, (b - NCOMM) >> 3);
}

}

extern "C" void kernel_launch(int rank, int nranks, ncclComm_t comm, hipStream_t stream,
                              void* const* d_in, const int* in_sizes, int n_in,
                              void* d_out, int out_size,
                              const int* in_dtypes, int out_dtype,
                              void* const* d_out_all,
                              void* const* d_ws_all, size_t ws_bytes) {
    (void)comm; (void)in_sizes; (void)n_in; (void)d_out; (void)out_size; (void)in_dtypes; (void)out_dtype;
    if (nranks != NR || (unsigned)rank >= (unsigned)NR || ws_bytes < WS_NEED) {
        fprintf(stderr, "unsupported configuration\n");
        return;
    }
    Params P;
    P.x = (const u16*)d_in[0]; P.w = (const u16*)d_in[1];
    for (int p = 0; p < NR; ++p) { P.ws[p] = (char*)d_ws_all[p]; P.out[p] = (char*)d_out_all[p]; }
    P.rank = rank;
    P.ws_bytes = (unsigned)WS_NEED;
    hipLaunchKernelGGL(fused_kernel, dim3(GRID), dim3(512), 0, stream, P);
}
```

```cpp
#include <hip/hip_runtime.h>
#include <rccl/rccl.h>
#include <cstdio>

namespace {

typedef unsigned short u16;
typedef short bf16x8 __attribute__((ext_vector_type(8)));
typedef float f32x4 __attribute__((ext_vector_type(4)));
typedef unsigned u32x4 __attribute__((ext_vector_type(4)));
#define LAS __attribute__((address_space(3)))
#define GAS __attribute__((address_space(1)))

constexpr int NR = 8;
constexpr int MM = 4096, NN = 32768, KSH = 512;
#define A_OWN 7
#define CWAVES 3
#define NCOMM 32
constexpr int NCT = 128 - 7 * A_OWN;
constexpr int GRID = 256;
constexpr int WG_PER_GRP = (GRID - NCOMM) / 8;
static_assert(NCOMM % 8 == 0, "comm workgroups: a multiple of 8");

constexpr unsigned OFF_TICKET = 0, OFF_TMO = 128;
constexpr unsigned OFF_OQ = 256;
constexpr unsigned OFF_TDALL = 512;
constexpr unsigned OFF_FX = 4096;
constexpr unsigned OFF_FW = 8192;
constexpr unsigned OFF_IC = 49152;
constexpr unsigned OFF_TD = 32768;
constexpr unsigned OFF_FO = 40960;
constexpr unsigned OFF_DATA = 1u << 20;
constexpr unsigned XA_BYTES = 32u << 20;
constexpr unsigned PANEL_BYTES = 2u << 20;
constexpr unsigned BUF_BYTES = XA_BYTES + (unsigned)NCT * PANEL_BYTES;
constexpr size_t WS_NEED = (size_t)OFF_DATA + 2 * (size_t)BUF_BYTES;
constexpr unsigned OUT_BYTES = 256u << 20;

constexpr int NW_ITEMS = NCT * 4;
constexpr int NO_PIECES = A_OWN * 16 * 16;
constexpr int N_XW = 64 + NW_ITEMS;
constexpr int NP_XW = N_XW * 8, NP_ALL = NP_XW + NO_PIECES;
static_assert(A_OWN >= 1 && A_OWN <= 14, "own tiles alternate with shared tiles at the head of the list");
__host__ __device__ constexpr bool ct_is_own(int j) { return j < 2 * A_OWN && !(j & 1); }
__host__ __device__ constexpr int ct_shared(int j) { return 8 * A_OWN + (j < 2 * A_OWN ? (j >> 1) : j - A_OWN); }

constexpr unsigned long long DEADLINE_TICKS = 100000000ull;
constexpr int STAGE_BYTES = 131072;
constexpr int LDS_BYTES = STAGE_BYTES;

struct Params {
    const u16* x; const u16* w;
    char* ws[NR]; char* out[NR];
    int rank; unsigned ws_bytes;
};

__device__ __forceinline__ int rfl(int v) { return __builtin_amdgcn_readfirstlane(v); }

__device__ __forceinline__ __amdgpu_buffer_rsrc_t mk_rsrc(const void* p, unsigned bytes) {
    return __builtin_amdgcn_make_buffer_rsrc(const_cast<void*>(p), 0, (int)bytes, 0x00020000);
}

template <int SCOPE>
__device__ __forceinline__ bool wait_ge(const unsigned* p, bool active, unsigned target, unsigned* tmo, unsigned call) {
    const unsigned long long t0 = __builtin_amdgcn_s_memrealtime();
    for (;;) {
        unsigned v = target;
        if (active) v = __hip_atomic_load(p, __ATOMIC_RELAXED, SCOPE);
        if (__all(v >= target)) return true;
        const unsigned tv = (unsigned)rfl((int)__hip_atomic_load(tmo, __ATOMIC_RELAXED, __HIP_MEMORY_SCOPE_AGENT));
        if (tv == call || __builtin_amdgcn_s_memrealtime() - t0 > DEADLINE_TICKS) {
            __hip_atomic_store(tmo, call, __ATOMIC_RELAXED, __HIP_MEMORY_SCOPE_AGENT);
            return false;
        }
        __builtin_amdgcn_s_sleep(4);
    }
}

#define WAIT_V(n) asm volatile("s_waitcnt vmcnt(" #n ")" ::: "memory")
#define WAIT_L(n) asm volatile("s_waitcnt lgkmcnt(" #n ")" ::: "memory")
#define DRAIN_V() do { __builtin_amdgcn_sched_barrier(0); __builtin_amdgcn_s_waitcnt(0x0F70); WAIT_V(0); __builtin_amdgcn_sched_barrier(0); } while (0)

#define STAMP_DECL
#define STAMP() do {} while (0)
#define STAMP_DUMP() do {} while (0)

constexpr int AUX_SYS = 17;
constexpr int AUX_AGT = 16;

__device__ __forceinline__ void transpose8(const u32x4 (&in)[8], u32x4 (&o)[8]) {
#pragma unroll
    for (int j = 0; j < 8; ++j)
#pragma unroll
        for (int e = 0; e < 4; ++e)
            o[j][e] = __builtin_amdgcn_perm(in[2 * e + 1][j >> 1], in[2 * e][j >> 1], (j & 1) ? 0x07060302u : 0x05040100u);
}

__device__ __forceinline__ void comm_role(const Params& P, LAS unsigned* lcnt, unsigned call, int c) {
    const int tid = threadIdx.x, wid = rfl(tid >> 6), lane = tid & 63;
    const int rank = P.rank;
    const unsigned dbase = OFF_DATA + (call & 1u) * BUF_BYTES;
    unsigned* tmo = (unsigned*)(P.ws[7] + OFF_TMO);
    unsigned* oq = (unsigned*)(P.ws[7] + OFF_OQ + (call & 1u) * 128u);
    if (tid == 0) lcnt[0] = 0u;
    __syncthreads();
    STAMP_DECL;
    STAMP();
    if (wid >= CWAVES) return;
    unsigned idv = (unsigned)(wid * NCOMM + c);
    char* wsl = P.ws[0];
#pragma unroll
    for (int p = 1; p < NR; ++p) wsl = (lane == p) ? P.ws[p] : wsl;

    int prev_flag = -1, prev_n = 0;
    unsigned oldv = 0;
    int zv = 0;
    asm volatile("" : "+v"(zv));
    bool bad = false, all_ready = false;
    auto sig_begin = [&]() {
        if (prev_flag >= 0 && lane == 0)
            oldv = __hip_atomic_fetch_add((unsigned*)(P.ws[7] + OFF_IC + prev_n * 16) + zv, 1u, __ATOMIC_RELAXED, __HIP_MEMORY_SCOPE_AGENT);
        __builtin_amdgcn_sched_barrier(0);
    };
    auto sig_end = [&]() {
        __builtin_amdgcn_sched_barrier(0);
        if (prev_flag >= 0 && ((unsigned)rfl((int)oldv) & 7u) == 7u && lane < NR)
            __hip_atomic_store((unsigned*)(wsl + prev_flag), call, __ATOMIC_RELAXED, __HIP_MEMORY_SCOPE_SYSTEM);
        __builtin_amdgcn_sched_barrier(0);
    };
    auto fetch_next = [&]() {
        __builtin_amdgcn_sched_barrier(0);
        if (lane == 0) idv = __hip_atomic_fetch_add(oq, 1u, __ATOMIC_RELAXED, __HIP_MEMORY_SCOPE_AGENT) + (unsigned)(NCOMM * CWAVES);
    };
    const int qi = 4 * (lane & 3) + ((lane & 15) >> 2);

    for (;;) {
        const int pc = rfl((int)idv);
        if (pc >= NP_ALL) break;
        const int n = pc >> 3, sub = pc & 7;
        u32x4 v[8];
        if (n < 128 && !(n & 1)) {
            const int u = n >> 1, mbi = u >> 2, kb = u & 3, mb = (mbi & 3) * 4 + (mbi >> 2);
            const u16* src = P.x + (size_t)(mb * 256 + sub * 32 + (lane & 15)) * KSH + kb * 128 + (lane >> 4) * 8;
#pragma unroll
            for (int t = 0; t < 8; ++t) v[t] = *(const u32x4*)(src + (t >> 2) * 16 * KSH + (t & 3) * 32);
            fetch_next();
            DRAIN_V();
            sig_begin();
            const unsigned dst = dbase + (unsigned)(mb * 64 + rank * 8 + kb * 2) * 32768u + (sub >> 2) * 16384u + (sub & 3) * 4096u + lane * 16u;
#pragma unroll
            for (int t = 0; t < 8; ++t) {
                if (t == 2) sig_end();
#pragma unroll
                for (int q = 0; q < NR; ++q)
                    __builtin_amdgcn_raw_buffer_store_b128(v[t], mk_rsrc(P.ws[q], P.ws_bytes), dst + (t >> 2) * 2048 + ((t >> 1) & 1) * 32768 + (t & 1) * 1024, 0, AUX_SYS);
            }
            prev_flag = OFF_FX + (mb * 32 + rank * 4 + kb) * 4;
        } else if (n < N_XW) {
            const int u = n < 128 ? (n >> 1) : (n - 64), j = u >> 2, kb = u & 3;
            const int ktl = sub >> 2, kh = (sub >> 1) & 1, wr = sub & 1;
            const u16* src0 = P.w + (size_t)(kb * 128 + ktl * 64 + kh * 32 + (lane >> 4) * 8) * NN + wr * 128 + qi * 8;
            const unsigned dst = dbase + XA_BYTES + (unsigned)(j * 64 + rank * 8 + kb * 2 + ktl) * 32768u + (wr * 8 + kh) * 1024u + lane * 16u;
            if (!ct_is_own(j)) {
                const u16* src = src0 + (size_t)ct_shared(j) * 256;
#pragma unroll
                for (int t = 0; t < 8; ++t) v[t] = *(const u32x4*)(src + (size_t)t * NN);
                fetch_next();
                DRAIN_V();
                sig_begin();
                u32x4 o[8];
                transpose8(v, o);
#pragma unroll
                for (int t = 0; t < 8; ++t) {
                    if (t == 2) sig_end();
#pragma unroll
                    for (int q = 0; q < NR; ++q)
                        __builtin_amdgcn_raw_buffer_store_b128(o[t], mk_rsrc(P.ws[q], P.ws_bytes), dst + (t >> 2) * 16384 + (t & 3) * 2048, 0, AUX_SYS);
                }
            } else {
#pragma unroll 1
                for (int hb = 0; hb < 2; ++hb) {
                    const int q0 = ((hb ^ sub) & 1) * 4;
                    u32x4 vv[4][8];
#pragma unroll
                    for (int q = 0; q < 4; ++q) {
                        const int p = (rank + 1 + q0 + q) & 7;
                        const u16* src = src0 + (size_t)(p * A_OWN + (j >> 1)) * 256;
#pragma unroll
                        for (int t = 0; t < 8; ++t) vv[q][t] = *(const u32x4*)(src + (size_t)t * NN);
                    }
                    if (hb == 1) fetch_next();
                    DRAIN_V();
                    sig_begin();
#pragma unroll
                    for (int t = 0; t < 8; ++t) {
                        if (t == 4) { sig_end(); prev_flag = -1; }
#pragma unroll
                        for (int q = 0; q < 4; ++q) {
                            u32x4 o;
#pragma unroll
                            for (int e = 0; e < 4; ++e)
                                o[e] = __builtin_amdgcn_perm(vv[q][2 * e + 1][t >> 1], vv[q][2 * e][t >> 1], (t & 1) ? 0x07060302u : 0x05040100u);
                            __builtin_amdgcn_raw_buffer_store_b128(o, mk_rsrc(q0 ? P.ws[4 + q] : P.ws[q], P.ws_bytes), dst + (t >> 2) * 16384 + (t & 3) * 2048, 0, AUX_SYS);
                        }
                    }
                }
            }
            prev_flag = OFF_FW + (j * 32 + rank * 4 + kb) * 4;
        } else {
            const int o = pc - NP_XW, mb = o / (16 * A_OWN);
            if (!all_ready) {
                DRAIN_V();
                sig_begin(); sig_end();
                prev_flag = -1;
                if (!wait_ge<__HIP_MEMORY_SCOPE_AGENT>((const unsigned*)(P.ws[7] + OFF_TD + mb * 128), true, 8u * A_OWN * call, tmo, call)) bad = true;
                all_ready = (unsigned)rfl((int)__hip_atomic_load((const unsigned*)(P.ws[7] + OFF_TDALL), __ATOMIC_RELAXED, __HIP_MEMORY_SCOPE_AGENT)) >= 8u * 16u * A_OWN * call;
                __builtin_amdgcn_fence(__ATOMIC_ACQUIRE, "agent");
            }
            unsigned off[8];
#pragma unroll
            for (int t = 0; t < 8; ++t) {
                const unsigned sg = (unsigned)(o * 16 + t * 2 + (lane >> 5)), row = sg / A_OWN, jj = sg - row * A_OWN;
                off[t] = row * 65536u + (unsigned)(rank * A_OWN + jj) * 512u + (lane & 31) * 16u;
            }
#pragma unroll
            for (int t = 0; t < 8; ++t) v[t] = __builtin_amdgcn_raw_buffer_load_b128(mk_rsrc(P.out[7], OUT_BYTES), off[t], 0, AUX_AGT);
            fetch_next();
            DRAIN_V();
            if (bad) {
#pragma unroll
                for (int t = 0; t < 8; ++t) v[t] = (u32x4){0x7fc07fc0u, 0x7fc07fc0u, 0x7fc07fc0u, 0x7fc07fc0u};
            }
#pragma unroll
            for (int t = 0; t < 8; ++t)
#pragma unroll
                for (int q = 0; q < NR - 1; ++q)
                    __builtin_amdgcn_raw_buffer_store_b128(v[t], mk_rsrc(P.out[q], OUT_BYTES), off[t], 0, AUX_SYS);
        }
        prev_n = n;
    }
    STAMP();
    DRAIN_V();
    STAMP();
    sig_begin(); sig_end();
    unsigned old = 0;
    if (lane == 0) old = __hip_atomic_fetch_add(lcnt, 1u, __ATOMIC_RELAXED, __HIP_MEMORY_SCOPE_WORKGROUP);
    old = (unsigned)rfl((int)old);
    if (old == CWAVES - 1u) {
        if (lane < NR - 1)
            __hip_atomic_store((unsigned*)(wsl + OFF_FO + c * 128 + rank * 4), call, __ATOMIC_RELAXED, __HIP_MEMORY_SCOPE_SYSTEM);
        const bool ok = wait_ge<__HIP_MEMORY_SCOPE_SYSTEM>((const unsigned*)(P.ws[7] + OFF_FO + c * 128) + (lane & 7), lane < NR && lane != rank, call, tmo, call);
        __builtin_amdgcn_fence(__ATOMIC_ACQUIRE, "");
        if (!ok && lane == 0)
            __hip_atomic_store((unsigned*)(P.out[7] + (size_t)rank * A_OWN * 512), 0x7fc07fc0u, __ATOMIC_RELAXED, __HIP_MEMORY_SCOPE_SYSTEM);
    }
}

struct Unit { int mb, j; };

__device__ __forceinline__ unsigned cvt_pk_bf16(float lo, float hi) {
    unsigned r; asm volatile("v_cvt_pk_bf16_f32 %0, %1, %2" : "=v"(r) : "v"(lo), "v"(hi)); return r;
}
__device__ __forceinline__ float silu(float y) {
    const float t = __builtin_amdgcn_fmed3f(y, -60.0f, 60.0f);
    const float e = __builtin_amdgcn_exp2f(t * -1.44269504088896341f);
    return y * __builtin_amdgcn_rcpf(1.0f + e);
}

__device__ __forceinline__ void gemm_role(const Params& P, LAS unsigned char* lds, unsigned call, int g, int lw) {
    const int tid = threadIdx.x, wid = rfl(tid >> 6), lane = tid & 63, wr = wid >> 2, wc = wid & 3, fr = lane & 15, fq = lane >> 4;
    const int rank = P.rank;
    char* const ws = P.ws[7];
    unsigned* tmo = (unsigned*)(ws + OFF_TMO);
    const char* XA = ws + OFF_DATA + (size_t)(call & 1u) * BUF_BYTES;
    const char* WB = XA + XA_BYTES;
    constexpr int nt = 64, HTB = 16384;
    constexpr size_t kstep = 32768, hstep = 16384;
    const unsigned voff0 = tid * 16u, voff1 = tid * 16u + 8192u;
    const unsigned ldsw = (unsigned)wid * 1024u;
    const int aoff = wr * 8192 + lane * 16, boff = wc * 4096 + lane * 16;
    bool bad = false;
    STAMP_DECL;
    STAMP();

    auto next = [&](int i, Unit& u) -> bool {
        const int e = lw + i * WG_PER_GRP;
        u.j = 2 * (e >> 2) + (g >> 2); u.mb = 4 * (g & 3) + (e & 3);
        return u.j < NCT;
    };
    auto ready = [&](const Unit& u) {
        if (wid == 0) {
            const unsigned* f = (const unsigned*)(ws + (lane < 32 ? OFF_FX + u.mb * 128 : OFF_FW + u.j * 128 - 128)) + lane;
            if (!wait_ge<__HIP_MEMORY_SCOPE_SYSTEM>(f, true, call, tmo, call)) bad = true;
            __builtin_amdgcn_fence(__ATOMIC_ACQUIRE, "");
            WAIT_V(0);
        }
        asm volatile("" ::: "memory"); __builtin_amdgcn_s_barrier(); asm volatile("" ::: "memory");
    };

#define SA(b, h) (((b) * 2 + (h)) * HTB)
#define SB(b, h) ((4 + (b) * 2 + (h)) * HTB)
#define STAGE(bufoff, gbase) do { \
        __builtin_amdgcn_global_load_lds((const GAS unsigned*)((const char*)(gbase) + voff0), (LAS unsigned*)(lds + (bufoff) + ldsw), 16, 0, 0); \
        __builtin_amdgcn_global_load_lds((const GAS unsigned*)((const char*)(gbase) + voff1), (LAS unsigned*)(lds + (bufoff) + ldsw + 8192), 16, 0, 0); } while (0)
#define LDA(dst, b, h) do { _Pragma("unroll") for (int m = 0; m < 4; ++m) _Pragma("unroll") for (int k = 0; k < 2; ++k) dst[m][k] = *(const LAS bf16x8*)(lds + SA(b, h) + aoff + m * 2048 + k * 1024); } while (0)
#define LDB(dst, b, h) do { _Pragma("unroll") for (int n = 0; n < 2; ++n) _Pragma("unroll") for (int k = 0; k < 2; ++k) dst[n][k] = *(const LAS bf16x8*)(lds + SB(b, h) + boff + n * 2048 + k * 1024); } while (0)
#define MMA(ai, bj, At, Bt) do { __builtin_amdgcn_s_setprio(1); _Pragma("unroll") for (int m = 0; m < 4; ++m) _Pragma("unroll") for (int n = 0; n < 2; ++n) _Pragma("unroll") for (int k = 0; k < 2; ++k) \
        acc[ai][bj][m][n] = __builtin_amdgcn_mfma_f32_16x16x32_bf16(At[m][k], Bt[n][k], acc[ai][bj][m][n], 0, 0, 0); __builtin_amdgcn_s_setprio(0); } while (0)
#define BAR __builtin_amdgcn_s_barrier()
#define SCHED __builtin_amdgcn_sched_barrier(0)

    Unit cur, nxt; int ui = 0;
    if (!next(0, cur)) return;
    f32x4 acc[2][2][4][2];
#pragma unroll
    for (int a = 0; a < 2; ++a)
#pragma unroll
        for (int b = 0; b < 2; ++b)
#pragma unroll
            for (int m = 0; m < 4; ++m)
#pragma unroll
                for (int n = 0; n < 2; ++n) acc[a][b][m][n] = (f32x4){0.f, 0.f, 0.f, 0.f};
    bf16x8 At[4][2], B0[2][2], B1[2][2];
    const char* cA = WB + (size_t)cur.j * PANEL_BYTES; const char* cB = XA + (size_t)cur.mb * PANEL_BYTES;
    ready(cur);
    STAMP();
    STAGE(SB(0, 0), cB); STAGE(SA(0, 0), cA); STAGE(SB(0, 1), cB + hstep); STAGE(SA(0, 1), cA + hstep);
    if (wr == 1) BAR;
    WAIT_V(4); BAR;
    STAGE(SB(1, 0), cB + kstep); STAGE(SA(1, 0), cA + kstep); STAGE(SB(1, 1), cB + hstep + kstep);
    WAIT_V(6); BAR;
    const __amdgpu_buffer_rsrc_t ro = mk_rsrc(P.out[7], OUT_BYTES);
    for (;;) {
        const bool has_next = next(ui + 1, nxt);
        const char* nA = has_next ? WB + (size_t)nxt.j * PANEL_BYTES : cA; const char* nB = has_next ? XA + (size_t)nxt.mb * PANEL_BYTES : cB;
        for (int t = 0; t < nt; t += 2) {
            const bool last = (t == nt - 2);
            const char* a1 = cA + (size_t)(t + 1) * kstep;
            const char* a2 = last ? nA : cA + (size_t)(t + 2) * kstep; const char* b2 = last ? nB : cB + (size_t)(t + 2) * kstep;
            const char* a3 = a2 + kstep; const char* b3 = b2 + kstep;
            if (last && has_next) { STAMP(); ready(nxt); STAMP(); }
            LDB(B0, 0, 0); SCHED; LDA(At, 0, 0); STAGE(SA(1, 1), a1 + hstep);
            WAIT_L(8); BAR; WAIT_L(0); MMA(0, 0, At, B0); BAR; SCHED;
            LDB(B1, 0, 1); STAGE(SB(0, 0), b2);
            BAR; WAIT_L(0); MMA(0, 1, At, B1); BAR;
            LDA(At, 0, 1); STAGE(SA(0, 0), a2);
            BAR; WAIT_L(0); MMA(1, 0, At, B0); BAR; SCHED;
            STAGE(SB(0, 1), b2 + hstep);
            WAIT_V(6); BAR; MMA(1, 1, At, B1); BAR;
            LDB(B0, 1, 0); SCHED; LDA(At, 1, 0); STAGE(SA(0, 1), a2 + hstep);
            WAIT_L(8); BAR; WAIT_L(0); MMA(0, 0, At, B0); BAR; SCHED;
            LDB(B1, 1, 1); STAGE(SB(1, 0), b3);
            BAR; WAIT_L(0); MMA(0, 1, At, B1); BAR;
            LDA(At, 1, 1); STAGE(SA(1, 0), a3);
            BAR; WAIT_L(0); MMA(1, 0, At, B0); BAR; SCHED;
            STAGE(SB(1, 1), b3 + hstep);
            WAIT_V(6); BAR; MMA(1, 1, At, B1); BAR;
        }
        STAMP();
        {
            const bool own = ct_is_own(cur.j);
            const int ct = own ? rank * A_OWN + (cur.j >> 1) : ct_shared(cur.j);
            const unsigned obase = (unsigned)(cur.mb * 256 + wc * 32 + fr) * 65536u + (unsigned)(ct * 256 + wr * 128 + fq * 8) * 2u;
#pragma unroll
            for (int bj = 0; bj < 2; ++bj)
#pragma unroll
                for (int n = 0; n < 2; ++n)
#pragma unroll
                    for (int r = 0; r < 4; ++r) {
                        u32x4 o;
                        o.x = cvt_pk_bf16(silu(acc[0][bj][0][n][r]), silu(acc[0][bj][1][n][r]));
                        o.y = cvt_pk_bf16(silu(acc[0][bj][2][n][r]), silu(acc[0][bj][3][n][r]));
                        o.z = cvt_pk_bf16(silu(acc[1][bj][0][n][r]), silu(acc[1][bj][1][n][r]));
                        o.w = cvt_pk_bf16(silu(acc[1][bj][2][n][r]), silu(acc[1][bj][3][n][r]));
                        if (bad) o = (u32x4){0x7fc07fc0u, 0x7fc07fc0u, 0x7fc07fc0u, 0x7fc07fc0u};
                        __builtin_amdgcn_raw_buffer_store_b128(o, ro, obase + r * 64u + (bj * 128 + n * 16) * 65536u, 0, AUX_AGT);
                    }
            if (own) {
                WAIT_V(0);
                if (lane < 2) __hip_atomic_fetch_add((unsigned*)(ws + (lane == 0 ? OFF_TD + cur.mb * 128 : OFF_TDALL)), 1u, __ATOMIC_RELAXED, __HIP_MEMORY_SCOPE_AGENT);
            }
        }
        STAMP();
        if (!has_next) break;
#pragma unroll
        for (int a = 0; a < 2; ++a)
#pragma unroll
            for (int b = 0; b < 2; ++b)
#pragma unroll
                for (int m = 0; m < 4; ++m)
#pragma unroll
                    for (int n = 0; n < 2; ++n) acc[a][b][m][n] = (f32x4){0.f, 0.f, 0.f, 0.f};
        cur = nxt; cA = nA; cB = nB; ++ui;
    }
    WAIT_V(0);
    if (wr == 0) BAR;
    STAMP();
    STAMP_DUMP();
}

__global__ __launch_bounds__(512, 2) void fused_kernel(Params P) {
    __shared__ __attribute__((aligned(16))) unsigned char smem[LDS_BYTES];
    LAS unsigned char* lds = (LAS unsigned char*)smem;
    unsigned tk = 0;
    if (threadIdx.x == 0) {
        tk = __hip_atomic_fetch_add((unsigned*)(P.ws[7] + OFF_TICKET), 1u, __ATOMIC_RELAXED, __HIP_MEMORY_SCOPE_AGENT);
        *(LAS unsigned*)(lds + 1024) = tk;
    }
    __syncthreads();
    const unsigned call = (unsigned)rfl((int)*(LAS unsigned*)(lds + 1024)) / GRID + 1u;
    __syncthreads();
    const int b = blockIdx.x;
    if (b == 0 && threadIdx.x == 0)
        __hip_atomic_store((unsigned*)(P.ws[7] + OFF_OQ + ((call + 1u) & 1u) * 128u), 0u, __ATOMIC_RELAXED, __HIP_MEMORY_SCOPE_AGENT);
    if (b < NCOMM) comm_role(P, (LAS unsigned*)lds, call, b);
    else gemm_role(P, lds, call, b & 7, (b - NCOMM) >> 3);
}

}

extern "C" void kernel_launch(int rank, int nranks, ncclComm_t comm, hipStream_t stream,
                              void* const* d_in, const int* in_sizes, int n_in,
                              void* d_out, int out_size,
                              const int* in_dtypes, int out_dtype,
                              void* const* d_out_all,
                              void* const* d_ws_all, size_t ws_bytes) {
    (void)comm; (void)in_sizes; (void)n_in; (void)d_out; (void)out_size; (void)in_dtypes; (void)out_dtype;
    if (nranks != NR || (unsigned)rank >= (unsigned)NR || ws_bytes < WS_NEED) {
        fprintf(stderr, "unsupported configuration\n");
        return;
    }
    Params P;
    P.x = (const u16*)d_in[0]; P.w = (const u16*)d_in[1];
    for (int q = 0; q < NR; ++q) { const int p = (rank + 1 + q) % NR; P.ws[q] = (char*)d_ws_all[p]; P.out[q] = (char*)d_out_all[p]; }
    P.rank = rank;
    P.ws_bytes = (unsigned)WS_NEED;
    hipLaunchKernelGGL(fused_kernel, dim3(GRID), dim3(512), 0, stream, P);
}
```

```cpp
#include <hip/hip_runtime.h>
#include <rccl/rccl.h>
#include <cstdio>

namespace {

typedef unsigned short u16;
typedef short bf16x8 __attribute__((ext_vector_type(8)));
typedef float f32x4 __attribute__((ext_vector_type(4)));
typedef unsigned u32x4 __attribute__((ext_vector_type(4)));
#define LAS __attribute__((address_space(3)))
#define GAS __attribute__((address_space(1)))

constexpr int NR = 8;
constexpr int MM = 4096, NN = 32768, KSH = 512;
#define A_OWN 7
#define PD 14
#define NS 5
#define PU 1
constexpr int NCOMM = 32;
constexpr int NCT = 128 - 7 * A_OWN;
constexpr int GRID = 256;
constexpr int NGEMM = GRID - NCOMM;
constexpr int WG_PER_GRP = NGEMM / 8;
static_assert(WG_PER_GRP == 28, "a round is 14 column tiles x 16 row blocks");
static_assert(A_OWN >= 1 && A_OWN <= 14, "own tiles sit in the first round");
constexpr int NSLOT = 16;
#define CHU 4
#define QAHEAD 1
#define EPI_AUX 0
static_assert(NS >= 1 && NS <= 5 && PD >= 7 && PD <= 63 && 64 % CHU == 0 && CHU >= 2, "waves of a communication workgroup");

constexpr unsigned OFF_TICKET = 0, OFF_TMO = 128;
constexpr unsigned OFF_PROG = 256;
constexpr unsigned OFF_WQ = 512;
constexpr unsigned OFF_GP = 1024;
constexpr unsigned OFF_TD = 4096;
constexpr unsigned OFF_FO = 8192;
constexpr unsigned OFF_FL = 16384;
constexpr unsigned OFF_XA = 1u << 20;
constexpr unsigned PANEL_BYTES = 2u << 20;
constexpr unsigned OFF_WB = OFF_XA + 16u * PANEL_BYTES;
constexpr unsigned OFF_SEND = OFF_WB + (unsigned)NCT * PANEL_BYTES;
constexpr unsigned WS_NEED = OFF_SEND + 8u * A_OWN * 262144u;
constexpr unsigned OUT_BYTES = 256u << 20;

constexpr int N_XW = 64 + NCT * 4;
constexpr int N_ALL = N_XW + A_OWN * 32;
constexpr int NCHUNK = N_ALL * 64 / CHU;
constexpr int N_GRP = N_XW + 16;
constexpr int N_STAGE = N_XW + 28 * A_OWN;
constexpr int LASTR = (NCT - 1) / 14, LASTSZ = NCT - 14 * LASTR;
static_assert(N_ALL * 64 + 256 < 65535 && NCHUNK < 32000, "progress fits 16 bits");
static_assert(OFF_FL + (N_GRP + 8) * 32 <= 65536, "flag table");
__host__ __device__ constexpr int ct_wi(int j) { return (j & 1) * 7 + (j >> 1); }
__host__ __device__ constexpr bool ct_is_own(int j) { return j < 14 && ct_wi(j) < A_OWN; }
__host__ __device__ constexpr int ct_shared(int j) { return 8 * A_OWN + (j < 14 ? ct_wi(j) : j) - A_OWN; }
__device__ __forceinline__ void item_decode(int n, bool& isx, int& idx, int& kb) {
    if (n < 120) {
        kb = n / 30; const int r = n - kb * 30;
        isx = r < 4 || (r >= 11 && r < 23);
        idx = r < 4 ? r : r < 11 ? 2 * (r - 4) : r < 23 ? r - 7 : 2 * (r - 23) + 1;
    } else {
        const int m = n - 120, i = 1 + m / 56, rem = m - (i - 1) * 56;
        const int sz = i == LASTR ? LASTSZ : 14, ne = (sz + 1) >> 1;
        kb = i == LASTR ? rem / LASTSZ : rem / 14; const int r = rem - kb * sz;
        isx = false;
        idx = 14 * i + (r < ne ? 2 * r : 2 * (r - ne) + 1);
    }
}
__device__ __forceinline__ int item_need(int i, int kb, int g) {
    if (i == 0) return 30 * kb + (g < 4 ? 11 + 4 * g : 30);
    const int sz = i == LASTR ? LASTSZ : 14;
    return 120 + 56 * (i - 1) + sz * kb + (g < 4 ? (sz + 1) >> 1 : sz);
}

constexpr unsigned long long DEADLINE_TICKS = 100000000ull;
constexpr int STAGE_BYTES = 131072;
constexpr int LDS_BYTES = STAGE_BYTES + 256;

struct Params {
    const u16* x; const u16* w;
    char* ws[NR]; char* out[NR];
    int rank;
};

__device__ __forceinline__ int rfl(int v) { return __builtin_amdgcn_readfirstlane(v); }

__device__ __forceinline__ __amdgpu_buffer_rsrc_t mk_rsrc(const void* p, unsigned bytes) {
    return __builtin_amdgcn_make_buffer_rsrc(const_cast<void*>(p), 0, (int)bytes, 0x00020000);
}
__device__ __forceinline__ bool timed_out(unsigned long long t0, unsigned* tmo, unsigned call) {
    if (__builtin_amdgcn_s_memrealtime() - t0 > DEADLINE_TICKS) {
        __hip_atomic_store(tmo, call, __ATOMIC_RELAXED, __HIP_MEMORY_SCOPE_AGENT);
        return true;
    }
    return false;
}
__device__ __forceinline__ unsigned lds_rd(unsigned addr) {
    unsigned v; asm volatile("ds_read_b32 %0, %1\n\ts_waitcnt lgkmcnt(0)" : "=v"(v) : "v"(addr) : "memory"); return v;
}
__device__ __forceinline__ void lds_wr(unsigned addr, unsigned v) { asm volatile("ds_write_b32 %0, %1" :: "v"(addr), "v"(v) : "memory"); }

#define WAIT_V(n) asm volatile("s_waitcnt vmcnt(" #n ")" ::: "memory")
#define WAIT_L(n) asm volatile("s_waitcnt lgkmcnt(" #n ")" ::: "memory")
#define SCHED __builtin_amdgcn_sched_barrier(0)
#define BAR __builtin_amdgcn_s_barrier()
#define DRAIN_V() do { SCHED; __builtin_amdgcn_s_waitcnt(0x0F70); WAIT_V(0); SCHED; } while (0)
template <int N> __device__ __forceinline__ void wait_v() { asm volatile("s_waitcnt vmcnt(%0)" :: "n"(N) : "memory"); }

#define STAMP_DECL
#define STAMP() do {} while (0)
#define STAMP_DUMP() do {} while (0)

constexpr int AUX_SYS = 17;
constexpr int AUX_AGT = 16;

__device__ __forceinline__ char* lane_ws(const Params& P, int lane) {
    char* wsl = P.ws[0];
#pragma unroll
    for (int p = 1; p < NR; ++p) wsl = (lane == p) ? P.ws[p] : wsl;
    return wsl;
}

template <bool REMOTE>
__device__ __forceinline__ void chunk_addr(int unit, int rank, bool& isout, unsigned& base, unsigned& stride) {
    const int n = unit >> 6;
    isout = n >= N_XW;
    bool isx; int idx, kb;
    item_decode(isout ? 0 : n, isx, idx, kb);
    const bool own = REMOTE && !isx && ct_is_own(idx);
    base = own ? OFF_SEND + (unsigned)(rank * A_OWN + ct_wi(idx)) * 262144u + (unsigned)kb * 65536u
               : (isx ? OFF_XA : OFF_WB) + (unsigned)idx * PANEL_BYTES + (unsigned)kb * 524288u;
    stride = isout ? A_OWN * 512u : own ? 0u : 65536u;
}
__device__ __forceinline__ unsigned unit_off(int unit, int lane, bool isout, unsigned base) {
    const unsigned sg = (unsigned)((isout ? unit - N_XW * 64 : 0) * 2 + (lane >> 5)), row = sg / A_OWN, jj = sg - row * A_OWN;
    return isout ? row * 65536u + jj * 512u + (lane & 31) * 16u : base + (unit & 63) * 1024u + lane * 16u;
}
__device__ __forceinline__ int lead_batches(unsigned cnt) {
    int r = __builtin_amdgcn_readlane((int)cnt, 0) * NS;
#pragma unroll
    for (int s = 1; s < NS; ++s) r = min(r, __builtin_amdgcn_readlane((int)cnt, s) * NS + s);
    return r;
}

__device__ __forceinline__ void comm_role(const Params& P, LAS unsigned char* lds, unsigned call, int c) {
    const int tid = threadIdx.x, wid = rfl(tid >> 6), lane = tid & 63;
    const int rank = P.rank;
    unsigned* tmo = (unsigned*)(P.ws[7] + OFF_TMO);
    const unsigned ctl = (unsigned)(uintptr_t)(lds + STAGE_BYTES);
    constexpr unsigned L_AVAIL = 0, L_FULL = 4, L_USED = 12, L_FREED = 32, L_DONE = 64, L_CQ = 128;
    constexpr int NCQ = 32;
    constexpr unsigned BADV = 0xffffu;
    const unsigned pbase = call << 16;
    auto chunk_id = [&](int k) -> int {
        const int e = rfl((int)lds_rd(ctl + L_CQ + (k & (NCQ - 1)) * 4));
        return (e >> 16) == k + 1 ? (e & 0xffff) : -1;
    };
    if (tid < 64) lds_wr(ctl + tid * 4, tid == L_CQ / 4 ? 0x10000u + (unsigned)c : 0u);
    WAIT_L(0);
    __syncthreads();
    STAMP_DECL;
    STAMP();
    if (wid == 0) {
        int avail = 0, freed = 0, i = 0;
        unsigned cur_stride = 0;
        __amdgpu_buffer_rsrc_t r[7];
#pragma unroll
        for (int q = 0; q < 7; ++q) r[q] = mk_rsrc(P.ws[q], WS_NEED);
#pragma unroll 1
        for (int k = 0;; ++k) {
            int id;
            while ((id = chunk_id(k)) < 0) __builtin_amdgcn_s_sleep(1);
            if (id >= NCHUNK) break;
            lds_wr(ctl + L_USED, (unsigned)(k + 1));
            const int unit0 = id * CHU;
            while ((unit0 >> 6) >= avail) {
                avail = rfl((int)lds_rd(ctl + L_AVAIL));
                if ((unit0 >> 6) >= avail) { WAIT_V(0); lds_wr(ctl + L_FULL, (unsigned)i); __builtin_amdgcn_s_sleep(1); }
            }
            bool isout; unsigned base, stride;
            chunk_addr<true>(unit0, rank, isout, base, stride);
            if (stride != cur_stride) {
                cur_stride = stride;
#pragma unroll
                for (int q = 0; q < 7; ++q) {
                    const unsigned so = (unsigned)((rank + 1 + q) & 7) * stride;
                    r[q] = mk_rsrc((isout ? P.out[q] : P.ws[q]) + so, (isout ? OUT_BYTES : WS_NEED) - so);
                }
            }
#pragma unroll 1
            for (int u = 0; u < CHU; ++u, ++i) {
                const unsigned off = unit_off(unit0 + u, lane, isout, base);
                while (i - NSLOT >= freed) { freed = lead_batches(lds_rd(ctl + L_FREED + (lane & 7) * 4)); if (i - NSLOT >= freed) __builtin_amdgcn_s_sleep(1); }
                LAS unsigned char* slot = lds + (i & (NSLOT - 1)) * 8192;
                SCHED;
#pragma unroll
                for (int q = 0; q < 7; ++q) {
                    wait_v<PD - 1>();
                    __builtin_amdgcn_raw_ptr_buffer_load_lds(r[q], (LAS void*)(slot + q * 1024), 16, off, 0, 0, AUX_SYS);
                    SCHED;
                    if (q == (PD - 1) % 7 && i >= (PD - 1) / 7) lds_wr(ctl + L_FULL, (unsigned)(i - (PD - 1) / 7));
                }
            }
        }
        STAMP();
#pragma unroll
        for (int k = (PD - 1) / 7; k >= 0; --k) {
            if (k == 8) wait_v<56>(); else if (k == 7) wait_v<49>(); else if (k == 6) wait_v<42>(); else if (k == 5) wait_v<35>(); else if (k == 4) wait_v<28>();
            else if (k == 3) wait_v<21>(); else if (k == 2) wait_v<14>(); else if (k == 1) wait_v<7>(); else wait_v<0>();
            if (i >= k) lds_wr(ctl + L_FULL, (unsigned)(i - k));
        }
        STAMP();
        if (lane < NR - 1)
            __hip_atomic_store((unsigned*)(lane_ws(P, lane) + OFF_FO + c * 128 + rank * 4), call, __ATOMIC_RELAXED, __HIP_MEMORY_SCOPE_SYSTEM);
        const unsigned* fo = (const unsigned*)(P.ws[7] + OFF_FO + c * 128) + (lane & 7);
        const unsigned long long t0 = __builtin_amdgcn_s_memrealtime();
        for (;;) {
            const unsigned v = __hip_atomic_load(fo, __ATOMIC_RELAXED, __HIP_MEMORY_SCOPE_SYSTEM);
            if (__all((int)(v - call) >= 0 || (lane & 7) == rank)) break;
            if (timed_out(t0, tmo, call)) break;
            __builtin_amdgcn_s_sleep(1);
        }
        STAMP();
        STAMP_DUMP();
    } else if (wid == 1) {
        const unsigned* fl = (const unsigned*)(P.ws[7] + OFF_FL) + lane;
        unsigned* prog = (unsigned*)(P.ws[7] + OFF_PROG);
        const unsigned long long t0 = __builtin_amdgcn_s_memrealtime();
        int pos = 0, mine = 0, all = c < 8 ? 0 : N_XW * 64;
        while (pos < N_GRP || mine < N_XW * 64 || all < N_XW * 64) {
            unsigned v = call, pv = 0;
            if (pos < N_GRP) v = __hip_atomic_load(fl + pos * 8, __ATOMIC_RELAXED, __HIP_MEMORY_SCOPE_SYSTEM);
            if (all < N_XW * 64) pv = __hip_atomic_load(prog + (lane & 31), __ATOMIC_RELAXED, __HIP_MEMORY_SCOPE_AGENT);
            if (mine < N_XW * 64) {
                const int lead = lead_batches(lds_rd(ctl + L_DONE + (lane & 7) * 4)), k = lead / CHU;
                const int id = chunk_id(k);
                if (id >= 0) {
                    const int m = id * CHU + lead - k * CHU;
                    if (m != mine && lane == 0) __hip_atomic_store(prog + c, pbase + (unsigned)m, __ATOMIC_RELAXED, __HIP_MEMORY_SCOPE_AGENT);
                    mine = m;
                }
            }
            int adv = 0;
            if (pos < N_GRP) {
                const unsigned long long miss = __ballot((int)(v - call) < 0 && pos + (lane >> 3) < N_GRP);
                adv = miss ? (int)(__builtin_ctzll(miss) >> 3) : 8;
                pos += adv;
                if (adv) lds_wr(ctl + L_AVAIL, (unsigned)(pos <= N_XW ? pos : N_XW + (pos - N_XW) * 2 * A_OWN));
            }
            if (all < N_XW * 64) {
                int m = (int)(pv - pbase);
#pragma unroll
                for (int o = 16; o >= 1; o >>= 1) m = min(m, __shfl_xor(m, o));
                m = rfl(m);
                if (m > all) {
                    if (lane == 0) __hip_atomic_store((unsigned*)(P.ws[7] + OFF_GP + c * 128), pbase + (unsigned)m, __ATOMIC_RELAXED, __HIP_MEMORY_SCOPE_AGENT);
                    all = m;
                }
            }
            if (timed_out(t0, tmo, call)) { lds_wr(ctl + L_AVAIL, BADV); break; }
            if (!adv) __builtin_amdgcn_s_sleep(4);
        }
    } else if (wid == 2) {
        unsigned* wq = (unsigned*)(P.ws[7] + OFF_WQ);
        const unsigned qbase = (call - 1u) * (unsigned)NCHUNK - (unsigned)NCOMM;
#pragma unroll 1
        for (int k = 1;; ++k) {
            while (rfl((int)lds_rd(ctl + L_USED)) + QAHEAD <= k) __builtin_amdgcn_s_sleep(1);
            unsigned id = 0;
            if (lane == 0) id = __hip_atomic_fetch_add(wq, 1u, __ATOMIC_RELAXED, __HIP_MEMORY_SCOPE_AGENT) - qbase;
            id = (unsigned)rfl((int)id);
            if (id >= (unsigned)NCHUNK) {
                if (lane < 4) lds_wr(ctl + L_CQ + ((k + lane) & (NCQ - 1)) * 4, (unsigned)((k + lane + 1) << 16 | NCHUNK));
                break;
            }
            lds_wr(ctl + L_CQ + (k & (NCQ - 1)) * 4, (unsigned)(k + 1) << 16 | id);
        }
    } else if (wid < 3 + NS) {
        const int s = wid - 3;
        int full = 0;
        unsigned cnt = 0;
        bool bad = false;
#pragma unroll 1
        for (int i = s;; i += NS) {
            const int k = i / CHU;
            int id;
            while ((id = chunk_id(k)) < 0) __builtin_amdgcn_s_sleep(1);
            if (id >= NCHUNK) break;
            const int unit = id * CHU + i - k * CHU;
            bool isout; unsigned base, stride;
            chunk_addr<false>(unit, rank, isout, base, stride);
            const unsigned off = unit_off(unit, lane, isout, base);
            const __amdgpu_buffer_rsrc_t r = mk_rsrc(isout ? P.out[7] : P.ws[7], isout ? OUT_BYTES : WS_NEED);
            while (i >= full) {
                const unsigned cw = lds_rd(ctl + (lane & 1) * 4);
                full = __builtin_amdgcn_readlane((int)cw, 1);
                bad = (unsigned)__builtin_amdgcn_readlane((int)cw, 0) == BADV;
                if (i >= full) __builtin_amdgcn_s_sleep(1);
            }
            u32x4 v[7];
#pragma unroll
            for (int q = 0; q < 7; ++q) v[q] = *(const LAS u32x4*)(lds + (i & (NSLOT - 1)) * 8192 + q * 1024 + lane * 16);
            SCHED; WAIT_L(0); SCHED;
            ++cnt;
            lds_wr(ctl + L_FREED + s * 4, cnt);
#pragma unroll
            for (int q = 0; q < 7; ++q) {
                if (bad) v[q] = (u32x4){0x7fc07fc0u, 0x7fc07fc0u, 0x7fc07fc0u, 0x7fc07fc0u};
                __builtin_amdgcn_raw_buffer_store_b128(v[q], r, off + (unsigned)((rank + 1 + q) & 7) * stride, 0, AUX_AGT);
            }
            if (!isout) { DRAIN_V(); } else wait_v<14>();
            lds_wr(ctl + L_DONE + s * 4, cnt);
        }
    }
}

__device__ __forceinline__ void stage_role(const Params& P, unsigned call, int s0) {
    const int tid = threadIdx.x, sub = rfl(tid >> 6), lane = tid & 63;
    const int rank = P.rank;
    const int qi = 4 * (lane & 3) + ((lane & 15) >> 2);
    const __amdgpu_buffer_rsrc_t rw = mk_rsrc(P.ws[7], WS_NEED);
    char* const wsl = lane_ws(P, lane);
    constexpr int G = 30 + 7 * A_OWN;
#pragma unroll 1
    for (int s = s0; s < N_STAGE; s += NGEMM) {
        int n, q = 7;
        if (s < 4 * G) {
            const int kb = s / G, r = s - kb * G;
            if (r < 30) n = kb * 30 + r;
            else { const int jo = (r - 30) / 7; q = r - 30 - jo * 7; n = kb * 30 + (jo < 7 ? 4 + jo : 16 + jo); }
        } else n = 120 + s - 4 * G;
        bool isx; int idx, kb;
        item_decode(n, isx, idx, kb);
        const bool own = !isx && ct_is_own(idx);
        u32x4 v[8];
        if (isx) {
            const u16* src = P.x + (size_t)(idx * 256 + sub * 32 + (lane & 15)) * KSH + kb * 128 + (lane >> 4) * 8;
#pragma unroll
            for (int t = 0; t < 8; ++t) v[t] = *(const u32x4*)(src + (t >> 2) * 16 * KSH + (t & 3) * 32);
            const unsigned dst = OFF_XA + (unsigned)idx * PANEL_BYTES + (unsigned)(kb * 8 + rank) * 65536u + (sub >> 2) * 16384u + (sub & 3) * 4096u + lane * 16u;
#pragma unroll
            for (int t = 0; t < 8; ++t)
                __builtin_amdgcn_raw_buffer_store_b128(v[t], rw, dst + (t >> 2) * 2048 + ((t >> 1) & 1) * 32768 + (t & 1) * 1024, 0, AUX_SYS);
        } else {
            const int ktl = sub >> 2, kh = (sub >> 1) & 1, wr = sub & 1;
            const int p = (rank + 1 + q) & 7;
            const int ct = own ? p * A_OWN + ct_wi(idx) : ct_shared(idx);
            const u16* src = P.w + (size_t)(kb * 128 + ktl * 64 + kh * 32 + (lane >> 4) * 8) * NN + ct * 256 + wr * 128 + qi * 8;
#pragma unroll
            for (int t = 0; t < 8; ++t) v[t] = *(const u32x4*)(src + (size_t)t * NN);
            const unsigned base = q < 7 ? OFF_SEND + (unsigned)ct * 262144u + (unsigned)kb * 65536u
                                        : OFF_WB + (unsigned)idx * PANEL_BYTES + (unsigned)(kb * 8 + rank) * 65536u;
            const unsigned dst = base + ktl * 32768u + (wr * 8 + kh) * 1024u + lane * 16u;
#pragma unroll
            for (int t = 0; t < 8; ++t) {
                u32x4 o;
#pragma unroll
                for (int e = 0; e < 4; ++e) o[e] = __builtin_amdgcn_perm(v[2 * e + 1][t >> 1], v[2 * e][t >> 1], (t & 1) ? 0x07060302u : 0x05040100u);
                __builtin_amdgcn_raw_buffer_store_b128(o, rw, dst + (t >> 2) * 16384 + (t & 3) * 2048, 0, AUX_SYS);
            }
        }
        DRAIN_V();
        BAR;
        if (sub == 0 && (own ? lane == q : lane < NR))
            __hip_atomic_store((unsigned*)(wsl + OFF_FL + n * 32 + rank * 4), call, __ATOMIC_RELAXED, __HIP_MEMORY_SCOPE_SYSTEM);
        SCHED;
    }
}

struct Unit { int mb, j; };

__device__ __forceinline__ unsigned cvt_pk_bf16(float lo, float hi) {
    unsigned r; asm volatile("v_cvt_pk_bf16_f32 %0, %1, %2" : "=v"(r) : "v"(lo), "v"(hi)); return r;
}
__device__ __forceinline__ float silu(float y) {
    const float t = __builtin_amdgcn_fmed3f(y, -60.0f, 60.0f);
    const float e = __builtin_amdgcn_exp2f(t * -1.44269504088896341f);
    return y * __builtin_amdgcn_rcpf(1.0f + e);
}

__device__ __forceinline__ void gemm_role(const Params& P, LAS unsigned char* lds, unsigned call, int g, int lw) {
    const int tid = threadIdx.x, wid = rfl(tid >> 6), lane = tid & 63, wr = wid >> 2, wc = wid & 3, fr = lane & 15, fq = lane >> 4;
    const int rank = P.rank;
    char* const ws = P.ws[7];
    unsigned* tmo = (unsigned*)(ws + OFF_TMO);
    const char* XA = ws + OFF_XA;
    const char* WB = ws + OFF_WB;
    constexpr int nt = 64, HTB = 16384;
    constexpr size_t kstep = 32768, hstep = 16384;
    const unsigned voff0 = tid * 16u, voff1 = tid * 16u + 8192u;
    const unsigned ldsw = (unsigned)wid * 1024u;
    const int aoff = wr * 8192 + lane * 16, boff = wc * 4096 + lane * 16;
    volatile LAS int* lknown = (volatile LAS int*)(lds + STAGE_BYTES);
    int known = 0;
    STAMP_DECL;
    STAMP();
    stage_role(P, call, lw * 8 + g);
    STAMP();

    auto next = [&](int i, Unit& u) -> bool {
        u.j = 14 * i + 2 * (lw >> 2) + (g >> 2); u.mb = 4 * (g & 3) + (lw & 3);
        return u.j < NCT;
    };
    auto ready = [&](int i, int kb) {
        const int need = item_need(i, kb, g) * 64;
        if ((unsigned)known >= (unsigned)need) return;
        if (wid == 0) {
            const unsigned* f = (const unsigned*)(ws + OFF_GP + g * 128);
            const unsigned long long t0 = __builtin_amdgcn_s_memrealtime();
            int m;
            for (;;) {
                m = rfl((int)(__hip_atomic_load(f, __ATOMIC_RELAXED, __HIP_MEMORY_SCOPE_AGENT) - (call << 16)));
                if (m >= need) break;
                if (timed_out(t0, tmo, call)) { m = -1; break; }
                __builtin_amdgcn_s_sleep(8);
            }
            if (lane == 0) lknown[0] = m;
            if (known == 0) __builtin_amdgcn_fence(__ATOMIC_ACQUIRE, "agent");
            WAIT_V(0); WAIT_L(0);
        }
        asm volatile("" ::: "memory"); BAR; asm volatile("" ::: "memory");
        known = rfl(lknown[0]);
    };

#define SA(b, h) (((b) * 2 + (h)) * HTB)
#define SB(b, h) ((4 + (b) * 2 + (h)) * HTB)
#define STAGE(bufoff, gbase) do { \
        __builtin_amdgcn_global_load_lds((const GAS unsigned*)((const char*)(gbase) + voff0), (LAS unsigned*)(lds + (bufoff) + ldsw), 16, 0, 0); \
        __builtin_amdgcn_global_load_lds((const GAS unsigned*)((const char*)(gbase) + voff1), (LAS unsigned*)(lds + (bufoff) + ldsw + 8192), 16, 0, 0); } while (0)
#define LDA(dst, b, h) do { _Pragma("unroll") for (int m = 0; m < 4; ++m) _Pragma("unroll") for (int k = 0; k < 2; ++k) dst[m][k] = *(const LAS bf16x8*)(lds + SA(b, h) + aoff + m * 2048 + k * 1024); } while (0)
#define LDB(dst, b, h) do { _Pragma("unroll") for (int n = 0; n < 2; ++n) _Pragma("unroll") for (int k = 0; k < 2; ++k) dst[n][k] = *(const LAS bf16x8*)(lds + SB(b, h) + boff + n * 2048 + k * 1024); } while (0)
#define MMA(ai, bj, At, Bt) do { __builtin_amdgcn_s_setprio(1); _Pragma("unroll") for (int m = 0; m < 4; ++m) _Pragma("unroll") for (int n = 0; n < 2; ++n) _Pragma("unroll") for (int k = 0; k < 2; ++k) \
        acc[ai][bj][m][n] = __builtin_amdgcn_mfma_f32_16x16x32_bf16(At[m][k], Bt[n][k], acc[ai][bj][m][n], 0, 0, 0); __builtin_amdgcn_s_setprio(0); } while (0)

    auto own_done = [&](int mb) {
        unsigned old = 0;
        if (lane == 0) old = __hip_atomic_fetch_add((unsigned*)(ws + OFF_TD + mb * 128), 1u, __ATOMIC_RELAXED, __HIP_MEMORY_SCOPE_AGENT);
        old = (unsigned)rfl((int)old) + 1u;
        if (old % (8u * A_OWN) == 0u && lane < NR)
            __hip_atomic_store((unsigned*)(lane_ws(P, lane) + OFF_FL + (N_XW + mb) * 32 + rank * 4), call, __ATOMIC_RELAXED, __HIP_MEMORY_SCOPE_SYSTEM);
    };
    Unit cur, nxt; int ui = 0;
    next(0, cur);
    f32x4 acc[2][2][4][2];
#pragma unroll
    for (int a = 0; a < 2; ++a)
#pragma unroll
        for (int b = 0; b < 2; ++b)
#pragma unroll
            for (int m = 0; m < 4; ++m)
#pragma unroll
                for (int n = 0; n < 2; ++n) acc[a][b][m][n] = (f32x4){0.f, 0.f, 0.f, 0.f};
    bf16x8 At[4][2], B0[2][2], B1[2][2];
    const char* cA = WB + (size_t)cur.j * PANEL_BYTES; const char* const cB = XA + (size_t)cur.mb * PANEL_BYTES;
    ready(0, 0);
    STAMP();
    STAGE(SB(0, 0), cB); STAGE(SA(0, 0), cA); STAGE(SB(0, 1), cB + hstep); STAGE(SA(0, 1), cA + hstep);
    if (wr == 1) BAR;
    WAIT_V(4); BAR;
    STAGE(SB(1, 0), cB + kstep); STAGE(SA(1, 0), cA + kstep); STAGE(SB(1, 1), cB + hstep + kstep);
    WAIT_V(6); BAR;
    const __amdgpu_buffer_rsrc_t ro = mk_rsrc(P.out[7], OUT_BYTES);
    for (;;) {
        const bool has_next = next(ui + 1, nxt);
        const char* nA = has_next ? WB + (size_t)nxt.j * PANEL_BYTES : cA;
        for (int t = 0; t < nt; t += 2) {
            const bool last = (t == nt - 2);
            const char* a1 = cA + (size_t)(t + 1) * kstep;
            const char* a2 = last ? nA : cA + (size_t)(t + 2) * kstep; const char* b2 = last ? cB : cB + (size_t)(t + 2) * kstep;
            const char* a3 = a2 + kstep; const char* b3 = b2 + kstep;
            if ((t & 15) == 14) {
                if (!last) { ready(ui, (t + 2) >> 4); STAMP(); }
                else if (has_next) { STAMP(); ready(ui + 1, 0); STAMP(); }
            }
            LDB(B0, 0, 0); SCHED; LDA(At, 0, 0); STAGE(SA(1, 1), a1 + hstep);
            WAIT_L(8); BAR; WAIT_L(0); MMA(0, 0, At, B0); BAR; SCHED;
            LDB(B1, 0, 1); STAGE(SB(0, 0), b2);
            BAR; WAIT_L(0); MMA(0, 1, At, B1); BAR;
            LDA(At, 0, 1); STAGE(SA(0, 0), a2);
            BAR; WAIT_L(0); MMA(1, 0, At, B0); BAR; SCHED;
            STAGE(SB(0, 1), b2 + hstep);
            WAIT_V(6); BAR; MMA(1, 1, At, B1); BAR;
            LDB(B0, 1, 0); SCHED; LDA(At, 1, 0); STAGE(SA(0, 1), a2 + hstep);
            WAIT_L(8); BAR; WAIT_L(0); MMA(0, 0, At, B0); BAR; SCHED;
            LDB(B1, 1, 1); STAGE(SB(1, 0), b3);
            BAR; WAIT_L(0); MMA(0, 1, At, B1); BAR;
            LDA(At, 1, 1); STAGE(SA(1, 0), a3);
            BAR; WAIT_L(0); MMA(1, 0, At, B0); BAR; SCHED;
            STAGE(SB(1, 1), b3 + hstep);
            WAIT_V(6); BAR; MMA(1, 1, At, B1); BAR;
        }
        STAMP();
        {
            const bool own = ct_is_own(cur.j);
            const int ct = own ? rank * A_OWN + ct_wi(cur.j) : ct_shared(cur.j);
            const unsigned obase = (unsigned)(cur.mb * 256 + wc * 32 + fr) * 65536u + (unsigned)(ct * 256 + wr * 128 + fq * 8) * 2u;
            const bool bad = known < 0;
#pragma unroll
            for (int bj = 0; bj < 2; ++bj)
#pragma unroll
                for (int n = 0; n < 2; ++n)
#pragma unroll
                    for (int r = 0; r < 4; ++r) {
                        u32x4 o;
                        o.x = cvt_pk_bf16(silu(acc[0][bj][0][n][r]), silu(acc[0][bj][1][n][r]));
                        o.y = cvt_pk_bf16(silu(acc[0][bj][2][n][r]), silu(acc[0][bj][3][n][r]));
                        o.z = cvt_pk_bf16(silu(acc[1][bj][0][n][r]), silu(acc[1][bj][1][n][r]));
                        o.w = cvt_pk_bf16(silu(acc[1][bj][2][n][r]), silu(acc[1][bj][3][n][r]));
                        if (bad) o = (u32x4){0x7fc07fc0u, 0x7fc07fc0u, 0x7fc07fc0u, 0x7fc07fc0u};
                        if (EPI_AUX == AUX_SYS || ui == 0) __builtin_amdgcn_raw_buffer_store_b128(o, ro, obase + r * 64u + (bj * 128 + n * 16) * 65536u, 0, AUX_SYS);
                        else __builtin_amdgcn_raw_buffer_store_b128(o, ro, obase + r * 64u + (bj * 128 + n * 16) * 65536u, 0, EPI_AUX);
                    }
            if (own) { WAIT_V(0); own_done(cur.mb); }
        }
        STAMP();
        if (!has_next) break;
#pragma unroll
        for (int a = 0; a < 2; ++a)
#pragma unroll
            for (int b = 0; b < 2; ++b)
#pragma unroll
                for (int m = 0; m < 4; ++m)
#pragma unroll
                    for (int n = 0; n < 2; ++n) acc[a][b][m][n] = (f32x4){0.f, 0.f, 0.f, 0.f};
        cur = nxt; cA = nA; ++ui;
    }
    WAIT_V(0);
    if (wr == 0) BAR;
    STAMP();
    STAMP_DUMP();
}

__global__ __launch_bounds__(512, 2) void fused_kernel(Params P) {
    __shared__ __attribute__((aligned(16))) unsigned char smem[LDS_BYTES];
    LAS unsigned char* lds = (LAS unsigned char*)smem;
    unsigned tk = 0;
    if (threadIdx.x == 0) {
        tk = __hip_atomic_fetch_add((unsigned*)(P.ws[7] + OFF_TICKET), 1u, __ATOMIC_RELAXED, __HIP_MEMORY_SCOPE_AGENT);
        *(LAS unsigned*)(lds + 1024) = tk;
    }
    __syncthreads();
    const unsigned call = (unsigned)rfl((int)*(LAS unsigned*)(lds + 1024)) / GRID + 1u;
    __syncthreads();
    const int b = blockIdx.x;
    if (b < NCOMM) comm_role(P, lds, call, b);
    else gemm_role(P, lds, call, b & 7, (b - NCOMM) >> 3);
}

}

extern "C" void kernel_launch(int rank, int nranks, ncclComm_t comm, hipStream_t stream,
                              void* const* d_in, const int* in_sizes, int n_in,
                              void* d_out, int out_size,
                              const int* in_dtypes, int out_dtype,
                              void* const* d_out_all,
                              void* const* d_ws_all, size_t ws_bytes) {
    (void)comm; (void)in_sizes; (void)n_in; (void)d_out; (void)out_size; (void)in_dtypes; (void)out_dtype;
    if (nranks != NR || (unsigned)rank >= (unsigned)NR || ws_bytes < WS_NEED) {
        fprintf(stderr, "unsupported configuration\n");
        return;
    }
    Params P;
    P.x = (const u16*)d_in[0]; P.w = (const u16*)d_in[1];
    for (int q = 0; q < NR; ++q) { const int p = (rank + 1 + q) % NR; P.ws[q] = (char*)d_ws_all[p]; P.out[q] = (char*)d_out_all[p]; }
    P.rank = rank;
    hipLaunchKernelGGL(fused_kernel, dim3(GRID), dim3(512), 0, stream, P);
}
```

```cpp
#include <hip/hip_runtime.h>
#include <rccl/rccl.h>
#include <cstdio>

namespace {

typedef unsigned short u16;
typedef short bf16x8 __attribute__((ext_vector_type(8)));
typedef float f32x4 __attribute__((ext_vector_type(4)));
typedef unsigned u32x4 __attribute__((ext_vector_type(4)));
#define LAS __attribute__((address_space(3)))
#define GAS __attribute__((address_space(1)))

constexpr int NR = 8;
constexpr int MM = 4096, NN = 32768, KSH = 512;
#define A_OWN 7
#define PD 14
#define NS 5
#define PU 1
constexpr int NCOMM = 32;
constexpr int NCT = 128 - 7 * A_OWN;
constexpr int GRID = 256;
constexpr int NGEMM = GRID - NCOMM;
constexpr int WG_PER_GRP = NGEMM / 8;
static_assert(WG_PER_GRP == 28, "a round is 14 column tiles x 16 row blocks");
static_assert(A_OWN >= 1 && A_OWN <= 14, "own tiles sit in the first round");
constexpr int NSLOT = 16;
#define CHU 4
#define QAHEAD 1
#define NPUSH 6
#define ST_AUX_OUT 18
#define OUT_PACK
#define OUT_ROT
#define ST_AUX 16
#define RUN0 4
#define RUN1 8
#define RUN_A 70
#define RUN_B 140
#define RUN_DIV 96
#define EPI_AUX 2
static_assert(NS >= 1 && NS <= 5 && PD >= 7 && PD <= 63 && 64 % CHU == 0 && CHU >= 2, "waves of a communication workgroup");

constexpr unsigned OFF_TMO = 128;
constexpr unsigned OFF_PROG = 256;
constexpr unsigned OFF_WQ = 512;
constexpr unsigned OFF_GP = 1024;
constexpr unsigned OFF_TICKET = 2048;
constexpr unsigned OFF_TD = 4096;
constexpr unsigned OFF_FO = 8192;
constexpr unsigned OFF_FL = 16384;
constexpr unsigned OFF_XA = 1u << 20;
constexpr unsigned PANEL_BYTES = 2u << 20;
constexpr unsigned OFF_WB = OFF_XA + 16u * PANEL_BYTES;
constexpr unsigned OFF_SEND = OFF_WB + (unsigned)NCT * PANEL_BYTES;
constexpr unsigned OFF_OS = OFF_SEND + 8u * A_OWN * 262144u;
constexpr unsigned WS_NEED = OFF_OS + A_OWN * PANEL_BYTES;
constexpr unsigned OUT_BYTES = 256u << 20;

constexpr int N_XW = 64 + NCT * 4;
constexpr int N_ALL = N_XW + A_OWN * 32;
constexpr int HOLE = 94;
static_assert(NPUSH >= 0 && NPUSH <= 7 && NPUSH <= A_OWN, "pushed items belong to own tiles");
constexpr int NCHUNK = (N_ALL - NPUSH) * 64 / CHU;
constexpr int N_GRP = N_XW + 16;
constexpr int N_STAGE = N_XW + 28 * A_OWN;
constexpr int LASTR = (NCT - 1) / 14, LASTSZ = NCT - 14 * LASTR;
static_assert(N_ALL * 64 + 256 < 65535 && NCHUNK < 32000, "progress fits 16 bits");
static_assert(OFF_FL + (N_GRP + 8) * 32 <= 65536, "flag table");
__host__ __device__ constexpr int ct_wi(int j) { return (j & 1) * 7 + (j >> 1); }
__host__ __device__ constexpr bool ct_is_own(int j) { return j < 14 && ct_wi(j) < A_OWN; }
__host__ __device__ constexpr int ct_shared(int j) { return 8 * A_OWN + (j < 14 ? ct_wi(j) : j) - A_OWN; }
__device__ __forceinline__ void item_decode(int n, bool& isx, int& idx, int& kb) {
    if (n < 120) {
        kb = n / 30; const int r = n - kb * 30;
        isx = r < 4 || (r >= 11 && r < 23);
        idx = r < 4 ? r : r < 11 ? 2 * (r - 4) : r < 23 ? r - 7 : 2 * (r - 23) + 1;
    } else {
        const int m = n - 120, i = 1 + m / 56, rem = m - (i - 1) * 56;
        const int sz = i == LASTR ? LASTSZ : 14, ne = (sz + 1) >> 1;
        kb = i == LASTR ? rem / LASTSZ : rem / 14; const int r = rem - kb * sz;
        isx = false;
        idx = 14 * i + (r < ne ? 2 * r : 2 * (r - ne) + 1);
    }
}
__device__ __forceinline__ int item_need(int i, int kb, int g) {
    if (i == 0) return 30 * kb + (g < 4 ? 11 + 4 * g : 30);
    const int sz = i == LASTR ? LASTSZ : 14;
    return 120 + 56 * (i - 1) + sz * kb + (g < 4 ? (sz + 1) >> 1 : sz);
}

constexpr unsigned long long DEADLINE_TICKS = 100000000ull;
constexpr int STAGE_BYTES = 131072;
constexpr int LDS_BYTES = STAGE_BYTES + 1024;

struct Params {
    const u16* x; const u16* w;
    char* ws[NR]; char* out[NR];
    int rank;
};

__device__ __forceinline__ int rfl(int v) { return __builtin_amdgcn_readfirstlane(v); }

__device__ __forceinline__ __amdgpu_buffer_rsrc_t mk_rsrc(const void* p, unsigned bytes) {
    return __builtin_amdgcn_make_buffer_rsrc(const_cast<void*>(p), 0, (int)bytes, 0x00020000);
}
__device__ __forceinline__ bool timed_out(unsigned long long t0, unsigned* tmo, unsigned call) {
    if (__builtin_amdgcn_s_memrealtime() - t0 > DEADLINE_TICKS) {
        __hip_atomic_store(tmo, call, __ATOMIC_RELAXED, __HIP_MEMORY_SCOPE_AGENT);
        return true;
    }
    return false;
}
__device__ __forceinline__ unsigned lds_rd(unsigned addr) {
    unsigned v; asm volatile("ds_read_b32 %0, %1\n\ts_waitcnt lgkmcnt(0)" : "=v"(v) : "v"(addr) : "memory"); return v;
}
__device__ __forceinline__ u32x4 lds_rd4(unsigned addr) {
    u32x4 v; asm volatile("ds_read_b128 %0, %1\n\ts_waitcnt lgkmcnt(0)" : "=v"(v) : "v"(addr) : "memory"); return v;
}
__device__ __forceinline__ void lds_wr4(unsigned addr, u32x4 v) { asm volatile("ds_write_b128 %0, %1" :: "v"(addr), "v"(v) : "memory"); }
__device__ __forceinline__ void lds_wr(unsigned addr, unsigned v) { asm volatile("ds_write_b32 %0, %1" :: "v"(addr), "v"(v) : "memory"); }

#define WAIT_V(n) asm volatile("s_waitcnt vmcnt(" #n ")" ::: "memory")
#define WAIT_L(n) asm volatile("s_waitcnt lgkmcnt(" #n ")" ::: "memory")
#define SCHED __builtin_amdgcn_sched_barrier(0)
#define BAR __builtin_amdgcn_s_barrier()
#define DRAIN_V() do { SCHED; __builtin_amdgcn_s_waitcnt(0x0F70); WAIT_V(0); SCHED; } while (0)
template <int N> __device__ __forceinline__ void wait_v() { asm volatile("s_waitcnt vmcnt(%0)" :: "n"(N) : "memory"); }

#define STAMP_DECL
#define STAMP() do {} while (0)
#define STAMP_DUMP() do {} while (0)

constexpr int AUX_SYS = 17;
constexpr int AUX_AGT = 16;

__device__ __forceinline__ GAS char* lane_ws(const Params& P, int lane) {
    char* wsl = P.ws[0];
#pragma unroll
    for (int p = 1; p < NR; ++p) {
        char* t = P.ws[p];
        asm volatile("" : "+s"(t));
        wsl = (lane == p) ? t : wsl;
    }
    return (GAS char*)wsl;
}

constexpr unsigned M_OWN = 1, M_OUT = 2;
__device__ __forceinline__ u32x4 chunk_entry(int k, unsigned id, int rank) {
    if (id >= (unsigned)NCHUNK) return (u32x4){(unsigned)(k + 1) << 16 | 0xffffu, 0u, 0u, 0u};
    const int unit = (int)id * CHU + ((int)id * CHU >= HOLE * 64 ? NPUSH * 64 : 0), n = unit >> 6;
    const bool isout = n >= N_XW;
    bool isx; int idx, kb;
    item_decode(isout ? 0 : n, isx, idx, kb);
    const bool own = !isx && ct_is_own(idx);
    const unsigned lb = (isx ? OFF_XA : OFF_WB) + (unsigned)idx * PANEL_BYTES + (unsigned)kb * 524288u + (unit & 63) * 1024u;
    const unsigned rb = own ? OFF_SEND + (unsigned)(rank * A_OWN + ct_wi(idx)) * 262144u + (unsigned)kb * 65536u + (unit & 63) * 1024u + M_OWN : lb;
    return (u32x4){(unsigned)(k + 1) << 16 | (unsigned)unit, isout ? M_OUT : rb, lb, 0u};
}
__device__ __forceinline__ unsigned out_idx(int unit, int rank) {
    unsigned o = (unsigned)(unit - N_XW * 64);
    o += (unsigned)rank * (A_OWN * 256u);
    if (o >= A_OWN * 2048u) o -= A_OWN * 2048u;
    return o;
}
__device__ __forceinline__ unsigned out_off(unsigned o, int lane) {
    const unsigned sg = o * 2u + (lane >> 5), row = sg / A_OWN, jj = sg - row * A_OWN;
    return row * 65536u + jj * 512u + (lane & 31) * 16u;
}
__device__ __forceinline__ int lead_batches(unsigned cnt) {
    int r = __builtin_amdgcn_readlane((int)cnt, 0) * NS;
#pragma unroll
    for (int s = 1; s < NS; ++s) r = min(r, __builtin_amdgcn_readlane((int)cnt, s) * NS + s);
    return r;
}

__device__ __forceinline__ void comm_role(const Params& P, LAS unsigned char* lds, unsigned call, int c) {
    const int tid = threadIdx.x, wid = rfl(tid >> 6), lane = tid & 63;
    const int rank = P.rank;
    unsigned* tmo = (unsigned*)(P.ws[7] + OFF_TMO);
    const unsigned ctl = (unsigned)(uintptr_t)(lds + STAGE_BYTES);
    constexpr unsigned L_AVAIL = 0, L_FULL = 4, L_USED = 12, L_FREED = 32, L_DONE = 64, L_CQ = 512;
    constexpr int NCQ = 32;
    constexpr unsigned BADV = 0xffffu;
    const unsigned pbase = call << 16;
    auto chunk = [&](int k, unsigned& rb, unsigned& lb) -> int {
        const u32x4 e = lds_rd4(ctl + L_CQ + (k & (NCQ - 1)) * 16);
        const int e0 = rfl((int)e.x);
        rb = (unsigned)rfl((int)e.y); lb = (unsigned)rfl((int)e.z);
        return (e0 >> 16) == k + 1 ? (e0 & 0xffff) : -1;
    };
    if (tid < 64) lds_wr4(ctl + tid * 16, tid == L_CQ / 16 ? chunk_entry(0, (unsigned)c, rank) : (u32x4){0u, 0u, 0u, 0u});
    WAIT_L(0);
    __syncthreads();
    STAMP_DECL;
    STAMP();
    if (wid == 0) {
        __builtin_amdgcn_s_setprio(3);
        int avail = 0, freed = 0, i = 0;
        unsigned cur_mode = M_OWN;
        __amdgpu_buffer_rsrc_t r[7];
#pragma unroll
        for (int q = 0; q < 7; ++q) r[q] = mk_rsrc(P.ws[q], WS_NEED);
#pragma unroll 1
        for (int k = 0;; ++k) {
            int unit0; unsigned rb, lb;
            while ((unit0 = chunk(k, rb, lb)) < 0) __builtin_amdgcn_s_sleep(1);
            if (unit0 == 0xffff) break;
            lds_wr(ctl + L_USED, (unsigned)(k + 1));
            const unsigned mode = rb & 3u;
            const unsigned stride = mode ? 0u : 65536u;
            if (false) {
                cur_mode = M_OUT;
#pragma unroll
                for (int q = 0; q < 7; ++q) r[q] = mk_rsrc(P.out[q], OUT_BYTES);
            }
            while ((unit0 >> 6) >= avail) {
                avail = rfl((int)lds_rd(ctl + L_AVAIL));
                if ((unit0 >> 6) >= avail) { WAIT_V(0); lds_wr(ctl + L_FULL, (unsigned)i); __builtin_amdgcn_s_sleep(1); }
            }
#pragma unroll 1
            for (int u = 0; u < CHU; ++u, ++i) {
                const unsigned off = (mode == M_OUT ? OFF_OS + out_idx(unit0 + u, rank) * 1024u : (rb & ~3u) + u * 1024u) + lane * 16u;
                while (i - NSLOT >= freed) { freed = lead_batches(lds_rd(ctl + L_FREED + (lane & 7) * 4)); if (i - NSLOT >= freed) __builtin_amdgcn_s_sleep(1); }
                LAS unsigned char* slot = lds + (i & (NSLOT - 1)) * 8192;
                SCHED;
#pragma unroll
                for (int q = 0; q < 7; ++q) {
                    wait_v<PD - 1>();
                    __builtin_amdgcn_raw_ptr_buffer_load_lds(r[q], (LAS void*)(slot + q * 1024), 16, off + (unsigned)((rank + 1 + q) & 7) * stride, 0, 0, AUX_SYS);
                    SCHED;
                    if (q == (PD - 1) % 7 && i >= (PD - 1) / 7) lds_wr(ctl + L_FULL, (unsigned)(i - (PD - 1) / 7));
                }
            }
        }
        STAMP();
#pragma unroll
        for (int k = (PD - 1) / 7; k >= 0; --k) {
            if (k == 8) wait_v<56>(); else if (k == 7) wait_v<49>(); else if (k == 6) wait_v<42>(); else if (k == 5) wait_v<35>(); else if (k == 4) wait_v<28>();
            else if (k == 3) wait_v<21>(); else if (k == 2) wait_v<14>(); else if (k == 1) wait_v<7>(); else wait_v<0>();
            if (i >= k) lds_wr(ctl + L_FULL, (unsigned)(i - k));
        }
        STAMP();
        if (lane < NR - 1)
            __hip_atomic_store((GAS unsigned*)(lane_ws(P, lane) + OFF_FO + c * 128 + rank * 4), call, __ATOMIC_RELAXED, __HIP_MEMORY_SCOPE_SYSTEM);
        const unsigned* fo = (const unsigned*)(P.ws[7] + OFF_FO + c * 128) + (lane & 7);
        const unsigned long long t0 = __builtin_amdgcn_s_memrealtime();
        for (;;) {
            const unsigned v = __hip_atomic_load(fo, __ATOMIC_RELAXED, __HIP_MEMORY_SCOPE_SYSTEM);
            if (__all((int)(v - call) >= 0 || (lane & 7) == rank)) break;
            if (timed_out(t0, tmo, call)) break;
            __builtin_amdgcn_s_sleep(1);
        }
        STAMP();
        STAMP_DUMP();
    } else if (wid == 1) {
        const unsigned* fl = (const unsigned*)(P.ws[7] + OFF_FL) + lane;
        unsigned* prog = (unsigned*)(P.ws[7] + OFF_PROG);
        const unsigned long long t0 = __builtin_amdgcn_s_memrealtime();
        int pos = 0, mine = 0, all = c < 8 ? 0 : N_XW * 64;
        while (pos < N_GRP || mine < N_XW * 64 || all < N_XW * 64) {
            unsigned v = call, pv = 0;
            if (pos < N_GRP) v = __hip_atomic_load(fl + pos * 8, __ATOMIC_RELAXED, __HIP_MEMORY_SCOPE_SYSTEM);
            if (all < N_XW * 64) pv = __hip_atomic_load(prog + (lane & 31), __ATOMIC_RELAXED, __HIP_MEMORY_SCOPE_AGENT);
            if (mine < N_XW * 64) {
                const int lead = lead_batches(lds_rd(ctl + L_DONE + (lane & 7) * 4)), k = lead / CHU;
                unsigned rb, lb;
                const int unit0 = chunk(k, rb, lb);
                if (unit0 >= 0) {
                    const int m = min(unit0 + lead - k * CHU, 0xffff);
                    if (m != mine && lane == 0) __hip_atomic_store(prog + c, pbase + (unsigned)m, __ATOMIC_RELAXED, __HIP_MEMORY_SCOPE_AGENT);
                    mine = m;
                }
            }
            int adv = 0;
            if (pos < N_GRP) {
                const unsigned long long miss = __ballot((int)(v - call) < 0 && pos + (lane >> 3) < N_GRP);
                adv = miss ? (int)(__builtin_ctzll(miss) >> 3) : 8;
                pos += adv;
                if (adv) lds_wr(ctl + L_AVAIL, (unsigned)(pos <= N_XW ? pos : pos < N_GRP ? N_XW : N_ALL));
            }
            if (all < N_XW * 64) {
                int m = (int)(pv - pbase);
#pragma unroll
                for (int o = 16; o >= 1; o >>= 1) m = min(m, __shfl_xor(m, o));
                m = rfl(m);
                if (pos < HOLE + NPUSH) m = min(m, HOLE * 64);
                if (m > all) {
                    if (lane == 0) __hip_atomic_store((unsigned*)(P.ws[7] + OFF_GP + c * 128), pbase + (unsigned)m, __ATOMIC_RELAXED, __HIP_MEMORY_SCOPE_AGENT);
                    all = m;
                }
            }
            if (timed_out(t0, tmo, call)) { lds_wr(ctl + L_AVAIL, BADV); break; }
            if (!adv) __builtin_amdgcn_s_sleep(4);
        }
    } else if (wid == 2) {
        unsigned* wq = (unsigned*)(P.ws[7] + OFF_WQ + (call & 1u) * 128u);
        if (c == 0 && lane == 0)
            __hip_atomic_store((unsigned*)(P.ws[7] + OFF_WQ + ((call + 1u) & 1u) * 128u), 0u, __ATOMIC_RELAXED, __HIP_MEMORY_SCOPE_AGENT);
        unsigned id = (unsigned)c, left = 0;
#pragma unroll 1
        for (int k = 1;; ++k) {
            while (rfl((int)lds_rd(ctl + L_USED)) + QAHEAD <= k) __builtin_amdgcn_s_sleep(1);
            if (left) { ++id; --left; }
            else {
                const int item = (int)id * CHU >> 6, rest = NCHUNK - (int)id;
                const unsigned m = item < RUN_A ? RUN0 : item < RUN_B ? 1u : (unsigned)max(1, min(RUN1, rest / RUN_DIV));
                if (lane == 0) id = __hip_atomic_fetch_add(wq, m, __ATOMIC_RELAXED, __HIP_MEMORY_SCOPE_AGENT) + (unsigned)NCOMM;
                id = (unsigned)rfl((int)id);
                left = m - 1u;
            }
            if (id >= (unsigned)NCHUNK) {
                if (lane < 4) lds_wr(ctl + L_CQ + ((k + lane) & (NCQ - 1)) * 16, (unsigned)(k + lane + 1) << 16 | 0xffffu);
                break;
            }
            lds_wr4(ctl + L_CQ + (k & (NCQ - 1)) * 16, chunk_entry(k, id, rank));
        }
    } else if (wid < 3 + NS) {
        const int s = wid - 3;
        int full = 0;
        unsigned cnt = 0;
        bool bad = false;
#pragma unroll 1
        for (int i = s;; i += NS) {
            const int k = i / CHU;
            int unit0; unsigned rb, lb;
            while ((unit0 = chunk(k, rb, lb)) < 0) __builtin_amdgcn_s_sleep(1);
            if (unit0 == 0xffff) break;
            const bool isout = rb == M_OUT;
            const unsigned off = isout ? out_off(out_idx(unit0 + i - k * CHU, rank), lane) : lb + (i - k * CHU) * 1024u + lane * 16u;
            const unsigned stride = isout ? A_OWN * 512u : 65536u;
            const __amdgpu_buffer_rsrc_t r = mk_rsrc(isout ? P.out[7] : P.ws[7], isout ? OUT_BYTES : WS_NEED);
            while (i >= full) {
                const unsigned cw = lds_rd(ctl + (lane & 1) * 4);
                full = __builtin_amdgcn_readlane((int)cw, 1);
                bad = (unsigned)__builtin_amdgcn_readlane((int)cw, 0) == BADV;
                if (i >= full) __builtin_amdgcn_s_sleep(1);
            }
            u32x4 v[7];
#pragma unroll
            for (int q = 0; q < 7; ++q) v[q] = *(const LAS u32x4*)(lds + (i & (NSLOT - 1)) * 8192 + q * 1024 + lane * 16);
            SCHED; WAIT_L(0); SCHED;
            ++cnt;
            lds_wr(ctl + L_FREED + s * 4, cnt);
#pragma unroll
            for (int q = 0; q < 7; ++q) {
                if (bad) v[q] = (u32x4){0x7fc07fc0u, 0x7fc07fc0u, 0x7fc07fc0u, 0x7fc07fc0u};
                if (ST_AUX == ST_AUX_OUT || !isout) __builtin_amdgcn_raw_buffer_store_b128(v[q], r, off + (unsigned)((rank + 1 + q) & 7) * stride, 0, ST_AUX);
                else __builtin_amdgcn_raw_buffer_store_b128(v[q], r, off + (unsigned)((rank + 1 + q) & 7) * stride, 0, ST_AUX_OUT);
            }
            if (!isout) { DRAIN_V(); } else wait_v<14>();
            lds_wr(ctl + L_DONE + s * 4, cnt);
        }
    }
}

__device__ __forceinline__ unsigned take_ticket(const Params& P, int zv) {
    return __hip_atomic_fetch_add((unsigned*)(P.ws[7] + OFF_TICKET + (blockIdx.x & 15) * 128) + zv, 1u, __ATOMIC_RELAXED, __HIP_MEMORY_SCOPE_AGENT);
}
__device__ __forceinline__ unsigned stage_role(const Params& P, GAS char* wsl, unsigned ctl, int s0) {
    const int tid = threadIdx.x, sub = rfl(tid >> 6), lane = tid & 63;
    const int rank = P.rank;
    const int qi = 4 * (lane & 3) + ((lane & 15) >> 2);
    constexpr int G = 30 + 7 * A_OWN, R0 = 3 * G + 30, NP7 = 7 * NPUSH;
    unsigned call = 0, tk = 0;
    int zv = 0;
    asm volatile("" : "+v"(zv));
#pragma unroll 1
    for (int s1 = s0; s1 < N_STAGE; s1 += NGEMM) {
        const bool push = s1 < NP7;
        const int s = push ? R0 + s1 : s1 - NP7 < R0 ? s1 - NP7 : s1;
        int n, q = 7;
        if (s < 4 * G) {
            const int kb = s / G, r = s - kb * G;
            if (r < 30) n = kb * 30 + r;
            else { const int jo = (r - 30) / 7; q = r - 30 - jo * 7; n = kb * 30 + (jo < 7 ? 4 + jo : 16 + jo); }
        } else n = 120 + s - 4 * G;
        const __amdgpu_buffer_rsrc_t rw = mk_rsrc(P.ws[push ? q : 7], WS_NEED);
        bool isx; int idx, kb;
        item_decode(n, isx, idx, kb);
        const bool own = !isx && ct_is_own(idx);
        u32x4 v[8];
        if (isx) {
            const u16* src = P.x + (size_t)(idx * 256 + sub * 32 + (lane & 15)) * KSH + kb * 128 + (lane >> 4) * 8;
#pragma unroll
            for (int t = 0; t < 8; ++t) v[t] = *(const u32x4*)(src + (t >> 2) * 16 * KSH + (t & 3) * 32);
            SCHED;
            if (s1 == s0 && tid == 0) tk = take_ticket(P, zv);
            SCHED;
            const unsigned dst = OFF_XA + (unsigned)idx * PANEL_BYTES + (unsigned)(kb * 8 + rank) * 65536u + (sub >> 2) * 16384u + (sub & 3) * 4096u + lane * 16u;
#pragma unroll
            for (int t = 0; t < 8; ++t)
                __builtin_amdgcn_raw_buffer_store_b128(v[t], rw, dst + (t >> 2) * 2048 + ((t >> 1) & 1) * 32768 + (t & 1) * 1024, 0, AUX_SYS);
        } else {
            const int ktl = sub >> 2, kh = (sub >> 1) & 1, wr = sub & 1;
            const int p = (rank + 1 + q) & 7;
            const int ct = own ? p * A_OWN + ct_wi(idx) : ct_shared(idx);
            const u16* src = P.w + (size_t)(kb * 128 + ktl * 64 + kh * 32 + (lane >> 4) * 8) * NN + ct * 256 + wr * 128 + qi * 8;
#pragma unroll
            for (int t = 0; t < 8; ++t) v[t] = *(const u32x4*)(src + (size_t)t * NN);
            SCHED;
            if (s1 == s0 && tid == 0) tk = take_ticket(P, zv);
            SCHED;
            const unsigned base = q < 7 && !push ? OFF_SEND + (unsigned)ct * 262144u + (unsigned)kb * 65536u
                                        : OFF_WB + (unsigned)idx * PANEL_BYTES + (unsigned)(kb * 8 + rank) * 65536u;
            const unsigned dst = base + ktl * 32768u + (wr * 8 + kh) * 1024u + lane * 16u;
#pragma unroll
            for (int t = 0; t < 8; ++t) {
                u32x4 o;
#pragma unroll
                for (int e = 0; e < 4; ++e) o[e] = __builtin_amdgcn_perm(v[2 * e + 1][t >> 1], v[2 * e][t >> 1], (t & 1) ? 0x07060302u : 0x05040100u);
                __builtin_amdgcn_raw_buffer_store_b128(o, rw, dst + (t >> 2) * 16384 + (t & 3) * 2048, 0, AUX_SYS);
            }
        }
        DRAIN_V();
        if (s1 == s0 && tid == 0) { lds_wr(ctl + 4, tk); WAIT_L(0); }
        BAR;
        if (s1 == s0) call = (unsigned)rfl((int)lds_rd(ctl + 4)) / (GRID / 16) + 1u;
        if (sub == 0 && (own ? lane == q : lane < NR))
            __hip_atomic_store((GAS unsigned*)(wsl + OFF_FL + n * 32 + rank * 4), call, __ATOMIC_RELAXED, __HIP_MEMORY_SCOPE_SYSTEM);
        SCHED;
    }
    return call;
}

struct Unit { int mb, j; };

__device__ __forceinline__ unsigned cvt_pk_bf16(float lo, float hi) {
    unsigned r; asm volatile("v_cvt_pk_bf16_f32 %0, %1, %2" : "=v"(r) : "v"(lo), "v"(hi)); return r;
}
__device__ __forceinline__ float silu(float y) {
    const float t = __builtin_amdgcn_fmed3f(y, -60.0f, 60.0f);
    const float e = __builtin_amdgcn_exp2f(t * -1.44269504088896341f);
    return y * __builtin_amdgcn_rcpf(1.0f + e);
}

__device__ __forceinline__ void gemm_role(const Params& P, LAS unsigned char* lds, int g, int lw) {
    const int tid = threadIdx.x, wid = rfl(tid >> 6), lane = tid & 63, wr = wid >> 2, wc = wid & 3, fr = lane & 15, fq = lane >> 4;
    const int rank = P.rank;
    char* const ws = P.ws[7];
    unsigned* tmo = (unsigned*)(ws + OFF_TMO);
    const char* XA = ws + OFF_XA;
    const char* WB = ws + OFF_WB;
    constexpr int nt = 64, HTB = 16384;
    constexpr size_t kstep = 32768, hstep = 16384;
    const unsigned voff0 = tid * 16u, voff1 = tid * 16u + 8192u;
    const unsigned ldsw = (unsigned)wid * 1024u;
    const int aoff = wr * 8192 + lane * 16, boff = wc * 4096 + lane * 16;
    volatile LAS int* lknown = (volatile LAS int*)(lds + STAGE_BYTES);
    int known = 0;
    STAMP_DECL;
    STAMP();
    GAS char* wsl = lane_ws(P, lane);
    asm volatile("" : "+v"(wsl));
    const unsigned call = stage_role(P, wsl, (unsigned)(uintptr_t)(lds + STAGE_BYTES), lw * 8 + g);
    STAMP();

    auto next = [&](int i, Unit& u) -> bool {
        u.j = 14 * i + 2 * (lw >> 2) + (g >> 2); u.mb = 4 * (g & 3) + (lw & 3);
        return u.j < NCT;
    };
    auto ready = [&](int i, int kb) {
        const int need = item_need(i, kb, g) * 64;
        if ((unsigned)known >= (unsigned)need) return;
        if (wid == 0) {
            const unsigned* f = (const unsigned*)(ws + OFF_GP + g * 128);
            const unsigned long long t0 = __builtin_amdgcn_s_memrealtime();
            int m;
            for (;;) {
                m = rfl((int)(__hip_atomic_load(f, __ATOMIC_RELAXED, __HIP_MEMORY_SCOPE_AGENT) - (call << 16)));
                if (m >= need) break;
                if (timed_out(t0, tmo, call)) { m = -1; break; }
                __builtin_amdgcn_s_sleep(8);
            }
            if (lane == 0) lknown[0] = m;
            if (known == 0) __builtin_amdgcn_fence(__ATOMIC_ACQUIRE, "");
            WAIT_V(0); WAIT_L(0);
        }
        asm volatile("" ::: "memory"); BAR; asm volatile("" ::: "memory");
        known = rfl(lknown[0]);
    };

#define SA(b, h) (((b) * 2 + (h)) * HTB)
#define SB(b, h) ((4 + (b) * 2 + (h)) * HTB)
#define STAGE(bufoff, gbase) do { \
        __builtin_amdgcn_global_load_lds((const GAS unsigned*)((const char*)(gbase) + voff0), (LAS unsigned*)(lds + (bufoff) + ldsw), 16, 0, 0); \
        __builtin_amdgcn_global_load_lds((const GAS unsigned*)((const char*)(gbase) + voff1), (LAS unsigned*)(lds + (bufoff) + ldsw + 8192), 16, 0, 0); } while (0)
#define LDA(dst, b, h) do { _Pragma("unroll") for (int m = 0; m < 4; ++m) _Pragma("unroll") for (int k = 0; k < 2; ++k) dst[m][k] = *(const LAS bf16x8*)(lds + SA(b, h) + aoff + m * 2048 + k * 1024); } while (0)
#define LDB(dst, b, h) do { _Pragma("unroll") for (int n = 0; n < 2; ++n) _Pragma("unroll") for (int k = 0; k < 2; ++k) dst[n][k] = *(const LAS bf16x8*)(lds + SB(b, h) + boff + n * 2048 + k * 1024); } while (0)
#define MMA(ai, bj, At, Bt) do { __builtin_amdgcn_s_setprio(1); _Pragma("unroll") for (int m = 0; m < 4; ++m) _Pragma("unroll") for (int n = 0; n < 2; ++n) _Pragma("unroll") for (int k = 0; k < 2; ++k) \
        acc[ai][bj][m][n] = __builtin_amdgcn_mfma_f32_16x16x32_bf16(At[m][k], Bt[n][k], acc[ai][bj][m][n], 0, 0, 0); __builtin_amdgcn_s_setprio(0); } while (0)

    auto own_done = [&](int mb) {
        unsigned old = 0;
        if (lane == 0) old = __hip_atomic_fetch_add((unsigned*)(ws + OFF_TD + mb * 128), 1u, __ATOMIC_RELAXED, __HIP_MEMORY_SCOPE_AGENT);
        old = (unsigned)rfl((int)old) + 1u;
        if (old % (8u * A_OWN) == 0u && lane < NR)
            __hip_atomic_store((GAS unsigned*)(wsl + OFF_FL + (N_XW + mb) * 32 + rank * 4), call, __ATOMIC_RELAXED, __HIP_MEMORY_SCOPE_SYSTEM);
    };
    Unit cur, nxt; int ui = 0;
    next(0, cur);
    f32x4 acc[2][2][4][2];
#pragma unroll
    for (int a = 0; a < 2; ++a)
#pragma unroll
        for (int b = 0; b < 2; ++b)
#pragma unroll
            for (int m = 0; m < 4; ++m)
#pragma unroll
                for (int n = 0; n < 2; ++n) acc[a][b][m][n] = (f32x4){0.f, 0.f, 0.f, 0.f};
    bf16x8 At[4][2], B0[2][2], B1[2][2];
    const char* cA = WB + (size_t)cur.j * PANEL_BYTES; const char* const cB = XA + (size_t)cur.mb * PANEL_BYTES;
    ready(0, 0);
    STAMP();
    STAGE(SB(0, 0), cB); STAGE(SA(0, 0), cA); STAGE(SB(0, 1), cB + hstep); STAGE(SA(0, 1), cA + hstep);
    if (wr == 1) BAR;
    WAIT_V(4); BAR;
    STAGE(SB(1, 0), cB + kstep); STAGE(SA(1, 0), cA + kstep); STAGE(SB(1, 1), cB + hstep + kstep);
    WAIT_V(6); BAR;
    const __amdgpu_buffer_rsrc_t ro = mk_rsrc(P.out[7], OUT_BYTES);
    for (;;) {
        const bool has_next = next(ui + 1, nxt);
        const char* nA = has_next ? WB + (size_t)nxt.j * PANEL_BYTES : cA;
        for (int t = 0; t < nt; t += 2) {
            const bool last = (t == nt - 2);
            const char* a1 = cA + (size_t)(t + 1) * kstep;
            const char* a2 = last ? nA : cA + (size_t)(t + 2) * kstep; const char* b2 = last ? cB : cB + (size_t)(t + 2) * kstep;
            const char* a3 = a2 + kstep; const char* b3 = b2 + kstep;
            if ((t & 15) == 14) {
                if (!last) { ready(ui, (t + 2) >> 4); STAMP(); }
                else if (has_next) { STAMP(); ready(ui + 1, 0); STAMP(); }
            }
            LDB(B0, 0, 0); SCHED; LDA(At, 0, 0); STAGE(SA(1, 1), a1 + hstep);
            WAIT_L(8); BAR; WAIT_L(0); MMA(0, 0, At, B0); BAR; SCHED;
            LDB(B1, 0, 1); STAGE(SB(0, 0), b2);
            BAR; WAIT_L(0); MMA(0, 1, At, B1); BAR;
            LDA(At, 0, 1); STAGE(SA(0, 0), a2);
            BAR; WAIT_L(0); MMA(1, 0, At, B0); BAR; SCHED;
            STAGE(SB(0, 1), b2 + hstep);
            WAIT_V(6); BAR; MMA(1, 1, At, B1); BAR;
            LDB(B0, 1, 0); SCHED; LDA(At, 1, 0); STAGE(SA(0, 1), a2 + hstep);
            WAIT_L(8); BAR; WAIT_L(0); MMA(0, 0, At, B0); BAR; SCHED;
            LDB(B1, 1, 1); STAGE(SB(1, 0), b3);
            BAR; WAIT_L(0); MMA(0, 1, At, B1); BAR;
            LDA(At, 1, 1); STAGE(SA(1, 0), a3);
            BAR; WAIT_L(0); MMA(1, 0, At, B0); BAR; SCHED;
            STAGE(SB(1, 1), b3 + hstep);
            WAIT_V(6); BAR; MMA(1, 1, At, B1); BAR;
        }
        STAMP();
        {
            const bool own = ct_is_own(cur.j);
            const int ct = own ? rank * A_OWN + ct_wi(cur.j) : ct_shared(cur.j);
            const unsigned obase = (unsigned)(cur.mb * 256 + wc * 32 + fr) * 65536u + (unsigned)(ct * 256 + wr * 128 + fq * 8) * 2u;
            const unsigned pbase = OFF_OS + (unsigned)(cur.mb * 256 + wc * 32 + fr) * (A_OWN * 512u) + (unsigned)(ct_wi(cur.j) * 256 + wr * 128 + fq * 8) * 2u;
            const bool bad = known < 0;
#pragma unroll
            for (int bj = 0; bj < 2; ++bj)
#pragma unroll
                for (int n = 0; n < 2; ++n)
#pragma unroll
                    for (int r = 0; r < 4; ++r) {
                        u32x4 o;
                        o.x = cvt_pk_bf16(silu(acc[0][bj][0][n][r]), silu(acc[0][bj][1][n][r]));
                        o.y = cvt_pk_bf16(silu(acc[0][bj][2][n][r]), silu(acc[0][bj][3][n][r]));
                        o.z = cvt_pk_bf16(silu(acc[1][bj][0][n][r]), silu(acc[1][bj][1][n][r]));
                        o.w = cvt_pk_bf16(silu(acc[1][bj][2][n][r]), silu(acc[1][bj][3][n][r]));
                        if (bad) o = (u32x4){0x7fc07fc0u, 0x7fc07fc0u, 0x7fc07fc0u, 0x7fc07fc0u};
                        __builtin_amdgcn_raw_buffer_store_b128(o, ro, obase + r * 64u + (bj * 128 + n * 16) * 65536u, 0, EPI_AUX);
                        if (own) __builtin_amdgcn_raw_buffer_store_b128(o, mk_rsrc(ws, WS_NEED), pbase + r * 64u + (bj * 128 + n * 16) * (A_OWN * 512u), 0, AUX_SYS);
                    }
            if (own) { WAIT_V(0); own_done(cur.mb); }
        }
        STAMP();
        if (!has_next) break;
#pragma unroll
        for (int a = 0; a < 2; ++a)
#pragma unroll
            for (int b = 0; b < 2; ++b)
#pragma unroll
                for (int m = 0; m < 4; ++m)
#pragma unroll
                    for (int n = 0; n < 2; ++n) acc[a][b][m][n] = (f32x4){0.f, 0.f, 0.f, 0.f};
        cur = nxt; cA = nA; ++ui;
    }
    WAIT_V(0);
    if (wr == 0) BAR;
    BAR;
    if (wid == 0) {
        unsigned old = 0;
        if (lane == 0) old = __hip_atomic_fetch_add((unsigned*)(ws + OFF_GP + g * 128 + 64), 1u, __ATOMIC_RELAXED, __HIP_MEMORY_SCOPE_AGENT);
        if (((unsigned)rfl((int)old) + 1u) % WG_PER_GRP == 0u) __builtin_amdgcn_fence(__ATOMIC_RELEASE, "agent");
    }
    STAMP();
    STAMP_DUMP();
}

__global__ __launch_bounds__(512, 2) void fused_kernel(Params P) {
    __shared__ __attribute__((aligned(16))) unsigned char smem[LDS_BYTES];
    LAS unsigned char* lds = (LAS unsigned char*)smem;
    const int b = blockIdx.x;
    if (b < NCOMM) {
        if (threadIdx.x == 0) *(LAS unsigned*)(lds + 1024) = take_ticket(P, 0);
        __syncthreads();
        const unsigned call = (unsigned)rfl((int)*(LAS unsigned*)(lds + 1024)) / (GRID / 16) + 1u;
        comm_role(P, lds, call, b);
    } else gemm_role(P, lds, b & 7, (b - NCOMM) >> 3);
}

}

extern "C" void kernel_launch(int rank, int nranks, ncclComm_t comm, hipStream_t stream,
                              void* const* d_in, const int* in_sizes, int n_in,
                              void* d_out, int out_size,
                              const int* in_dtypes, int out_dtype,
                              void* const* d_out_all,
                              void* const* d_ws_all, size_t ws_bytes) {
    (void)comm; (void)in_sizes; (void)n_in; (void)d_out; (void)out_size; (void)in_dtypes; (void)out_dtype;
    if (nranks != NR || (unsigned)rank >= (unsigned)NR || ws_bytes < WS_NEED) {
        fprintf(stderr, "unsupported configuration\n");
        return;
    }
    Params P;
    P.x = (const u16*)d_in[0]; P.w = (const u16*)d_in[1];
    for (int q = 0; q < NR; ++q) { const int p = (rank + 1 + q) % NR; P.ws[q] = (char*)d_ws_all[p]; P.out[q] = (char*)d_out_all[p]; }
    P.rank = rank;
    hipLaunchKernelGGL(fused_kernel, dim3(GRID), dim3(512), 0, stream, P);
}
```

```cpp
#include <hip/hip_runtime.h>
#include <rccl/rccl.h>
#include <cstdio>

namespace {

typedef unsigned short u16;
typedef short bf16x8 __attribute__((ext_vector_type(8)));
typedef float f32x4 __attribute__((ext_vector_type(4)));
typedef unsigned u32x4 __attribute__((ext_vector_type(4)));
#define LAS __attribute__((address_space(3)))
#define GAS __attribute__((address_space(1)))

constexpr int NR = 8;
constexpr int NN = 32768, KSH = 512;
constexpr int GRID = 256;
constexpr int NCOMM = 24;
constexpr int NGEMM = GRID - NCOMM;
constexpr int WG_PER_GRP = NGEMM / 8;
constexpr int RC = 14, NROUND = 6;
constexpr int NCT = RC * NROUND + 2;
constexpr int NOWN = 6;
constexpr int OWNB = NOWN * 512;
constexpr int SH0 = 8 * NOWN;
static_assert(WG_PER_GRP == 29 && NCT == 128 - 7 * NOWN, "tile map");
#define PD 21
#define NS 5
#define NSLOT 18
#define CHU 8
#define QAHEAD 1
#define NHOLE 6
#define QS 4
#define SL0 3
#define NSL1 2
#define DLY_L 300u
#define NEARLY 72
#define ST_AUX_OUT 18
#define ST_AUX 16
#define RUN0 2
#define RUNM 1
#define RUN1 4
#define RUN_A 70
#define RUN_B 150
#define RUN_DIV 72
#define EPI_AUX 2
static_assert(NS >= 1 && NS <= 5 && PD >= 7 && PD <= 63 && CHU >= 2 && (64 / QS) % CHU == 0, "waves of a communication workgroup");
static_assert(NHOLE >= 0 && NHOLE <= NOWN && (QS == 1 || QS == 2 || QS == 4) && NSL1 >= 0 && NSL1 <= 4, "pushed items");

constexpr unsigned OFF_TMO = 128;
constexpr unsigned OFF_PROG = 256;
constexpr unsigned OFF_WQ = 512;
constexpr unsigned OFF_GP = 1024;
constexpr unsigned OFF_TICKET = 2048;
constexpr unsigned OFF_TD = 4096;
constexpr unsigned OFF_FO = 8192;
constexpr unsigned OFF_FL = 16384;
constexpr unsigned OFF_XA = 1u << 20;
constexpr unsigned PANEL_BYTES = 2u << 20;
constexpr unsigned OFF_WB = OFF_XA + 16u * PANEL_BYTES;
constexpr unsigned OFF_SEND = OFF_WB + (unsigned)NCT * PANEL_BYTES;
constexpr unsigned OFF_OS = OFF_SEND + 8u * NOWN * 262144u;
constexpr unsigned WS_NEED = OFF_OS + 4096u * OWNB;
constexpr unsigned OUT_BYTES = 256u << 20;

constexpr int N_R0 = 120, N_R1 = N_R0 + 4 * RC;
constexpr int N_XW = 64 + NCT * 4;
constexpr int HOLE = 101 - NHOLE;
constexpr int K3N = 30 - NHOLE;
constexpr int V_R0 = N_R0 - NHOLE;
constexpr int V_XW = N_XW - NHOLE;
constexpr int V_ALL = V_XW + OWNB / 16;
constexpr int NCHUNK = V_ALL * 64 / CHU;
constexpr int N_GRP = N_XW + 16;
static_assert(V_ALL * 64 + 256 < 65535 && NCHUNK < 32000, "progress fits 16 bits");
static_assert(OFF_FL + (N_GRP + 8) * 32 <= 65536, "flag table");
__host__ __device__ constexpr int ct_wi(int j) { return j >= RC ? j : (j & 1) * 7 + (j >> 1); }
__device__ __forceinline__ void item_decode(int n, bool& isx, int& idx, int& kb) {
    if (n < N_R0) {
        kb = n / 30; const int r = n - kb * 30;
        isx = r < 4 || (r >= 11 && r < 23);
        idx = r < 4 ? r : r < 11 ? 20 - 2 * r : r < 23 ? r - 7 : 2 * (r - 23) + 1;
    } else {
        const bool extra = n >= N_R1 && n < N_R1 + 8;
        const int m = n - (n < N_R1 ? N_R0 : N_R0 + 8), i = m / (4 * RC), rem = m - i * (4 * RC);
        kb = rem / RC; const int r = rem - kb * RC;
        isx = false;
        idx = RC * (i + 1) + (r < 7 ? 2 * r : 2 * (r - 7) + 1);
        if (extra) { idx = RC * NROUND + ((n - N_R1) >> 2); kb = (n - N_R1) & 3; }
    }
}
__device__ __forceinline__ int unit_need(int i, int ks, int g) {
    const int kb = ks >> 2, sl = (ks & 3) * QS >> 2;
    if (i == 0) {
        if (kb < SL0) return (30 * kb + (g < 4 ? 11 + 4 * g : 30)) * 64;
        if (kb < 3) return 30 * kb * 64 + (sl * 30 + (g < 4 ? 11 + 4 * g : 30)) * (64 / QS);
        return 90 * 64 + (sl * K3N + (g < 4 ? 11 - NHOLE + 4 * g : K3N)) * (64 / QS);
    }
    if (i == 1 && kb < NSL1) return (V_R0 + RC * kb) * 64 + (sl * RC + (g < 4 ? 7 : RC)) * (64 / QS);
    return (V_R0 + 4 * RC * (i - 1) + (i > 1 ? 8 : 0) + RC * kb + (g < 4 ? 7 : RC)) * 64;
}

constexpr unsigned long long DEADLINE_TICKS = 100000000ull;
constexpr int STAGE_BYTES = NSLOT * 7168 > 131072 ? NSLOT * 7168 : 131072;
constexpr int LDS_BYTES = STAGE_BYTES + 1024;

struct Params {
    const u16* x; const u16* w;
    char* ws[NR];
    char* out;
    int rank;
};

__device__ __forceinline__ int rfl(int v) { return __builtin_amdgcn_readfirstlane(v); }

__device__ __forceinline__ __amdgpu_buffer_rsrc_t mk_rsrc(const void* p, unsigned bytes) {
    return __builtin_amdgcn_make_buffer_rsrc(const_cast<void*>(p), 0, (int)bytes, 0x00020000);
}
__device__ __forceinline__ bool timed_out(unsigned long long t0, unsigned* tmo, unsigned call) {
    if (__builtin_amdgcn_s_memrealtime() - t0 > DEADLINE_TICKS) {
        __hip_atomic_store(tmo, call, __ATOMIC_RELAXED, __HIP_MEMORY_SCOPE_AGENT);
        return true;
    }
    return false;
}
__device__ __forceinline__ unsigned lds_rd(unsigned addr) {
    unsigned v; asm volatile("ds_read_b32 %0, %1\n\ts_waitcnt lgkmcnt(0)" : "=v"(v) : "v"(addr) : "memory"); return v;
}
__device__ __forceinline__ u32x4 lds_rd4(unsigned addr) {
    u32x4 v; asm volatile("ds_read_b128 %0, %1\n\ts_waitcnt lgkmcnt(0)" : "=v"(v) : "v"(addr) : "memory"); return v;
}
__device__ __forceinline__ void lds_wr4(unsigned addr, u32x4 v) { asm volatile("ds_write_b128 %0, %1" :: "v"(addr), "v"(v) : "memory"); }
__device__ __forceinline__ void lds_wr(unsigned addr, unsigned v) { asm volatile("ds_write_b32 %0, %1" :: "v"(addr), "v"(v) : "memory"); }

#define WAIT_V(n) asm volatile("s_waitcnt vmcnt(" #n ")" ::: "memory")
#define WAIT_L(n) asm volatile("s_waitcnt lgkmcnt(" #n ")" ::: "memory")
#define SCHED __builtin_amdgcn_sched_barrier(0)
#define BAR __builtin_amdgcn_s_barrier()
#define DRAIN_V() do { SCHED; __builtin_amdgcn_s_waitcnt(0x0F70); WAIT_V(0); SCHED; } while (0)
template <int N> __device__ __forceinline__ void wait_v() { asm volatile("s_waitcnt vmcnt(%0)" :: "n"(N) : "memory"); }

#define STAMP_DECL
#define STAMP() do {} while (0)
#define STAMP_DUMP() do {} while (0)

constexpr int AUX_SYS = 17;

__device__ __forceinline__ GAS char* lane_ws(const Params& P, int lane) {
    char* wsl = P.ws[0];
#pragma unroll
    for (int p = 1; p < NR; ++p) {
        char* t = P.ws[p];
        asm volatile("" : "+s"(t));
        wsl = (lane == p) ? t : wsl;
    }
    return (GAS char*)wsl;
}

constexpr unsigned M_OWN = 1, M_OUT = 2;
__device__ __forceinline__ u32x4 chunk_entry(int k, unsigned id, int rank) {
    if (id >= (unsigned)NCHUNK) return (u32x4){(unsigned)(k + 1) << 16 | 0xffffu, 0u, 0u, 0u};
    const int v = (int)id * CHU;
    int n = v >> 6, u = v & 63;
    if (n >= V_XW) {
        unsigned o = (unsigned)(v - V_XW * 64) + (unsigned)rank * (OWNB / 2u);
        if (o >= OWNB * 4u) o -= OWNB * 4u;
        return (u32x4){(unsigned)(k + 1) << 16 | (unsigned)v, OFF_OS + o * 1024u + M_OUT, o, (unsigned)N_XW};
    }
    if (n >= V_R0 + NSL1 * RC) n += NHOLE;
    else if (n >= V_R0) {
        const int v1 = v - V_R0 * 64, sl = v1 / (RC * 64 / QS), rem = v1 - sl * (RC * 64 / QS);
        n = N_R0 + sl / QS * RC + rem / (64 / QS); u = sl % QS * (64 / QS) + rem % (64 / QS);
    } else if (n >= 90) {
        const int v1 = v - 90 * 64, sl = v1 / (K3N * 64 / QS), rem = v1 - sl * (K3N * 64 / QS);
        n = 90 + rem / (64 / QS); u = sl * (64 / QS) + rem % (64 / QS);
        if (n >= HOLE) n += NHOLE;
    } else if (n >= SL0 * 30) {
        const int sl = v / (30 * 64 / QS), rem = v - sl * (30 * 64 / QS);
        n = sl / QS * 30 + rem / (64 / QS); u = sl % QS * (64 / QS) + rem % (64 / QS);
    }
    bool isx; int idx, kb;
    item_decode(n, isx, idx, kb);
    const bool own = !isx && ct_wi(idx) < NOWN;
    const unsigned lb = (isx ? OFF_XA : OFF_WB) + (unsigned)idx * PANEL_BYTES + (unsigned)kb * 524288u + u * 1024u;
    const unsigned rb = own ? OFF_SEND + (unsigned)(rank * NOWN + ct_wi(idx)) * 262144u + (unsigned)kb * 65536u + u * 1024u + M_OWN : lb;
    return (u32x4){(unsigned)(k + 1) << 16 | (unsigned)v, rb, lb, (unsigned)n};
}
__device__ __forceinline__ unsigned out_off(unsigned o, int lane) {
    const unsigned row = o / (OWNB / 1024u), jj = o - row * (OWNB / 1024u);
    return row * 65536u + jj * 1024u + lane * 16u;
}
__device__ __forceinline__ int lead_batches(unsigned cnt) {
    int r = __builtin_amdgcn_readlane((int)cnt, 0) * NS;
#pragma unroll
    for (int s = 1; s < NS; ++s) r = min(r, __builtin_amdgcn_readlane((int)cnt, s) * NS + s);
    return r;
}

__device__ __forceinline__ void comm_role(const Params& P, LAS unsigned char* lds, unsigned call, int c) {
    const int tid = threadIdx.x, wid = rfl(tid >> 6), lane = tid & 63;
    const int rank = P.rank;
    unsigned* tmo = (unsigned*)(P.ws[7] + OFF_TMO);
    const unsigned ctl = (unsigned)(uintptr_t)(lds + STAGE_BYTES);
    constexpr unsigned L_AVAIL = 0, L_FULL = 4, L_USED = 12, L_FREED = 32, L_DONE = 64, L_CQ = 512;
    constexpr int NCQ = 32;
    constexpr unsigned BADV = 0xffffu;
    const unsigned pbase = call << 16;
    auto chunk = [&](int k, unsigned& rb, unsigned& lb, int& n) -> int {
        const u32x4 e = lds_rd4(ctl + L_CQ + (k & (NCQ - 1)) * 16);
        const int e0 = rfl((int)e.x);
        rb = (unsigned)rfl((int)e.y); lb = (unsigned)rfl((int)e.z); n = rfl((int)e.w);
        return (e0 >> 16) == k + 1 ? (e0 & 0xffff) : -1;
    };
    if (tid < 64) lds_wr4(ctl + tid * 16, tid == L_CQ / 16 ? chunk_entry(0, (unsigned)c, rank) : (u32x4){0u, 0u, 0u, 0u});
    WAIT_L(0);
    __syncthreads();
    STAMP_DECL;
    STAMP();
    if (wid == 0) {
        __builtin_amdgcn_s_setprio(3);
        int avail = 0, freed = 0, i = 0;
        __amdgpu_buffer_rsrc_t r[7];
#pragma unroll
        for (int q = 0; q < 7; ++q) r[q] = mk_rsrc(P.ws[q], WS_NEED);
#pragma unroll 1
        for (int k = 0;; ++k) {
            int unit0, n; unsigned rb, lb;
            while ((unit0 = chunk(k, rb, lb, n)) < 0) __builtin_amdgcn_s_sleep(1);
            if (unit0 == 0xffff) break;
            lds_wr(ctl + L_USED, (unsigned)(k + 1));
            const unsigned stride = (rb & 3u) ? 0u : 65536u;
            while (n >= avail) {
                avail = rfl((int)lds_rd(ctl + L_AVAIL));
                if (n >= avail) { WAIT_V(0); lds_wr(ctl + L_FULL, (unsigned)i); __builtin_amdgcn_s_sleep(1); }
            }
#pragma unroll 1
            for (int u = 0; u < CHU; ++u, ++i) {
                const unsigned off = (rb & ~3u) + u * 1024u + lane * 16u;
                while (i - NSLOT >= freed) { freed = lead_batches(lds_rd(ctl + L_FREED + (lane & 7) * 4)); if (i - NSLOT >= freed) __builtin_amdgcn_s_sleep(1); }
                LAS unsigned char* slot = lds + (unsigned)i % NSLOT * 7168u;
                SCHED;
#pragma unroll
                for (int q = 0; q < 7; ++q) {
                    wait_v<PD - 1>();
                    __builtin_amdgcn_raw_ptr_buffer_load_lds(r[q], (LAS void*)(slot + q * 1024), 16, off + (unsigned)((rank + 1 + q) & 7) * stride, 0, 0, AUX_SYS);
                    SCHED;
                    if (q == (PD - 1) % 7 && i >= (PD - 1) / 7) lds_wr(ctl + L_FULL, (unsigned)(i - (PD - 1) / 7));
                }
            }
        }
        STAMP();
#pragma unroll
        for (int k = (PD - 1) / 7; k >= 0; --k) {
            if (k == 8) wait_v<56>(); else if (k == 7) wait_v<49>(); else if (k == 6) wait_v<42>(); else if (k == 5) wait_v<35>(); else if (k == 4) wait_v<28>();
            else if (k == 3) wait_v<21>(); else if (k == 2) wait_v<14>(); else if (k == 1) wait_v<7>(); else wait_v<0>();
            if (i >= k) lds_wr(ctl + L_FULL, (unsigned)(i - k));
        }
        STAMP();
        STAMP();
        STAMP_DUMP();
    } else if (wid == 1) {
        const unsigned* fl = (const unsigned*)(P.ws[7] + OFF_FL) + lane;
        unsigned* prog = (unsigned*)(P.ws[7] + OFF_PROG);
        const unsigned long long t0 = __builtin_amdgcn_s_memrealtime();
        int pos = 0, mine = 0, all = c < 8 ? 0 : V_XW * 64;
        bool pushed = c >= 8;
        while (pos < N_GRP || mine < V_XW * 64 || all < V_XW * 64) {
            unsigned v = call, pv = 0, hv = call;
            if (pos < N_GRP) v = __hip_atomic_load(fl + pos * 8, __ATOMIC_RELAXED, __HIP_MEMORY_SCOPE_SYSTEM);
            if (!pushed) hv = __hip_atomic_load(fl + HOLE * 8, __ATOMIC_RELAXED, __HIP_MEMORY_SCOPE_SYSTEM);
            if (all < V_XW * 64) pv = __hip_atomic_load(prog + min(lane & 31, NCOMM - 1), __ATOMIC_RELAXED, __HIP_MEMORY_SCOPE_AGENT);
            if (mine < V_XW * 64) {
                const int lead = lead_batches(lds_rd(ctl + L_DONE + (lane & 7) * 4)), k = lead / CHU;
                unsigned rb, lb; int n;
                const int unit0 = chunk(k, rb, lb, n);
                if (unit0 >= 0) {
                    const int m = min(unit0 + lead - k * CHU, 0xffff);
                    if (m != mine && lane == 0) __hip_atomic_store(prog + c, pbase + (unsigned)m, __ATOMIC_RELAXED, __HIP_MEMORY_SCOPE_AGENT);
                    mine = m;
                }
            }
            int adv = 0;
            if (pos < N_GRP) {
                const int it = pos + (lane >> 3);
                const unsigned long long miss = __ballot((int)(v - call) < 0 && it < N_GRP && (it < HOLE || it >= HOLE + NHOLE));
                adv = miss ? (int)(__builtin_ctzll(miss) >> 3) : 8;
                pos += adv;
                if (adv) lds_wr(ctl + L_AVAIL, (unsigned)(pos <= N_XW ? pos : pos < N_GRP ? N_XW : N_XW + 1));
            }
            if (!pushed) pushed = __all((int)(hv - call) >= 0 || lane >= NHOLE * 8);
            if (all < V_XW * 64) {
                int m = (int)(pv - pbase);
#pragma unroll
                for (int o = 16; o >= 1; o >>= 1) m = min(m, __shfl_xor(m, o));
                m = rfl(m);
                if (!pushed) m = min(m, 90 * 64);
                if (m > all) {
                    if (lane == 0) __hip_atomic_store((unsigned*)(P.ws[7] + OFF_GP + c * 128), pbase + (unsigned)m, __ATOMIC_RELAXED, __HIP_MEMORY_SCOPE_AGENT);
                    all = m;
                }
            }
            if (timed_out(t0, tmo, call)) { lds_wr(ctl + L_AVAIL, BADV); break; }
            if (!adv) __builtin_amdgcn_s_sleep(4);
        }
        if (lane < NR - 1)
            __hip_atomic_store((GAS unsigned*)(lane_ws(P, lane) + OFF_FO + c * 128 + rank * 4), call, __ATOMIC_RELAXED, __HIP_MEMORY_SCOPE_SYSTEM);
        const unsigned* fo = (const unsigned*)(P.ws[7] + OFF_FO + c * 128) + (lane & 7);
        for (;;) {
            const unsigned v = __hip_atomic_load(fo, __ATOMIC_RELAXED, __HIP_MEMORY_SCOPE_SYSTEM);
            if (__all((int)(v - call) >= 0 || (lane & 7) == rank)) break;
            if (timed_out(t0, tmo, call)) break;
            __builtin_amdgcn_s_sleep(16);
        }
    } else if (wid == 2) {
        unsigned* wq = (unsigned*)(P.ws[7] + OFF_WQ + (call & 1u) * 128u);
        if (c == 0 && lane == 0)
            __hip_atomic_store((unsigned*)(P.ws[7] + OFF_WQ + ((call + 1u) & 1u) * 128u), 0u, __ATOMIC_RELAXED, __HIP_MEMORY_SCOPE_AGENT);
        unsigned id = (unsigned)c, left = 0;
#pragma unroll 1
        for (int k = 1;; ++k) {
            while (rfl((int)lds_rd(ctl + L_USED)) + QAHEAD <= k) __builtin_amdgcn_s_sleep(1);
            if (left) { ++id; --left; }
            else {
                const int item = (int)id * CHU >> 6, rest = NCHUNK - (int)id;
                const unsigned m = item < RUN_A ? RUN0 : item < RUN_B ? RUNM : (unsigned)max(1, min(RUN1, rest / RUN_DIV));
                if (lane == 0) id = __hip_atomic_fetch_add(wq, m, __ATOMIC_RELAXED, __HIP_MEMORY_SCOPE_AGENT) + (unsigned)NCOMM;
                id = (unsigned)rfl((int)id);
                left = m - 1u;
            }
            if (id >= (unsigned)NCHUNK) {
                if (lane < 4) lds_wr(ctl + L_CQ + ((k + lane) & (NCQ - 1)) * 16, (unsigned)(k + lane + 1) << 16 | 0xffffu);
                break;
            }
            lds_wr4(ctl + L_CQ + (k & (NCQ - 1)) * 16, chunk_entry(k, id, rank));
        }
    } else if (wid < 3 + NS) {
        const int s = wid - 3;
        int full = 0;
        unsigned cnt = 0, done = 0;
        bool bad = false;
#pragma unroll 1
        for (int i = s;; i += NS) {
            const int k = i / CHU;
            int unit0, n; unsigned rb, lb;
            while ((unit0 = chunk(k, rb, lb, n)) < 0) {
                if (done != cnt) { DRAIN_V(); done = cnt; lds_wr(ctl + L_DONE + s * 4, done); }
                else __builtin_amdgcn_s_sleep(1);
            }
            if (unit0 == 0xffff) break;
            const bool isout = (rb & 3u) == M_OUT;
            const unsigned off = isout ? out_off(lb + (i - k * CHU), lane) : lb + (i - k * CHU) * 1024u + lane * 16u;
            const unsigned stride = isout ? OWNB : 65536u;
            const __amdgpu_buffer_rsrc_t r = mk_rsrc(isout ? P.out : P.ws[7], isout ? OUT_BYTES : WS_NEED);
            while (i >= full) {
                const unsigned cw = lds_rd(ctl + (lane & 1) * 4);
                full = __builtin_amdgcn_readlane((int)cw, 1);
                bad = (unsigned)__builtin_amdgcn_readlane((int)cw, 0) == BADV;
                if (i >= full) {
                    if (done != cnt) { DRAIN_V(); done = cnt; lds_wr(ctl + L_DONE + s * 4, done); }
                    else __builtin_amdgcn_s_sleep(1);
                }
            }
            u32x4 v[7];
#pragma unroll
            for (int q = 0; q < 7; ++q) v[q] = *(const LAS u32x4*)(lds + (unsigned)i % NSLOT * 7168u + q * 1024 + lane * 16);
            SCHED; WAIT_L(0); SCHED;
            ++cnt;
            lds_wr(ctl + L_FREED + s * 4, cnt);
#pragma unroll
            for (int q = 0; q < 7; ++q) {
                if (bad) v[q] = (u32x4){0x7fc07fc0u, 0x7fc07fc0u, 0x7fc07fc0u, 0x7fc07fc0u};
                if (ST_AUX == ST_AUX_OUT || !isout) __builtin_amdgcn_raw_buffer_store_b128(v[q], r, off + (unsigned)((rank + 1 + q) & 7) * stride, 0, ST_AUX);
                else __builtin_amdgcn_raw_buffer_store_b128(v[q], r, off + (unsigned)((rank + 1 + q) & 7) * stride, 0, ST_AUX_OUT);
            }
            SCHED;
            if (done + 1u != cnt) { wait_v<7>(); done = cnt - 1u; lds_wr(ctl + L_DONE + s * 4, done); }
        }
        DRAIN_V();
        lds_wr(ctl + L_DONE + s * 4, cnt);
    }
}

__device__ __forceinline__ unsigned take_ticket(const Params& P, int zv) {
    return __hip_atomic_fetch_add((unsigned*)(P.ws[7] + OFF_TICKET + (blockIdx.x & 15) * 128) + zv, 1u, __ATOMIC_RELAXED, __HIP_MEMORY_SCOPE_AGENT);
}
__device__ __forceinline__ unsigned stage_role(const Params& P, GAS char* wsl, unsigned ctl, int s0) {
    const int tid = threadIdx.x, sub = rfl(tid >> 6), lane = tid & 63;
    const int rank = P.rank;
    const int qi = 4 * (lane & 3) + ((lane & 15) >> 2);
    constexpr int NP8 = 7 * NHOLE, G = 30 + 7 * NOWN, S3 = 3 * G, S4 = S3 + 30, S5 = S4 + 7 * (NOWN - NHOLE), N_STAGE = NP8 + S5 + N_XW - N_R0;
    unsigned call = 0, tk = 0;
    int zv = 0;
    asm volatile("" : "+v"(zv));
    if (s0 >= NP8 + NEARLY) {
        const unsigned long long t0 = __builtin_amdgcn_s_memrealtime();
        while (__builtin_amdgcn_s_memrealtime() - t0 < DLY_L) __builtin_amdgcn_s_sleep(2);
    }
#pragma unroll 1
    for (int s1 = s0; s1 < N_STAGE; s1 += NGEMM) {
        const bool push = s1 < NP8;
        const int s = s1 - NP8;
        int n, q = 7;
        if (push) { const int jo = s1 / 7; q = s1 - jo * 7; n = HOLE + jo; }
        else if (s < S3) {
            const int kb = s / G, r = s - kb * G;
            if (r < 30) n = kb * 30 + r;
            else { const int jo = (r - 30) / 7; q = r - 30 - jo * 7; n = kb * 30 + 5 + jo; }
        } else if (s < S4) n = 90 + s - S3;
        else if (s < S5) { const int jo = (s - S4) / 7; q = s - S4 - jo * 7; n = 95 + jo; }
        else n = N_R0 + s - S5;
        const __amdgpu_buffer_rsrc_t rw = mk_rsrc(P.ws[push ? q : 7], WS_NEED);
        bool isx; int idx, kb;
        item_decode(n, isx, idx, kb);
        const int wi = ct_wi(idx);
        const bool own = !isx && wi < NOWN;
        u32x4 v[8];
        if (isx) {
            const u16* src = P.x + (size_t)(idx * 256 + sub * 32 + (lane & 15)) * KSH + kb * 128 + (lane >> 4) * 8;
#pragma unroll
            for (int t = 0; t < 8; ++t) v[t] = *(const u32x4*)(src + (t >> 2) * 16 * KSH + (t & 3) * 32);
            SCHED;
            if (s1 == s0 && tid == 0) tk = take_ticket(P, zv);
            SCHED;
            const unsigned dst = OFF_XA + (unsigned)idx * PANEL_BYTES + (unsigned)(kb * 8 + rank) * 65536u + sub * 2048u + lane * 16u;
#pragma unroll
            for (int t = 0; t < 8; ++t)
                __builtin_amdgcn_raw_buffer_store_b128(v[t], rw, dst + (t >> 2) * 1024 + (t & 3) * 16384, 0, AUX_SYS);
        } else {
            const int wr = sub & 1;
            const int p = (rank + 1 + q) & 7;
            const int col = own ? p * (OWNB / 2) + wi * 256 : (SH0 - NOWN + wi) * 256;
            const u16* src = P.w + (size_t)(kb * 128 + (sub >> 1) * 32 + (lane >> 4) * 8) * NN + col + wr * 128 + qi * 8;
#pragma unroll
            for (int t = 0; t < 8; ++t) v[t] = *(const u32x4*)(src + (size_t)t * NN);
            SCHED;
            if (s1 == s0 && tid == 0) tk = take_ticket(P, zv);
            SCHED;
            const unsigned base = q < 7 && !push ? OFF_SEND + (unsigned)(p * NOWN + wi) * 262144u + (unsigned)kb * 65536u
                                                 : OFF_WB + (unsigned)idx * PANEL_BYTES + (unsigned)(kb * 8 + rank) * 65536u;
            const unsigned dst = base + (sub >> 1) * 16384u + wr * 4096u + lane * 16u;
#pragma unroll
            for (int t = 0; t < 8; ++t) {
                u32x4 o;
#pragma unroll
                for (int e = 0; e < 4; ++e) o[e] = __builtin_amdgcn_perm(v[2 * e + 1][t >> 1], v[2 * e][t >> 1], (t & 1) ? 0x07060302u : 0x05040100u);
                __builtin_amdgcn_raw_buffer_store_b128(o, rw, dst + (t >> 2) * 8192 + (t & 3) * 1024, 0, AUX_SYS);
            }
        }
        DRAIN_V();
        if (s1 == s0 && tid == 0) { lds_wr(ctl + 4, tk); WAIT_L(0); }
        BAR;
        if (s1 == s0) call = (unsigned)rfl((int)lds_rd(ctl + 4)) / (GRID / 16) + 1u;
        if (sub == 0 && (own ? lane == q : lane < NR))
            __hip_atomic_store((GAS unsigned*)(wsl + OFF_FL + n * 32 + rank * 4), call, __ATOMIC_RELAXED, __HIP_MEMORY_SCOPE_SYSTEM);
        SCHED;
    }
    return call;
}

__device__ __forceinline__ unsigned cvt_pk_bf16(float lo, float hi) {
    unsigned r; asm volatile("v_cvt_pk_bf16_f32 %0, %1, %2" : "=v"(r) : "v"(lo), "v"(hi)); return r;
}
__device__ __forceinline__ float silu(float y) {
    const float t = __builtin_amdgcn_fmed3f(y, -60.0f, 60.0f);
    const float e = __builtin_amdgcn_exp2f(t * -1.44269504088896341f);
    return y * __builtin_amdgcn_rcpf(1.0f + e);
}

__device__ __forceinline__ void gemm_role(const Params& P, LAS unsigned char* lds, int g, int lw) {
    const int tid = threadIdx.x, wid = rfl(tid >> 6), lane = tid & 63, wr = wid >> 2, wc = wid & 3, fr = lane & 15, fq = lane >> 4;
    const int rank = P.rank;
    char* const ws = P.ws[7];
    unsigned* tmo = (unsigned*)(ws + OFF_TMO);
    const unsigned ldsw = (unsigned)wid * 1024u;
    volatile LAS int* lknown = (volatile LAS int*)(lds + STAGE_BYTES);
    int known = 0;
    STAMP_DECL;
    STAMP();
    GAS char* wsl = lane_ws(P, lane);
    asm volatile("" : "+v"(wsl));
    const unsigned call = stage_role(P, wsl, (unsigned)(uintptr_t)(lds + STAGE_BYTES), lw * 8 + g);
    STAMP();

    auto ready = [&](int i, int ks) {
        const int need = lw < 28 ? unit_need(i, ks, g) : (V_R0 + 4 * RC + 4 + (i >> 1) * 4) * 64;
        if ((unsigned)known >= (unsigned)need) return;
        if (wid == 0) {
            const unsigned* f = (const unsigned*)(ws + OFF_GP + g * 128);
            const unsigned long long t0 = __builtin_amdgcn_s_memrealtime();
            int m;
            for (;;) {
                m = rfl((int)(__hip_atomic_load(f, __ATOMIC_RELAXED, __HIP_MEMORY_SCOPE_AGENT) - (call << 16)));
                if (m >= need) break;
                if (timed_out(t0, tmo, call)) { m = -1; break; }
                __builtin_amdgcn_s_sleep(8);
            }
            if (lane == 0) lknown[0] = m;
            if (known == 0) __builtin_amdgcn_fence(__ATOMIC_ACQUIRE, "");
            WAIT_V(0); WAIT_L(0);
        }
        asm volatile("" ::: "memory"); BAR; asm volatile("" ::: "memory");
        known = rfl(lknown[0]);
    };

    const int ntile = lw < 28 ? NROUND : 4;
    int j = lw < 28 ? 2 * (lw >> 2) + (g >> 2) : RC * NROUND;
    int mb = 4 * (g & 3) + (lw < 28 ? lw & 3 : 2 * (g >> 2));
    f32x4 acc[2][2][4][2];
    bf16x8 Ar[2][2][4], Br[2][2][2];
    const __amdgpu_buffer_rsrc_t rw = mk_rsrc(ws, WS_NEED);
    unsigned gv = tid * 16u;
    unsigned la = (unsigned)(uintptr_t)lds + wr * 4096u + lane * 16u, lb = (unsigned)(uintptr_t)lds + 65536u + wc * 2048u + lane * 16u;
    asm volatile("" : "+v"(gv), "+v"(la), "+v"(lb));
    unsigned cA = OFF_WB + (unsigned)j * PANEL_BYTES, cB = OFF_XA + (unsigned)mb * PANEL_BYTES;
#define GL(b, i, sa, sb) __builtin_amdgcn_raw_ptr_buffer_load_lds(rw, (LAS void*)(lds + ((i) & 1) * 65536 + (b) * 16384 + ((i) >> 1) * 8192 + ldsw), 16, gv, \
        (((i) & 1) ? (sb) : (sa)) + (b) * 65536u + ((i) >> 1) * 8192u, 0, 0)
#define RDA(r, b, h, m) Ar[r][h][m] = *(const LAS bf16x8*)(uintptr_t)(la + (b) * 16384 + (h) * 8192 + (m) * 1024)
#define RDB(r, b, h, n) Br[r][h][n] = *(const LAS bf16x8*)(uintptr_t)(lb + (b) * 16384 + (h) * 8192 + (n) * 1024)
#define RD(r, b, i) do { if ((i) < 4) RDB(r, b, (i) >> 1, (i) & 1); else RDA(r, b, ((i) - 4) >> 2, ((i) - 4) & 3); } while (0)
#define ACC2(i) acc[(i) >> 4][((i) >> 1) & 1][((i) >> 2) & 3][(i) & 1]
#define MF2(z, r, i) do { \
        if (z) asm volatile("v_mfma_f32_16x16x32_bf16 %0, %1, %2, 0" : "=&v"(ACC2(i)) : "v"(Ar[r][(i) >> 4][((i) >> 2) & 3]), "v"(Br[r][((i) >> 1) & 1][(i) & 1])); \
        else asm volatile("v_mfma_f32_16x16x32_bf16 %0, %1, %2, %0" : "+v"(ACC2(i)) : "v"(Ar[r][(i) >> 4][((i) >> 2) & 3]), "v"(Br[r][((i) >> 1) & 1][(i) & 1])); } while (0)
#define GPOS 1
#define GSTR 8
#define RPOS 0
#define RSTR 2
#define STEP(b, z, vw, sa, sb) do { \
        SCHED; wait_v<vw>(); __builtin_amdgcn_s_waitcnt(0xC07F); BAR; SCHED; \
        _Pragma("unroll") for (int i = 0; i < 32; ++i) { \
            MF2(z, (b) & 1, i); SCHED; \
            if (i >= GPOS && (i - GPOS) % GSTR == 0 && (i - GPOS) / GSTR < 4) { GL(b, (i - GPOS) / GSTR, sa, sb); SCHED; } \
            if (i >= RPOS && (i - RPOS) % RSTR == 0 && (i - RPOS) / RSTR < 12) { RD(((b) + 1) & 1, ((b) + 1) & 3, (i - RPOS) / RSTR); SCHED; } \
        } } while (0)
    ready(0, 0);
    STAMP();
#pragma unroll
    for (int b = 0; b < 4; ++b)
#pragma unroll
        for (int i = 0; i < 4; ++i) GL(b, i, cA, cB);
    SCHED; WAIT_V(0); BAR; SCHED;
#pragma unroll
    for (int i = 0; i < 12; ++i) RD(0, 0, i);
    const __amdgpu_buffer_rsrc_t ro = mk_rsrc(P.out, OUT_BYTES);
#pragma unroll 1
    for (int ui = 0; ui < ntile; ++ui) {
        const bool has_next = ui + 1 < ntile;
        const int nj = !has_next ? j : lw < 28 ? j + RC : j + (ui & 1), nmb = lw < 28 ? mb : mb ^ 1;
        const unsigned nA = OFF_WB + (unsigned)nj * PANEL_BYTES, nB = OFF_XA + (unsigned)nmb * PANEL_BYTES;
        STEP(0, true, 24, cA + 262144u, cB + 262144u); STEP(1, false, 24, cA + 262144u, cB + 262144u); STEP(2, false, 24, cA + 262144u, cB + 262144u);
        STEP(3, false, 8, cA + 262144u, cB + 262144u);
#pragma unroll 1
        for (int t = 2; t <= 32; ++t) {
            const bool last = t == 32;
            const unsigned so = (unsigned)(t >> 3) * 524288u + (unsigned)((t >> 1) & 3) * 16384u + (unsigned)(t & 1) * 262144u;
            const unsigned sa = last ? nA : cA + so, sb = last ? nB : cB + so;
            if (!(t & 1)) {
                if (!last) { ready(ui, t >> 1); if (!(t & 7)) STAMP(); }
                else if (has_next) { STAMP(); ready(ui + 1, 0); STAMP(); }
            }
            STEP(0, false, 8, sa, sb); STEP(1, false, 8, sa, sb); STEP(2, false, 8, sa, sb); STEP(3, false, 8, sa, sb);
        }
        asm volatile("s_nop 15\n\ts_nop 15" ::: "memory");
        SCHED;
        STAMP();
        {
            const int wi = ct_wi(j);
            const bool own = wi < NOWN;
            const unsigned row = (unsigned)(mb * 256 + wc * 32 + fr), cb = (unsigned)(wi * 256 + wr * 128 + fq * 8) * 2u;
            const unsigned obase = row * 65536u + cb + (own ? (unsigned)rank * OWNB : (SH0 - NOWN) * 512u);
            const unsigned pbase = OFF_OS + row * OWNB + cb;
            const bool bad = known < 0;
            {
#pragma unroll
                for (int bj = 0; bj < 2; ++bj)
#pragma unroll
                    for (int n = 0; n < 2; ++n)
#pragma unroll
                        for (int r = 0; r < 4; ++r) {
                            u32x4 o;
                            o.x = cvt_pk_bf16(silu(acc[0][bj][0][n][r]), silu(acc[0][bj][1][n][r]));
                            o.y = cvt_pk_bf16(silu(acc[0][bj][2][n][r]), silu(acc[0][bj][3][n][r]));
                            o.z = cvt_pk_bf16(silu(acc[1][bj][0][n][r]), silu(acc[1][bj][1][n][r]));
                            o.w = cvt_pk_bf16(silu(acc[1][bj][2][n][r]), silu(acc[1][bj][3][n][r]));
                            if (bad) o = (u32x4){0x7fc07fc0u, 0x7fc07fc0u, 0x7fc07fc0u, 0x7fc07fc0u};
                            __builtin_amdgcn_raw_buffer_store_b128(o, ro, obase + r * 64u + (bj * 128 + n * 16) * 65536u, 0, EPI_AUX);
                            if (own) __builtin_amdgcn_raw_buffer_store_b128(o, rw, pbase + r * 64u + (bj * 128 + n * 16) * (unsigned)OWNB, 0, AUX_SYS);
                        }
            }
            if (own) {
                WAIT_V(0);
                unsigned old = 0;
                if (lane == 0) old = __hip_atomic_fetch_add((unsigned*)(ws + OFF_TD + mb * 128), 1u, __ATOMIC_RELAXED, __HIP_MEMORY_SCOPE_AGENT);
                old = (unsigned)rfl((int)old) + 1u;
                if (old % (8u * NOWN) == 0u && lane < NR)
                    __hip_atomic_store((GAS unsigned*)(wsl + OFF_FL + (N_XW + mb) * 32 + rank * 4), call, __ATOMIC_RELAXED, __HIP_MEMORY_SCOPE_SYSTEM);
            }
        }
        SCHED;
        STAMP();
        cA = nA; cB = nB; j = nj; mb = nmb;
    }
    WAIT_V(0);
    BAR;
    if (wid == 0) {
        unsigned old = 0;
        if (lane == 0) old = __hip_atomic_fetch_add((unsigned*)(ws + OFF_GP + g * 128 + 64), 1u, __ATOMIC_RELAXED, __HIP_MEMORY_SCOPE_AGENT);
        if (((unsigned)rfl((int)old) + 1u) % WG_PER_GRP == 0u) __builtin_amdgcn_fence(__ATOMIC_RELEASE, "agent");
    }
    STAMP();
    STAMP_DUMP();
}

__global__ __launch_bounds__(512, 2) void fused_kernel(Params P) {
    __shared__ __attribute__((aligned(16))) unsigned char smem[LDS_BYTES];
    LAS unsigned char* lds = (LAS unsigned char*)smem;
    const int b = blockIdx.x;
    if (b < NCOMM) {
        if (threadIdx.x == 0) *(LAS unsigned*)(lds + 1024) = take_ticket(P, 0);
        __syncthreads();
        const unsigned call = (unsigned)rfl((int)*(LAS unsigned*)(lds + 1024)) / (GRID / 16) + 1u;
        comm_role(P, lds, call, b);
    } else gemm_role(P, lds, b & 7, (b - NCOMM) >> 3);
}

}

extern "C" void kernel_launch(int rank, int nranks, ncclComm_t comm, hipStream_t stream,
                              void* const* d_in, const int* in_sizes, int n_in,
                              void* d_out, int out_size,
                              const int* in_dtypes, int out_dtype,
                              void* const* d_out_all,
                              void* const* d_ws_all, size_t ws_bytes) {
    (void)comm; (void)in_sizes; (void)n_in; (void)d_out_all; (void)out_size; (void)in_dtypes; (void)out_dtype;
    if (nranks != NR || (unsigned)rank >= (unsigned)NR || ws_bytes < WS_NEED) {
        fprintf(stderr, "unsupported configuration\n");
        return;
    }
    Params P;
    P.x = (const u16*)d_in[0]; P.w = (const u16*)d_in[1];
    for (int q = 0; q < NR; ++q) P.ws[q] = (char*)d_ws_all[(rank + 1 + q) % NR];
    P.out = (char*)d_out;
    P.rank = rank;
    hipLaunchKernelGGL(fused_kernel, dim3(GRID), dim3(512), 0, stream, P);
}
```

```cpp
#include <hip/hip_runtime.h>
#include <rccl/rccl.h>
#include <cstdio>

namespace {

typedef unsigned short u16;
typedef short bf16x8 __attribute__((ext_vector_type(8)));
typedef float f32x4 __attribute__((ext_vector_type(4)));
typedef unsigned u32x4 __attribute__((ext_vector_type(4)));
#define LAS __attribute__((address_space(3)))
#define GAS __attribute__((address_space(1)))

constexpr int NR = 8;
constexpr int NN = 32768, KSH = 512;
constexpr int GRID = 256;
constexpr int NCOMM = 24;
constexpr int NGEMM = GRID - NCOMM;
constexpr int WG_PER_GRP = NGEMM / 8;
constexpr int RC = 14, NROUND = 6;
constexpr int NCT = RC * NROUND + 2;
constexpr int NOWN = 6;
constexpr int OWNB = NOWN * 512;
constexpr int SH0 = 8 * NOWN;
static_assert(WG_PER_GRP == 29 && NCT == 128 - 7 * NOWN, "tile map");
#define PD 21
#define NS 5
#define NSLOT 18
#define AHEAD 5
#define NHOLE 6
#define QS 4
#define SL0 3
#define NSL1 2
#define DLY_L 300u
#define NEARLY 72
#define ST_AUX_OUT 18
#define ST_AUX 16
#define RUN0 16
#define RUNM 8
#define RUN1 32
#define RUNT 2
#define RUNF 8
#define RUN_A 70
#define RUN_B 150
#define RUN_DIV 72
#define GX_AUX 0
#define GW_AUX 0
#define LD_AUX 17
#define OS_AUX 17
#define EPI_AUX 18
static_assert(NS >= 1 && NS <= 5 && PD >= 7 && PD <= 63 && RUN0 <= 64 && RUN1 <= 64, "waves of a communication workgroup");
static_assert(NHOLE >= 0 && NHOLE <= NOWN && (QS == 1 || QS == 2 || QS == 4) && NSL1 >= 0 && NSL1 <= 4, "pushed items");

constexpr unsigned OFF_TMO = 128;
constexpr unsigned OFF_PROG = 256;
constexpr unsigned OFF_WQ = 512;
constexpr unsigned OFF_GP = 1024;
constexpr unsigned OFF_TICKET = 2048;
constexpr unsigned OFF_TD = 4096;
constexpr unsigned OFF_FO = 8192;
constexpr unsigned OFF_FL = 16384;
constexpr unsigned OFF_XA = 1u << 20;
constexpr unsigned PANEL_BYTES = 2u << 20;
constexpr unsigned OFF_WB = OFF_XA + 16u * PANEL_BYTES;
constexpr unsigned OFF_SEND = OFF_WB + (unsigned)NCT * PANEL_BYTES;
constexpr unsigned OFF_OS = OFF_SEND + 8u * NOWN * 262144u;
constexpr unsigned WS_NEED = OFF_OS + 4096u * OWNB;
constexpr unsigned OUT_BYTES = 256u << 20;

constexpr int N_R0 = 120, N_R1 = N_R0 + 4 * RC;
constexpr int N_XW = 64 + NCT * 4;
constexpr int HOLE = 101 - NHOLE;
constexpr int K3N = 30 - NHOLE;
constexpr int V_R0 = N_R0 - NHOLE;
constexpr int V_XW = N_XW - NHOLE;
constexpr int V_ALL = V_XW + OWNB / 16;
constexpr int NUNIT = V_ALL * 64;
constexpr int N_GRP = N_XW + 16;
static_assert(NUNIT + 256 < 65535, "progress fits 16 bits");
static_assert(OFF_FL + (N_GRP + 8) * 32 <= 65536, "flag table");
__host__ __device__ constexpr int ct_wi(int j) { return j >= RC ? j : (j & 1) * 7 + (j >> 1); }
__device__ __forceinline__ void item_decode(int n, bool& isx, int& idx, int& kb) {
    if (n < N_R0) {
        kb = n / 30; const int r = n - kb * 30;
        isx = r < 4 || (r >= 11 && r < 23);
        idx = r < 4 ? r : r < 11 ? 20 - 2 * r : r < 23 ? r - 7 : 2 * (r - 23) + 1;
    } else {
        const bool extra = n >= N_R1 && n < N_R1 + 8;
        const int m = n - (n < N_R1 ? N_R0 : N_R0 + 8), i = m / (4 * RC), rem = m - i * (4 * RC);
        kb = rem / RC; const int r = rem - kb * RC;
        isx = false;
        idx = RC * (i + 1) + (r < 7 ? 2 * r : 2 * (r - 7) + 1);
        if (extra) { idx = RC * NROUND + ((n - N_R1) >> 2); kb = (n - N_R1) & 3; }
    }
}
__device__ __forceinline__ int unit_need(int i, int ks, int g) {
    const int kb = ks >> 2, sl = (ks & 3) * QS >> 2;
    if (i == 0) {
        if (kb < SL0) return (30 * kb + (g < 4 ? 11 + 4 * g : 30)) * 64;
        if (kb < 3) return 30 * kb * 64 + (sl * 30 + (g < 4 ? 11 + 4 * g : 30)) * (64 / QS);
        return 90 * 64 + (sl * K3N + (g < 4 ? 11 - NHOLE + 4 * g : K3N)) * (64 / QS);
    }
    if (i == 1 && kb < NSL1) return (V_R0 + RC * kb) * 64 + (sl * RC + (g < 4 ? 7 : RC)) * (64 / QS);
    return (V_R0 + 4 * RC * (i - 1) + (i > 1 ? 8 : 0) + RC * kb + (g < 4 ? 7 : RC)) * 64;
}

constexpr unsigned long long DEADLINE_TICKS = 100000000ull;
constexpr int STAGE_BYTES = NSLOT * 7168 > 131072 ? NSLOT * 7168 : 131072;
constexpr int LDS_BYTES = STAGE_BYTES + 4096;

struct Params {
    const u16* x; const u16* w;
    char* ws[NR];
    char* out;
    int rank;
};

__device__ __forceinline__ int rfl(int v) { return __builtin_amdgcn_readfirstlane(v); }

__device__ __forceinline__ __amdgpu_buffer_rsrc_t mk_rsrc(const void* p, unsigned bytes) {
    return __builtin_amdgcn_make_buffer_rsrc(const_cast<void*>(p), 0, (int)bytes, 0x00020000);
}
__device__ __forceinline__ bool timed_out(unsigned long long t0, unsigned* tmo, unsigned call) {
    if (__builtin_amdgcn_s_memrealtime() - t0 > DEADLINE_TICKS) {
        __hip_atomic_store(tmo, call, __ATOMIC_RELAXED, __HIP_MEMORY_SCOPE_AGENT);
        return true;
    }
    return false;
}
__device__ __forceinline__ unsigned lds_rd(unsigned addr) {
    unsigned v; asm volatile("ds_read_b32 %0, %1\n\ts_waitcnt lgkmcnt(0)" : "=v"(v) : "v"(addr) : "memory"); return v;
}
__device__ __forceinline__ u32x4 lds_rd4(unsigned addr) {
    u32x4 v; asm volatile("ds_read_b128 %0, %1\n\ts_waitcnt lgkmcnt(0)" : "=v"(v) : "v"(addr) : "memory"); return v;
}
__device__ __forceinline__ void lds_wr4(unsigned addr, u32x4 v) { asm volatile("ds_write_b128 %0, %1" :: "v"(addr), "v"(v) : "memory"); }
__device__ __forceinline__ void lds_wr(unsigned addr, unsigned v) { asm volatile("ds_write_b32 %0, %1" :: "v"(addr), "v"(v) : "memory"); }

#define WAIT_V(n) asm volatile("s_waitcnt vmcnt(" #n ")" ::: "memory")
#define WAIT_L(n) asm volatile("s_waitcnt lgkmcnt(" #n ")" ::: "memory")
#define SCHED __builtin_amdgcn_sched_barrier(0)
#define BAR __builtin_amdgcn_s_barrier()
#define DRAIN_V() do { SCHED; __builtin_amdgcn_s_waitcnt(0x0F70); WAIT_V(0); SCHED; } while (0)
template <int N> __device__ __forceinline__ void wait_v() { asm volatile("s_waitcnt vmcnt(%0)" :: "n"(N) : "memory"); }

#define STAMP_DECL
#define STAMP() do {} while (0)
#define STAMP_DUMP() do {} while (0)

constexpr int AUX_SYS = 17;

__device__ __forceinline__ GAS char* lane_ws(const Params& P, int lane) {
    char* wsl = P.ws[0];
#pragma unroll
    for (int p = 1; p < NR; ++p) {
        char* t = P.ws[p];
        asm volatile("" : "+s"(t));
        wsl = (lane == p) ? t : wsl;
    }
    return (GAS char*)wsl;
}

constexpr unsigned M_OWN = 1, M_OUT = 2;
__device__ __forceinline__ u32x4 unit_entry(int k, int v, int rank) {
    if (v >= NUNIT) return (u32x4){(unsigned)(k + 1) << 16 | 0xffffu, 0u, 0u, 0u};
    int n = v >> 6, u = v & 63;
    if (n >= V_XW) {
        unsigned o = (unsigned)(v - V_XW * 64) + (unsigned)rank * (OWNB / 2u);
        if (o >= OWNB * 4u) o -= OWNB * 4u;
        const unsigned row = o / (OWNB / 1024u);
        return (u32x4){(unsigned)(k + 1) << 16 | (unsigned)v, OFF_OS + o * 1024u + M_OUT, row * 65536u + (o - row * (OWNB / 1024u)) * 1024u, (unsigned)N_XW};
    }
    if (n >= V_R0 + NSL1 * RC) n += NHOLE;
    else if (n >= V_R0) {
        const int v1 = v - V_R0 * 64, sl = v1 / (RC * 64 / QS), rem = v1 - sl * (RC * 64 / QS);
        n = N_R0 + sl / QS * RC + rem / (64 / QS); u = sl % QS * (64 / QS) + rem % (64 / QS);
    } else if (n >= 90) {
        const int v1 = v - 90 * 64, sl = v1 / (K3N * 64 / QS), rem = v1 - sl * (K3N * 64 / QS);
        n = 90 + rem / (64 / QS); u = sl * (64 / QS) + rem % (64 / QS);
        if (n >= HOLE) n += NHOLE;
    } else if (n >= SL0 * 30) {
        const int sl = v / (30 * 64 / QS), rem = v - sl * (30 * 64 / QS);
        n = sl / QS * 30 + rem / (64 / QS); u = sl % QS * (64 / QS) + rem % (64 / QS);
    }
    bool isx; int idx, kb;
    item_decode(n, isx, idx, kb);
    const bool own = !isx && ct_wi(idx) < NOWN;
    const unsigned lb = (isx ? OFF_XA : OFF_WB) + (unsigned)idx * PANEL_BYTES + (unsigned)kb * 524288u + u * 1024u;
    const unsigned rb = own ? OFF_SEND + (unsigned)(rank * NOWN + ct_wi(idx)) * 262144u + (unsigned)kb * 65536u + u * 1024u + M_OWN : lb;
    return (u32x4){(unsigned)(k + 1) << 16 | (unsigned)v, rb, lb, (unsigned)n};
}
__device__ __forceinline__ int lead_batches(unsigned cnt) {
    int r = __builtin_amdgcn_readlane((int)cnt, 0) * NS;
#pragma unroll
    for (int s = 1; s < NS; ++s) r = min(r, __builtin_amdgcn_readlane((int)cnt, s) * NS + s);
    return r;
}

__device__ __forceinline__ void comm_role(const Params& P, LAS unsigned char* lds, unsigned call, int c) {
    const int tid = threadIdx.x, wid = rfl(tid >> 6), lane = tid & 63;
    const int rank = P.rank;
    unsigned* tmo = (unsigned*)(P.ws[7] + OFF_TMO);
    const unsigned ctl = (unsigned)(uintptr_t)(lds + STAGE_BYTES);
    constexpr unsigned L_AVAIL = 0, L_FULL = 4, L_USED = 12, L_FREED = 32, L_DONE = 64, L_DQ = 2048;
    constexpr int NDQ = 128;
    static_assert(2 * NS + NSLOT + AHEAD + 64 + NS <= NDQ, "ring of claimed units");
    constexpr unsigned BADV = 0xffffu;
    const unsigned pbase = call << 16;
    auto desc = [&](int k, unsigned& rb, unsigned& lb) -> int {
        const u32x4 e = lds_rd4(ctl + L_DQ + (k & (NDQ - 1)) * 16);
        const unsigned e0 = (unsigned)rfl((int)e.x);
        rb = (unsigned)rfl((int)e.y); lb = (unsigned)rfl((int)e.z);
        return (e0 >> 16) == (unsigned)(k + 1) ? (int)(e0 & 0xffffu) : -1;
    };
    if (tid < 256) lds_wr4(ctl + tid * 16, tid >= 128 && tid < 128 + RUNF ? unit_entry(tid - 128, c * RUNF + tid - 128, rank) : (u32x4){0u, 0u, 0u, 0u});
    WAIT_L(0);
    __syncthreads();
    STAMP_DECL;
    STAMP();
    if (wid == 0) {
        __builtin_amdgcn_s_setprio(3);
        int avail = 0, freed = 0, i = 0;
        __amdgpu_buffer_rsrc_t r[7];
#pragma unroll
        for (int q = 0; q < 7; ++q) r[q] = mk_rsrc(P.ws[q], WS_NEED);
        bool more = true;
#pragma unroll 1
        while (more) {
            const u32x4 e = lds_rd4(ctl + L_DQ + ((i + lane) & (NDQ - 1)) * 16);
            const int nv = (int)__builtin_ctzll(~__ballot((e.x >> 16) == (unsigned)(i + lane + 1)) | 1ull << 63);
            if (!nv) __builtin_amdgcn_s_sleep(1);
#pragma unroll 1
            for (int u = 0; u < nv; ++u, ++i) {
                const int unit0 = __builtin_amdgcn_readlane((int)e.x, u) & 0xffff, n = __builtin_amdgcn_readlane((int)e.w, u);
                const unsigned rb = (unsigned)__builtin_amdgcn_readlane((int)e.y, u);
                if (unit0 == 0xffff) { more = false; break; }
                lds_wr(ctl + L_USED, (unsigned)(i + 1));
                const unsigned stride = (rb & 3u) ? 0u : 65536u;
                while (n >= avail) {
                    avail = rfl((int)lds_rd(ctl + L_AVAIL));
                    if (n >= avail) { WAIT_V(0); lds_wr(ctl + L_FULL, (unsigned)i); __builtin_amdgcn_s_sleep(1); }
                }
                const unsigned off = (rb & ~3u) + lane * 16u;
                while (i - NSLOT >= freed) { freed = lead_batches(lds_rd(ctl + L_FREED + (lane & 7) * 4)); if (i - NSLOT >= freed) __builtin_amdgcn_s_sleep(1); }
                LAS unsigned char* slot = lds + (unsigned)i % NSLOT * 7168u;
                SCHED;
#pragma unroll
                for (int q = 0; q < 7; ++q) {
                    wait_v<PD - 1>();
                    __builtin_amdgcn_raw_ptr_buffer_load_lds(r[q], (LAS void*)(slot + q * 1024), 16, off + (unsigned)((rank + 1 + q) & 7) * stride, 0, 0, LD_AUX);
                    SCHED;
                    if (q == (PD - 1) % 7 && i >= (PD - 1) / 7) lds_wr(ctl + L_FULL, (unsigned)(i - (PD - 1) / 7));
                }
            }
        }
        STAMP();
#pragma unroll
        for (int k = (PD - 1) / 7; k >= 0; --k) {
            if (k == 8) wait_v<56>(); else if (k == 7) wait_v<49>(); else if (k == 6) wait_v<42>(); else if (k == 5) wait_v<35>(); else if (k == 4) wait_v<28>();
            else if (k == 3) wait_v<21>(); else if (k == 2) wait_v<14>(); else if (k == 1) wait_v<7>(); else wait_v<0>();
            if (i >= k) lds_wr(ctl + L_FULL, (unsigned)(i - k));
        }
        STAMP();
        STAMP();
        STAMP_DUMP();
    } else if (wid == 1) {
        const unsigned* fl = (const unsigned*)(P.ws[7] + OFF_FL) + lane;
        unsigned* prog = (unsigned*)(P.ws[7] + OFF_PROG);
        const unsigned long long t0 = __builtin_amdgcn_s_memrealtime();
        int pos = 0, mine = 0, all = c < 8 ? 0 : V_XW * 64;
        bool pushed = c >= 8;
        while (pos < N_GRP || mine < V_XW * 64 || all < V_XW * 64) {
            unsigned v = call, pv = 0, hv = call;
            if (pos < N_GRP) v = __hip_atomic_load(fl + pos * 8, __ATOMIC_RELAXED, __HIP_MEMORY_SCOPE_SYSTEM);
            if (!pushed) hv = __hip_atomic_load(fl + HOLE * 8, __ATOMIC_RELAXED, __HIP_MEMORY_SCOPE_SYSTEM);
            if (all < V_XW * 64) pv = __hip_atomic_load(prog + min(lane & 31, NCOMM - 1), __ATOMIC_RELAXED, __HIP_MEMORY_SCOPE_AGENT);
            if (mine < V_XW * 64) {
                unsigned rb, lb;
                const int m = desc(lead_batches(lds_rd(ctl + L_DONE + (lane & 7) * 4)), rb, lb);
                if (m > mine) {
                    if (lane == 0) __hip_atomic_store(prog + c, pbase + (unsigned)m, __ATOMIC_RELAXED, __HIP_MEMORY_SCOPE_AGENT);
                    mine = m;
                }
            }
            int adv = 0;
            if (pos < N_GRP) {
                const int it = pos + (lane >> 3);
                const unsigned long long miss = __ballot((int)(v - call) < 0 && it < N_GRP && (it < HOLE || it >= HOLE + NHOLE));
                adv = miss ? (int)(__builtin_ctzll(miss) >> 3) : 8;
                pos += adv;
                if (adv) lds_wr(ctl + L_AVAIL, (unsigned)(pos <= N_XW ? pos : pos < N_GRP ? N_XW : N_XW + 1));
            }
            if (!pushed) pushed = __all((int)(hv - call) >= 0 || lane >= NHOLE * 8);
            if (all < V_XW * 64) {
                int m = (int)(pv - pbase);
#pragma unroll
                for (int o = 16; o >= 1; o >>= 1) m = min(m, __shfl_xor(m, o));
                m = rfl(m);
                if (!pushed) m = min(m, 90 * 64);
                if (m > all) {
                    if (lane == 0) __hip_atomic_store((unsigned*)(P.ws[7] + OFF_GP + c * 128), pbase + (unsigned)m, __ATOMIC_RELAXED, __HIP_MEMORY_SCOPE_AGENT);
                    all = m;
                }
            }
            if (timed_out(t0, tmo, call)) { lds_wr(ctl + L_AVAIL, BADV); break; }
            if (!adv) __builtin_amdgcn_s_sleep(4);
        }
        if (lane < NR - 1)
            __hip_atomic_store((GAS unsigned*)(lane_ws(P, lane) + OFF_FO + c * 128 + rank * 4), call, __ATOMIC_RELAXED, __HIP_MEMORY_SCOPE_SYSTEM);
        const unsigned* fo = (const unsigned*)(P.ws[7] + OFF_FO + c * 128) + (lane & 7);
        for (;;) {
            const unsigned v = __hip_atomic_load(fo, __ATOMIC_RELAXED, __HIP_MEMORY_SCOPE_SYSTEM);
            if (__all((int)(v - call) >= 0 || (lane & 7) == rank)) break;
            if (timed_out(t0, tmo, call)) break;
            __builtin_amdgcn_s_sleep(16);
        }
    } else if (wid == 2) {
        unsigned* wq = (unsigned*)(P.ws[7] + OFF_WQ + (call & 1u) * 128u);
        if (c == 0 && lane == 0)
            __hip_atomic_store((unsigned*)(P.ws[7] + OFF_WQ + ((call + 1u) & 1u) * 128u), 0u, __ATOMIC_RELAXED, __HIP_MEMORY_SCOPE_AGENT);
        int v = 0;
#pragma unroll 1
        for (int k = RUNF; v < NUNIT;) {
            while (rfl((int)lds_rd(ctl + L_USED)) + AHEAD < k) __builtin_amdgcn_s_sleep(1);
            int m = v < RUN_A * 64 ? RUN0 : v < RUN_B * 64 ? RUNM : max(RUNT, min(RUN1, (NUNIT - v) / RUN_DIV));
            if (lane == 0) v = (int)__hip_atomic_fetch_add(wq, (unsigned)m, __ATOMIC_RELAXED, __HIP_MEMORY_SCOPE_AGENT) + NCOMM * RUNF;
            v = rfl(v);
            if (lane < m || (v + m > NUNIT && lane < m + NS)) lds_wr4(ctl + L_DQ + ((k + lane) & (NDQ - 1)) * 16, unit_entry(k + lane, v + lane, rank));
            k += m;
            if (v + m > NUNIT) break;
        }
    } else if (wid < 3 + NS) {
        const int s = wid - 3;
        int full = 0;
        unsigned cnt = 0, done = 0;
        bool bad = false;
#pragma unroll 1
        for (int i = s;; i += NS) {
            int unit0; unsigned rb, lb;
            while ((unit0 = desc(i, rb, lb)) < 0) {
                if (done != cnt) { DRAIN_V(); done = cnt; lds_wr(ctl + L_DONE + s * 4, done); }
                else __builtin_amdgcn_s_sleep(1);
            }
            if (unit0 == 0xffff) break;
            const bool isout = (rb & 3u) == M_OUT;
            const unsigned off = lb + lane * 16u;
            const unsigned stride = isout ? OWNB : 65536u;
            const __amdgpu_buffer_rsrc_t r = mk_rsrc(isout ? P.out : P.ws[7], isout ? OUT_BYTES : WS_NEED);
            while (i >= full) {
                const unsigned cw = lds_rd(ctl + (lane & 1) * 4);
                full = __builtin_amdgcn_readlane((int)cw, 1);
                bad = (unsigned)__builtin_amdgcn_readlane((int)cw, 0) == BADV;
                if (i >= full) {
                    if (done != cnt) { DRAIN_V(); done = cnt; lds_wr(ctl + L_DONE + s * 4, done); }
                    else __builtin_amdgcn_s_sleep(1);
                }
            }
            u32x4 v[7];
#pragma unroll
            for (int q = 0; q < 7; ++q) v[q] = *(const LAS u32x4*)(lds + (unsigned)i % NSLOT * 7168u + q * 1024 + lane * 16);
            SCHED; WAIT_L(0); SCHED;
            ++cnt;
            lds_wr(ctl + L_FREED + s * 4, cnt);
#pragma unroll
            for (int q = 0; q < 7; ++q) {
                if (bad) v[q] = (u32x4){0x7fc07fc0u, 0x7fc07fc0u, 0x7fc07fc0u, 0x7fc07fc0u};
                if (ST_AUX == ST_AUX_OUT || !isout) __builtin_amdgcn_raw_buffer_store_b128(v[q], r, off + (unsigned)((rank + 1 + q) & 7) * stride, 0, ST_AUX);
                else __builtin_amdgcn_raw_buffer_store_b128(v[q], r, off + (unsigned)((rank + 1 + q) & 7) * stride, 0, ST_AUX_OUT);
            }
            SCHED;
            if (done + 1u != cnt) { wait_v<7>(); done = cnt - 1u; lds_wr(ctl + L_DONE + s * 4, done); }
        }
        DRAIN_V();
        lds_wr(ctl + L_DONE + s * 4, cnt);
    }
}

#define LDIN(p) __builtin_nontemporal_load(p)
__device__ __forceinline__ unsigned take_ticket(const Params& P, int zv) {
    return __hip_atomic_fetch_add((unsigned*)(P.ws[7] + OFF_TICKET + (blockIdx.x & 15) * 128) + zv, 1u, __ATOMIC_RELAXED, __HIP_MEMORY_SCOPE_AGENT);
}
__device__ __forceinline__ unsigned stage_role(const Params& P, GAS char* wsl, unsigned ctl, int s0) {
    const int tid = threadIdx.x, sub = rfl(tid >> 6), lane = tid & 63;
    const int rank = P.rank;
    const int qi = 4 * (lane & 3) + ((lane & 15) >> 2);
    constexpr int NP8 = 7 * NHOLE, G = 30 + 7 * NOWN, S3 = 3 * G, S4 = S3 + 30, S5 = S4 + 7 * (NOWN - NHOLE), N_STAGE = NP8 + S5 + N_XW - N_R0;
    unsigned call = 0, tk = 0;
#define SST(i)
    int zv = 0;
    asm volatile("" : "+v"(zv));
    if (s0 >= NP8 + NEARLY) {
        const unsigned long long t0 = __builtin_amdgcn_s_memrealtime();
        while (__builtin_amdgcn_s_memrealtime() - t0 < DLY_L) __builtin_amdgcn_s_sleep(2);
    }
#pragma unroll 1
    for (int s1 = s0; s1 < N_STAGE; s1 += NGEMM) {
        const bool push = s1 < NP8;
        const int s = s1 - NP8;
        int n, q = 7;
        if (push) { const int jo = s1 / 7; q = s1 - jo * 7; n = HOLE + jo; }
        else if (s < S3) {
            const int kb = s / G, r = s - kb * G;
            if (r < 30) n = kb * 30 + r;
            else { const int jo = (r - 30) / 7; q = r - 30 - jo * 7; n = kb * 30 + 5 + jo; }
        } else if (s < S4) n = 90 + s - S3;
        else if (s < S5) { const int jo = (s - S4) / 7; q = s - S4 - jo * 7; n = 95 + jo; }
        else n = N_R0 + s - S5;
        const __amdgpu_buffer_rsrc_t rw = mk_rsrc(P.ws[push ? q : 7], WS_NEED);
        bool isx; int idx, kb;
        item_decode(n, isx, idx, kb);
        const int wi = ct_wi(idx);
        const bool own = !isx && wi < NOWN;
        u32x4 v[8];
        if (isx) {
            const u16* src = P.x + (size_t)(idx * 256 + sub * 32 + (lane & 15)) * KSH + kb * 128 + (lane >> 4) * 8;
#pragma unroll
            for (int t = 0; t < 8; ++t) v[t] = LDIN((const u32x4*)(src + (t >> 2) * 16 * KSH + (t & 3) * 32));
            SCHED;
            if (s1 == s0 && tid == 0) tk = take_ticket(P, zv);
            SCHED;
            SST(1);
            const unsigned dst = OFF_XA + (unsigned)idx * PANEL_BYTES + (unsigned)(kb * 8 + rank) * 65536u + sub * 2048u + lane * 16u;
#pragma unroll
            for (int t = 0; t < 8; ++t)
                __builtin_amdgcn_raw_buffer_store_b128(v[t], rw, dst + (t >> 2) * 1024 + (t & 3) * 16384, 0, AUX_SYS);
        } else {
            const int wr = sub & 1;
            const int p = (rank + 1 + q) & 7;
            const int col = own ? p * (OWNB / 2) + wi * 256 : (SH0 - NOWN + wi) * 256;
            const u16* src = P.w + (size_t)(kb * 128 + (sub >> 1) * 32 + (lane >> 4) * 8) * NN + col + wr * 128 + qi * 8;
#pragma unroll
            for (int t = 0; t < 8; ++t) v[t] = LDIN((const u32x4*)(src + (size_t)t * NN));
            SCHED;
            if (s1 == s0 && tid == 0) tk = take_ticket(P, zv);
            SCHED;
            SST(1);
            const unsigned base = q < 7 && !push ? OFF_SEND + (unsigned)(p * NOWN + wi) * 262144u + (unsigned)kb * 65536u
                                                 : OFF_WB + (unsigned)idx * PANEL_BYTES + (unsigned)(kb * 8 + rank) * 65536u;
            const unsigned dst = base + (sub >> 1) * 16384u + wr * 4096u + lane * 16u;
#pragma unroll
            for (int t = 0; t < 8; ++t) {
                u32x4 o;
#pragma unroll
                for (int e = 0; e < 4; ++e) o[e] = __builtin_amdgcn_perm(v[2 * e + 1][t >> 1], v[2 * e][t >> 1], (t & 1) ? 0x07060302u : 0x05040100u);
                __builtin_amdgcn_raw_buffer_store_b128(o, rw, dst + (t >> 2) * 8192 + (t & 3) * 1024, 0, AUX_SYS);
            }
        }
        DRAIN_V();
        SST(2);
        if (s1 == s0 && tid == 0) { lds_wr(ctl + 4, tk); WAIT_L(0); }
        BAR;
        SST(3);
        if (s1 == s0) call = (unsigned)rfl((int)lds_rd(ctl + 4)) / (GRID / 16) + 1u;
        if (sub == 0 && (own ? lane == q : lane < NR))
            __hip_atomic_store((GAS unsigned*)(wsl + OFF_FL + n * 32 + rank * 4), call, __ATOMIC_RELAXED, __HIP_MEMORY_SCOPE_SYSTEM);
        SCHED;
    }
    return call;
}

__device__ __forceinline__ unsigned cvt_pk_bf16(float lo, float hi) {
    unsigned r; asm volatile("v_cvt_pk_bf16_f32 %0, %1, %2" : "=v"(r) : "v"(lo), "v"(hi)); return r;
}
__device__ __forceinline__ f32x4 silu4(f32x4 y, float one) {
    f32x4 t = y * -1.44269504088896341f;
    asm("" : "+v"(t));
    f32x4 e;
#pragma unroll
    for (int i = 0; i < 4; ++i) e[i] = __builtin_amdgcn_exp2f(__builtin_amdgcn_fmed3f(t[i], -86.5617024533378f, 86.5617024533378f));
    e = e + one;
#pragma unroll
    for (int i = 0; i < 4; ++i) e[i] = __builtin_amdgcn_rcpf(e[i]);
    return e * y;
}

__device__ __forceinline__ void gemm_role(const Params& P, LAS unsigned char* lds, int g, int lw) {
    const int tid = threadIdx.x, wid = rfl(tid >> 6), lane = tid & 63, wr = wid >> 2, wc = wid & 3, fr = lane & 15, fq = lane >> 4;
    const int rank = P.rank;
    char* const ws = P.ws[7];
    unsigned* tmo = (unsigned*)(ws + OFF_TMO);
    const unsigned ldsw = (unsigned)wid * 1024u;
    volatile LAS int* lknown = (volatile LAS int*)(lds + STAGE_BYTES);
    int known = 0;
    STAMP_DECL;
    STAMP();
    GAS char* wsl = lane_ws(P, lane);
    asm volatile("" : "+v"(wsl));
    const unsigned call = stage_role(P, wsl, (unsigned)(uintptr_t)(lds + STAGE_BYTES), lw * 8 + g);
    STAMP();

    auto ready = [&](int i, int ks) {
        const int need = lw < 28 ? unit_need(i, ks, g) : (V_R0 + 4 * RC + 4 + (i >> 1) * 4) * 64;
        if ((unsigned)known >= (unsigned)need) return;
        if (wid == 0) {
            const unsigned* f = (const unsigned*)(ws + OFF_GP + g * 128);
            const unsigned long long t0 = __builtin_amdgcn_s_memrealtime();
            int m;
            for (;;) {
                m = rfl((int)(__hip_atomic_load(f, __ATOMIC_RELAXED, __HIP_MEMORY_SCOPE_AGENT) - (call << 16)));
                if (m >= need) break;
                if (timed_out(t0, tmo, call)) { m = -1; break; }
                __builtin_amdgcn_s_sleep(8);
            }
            if (lane == 0) lknown[0] = m;
            if (known == 0) __builtin_amdgcn_fence(__ATOMIC_ACQUIRE, "");
            WAIT_V(0); WAIT_L(0);
        }
        asm volatile("" ::: "memory"); BAR; asm volatile("" ::: "memory");
        known = rfl(lknown[0]);
    };

    const int ntile = lw < 28 ? NROUND : 4;
    int j = lw < 28 ? 2 * (lw >> 2) + (g >> 2) : RC * NROUND;
    int mb = 4 * (g & 3) + (lw < 28 ? lw & 3 : 2 * (g >> 2));
    f32x4 acc[2][2][4][2];
    bf16x8 Ar[2][2][4], Br[2][2][2];
    const __amdgpu_buffer_rsrc_t rw = mk_rsrc(ws, WS_NEED);
    unsigned gv = tid * 16u;
    unsigned la = (unsigned)(uintptr_t)lds + wr * 4096u + lane * 16u, lb = (unsigned)(uintptr_t)lds + 65536u + wc * 2048u + lane * 16u;
    asm volatile("" : "+v"(gv), "+v"(la), "+v"(lb));
    unsigned cA = OFF_WB + (unsigned)j * PANEL_BYTES, cB = OFF_XA + (unsigned)mb * PANEL_BYTES;
#define SSTR 65536u
#define GL(b, i, sa, sb) do { \
        if ((i) & 1) __builtin_amdgcn_raw_ptr_buffer_load_lds(rw, (LAS void*)(lds + 65536 + (b) * 16384 + ((i) >> 1) * 8192 + ldsw), 16, gv, (sb) + (b) * SSTR + ((i) >> 1) * 8192u, 0, GX_AUX); \
        else __builtin_amdgcn_raw_ptr_buffer_load_lds(rw, (LAS void*)(lds + (b) * 16384 + ((i) >> 1) * 8192 + ldsw), 16, gv, (sa) + (b) * SSTR + ((i) >> 1) * 8192u, 0, GW_AUX); } while (0)
#define RDA(r, b, h, m) Ar[r][h][m] = *(const LAS bf16x8*)(uintptr_t)(la + (b) * 16384 + (h) * 8192 + (m) * 1024)
#define RDB(r, b, h, n) Br[r][h][n] = *(const LAS bf16x8*)(uintptr_t)(lb + (b) * 16384 + (h) * 8192 + (n) * 1024)
#define RD(r, b, i) do { if ((i) < 4) RDB(r, b, (i) >> 1, (i) & 1); else RDA(r, b, ((i) - 4) >> 2, ((i) - 4) & 3); } while (0)
#define ACC2(i) acc[(i) >> 4][((i) >> 1) & 1][((i) >> 2) & 3][(i) & 1]
#define MF2(z, r, i) do { \
        if (z) asm volatile("v_mfma_f32_16x16x32_bf16 %0, %1, %2, 0" : "=&v"(ACC2(i)) : "v"(Ar[r][(i) >> 4][((i) >> 2) & 3]), "v"(Br[r][((i) >> 1) & 1][(i) & 1])); \
        else asm volatile("v_mfma_f32_16x16x32_bf16 %0, %1, %2, %0" : "+v"(ACC2(i)) : "v"(Ar[r][(i) >> 4][((i) >> 2) & 3]), "v"(Br[r][((i) >> 1) & 1][(i) & 1])); } while (0)
#define MORD 1
#define MIDX(i) (((i) & 4) ? (i) ^ 3 : (i))
#define GPOS 1
#define GSTR 8
#define RPOS 0
#define RSTR 2
#define XBAR BAR
#define XGL 4
#define XRD 12
#define STEP(b, z, vw, sa, sb) do { \
        SCHED; wait_v<vw>(); __builtin_amdgcn_s_waitcnt(0xC07F); XBAR; SCHED; \
        _Pragma("unroll") for (int i = 0; i < 32; ++i) { \
            MF2(z, (b) & 1, MIDX(i)); SCHED; \
            if (i >= GPOS && (i - GPOS) % GSTR == 0 && (i - GPOS) / GSTR < XGL) { GL(b, (i - GPOS) / GSTR, sa, sb); SCHED; } \
            if (i >= RPOS && (i - RPOS) % RSTR == 0 && (i - RPOS) / RSTR < XRD) { RD(((b) + 1) & 1, ((b) + 1) & 3, (i - RPOS) / RSTR); SCHED; } \
        } } while (0)
    ready(0, 0);
    STAMP();
#pragma unroll
    for (int b = 0; b < 4; ++b)
#pragma unroll
        for (int i = 0; i < 4; ++i) GL(b, i, cA, cB);
    SCHED; WAIT_V(0); BAR; SCHED;
#pragma unroll
    for (int i = 0; i < 12; ++i) RD(0, 0, i);
    const __amdgpu_buffer_rsrc_t ro = mk_rsrc(P.out, OUT_BYTES);
#pragma unroll 1
    for (int ui = 0; ui < ntile; ++ui) {
        const bool has_next = ui + 1 < ntile;
        const int nj = !has_next ? j : lw < 28 ? j + RC : j + (ui & 1), nmb = lw < 28 ? mb : mb ^ 1;
        const unsigned nA = OFF_WB + (unsigned)nj * PANEL_BYTES, nB = OFF_XA + (unsigned)nmb * PANEL_BYTES;
        STEP(0, true, 24, cA + 4 * SSTR, cB + 4 * SSTR); STEP(1, false, 24, cA + 4 * SSTR, cB + 4 * SSTR); STEP(2, false, 24, cA + 4 * SSTR, cB + 4 * SSTR);
        STEP(3, false, 8, cA + 4 * SSTR, cB + 4 * SSTR);
        const unsigned lim = lw < 28 ? unit_need(ui, 15, g) : 0;
#pragma unroll 1
        for (int t = 2; t <= 32; ++t) {
            const bool last = t == 32;
            const unsigned so = (unsigned)(t >> 3) * 524288u + (unsigned)((t >> 1) & 3) * 16384u + (unsigned)(t & 1) * 262144u;
            const unsigned sa = last ? nA : cA + so, sb = last ? nB : cB + so;
            if (!(t & 1) && ((unsigned)known < lim || last)) {
                if (!last) { ready(ui, t >> 1); if (!(t & 7)) STAMP(); }
                else if (has_next) { STAMP(); ready(ui + 1, 0); STAMP(); }
            }
            STEP(0, false, 8, sa, sb); STEP(1, false, 8, sa, sb); STEP(2, false, 8, sa, sb); STEP(3, false, 8, sa, sb);
        }
        asm volatile("s_nop 15\n\ts_nop 15" ::: "memory");
        SCHED;
        STAMP();
        {
            const int wi = ct_wi(j);
            const bool own = wi < NOWN;
            const unsigned row = (unsigned)(mb * 256 + wc * 32 + fr), cb = (unsigned)(wi * 256 + wr * 128 + fq * 8) * 2u;
            const unsigned obase = row * 65536u + cb + (own ? (unsigned)rank * OWNB : (SH0 - NOWN) * 512u);
            const unsigned pbase = OFF_OS + row * OWNB + cb;
            const float one = known < 0 ? __builtin_nanf("") : 1.0f;
            {
#pragma unroll
                for (int bj = 0; bj < 2; ++bj)
#pragma unroll
                    for (int n = 0; n < 2; ++n) {
                        f32x4 sv[8];
#pragma unroll
                        for (int m = 0; m < 8; ++m) sv[m] = silu4(acc[m >> 2][bj][m & 3][n], one);
#pragma unroll
                        for (int r = 0; r < 4; ++r) {
                            u32x4 o;
#pragma unroll
                            for (int e = 0; e < 4; ++e) o[e] = cvt_pk_bf16(sv[2 * e][r], sv[2 * e + 1][r]);
                            __builtin_amdgcn_raw_buffer_store_b128(o, ro, obase + r * 64u + (bj * 128 + n * 16) * 65536u, 0, EPI_AUX);
                            if (own) __builtin_amdgcn_raw_buffer_store_b128(o, rw, pbase + r * 64u + (bj * 128 + n * 16) * (unsigned)OWNB, 0, OS_AUX);
                        }
                    }
            }
            if (own) {
                WAIT_V(0);
                unsigned old = 0;
                if (lane == 0) old = __hip_atomic_fetch_add((unsigned*)(ws + OFF_TD + mb * 128), 1u, __ATOMIC_RELAXED, __HIP_MEMORY_SCOPE_AGENT);
                old = (unsigned)rfl((int)old) + 1u;
                if (old % (8u * NOWN) == 0u && lane < NR)
                    __hip_atomic_store((GAS unsigned*)(wsl + OFF_FL + (N_XW + mb) * 32 + rank * 4), call, __ATOMIC_RELAXED, __HIP_MEMORY_SCOPE_SYSTEM);
            }
        }
        SCHED;
        STAMP();
        cA = nA; cB = nB; j = nj; mb = nmb;
    }
    WAIT_V(0);
    BAR;
    if (wid == 0) {
        unsigned old = 0;
        if (lane == 0) old = __hip_atomic_fetch_add((unsigned*)(ws + OFF_GP + g * 128 + 64), 1u, __ATOMIC_RELAXED, __HIP_MEMORY_SCOPE_AGENT);
        if (((unsigned)rfl((int)old) + 1u) % WG_PER_GRP == 0u) __builtin_amdgcn_fence(__ATOMIC_RELEASE, "agent");
    }
    STAMP();
    STAMP_DUMP();
}

__global__ __launch_bounds__(512, 2) void fused_kernel(Params P) {
    __shared__ __attribute__((aligned(16))) unsigned char smem[LDS_BYTES];
    LAS unsigned char* lds = (LAS unsigned char*)smem;
    const int b = blockIdx.x;
    if (b < NCOMM) {
        if (threadIdx.x == 0) *(LAS unsigned*)(lds + 1024) = take_ticket(P, 0);
        __syncthreads();
        const unsigned call = (unsigned)rfl((int)*(LAS unsigned*)(lds + 1024)) / (GRID / 16) + 1u;
        comm_role(P, lds, call, b);
    } else {
#define GMAP 0
        const int x = b & 7;
        gemm_role(P, lds, GMAP ? (~x & 1) * 4 + (x >> 1) : x, (b - NCOMM) >> 3);
    }
}

}

extern "C" void kernel_launch(int rank, int nranks, ncclComm_t comm, hipStream_t stream,
                              void* const* d_in, const int* in_sizes, int n_in,
                              void* d_out, int out_size,
                              const int* in_dtypes, int out_dtype,
                              void* const* d_out_all,
                              void* const* d_ws_all, size_t ws_bytes) {
    (void)comm; (void)in_sizes; (void)n_in; (void)d_out_all; (void)out_size; (void)in_dtypes; (void)out_dtype;
    if (nranks != NR || (unsigned)rank >= (unsigned)NR || ws_bytes < WS_NEED) {
        fprintf(stderr, "unsupported configuration\n");
        return;
    }
    Params P;
    P.x = (const u16*)d_in[0]; P.w = (const u16*)d_in[1];
    for (int q = 0; q < NR; ++q) P.ws[q] = (char*)d_ws_all[(rank + 1 + q) % NR];
    P.out = (char*)d_out;
    P.rank = rank;
    hipLaunchKernelGGL(fused_kernel, dim3(GRID), dim3(512), 0, stream, P);
}
```

```cpp
#include <hip/hip_runtime.h>
#include <rccl/rccl.h>
#include <cstdio>

namespace {

typedef unsigned short u16;
typedef short bf16x8 __attribute__((ext_vector_type(8)));
typedef float f32x4 __attribute__((ext_vector_type(4)));
typedef unsigned u32x4 __attribute__((ext_vector_type(4)));
#define LAS __attribute__((address_space(3)))
#define GAS __attribute__((address_space(1)))

constexpr int NR = 8;
constexpr int NN = 32768, KSH = 512;
constexpr int GRID = 256;
constexpr int NCOMM = 24;
constexpr int NGEMM = GRID - NCOMM;
constexpr int WG_PER_GRP = NGEMM / 8;
constexpr int RC = 14, NROUND = 6;
constexpr int NCT = RC * NROUND + 2;
constexpr int NOWN = 6;
constexpr int OWNB = NOWN * 512;
constexpr int SH0 = 8 * NOWN;
static_assert(WG_PER_GRP == 29 && NCT == 128 - 7 * NOWN, "tile map");
#define PD 21
#define NS 5
#define NSLOT 18
#define AHEAD 5
#define NHOLE 6
#define QS 4
#define SL0 3
#define NSL1 2
#define DLY_L 300u
#define NEARLY 72
#define ST_AUX_OUT 18
#define ST_AUX 16
#define RUN0 32
#define RUNM 8
#define RUN1 32
#define RUNT 2
#define RUNF 8
#define RUN_A 60
#define RUN_B 150
#define RUN_DIV 72
#define GX_AUX 0
#define GW_AUX 0
#define LD_AUX 17
#define OS_AUX 17
#define EPI_AUX 18
static_assert(NS >= 1 && NS <= 5 && PD >= 7 && PD <= 63 && RUN0 <= 64 && RUN1 <= 64, "waves of a communication workgroup");
static_assert(NHOLE >= 0 && NHOLE <= NOWN && (QS == 1 || QS == 2 || QS == 4) && NSL1 >= 0 && NSL1 <= 4, "pushed items");

constexpr unsigned OFF_TMO = 128;
constexpr unsigned OFF_PROG = 256;
constexpr unsigned OFF_WQ = 512;
constexpr unsigned OFF_GP = 1024;
constexpr unsigned OFF_TICKET = 2048;
constexpr unsigned OFF_TD = 4096;
constexpr unsigned OFF_FO = 8192;
constexpr unsigned OFF_FL = 16384;
constexpr unsigned OFF_XA = 1u << 20;
constexpr unsigned PANEL_BYTES = 2u << 20;
constexpr unsigned OFF_WB = OFF_XA + 16u * PANEL_BYTES;
constexpr unsigned OFF_SEND = OFF_WB + (unsigned)NCT * PANEL_BYTES;
constexpr unsigned OFF_OS = OFF_SEND + 8u * NOWN * 262144u;
constexpr unsigned WS_NEED = OFF_OS + 4096u * OWNB;
constexpr unsigned OUT_BYTES = 256u << 20;

constexpr int N_R0 = 120, N_R1 = N_R0 + 4 * RC;
constexpr int N_XW = 64 + NCT * 4;
constexpr int HOLE = 101 - NHOLE;
constexpr int K3N = 30 - NHOLE;
constexpr int V_R0 = N_R0 - NHOLE;
constexpr int V_XW = N_XW - NHOLE;
constexpr int V_ALL = V_XW + OWNB / 16;
constexpr int NUNIT = V_ALL * 64;
constexpr int N_GRP = N_XW + 16;
static_assert(NUNIT + 256 < 65535, "progress fits 16 bits");
static_assert(OFF_FL + (N_GRP + 8) * 32 <= 65536, "flag table");
__host__ __device__ constexpr int ct_wi(int j) { return j >= RC ? j : (j & 1) * 7 + (j >> 1); }
__device__ __forceinline__ void item_decode(int n, bool& isx, int& idx, int& kb) {
    if (n < N_R0) {
        kb = n / 30; const int r = n - kb * 30;
        isx = r < 4 || (r >= 11 && r < 23);
        idx = r < 4 ? r : r < 11 ? 20 - 2 * r : r < 23 ? r - 7 : 2 * (r - 23) + 1;
    } else {
        const bool extra = n >= N_R1 && n < N_R1 + 8;
        const int m = n - (n < N_R1 ? N_R0 : N_R0 + 8), i = m / (4 * RC), rem = m - i * (4 * RC);
        kb = rem / RC; const int r = rem - kb * RC;
        isx = false;
        idx = RC * (i + 1) + (r < 7 ? 2 * r : 2 * (r - 7) + 1);
        if (extra) { idx = RC * NROUND + ((n - N_R1) >> 2); kb = (n - N_R1) & 3; }
    }
}
__device__ __forceinline__ int unit_need(int i, int ks, int g) {
    const int kb = ks >> 2, sl = (ks & 3) * QS >> 2;
    if (i == 0) {
        if (kb < SL0) return (30 * kb + (g < 4 ? 11 + 4 * g : 30)) * 64;
        if (kb < 3) return 30 * kb * 64 + (sl * 30 + (g < 4 ? 11 + 4 * g : 30)) * (64 / QS);
        return 90 * 64 + (sl * K3N + (g < 4 ? 11 - NHOLE + 4 * g : K3N)) * (64 / QS);
    }
    if (i == 1 && kb < NSL1) return (V_R0 + RC * kb) * 64 + (sl * RC + (g < 4 ? 7 : RC)) * (64 / QS);
    return (V_R0 + 4 * RC * (i - 1) + (i > 1 ? 8 : 0) + RC * kb + (g < 4 ? 7 : RC)) * 64;
}

constexpr unsigned long long DEADLINE_TICKS = 100000000ull;
constexpr int STAGE_BYTES = NSLOT * 7168 > 131072 ? NSLOT * 7168 : 131072;
constexpr int LDS_BYTES = STAGE_BYTES + 4096;

struct Params {
    const u16* x; const u16* w;
    char* ws[NR];
    char* out;
    int rank;
};

__device__ __forceinline__ int rfl(int v) { return __builtin_amdgcn_readfirstlane(v); }

__device__ __forceinline__ __amdgpu_buffer_rsrc_t mk_rsrc(const void* p, unsigned bytes) {
    return __builtin_amdgcn_make_buffer_rsrc(const_cast<void*>(p), 0, (int)bytes, 0x00020000);
}
__device__ __forceinline__ bool timed_out(unsigned long long t0, unsigned* tmo, unsigned call) {
    if (__builtin_amdgcn_s_memrealtime() - t0 > DEADLINE_TICKS) {
        __hip_atomic_store(tmo, call, __ATOMIC_RELAXED, __HIP_MEMORY_SCOPE_AGENT);
        return true;
    }
    return false;
}
__device__ __forceinline__ unsigned lds_rd(unsigned addr) {
    unsigned v; asm volatile("ds_read_b32 %0, %1\n\ts_waitcnt lgkmcnt(0)" : "=v"(v) : "v"(addr) : "memory"); return v;
}
__device__ __forceinline__ u32x4 lds_rd4(unsigned addr) {
    u32x4 v; asm volatile("ds_read_b128 %0, %1\n\ts_waitcnt lgkmcnt(0)" : "=v"(v) : "v"(addr) : "memory"); return v;
}
__device__ __forceinline__ void lds_wr4(unsigned addr, u32x4 v) { asm volatile("ds_write_b128 %0, %1" :: "v"(addr), "v"(v) : "memory"); }
__device__ __forceinline__ void lds_wr(unsigned addr, unsigned v) { asm volatile("ds_write_b32 %0, %1" :: "v"(addr), "v"(v) : "memory"); }

#define WAIT_V(n) asm volatile("s_waitcnt vmcnt(" #n ")" ::: "memory")
#define WAIT_L(n) asm volatile("s_waitcnt lgkmcnt(" #n ")" ::: "memory")
#define SCHED __builtin_amdgcn_sched_barrier(0)
#define BAR __builtin_amdgcn_s_barrier()
#define DRAIN_V() do { SCHED; __builtin_amdgcn_s_waitcnt(0x0F70); WAIT_V(0); SCHED; } while (0)
template <int N> __device__ __forceinline__ void wait_v() { asm volatile("s_waitcnt vmcnt(%0)" :: "n"(N) : "memory"); }


constexpr int AUX_SYS = 17;

__device__ __forceinline__ GAS char* lane_ws(const Params& P, int lane) {
    char* wsl = P.ws[0];
#pragma unroll
    for (int p = 1; p < NR; ++p) {
        char* t = P.ws[p];
        asm volatile("" : "+s"(t));
        wsl = (lane == p) ? t : wsl;
    }
    return (GAS char*)wsl;
}

constexpr unsigned M_OWN = 1, M_OUT = 2;
__device__ __forceinline__ u32x4 unit_entry(int k, int v, int rank) {
    if (v >= NUNIT) return (u32x4){(unsigned)(k + 1) << 16 | 0xffffu, 0u, 0u, 0u};
    int n = v >> 6, u = v & 63;
    if (n >= V_XW) {
        unsigned o = (unsigned)(v - V_XW * 64) + (unsigned)rank * (OWNB / 2u);
        if (o >= OWNB * 4u) o -= OWNB * 4u;
        const unsigned row = o / (OWNB / 1024u);
        return (u32x4){(unsigned)(k + 1) << 16 | (unsigned)v, OFF_OS + o * 1024u + M_OUT, row * 65536u + (o - row * (OWNB / 1024u)) * 1024u, (unsigned)N_XW};
    }
    if (n >= V_R0 + NSL1 * RC) n += NHOLE;
    else if (n >= V_R0) {
        const int v1 = v - V_R0 * 64, sl = v1 / (RC * 64 / QS), rem = v1 - sl * (RC * 64 / QS);
        n = N_R0 + sl / QS * RC + rem / (64 / QS); u = sl % QS * (64 / QS) + rem % (64 / QS);
    } else if (n >= 90) {
        const int v1 = v - 90 * 64, sl = v1 / (K3N * 64 / QS), rem = v1 - sl * (K3N * 64 / QS);
        n = 90 + rem / (64 / QS); u = sl * (64 / QS) + rem % (64 / QS);
        if (n >= HOLE) n += NHOLE;
    } else if (n >= SL0 * 30) {
        const int sl = v / (30 * 64 / QS), rem = v - sl * (30 * 64 / QS);
        n = sl / QS * 30 + rem / (64 / QS); u = sl % QS * (64 / QS) + rem % (64 / QS);
    }
    bool isx; int idx, kb;
    item_decode(n, isx, idx, kb);
    const bool own = !isx && ct_wi(idx) < NOWN;
    const unsigned lb = (isx ? OFF_XA : OFF_WB) + (unsigned)idx * PANEL_BYTES + (unsigned)kb * 524288u + u * 1024u;
    const unsigned rb = own ? OFF_SEND + (unsigned)(rank * NOWN + ct_wi(idx)) * 262144u + (unsigned)kb * 65536u + u * 1024u + M_OWN : lb;
    return (u32x4){(unsigned)(k + 1) << 16 | (unsigned)v, rb, lb, (unsigned)n};
}
__device__ __forceinline__ int lead_batches(unsigned cnt) {
    int r = __builtin_amdgcn_readlane((int)cnt, 0) * NS;
#pragma unroll
    for (int s = 1; s < NS; ++s) r = min(r, __builtin_amdgcn_readlane((int)cnt, s) * NS + s);
    return r;
}

__device__ __forceinline__ void comm_role(const Params& P, LAS unsigned char* lds, unsigned call, int c) {
    const int tid = threadIdx.x, wid = rfl(tid >> 6), lane = tid & 63;
    const int rank = P.rank;
    unsigned* tmo = (unsigned*)(P.ws[7] + OFF_TMO);
    const unsigned ctl = (unsigned)(uintptr_t)(lds + STAGE_BYTES);
    constexpr unsigned L_AVAIL = 0, L_FULL = 4, L_USED = 12, L_FREED = 32, L_DONE = 64, L_DQ = 2048;
    constexpr int NDQ = 128;
    static_assert(2 * NS + NSLOT + AHEAD + 64 + NS <= NDQ, "ring of claimed units");
    constexpr unsigned BADV = 0xffffu;
    const unsigned pbase = call << 16;
    auto desc = [&](int k, unsigned& rb, unsigned& lb) -> int {
        const u32x4 e = lds_rd4(ctl + L_DQ + (k & (NDQ - 1)) * 16);
        const unsigned e0 = (unsigned)rfl((int)e.x);
        rb = (unsigned)rfl((int)e.y); lb = (unsigned)rfl((int)e.z);
        return (e0 >> 16) == (unsigned)(k + 1) ? (int)(e0 & 0xffffu) : -1;
    };
    if (tid < 256) lds_wr4(ctl + tid * 16, tid >= 128 && tid < 128 + RUNF ? unit_entry(tid - 128, c * RUNF + tid - 128, rank) : (u32x4){0u, 0u, 0u, 0u});
    WAIT_L(0);
    __syncthreads();
    if (wid == 0) {
        __builtin_amdgcn_s_setprio(3);
        int avail = 0, freed = 0, i = 0;
        __amdgpu_buffer_rsrc_t r[7];
#pragma unroll
        for (int q = 0; q < 7; ++q) r[q] = mk_rsrc(P.ws[q], WS_NEED);
        bool more = true;
#pragma unroll 1
        while (more) {
            const u32x4 e = lds_rd4(ctl + L_DQ + ((i + lane) & (NDQ - 1)) * 16);
            const int nv = (int)__builtin_ctzll(~__ballot((e.x >> 16) == (unsigned)(i + lane + 1)) | 1ull << 63);
            if (!nv) __builtin_amdgcn_s_sleep(1);
#pragma unroll 1
            for (int u = 0; u < nv; ++u, ++i) {
                const int unit0 = __builtin_amdgcn_readlane((int)e.x, u) & 0xffff, n = __builtin_amdgcn_readlane((int)e.w, u);
                const unsigned rb = (unsigned)__builtin_amdgcn_readlane((int)e.y, u);
                if (unit0 == 0xffff) { more = false; break; }
                lds_wr(ctl + L_USED, (unsigned)(i + 1));
                const unsigned stride = (rb & 3u) ? 0u : 65536u;
                while (n >= avail) {
                    avail = rfl((int)lds_rd(ctl + L_AVAIL));
                    if (n >= avail) { WAIT_V(0); lds_wr(ctl + L_FULL, (unsigned)i); __builtin_amdgcn_s_sleep(1); }
                }
                const unsigned off = (rb & ~3u) + lane * 16u;
                while (i - NSLOT >= freed) { freed = lead_batches(lds_rd(ctl + L_FREED + (lane & 7) * 4)); if (i - NSLOT >= freed) __builtin_amdgcn_s_sleep(1); }
                LAS unsigned char* slot = lds + (unsigned)i % NSLOT * 7168u;
                SCHED;
#pragma unroll
                for (int q = 0; q < 7; ++q) {
                    wait_v<PD - 1>();
                    __builtin_amdgcn_raw_ptr_buffer_load_lds(r[q], (LAS void*)(slot + q * 1024), 16, off + (unsigned)((rank + 1 + q) & 7) * stride, 0, 0, LD_AUX);
                    SCHED;
                    if (q == (PD - 1) % 7 && i >= (PD - 1) / 7) lds_wr(ctl + L_FULL, (unsigned)(i - (PD - 1) / 7));
                }
            }
        }
#pragma unroll
        for (int k = (PD - 1) / 7; k >= 0; --k) {
            if (k == 8) wait_v<56>(); else if (k == 7) wait_v<49>(); else if (k == 6) wait_v<42>(); else if (k == 5) wait_v<35>(); else if (k == 4) wait_v<28>();
            else if (k == 3) wait_v<21>(); else if (k == 2) wait_v<14>(); else if (k == 1) wait_v<7>(); else wait_v<0>();
            if (i >= k) lds_wr(ctl + L_FULL, (unsigned)(i - k));
        }
    } else if (wid == 1) {
        const unsigned* fl = (const unsigned*)(P.ws[7] + OFF_FL) + lane;
        unsigned* prog = (unsigned*)(P.ws[7] + OFF_PROG);
        const unsigned long long t0 = __builtin_amdgcn_s_memrealtime();
        int pos = 0, mine = 0, all = c < 8 ? 0 : V_XW * 64;
        bool pushed = c >= 8;
        while (pos < N_GRP || mine < V_XW * 64 || all < V_XW * 64) {
            unsigned v = call, pv = 0, hv = call;
            if (pos < N_GRP) v = __hip_atomic_load(fl + pos * 8, __ATOMIC_RELAXED, __HIP_MEMORY_SCOPE_SYSTEM);
            if (!pushed) hv = __hip_atomic_load(fl + HOLE * 8, __ATOMIC_RELAXED, __HIP_MEMORY_SCOPE_SYSTEM);
            if (all < V_XW * 64) pv = __hip_atomic_load(prog + min(lane & 31, NCOMM - 1), __ATOMIC_RELAXED, __HIP_MEMORY_SCOPE_AGENT);
            if (mine < V_XW * 64) {
                unsigned rb, lb;
                const int m = desc(lead_batches(lds_rd(ctl + L_DONE + (lane & 7) * 4)), rb, lb);
                if (m > mine) {
                    if (lane == 0) __hip_atomic_store(prog + c, pbase + (unsigned)m, __ATOMIC_RELAXED, __HIP_MEMORY_SCOPE_AGENT);
                    mine = m;
                }
            }
            int adv = 0;
            if (pos < N_GRP) {
                const int it = pos + (lane >> 3);
                const unsigned long long miss = __ballot((int)(v - call) < 0 && it < N_GRP && (it < HOLE || it >= HOLE + NHOLE));
                adv = miss ? (int)(__builtin_ctzll(miss) >> 3) : 8;
                pos += adv;
                if (adv) lds_wr(ctl + L_AVAIL, (unsigned)(pos <= N_XW ? pos : pos < N_GRP ? N_XW : N_XW + 1));
            }
            if (!pushed) pushed = __all((int)(hv - call) >= 0 || lane >= NHOLE * 8);
            if (all < V_XW * 64) {
                int m = (int)(pv - pbase);
#pragma unroll
                for (int o = 16; o >= 1; o >>= 1) m = min(m, __shfl_xor(m, o));
                m = rfl(m);
                if (!pushed) m = min(m, 90 * 64);
                if (m > all) {
                    if (lane == 0) __hip_atomic_store((unsigned*)(P.ws[7] + OFF_GP + c * 128), pbase + (unsigned)m, __ATOMIC_RELAXED, __HIP_MEMORY_SCOPE_AGENT);
                    all = m;
                }
            }
            if (timed_out(t0, tmo, call)) { lds_wr(ctl + L_AVAIL, BADV); break; }
            if (!adv) __builtin_amdgcn_s_sleep(4);
        }
        if (lane < NR - 1)
            __hip_atomic_store((GAS unsigned*)(lane_ws(P, lane) + OFF_FO + c * 128 + rank * 4), call, __ATOMIC_RELAXED, __HIP_MEMORY_SCOPE_SYSTEM);
        const unsigned* fo = (const unsigned*)(P.ws[7] + OFF_FO + c * 128) + (lane & 7);
        for (;;) {
            const unsigned v = __hip_atomic_load(fo, __ATOMIC_RELAXED, __HIP_MEMORY_SCOPE_SYSTEM);
            if (__all((int)(v - call) >= 0 || (lane & 7) == rank)) break;
            if (timed_out(t0, tmo, call)) break;
            __builtin_amdgcn_s_sleep(16);
        }
    } else if (wid == 2) {
        unsigned* wq = (unsigned*)(P.ws[7] + OFF_WQ + (call & 1u) * 128u);
        if (c == 0 && lane == 0)
            __hip_atomic_store((unsigned*)(P.ws[7] + OFF_WQ + ((call + 1u) & 1u) * 128u), 0u, __ATOMIC_RELAXED, __HIP_MEMORY_SCOPE_AGENT);
        int v = 0;
#pragma unroll 1
        for (int k = RUNF; v < NUNIT;) {
            while (rfl((int)lds_rd(ctl + L_USED)) + AHEAD < k) __builtin_amdgcn_s_sleep(1);
            int m = v < RUN_A * 64 ? RUN0 : v < RUN_B * 64 ? RUNM : max(RUNT, min(RUN1, (NUNIT - v) / RUN_DIV));
            if (lane == 0) v = (int)__hip_atomic_fetch_add(wq, (unsigned)m, __ATOMIC_RELAXED, __HIP_MEMORY_SCOPE_AGENT) + NCOMM * RUNF;
            v = rfl(v);
            if (lane < m || (v + m > NUNIT && lane < m + NS)) lds_wr4(ctl + L_DQ + ((k + lane) & (NDQ - 1)) * 16, unit_entry(k + lane, v + lane, rank));
            k += m;
            if (v + m > NUNIT) break;
        }
    } else if (wid < 3 + NS) {
        const int s = wid - 3;
        int full = 0;
        unsigned cnt = 0, done = 0;
        bool bad = false;
#pragma unroll 1
        for (int i = s;; i += NS) {
            int unit0; unsigned rb, lb;
            while ((unit0 = desc(i, rb, lb)) < 0) {
                if (done != cnt) { DRAIN_V(); done = cnt; lds_wr(ctl + L_DONE + s * 4, done); }
                else __builtin_amdgcn_s_sleep(1);
            }
            if (unit0 == 0xffff) break;
            const bool isout = (rb & 3u) == M_OUT;
            const unsigned off = lb + lane * 16u;
            const unsigned stride = isout ? OWNB : 65536u;
            const __amdgpu_buffer_rsrc_t r = mk_rsrc(isout ? P.out : P.ws[7], isout ? OUT_BYTES : WS_NEED);
            while (i >= full) {
                const unsigned cw = lds_rd(ctl + (lane & 1) * 4);
                full = __builtin_amdgcn_readlane((int)cw, 1);
                bad = (unsigned)__builtin_amdgcn_readlane((int)cw, 0) == BADV;
                if (i >= full) {
                    if (done != cnt) { DRAIN_V(); done = cnt; lds_wr(ctl + L_DONE + s * 4, done); }
                    else __builtin_amdgcn_s_sleep(1);
                }
            }
            u32x4 v[7];
#pragma unroll
            for (int q = 0; q < 7; ++q) v[q] = *(const LAS u32x4*)(lds + (unsigned)i % NSLOT * 7168u + q * 1024 + lane * 16);
            SCHED; WAIT_L(0); SCHED;
            ++cnt;
            lds_wr(ctl + L_FREED + s * 4, cnt);
#pragma unroll
            for (int q = 0; q < 7; ++q) {
                if (bad) v[q] = (u32x4){0x7fc07fc0u, 0x7fc07fc0u, 0x7fc07fc0u, 0x7fc07fc0u};
                if (ST_AUX == ST_AUX_OUT || !isout) __builtin_amdgcn_raw_buffer_store_b128(v[q], r, off + (unsigned)((rank + 1 + q) & 7) * stride, 0, ST_AUX);
                else __builtin_amdgcn_raw_buffer_store_b128(v[q], r, off + (unsigned)((rank + 1 + q) & 7) * stride, 0, ST_AUX_OUT);
            }
            SCHED;
            if (done + 1u != cnt) { wait_v<7>(); done = cnt - 1u; lds_wr(ctl + L_DONE + s * 4, done); }
            if (isout) { done = cnt; lds_wr(ctl + L_DONE + s * 4, done); }
        }
        if (done != cnt) { DRAIN_V(); lds_wr(ctl + L_DONE + s * 4, cnt); }
    }
}

#define LDIN(p) __builtin_nontemporal_load(p)
__device__ __forceinline__ unsigned take_ticket(const Params& P, int zv) {
    return __hip_atomic_fetch_add((unsigned*)(P.ws[7] + OFF_TICKET + (blockIdx.x & 15) * 128) + zv, 1u, __ATOMIC_RELAXED, __HIP_MEMORY_SCOPE_AGENT);
}
__device__ __forceinline__ unsigned stage_role(const Params& P, GAS char* wsl, unsigned ctl, int s0) {
    const int tid = threadIdx.x, sub = rfl(tid >> 6), lane = tid & 63;
    const int rank = P.rank;
    const int qi = 4 * (lane & 3) + ((lane & 15) >> 2);
    constexpr int NP8 = 7 * NHOLE, G = 30 + 7 * NOWN, S3 = 3 * G, S4 = S3 + 30, S5 = S4 + 7 * (NOWN - NHOLE), N_STAGE = NP8 + S5 + N_XW - N_R0;
    unsigned call = 0, tk = 0;
    int zv = 0;
    asm volatile("" : "+v"(zv));
    if (s0 >= NP8 + NEARLY) {
        const unsigned long long t0 = __builtin_amdgcn_s_memrealtime();
        while (__builtin_amdgcn_s_memrealtime() - t0 < DLY_L) __builtin_amdgcn_s_sleep(2);
    }
#pragma unroll 1
    for (int s1 = s0; s1 < N_STAGE; s1 += NGEMM) {
        const bool push = s1 < NP8;
        const int s = s1 - NP8;
        int n, q = 7;
        if (push) { const int jo = s1 / 7; q = s1 - jo * 7; n = HOLE + jo; }
        else if (s < S3) {
            const int kb = s / G, r = s - kb * G;
            if (r < 30) n = kb * 30 + r;
            else { const int jo = (r - 30) / 7; q = r - 30 - jo * 7; n = kb * 30 + 5 + jo; }
        } else if (s < S4) n = 90 + s - S3;
        else if (s < S5) { const int jo = (s - S4) / 7; q = s - S4 - jo * 7; n = 95 + jo; }
        else n = N_R0 + s - S5;
        const __amdgpu_buffer_rsrc_t rw = mk_rsrc(P.ws[push ? q : 7], WS_NEED);
        bool isx; int idx, kb;
        item_decode(n, isx, idx, kb);
        const int wi = ct_wi(idx);
        const bool own = !isx && wi < NOWN;
        u32x4 v[8];
        if (isx) {
            const u16* src = P.x + (size_t)(idx * 256 + sub * 32 + (lane & 15)) * KSH + kb * 128 + (lane >> 4) * 8;
#pragma unroll
            for (int t = 0; t < 8; ++t) v[t] = LDIN((const u32x4*)(src + (t >> 2) * 16 * KSH + (t & 3) * 32));
            SCHED;
            if (s1 == s0 && tid == 0) tk = take_ticket(P, zv);
            SCHED;
            const unsigned dst = OFF_XA + (unsigned)idx * PANEL_BYTES + (unsigned)(kb * 8 + rank) * 65536u + sub * 2048u + lane * 16u;
#pragma unroll
            for (int t = 0; t < 8; ++t)
                __builtin_amdgcn_raw_buffer_store_b128(v[t], rw, dst + (t >> 2) * 1024 + (t & 3) * 16384, 0, AUX_SYS);
        } else {
            const int wr = sub & 1;
            const int p = (rank + 1 + q) & 7;
            const int col = own ? p * (OWNB / 2) + wi * 256 : (SH0 - NOWN + wi) * 256;
            const u16* src = P.w + (size_t)(kb * 128 + (sub >> 1) * 32 + (lane >> 4) * 8) * NN + col + wr * 128 + qi * 8;
#pragma unroll
            for (int t = 0; t < 8; ++t) v[t] = LDIN((const u32x4*)(src + (size_t)t * NN));
            SCHED;
            if (s1 == s0 && tid == 0) tk = take_ticket(P, zv);
            SCHED;
            const unsigned base = q < 7 && !push ? OFF_SEND + (unsigned)(p * NOWN + wi) * 262144u + (unsigned)kb * 65536u
                                                 : OFF_WB + (unsigned)idx * PANEL_BYTES + (unsigned)(kb * 8 + rank) * 65536u;
            const unsigned dst = base + (sub >> 1) * 16384u + wr * 4096u + lane * 16u;
#pragma unroll
            for (int t = 0; t < 8; ++t) {
                u32x4 o;
#pragma unroll
                for (int e = 0; e < 4; ++e) o[e] = __builtin_amdgcn_perm(v[2 * e + 1][t >> 1], v[2 * e][t >> 1], (t & 1) ? 0x07060302u : 0x05040100u);
                __builtin_amdgcn_raw_buffer_store_b128(o, rw, dst + (t >> 2) * 8192 + (t & 3) * 1024, 0, AUX_SYS);
            }
        }
        DRAIN_V();
        if (s1 == s0 && tid == 0) { lds_wr(ctl + 4, tk); WAIT_L(0); }
        BAR;
        if (s1 == s0) call = (unsigned)rfl((int)lds_rd(ctl + 4)) / (GRID / 16) + 1u;
        if (sub == 0 && (own ? lane == q : lane < NR))
            __hip_atomic_store((GAS unsigned*)(wsl + OFF_FL + n * 32 + rank * 4), call, __ATOMIC_RELAXED, __HIP_MEMORY_SCOPE_SYSTEM);
        SCHED;
    }
    return call;
}

__device__ __forceinline__ unsigned cvt_pk_bf16(float lo, float hi) {
    unsigned r; asm volatile("v_cvt_pk_bf16_f32 %0, %1, %2" : "=v"(r) : "v"(lo), "v"(hi)); return r;
}
__device__ __forceinline__ f32x4 silu4(f32x4 y, float one) {
    f32x4 t = y * -1.44269504088896341f;
    asm("" : "+v"(t));
    f32x4 e;
#pragma unroll
    for (int i = 0; i < 4; ++i) e[i] = __builtin_amdgcn_exp2f(__builtin_amdgcn_fmed3f(t[i], -86.5617024533378f, 86.5617024533378f));
    e = e + one;
#pragma unroll
    for (int i = 0; i < 4; ++i) e[i] = __builtin_amdgcn_rcpf(e[i]);
    return e * y;
}

__device__ __forceinline__ void gemm_role(const Params& P, LAS unsigned char* lds, int g, int lw) {
    const int tid = threadIdx.x, wid = rfl(tid >> 6), lane = tid & 63, wr = wid >> 2, wc = wid & 3, fr = lane & 15, fq = lane >> 4;
    const int rank = P.rank;
    char* const ws = P.ws[7];
    unsigned* tmo = (unsigned*)(ws + OFF_TMO);
    const unsigned ldsw = (unsigned)wid * 1024u;
    volatile LAS int* lknown = (volatile LAS int*)(lds + STAGE_BYTES);
    int known = 0;
    GAS char* wsl = lane_ws(P, lane);
    asm volatile("" : "+v"(wsl));
    const unsigned call = stage_role(P, wsl, (unsigned)(uintptr_t)(lds + STAGE_BYTES), lw * 8 + g);

    auto ready = [&](int i, int ks) {
        const int need = lw < 28 ? unit_need(i, ks, g) : (V_R0 + 4 * RC + 4 + (i >> 1) * 4) * 64;
        if ((unsigned)known >= (unsigned)need) return;
        if (wid == 0) {
            const unsigned* f = (const unsigned*)(ws + OFF_GP + g * 128);
            const unsigned long long t0 = __builtin_amdgcn_s_memrealtime();
            int m;
            for (;;) {
                m = rfl((int)(__hip_atomic_load(f, __ATOMIC_RELAXED, __HIP_MEMORY_SCOPE_AGENT) - (call << 16)));
                if (m >= need) break;
                if (timed_out(t0, tmo, call)) { m = -1; break; }
                __builtin_amdgcn_s_sleep(8);
            }
            if (lane == 0) lknown[0] = m;
            if (known == 0) __builtin_amdgcn_fence(__ATOMIC_ACQUIRE, "");
            WAIT_V(0); WAIT_L(0);
        }
        asm volatile("" ::: "memory"); BAR; asm volatile("" ::: "memory");
        known = rfl(lknown[0]);
    };

    const int ntile = lw < 28 ? NROUND : 4;
    int j = lw < 28 ? 2 * (lw >> 2) + (g >> 2) : RC * NROUND;
    int mb = 4 * (g & 3) + (lw < 28 ? lw & 3 : 2 * (g >> 2));
    f32x4 acc[2][2][4][2];
    bf16x8 Ar[2][2][4], Br[2][2][2];
    const __amdgpu_buffer_rsrc_t rw = mk_rsrc(ws, WS_NEED);
    unsigned gv = tid * 16u;
    unsigned la = (unsigned)(uintptr_t)lds + wr * 4096u + lane * 16u, lb = (unsigned)(uintptr_t)lds + 65536u + wc * 2048u + lane * 16u;
    asm volatile("" : "+v"(gv), "+v"(la), "+v"(lb));
    unsigned cA = OFF_WB + (unsigned)j * PANEL_BYTES, cB = OFF_XA + (unsigned)mb * PANEL_BYTES;
#define SSTR 65536u
#define GL(b, i, sa, sb) do { \
        if ((i) & 1) __builtin_amdgcn_raw_ptr_buffer_load_lds(rw, (LAS void*)(lds + 65536 + (b) * 16384 + ((i) >> 1) * 8192 + ldsw), 16, gv, (sb) + (b) * SSTR + ((i) >> 1) * 8192u, 0, GX_AUX); \
        else __builtin_amdgcn_raw_ptr_buffer_load_lds(rw, (LAS void*)(lds + (b) * 16384 + ((i) >> 1) * 8192 + ldsw), 16, gv, (sa) + (b) * SSTR + ((i) >> 1) * 8192u, 0, GW_AUX); } while (0)
#define RDA(r, b, h, m) Ar[r][h][m] = *(const LAS bf16x8*)(uintptr_t)(la + (b) * 16384 + (h) * 8192 + (m) * 1024)
#define RDB(r, b, h, n) Br[r][h][n] = *(const LAS bf16x8*)(uintptr_t)(lb + (b) * 16384 + (h) * 8192 + (n) * 1024)
#define RD(r, b, i) do { if ((i) < 4) RDB(r, b, (i) >> 1, (i) & 1); else RDA(r, b, ((i) - 4) >> 2, ((i) - 4) & 3); } while (0)
#define ACC2(i) acc[(i) >> 4][((i) >> 1) & 1][((i) >> 2) & 3][(i) & 1]
#define MF2(z, r, i) do { \
        if (z) asm volatile("v_mfma_f32_16x16x32_bf16 %0, %1, %2, 0" : "=&v"(ACC2(i)) : "v"(Ar[r][(i) >> 4][((i) >> 2) & 3]), "v"(Br[r][((i) >> 1) & 1][(i) & 1])); \
        else asm volatile("v_mfma_f32_16x16x32_bf16 %0, %1, %2, %0" : "+v"(ACC2(i)) : "v"(Ar[r][(i) >> 4][((i) >> 2) & 3]), "v"(Br[r][((i) >> 1) & 1][(i) & 1])); } while (0)
#define MORD 1
#define MIDX(i) (((i) & 4) ? (i) ^ 3 : (i))
#define GPOS 1
#define GSTR 8
#define RPOS 0
#define RSTR 2
#define XBAR BAR
#define XGL 4
#define XRD 12
#define STEP(b, z, vw, sa, sb) do { \
        SCHED; wait_v<vw>(); __builtin_amdgcn_s_waitcnt(0xC07F); XBAR; SCHED; \
        _Pragma("unroll") for (int i = 0; i < 32; ++i) { \
            MF2(z, (b) & 1, MIDX(i)); SCHED; \
            if (i >= GPOS && (i - GPOS) % GSTR == 0 && (i - GPOS) / GSTR < XGL) { GL(b, (i - GPOS) / GSTR, sa, sb); SCHED; } \
            if (i >= RPOS && (i - RPOS) % RSTR == 0 && (i - RPOS) / RSTR < XRD) { RD(((b) + 1) & 1, ((b) + 1) & 3, (i - RPOS) / RSTR); SCHED; } \
        } } while (0)
    ready(0, 0);
#pragma unroll
    for (int b = 0; b < 4; ++b)
#pragma unroll
        for (int i = 0; i < 4; ++i) GL(b, i, cA, cB);
    SCHED; WAIT_V(0); BAR; SCHED;
#pragma unroll
    for (int i = 0; i < 12; ++i) RD(0, 0, i);
    const __amdgpu_buffer_rsrc_t ro = mk_rsrc(P.out, OUT_BYTES);
#pragma unroll 1
    for (int ui = 0; ui < ntile; ++ui) {
        const bool has_next = ui + 1 < ntile;
        const int nj = !has_next ? j : lw < 28 ? j + RC : j + (ui & 1), nmb = lw < 28 ? mb : mb ^ 1;
        const unsigned nA = OFF_WB + (unsigned)nj * PANEL_BYTES, nB = OFF_XA + (unsigned)nmb * PANEL_BYTES;
        STEP(0, true, 24, cA + 4 * SSTR, cB + 4 * SSTR); STEP(1, false, 24, cA + 4 * SSTR, cB + 4 * SSTR); STEP(2, false, 24, cA + 4 * SSTR, cB + 4 * SSTR);
        STEP(3, false, 8, cA + 4 * SSTR, cB + 4 * SSTR);
        const unsigned lim = lw < 28 ? unit_need(ui, 15, g) : 0;
#pragma unroll 1
        for (int t = 2; t <= 32; ++t) {
            const bool last = t == 32;
            const unsigned so = (unsigned)(t >> 3) * 524288u + (unsigned)((t >> 1) & 3) * 16384u + (unsigned)(t & 1) * 262144u;
            const unsigned sa = last ? nA : cA + so, sb = last ? nB : cB + so;
            if (!(t & 1) && ((unsigned)known < lim || last)) {
                if (!last) ready(ui, t >> 1);
                else if (has_next) ready(ui + 1, 0);
            }
            STEP(0, false, 8, sa, sb); STEP(1, false, 8, sa, sb); STEP(2, false, 8, sa, sb); STEP(3, false, 8, sa, sb);
        }
        asm volatile("s_nop 15\n\ts_nop 15" ::: "memory");
        SCHED;
        {
            const int wi = ct_wi(j);
            const bool own = wi < NOWN;
            const unsigned row = (unsigned)(mb * 256 + wc * 32 + fr), cb = (unsigned)(wi * 256 + wr * 128 + fq * 8) * 2u;
            const unsigned obase = row * 65536u + cb + (own ? (unsigned)rank * OWNB : (SH0 - NOWN) * 512u);
            const unsigned pbase = OFF_OS + row * OWNB + cb;
            const float one = known < 0 ? __builtin_nanf("") : 1.0f;
            {
#pragma unroll
                for (int bj = 0; bj < 2; ++bj)
#pragma unroll
                    for (int n = 0; n < 2; ++n) {
                        f32x4 sv[8];
#pragma unroll
                        for (int m = 0; m < 8; ++m) sv[m] = silu4(acc[m >> 2][bj][m & 3][n], one);
#pragma unroll
                        for (int r = 0; r < 4; ++r) {
                            u32x4 o;
#pragma unroll
                            for (int e = 0; e < 4; ++e) o[e] = cvt_pk_bf16(sv[2 * e][r], sv[2 * e + 1][r]);
                            __builtin_amdgcn_raw_buffer_store_b128(o, ro, obase + r * 64u + (bj * 128 + n * 16) * 65536u, 0, EPI_AUX);
                            if (own) __builtin_amdgcn_raw_buffer_store_b128(o, rw, pbase + r * 64u + (bj * 128 + n * 16) * (unsigned)OWNB, 0, OS_AUX);
                        }
                    }
            }
            if (own) {
                WAIT_V(0);
                unsigned old = 0;
                if (lane == 0) old = __hip_atomic_fetch_add((unsigned*)(ws + OFF_TD + mb * 128), 1u, __ATOMIC_RELAXED, __HIP_MEMORY_SCOPE_AGENT);
                old = (unsigned)rfl((int)old) + 1u;
                if (old % (8u * NOWN) == 0u && lane < NR)
                    __hip_atomic_store((GAS unsigned*)(wsl + OFF_FL + (N_XW + mb) * 32 + rank * 4), call, __ATOMIC_RELAXED, __HIP_MEMORY_SCOPE_SYSTEM);
            }
        }
        SCHED;
        cA = nA; cB = nB; j = nj; mb = nmb;
    }
    WAIT_V(0);
    BAR;
    if (wid == 0) {
        unsigned old = 0;
        if (lane == 0) old = __hip_atomic_fetch_add((unsigned*)(ws + OFF_GP + g * 128 + 64), 1u, __ATOMIC_RELAXED, __HIP_MEMORY_SCOPE_AGENT);
        if (((unsigned)rfl((int)old) + 1u) % WG_PER_GRP == 0u) __builtin_amdgcn_fence(__ATOMIC_RELEASE, "agent");
    }
}

__global__ __launch_bounds__(512, 2) void fused_kernel(Params P) {
    __shared__ __attribute__((aligned(16))) unsigned char smem[LDS_BYTES];
    LAS unsigned char* lds = (LAS unsigned char*)smem;
    const int b = blockIdx.x;
    if (b < NCOMM) {
        if (threadIdx.x == 0) *(LAS unsigned*)(lds + 1024) = take_ticket(P, 0);
        __syncthreads();
        const unsigned call = (unsigned)rfl((int)*(LAS unsigned*)(lds + 1024)) / (GRID / 16) + 1u;
        comm_role(P, lds, call, b);
    } else {
#define GMAP 0
        const int x = b & 7;
        gemm_role(P, lds, GMAP ? (~x & 1) * 4 + (x >> 1) : x, (b - NCOMM) >> 3);
    }
}

}

extern "C" void kernel_launch(int rank, int nranks, ncclComm_t comm, hipStream_t stream,
                              void* const* d_in, const int* in_sizes, int n_in,
                              void* d_out, int out_size,
                              const int* in_dtypes, int out_dtype,
                              void* const* d_out_all,
                              void* const* d_ws_all, size_t ws_bytes) {
    (void)comm; (void)in_sizes; (void)n_in; (void)d_out_all; (void)out_size; (void)in_dtypes; (void)out_dtype;
    if (nranks != NR || (unsigned)rank >= (unsigned)NR || ws_bytes < WS_NEED) {
        fprintf(stderr, "unsupported configuration\n");
        return;
    }
    Params P;
    P.x = (const u16*)d_in[0]; P.w = (const u16*)d_in[1];
    for (int q = 0; q < NR; ++q) P.ws[q] = (char*)d_ws_all[(rank + 1 + q) % NR];
    P.out = (char*)d_out;
    P.rank = rank;
    hipLaunchKernelGGL(fused_kernel, dim3(GRID), dim3(512), 0, stream, P);
}
```

```cpp
#include <hip/hip_runtime.h>
#include <rccl/rccl.h>
#include <cstdio>

namespace {

typedef unsigned short u16;
typedef short bf16x8 __attribute__((ext_vector_type(8)));
typedef float f32x4 __attribute__((ext_vector_type(4)));
typedef unsigned u32x4 __attribute__((ext_vector_type(4)));
#define LAS __attribute__((address_space(3)))
#define GAS __attribute__((address_space(1)))

constexpr int NR = 8;
constexpr int NN = 32768, KSH = 512;
constexpr int GRID = 256;
constexpr int NCOMM = 24;
constexpr int NGEMM = GRID - NCOMM;
constexpr int WG_PER_GRP = NGEMM / 8;
constexpr int RC = 14, NROUND = 6;
constexpr int NCT = RC * NROUND + 2;
constexpr int NOWN = 6;
constexpr int OWNB = NOWN * 512;
constexpr int SH0 = 8 * NOWN;
static_assert(WG_PER_GRP == 29 && NCT == 128 - 7 * NOWN, "tile map");
#define PD 21
#define NS 5
#define NSLOT 18
#define AHEAD 5
#define NHOLE 6
#define QS 4
#define SL0 3
#define NSL1 2
#define DLY_L 300u
#define NEARLY 72
#define ST_AUX_OUT 18
#define ST_AUX 16
#define RUN1 8
#define RUNT 2
#define RUNF 8
#define RUN_DIV 72
#define GX_AUX 0
#define GW_AUX 0
#define LD_AUX 17
#define OS_AUX 17
#define EPI_AUX 18
static_assert(NS >= 1 && NS <= 5 && PD >= 7 && PD <= 63 && RUN1 <= 64, "waves of a communication workgroup");
static_assert(NHOLE >= 0 && NHOLE <= NOWN && (QS == 1 || QS == 2 || QS == 4) && NSL1 >= 0 && NSL1 <= 4, "pushed items");

constexpr unsigned OFF_TMO = 128;
constexpr unsigned OFF_PROG = 256;
constexpr unsigned OFF_WQ = 512;
constexpr unsigned OFF_GP = 1024;
constexpr unsigned OFF_TICKET = 2048;
constexpr unsigned OFF_TD = 4096;
constexpr unsigned OFF_FO = 8192;
constexpr unsigned OFF_FL = 16384;
constexpr unsigned OFF_XA = 1u << 20;
constexpr unsigned SEND_BYTES = 262144u;
constexpr unsigned PANEL_BYTES = 2u << 20;
constexpr unsigned OFF_WB = OFF_XA + 16u * PANEL_BYTES;
constexpr unsigned OFF_SEND = OFF_WB + (unsigned)NCT * PANEL_BYTES;
constexpr unsigned OFF_OS = OFF_SEND + 8u * NOWN * SEND_BYTES;
constexpr unsigned OFF_MINE = OFF_OS + 4096u * OWNB;
constexpr unsigned WS_NEED = OFF_MINE + (64u + NCT * 4u) * 65536u;
constexpr unsigned OUT_BYTES = 256u << 20;

constexpr int N_R0 = 120, N_R1 = N_R0 + 4 * RC;
constexpr int N_XW = 64 + NCT * 4;
constexpr int HOLE = 101 - NHOLE;
constexpr int K3N = 30 - NHOLE;
constexpr int V_R0 = N_R0 - NHOLE;
constexpr int V_XW = N_XW - NHOLE;
constexpr int V_ALL = V_XW + OWNB / 16;
constexpr int NUNIT = V_ALL * 64;
constexpr int N_GRP = N_XW + 16;
static_assert(NUNIT + 256 < 65535, "progress fits 16 bits");
static_assert(OFF_FL + (N_GRP + 8) * 32 <= 65536, "flag table");
__host__ __device__ constexpr int ct_wi(int j) { return j >= RC ? j : (j & 1) * 7 + (j >> 1); }
__device__ __forceinline__ void item_decode(int n, bool& isx, int& idx, int& kb) {
    if (n < N_R0) {
        kb = n / 30; const int r = n - kb * 30;
        isx = r < 4 || (r >= 11 && r < 23);
        idx = r < 4 ? r : r < 11 ? 20 - 2 * r : r < 23 ? r - 7 : 2 * (r - 23) + 1;
    } else {
        const bool extra = n >= N_R1 && n < N_R1 + 8;
        const int m = n - (n < N_R1 ? N_R0 : N_R0 + 8), i = m / (4 * RC), rem = m - i * (4 * RC);
        kb = rem / RC; const int r = rem - kb * RC;
        isx = false;
        idx = RC * (i + 1) + (r < 7 ? 2 * r : 2 * (r - 7) + 1);
        if (extra) { idx = RC * NROUND + ((n - N_R1) >> 2); kb = (n - N_R1) & 3; }
    }
}
__device__ __forceinline__ int unit_need(int i, int ks, int g) {
    const int kb = ks >> 2, sl = (ks & 3) * QS >> 2;
    if (i == 0) {
        if (kb < SL0) return (30 * kb + (g < 4 ? 11 + 4 * g : 30)) * 64;
        if (kb < 3) return 30 * kb * 64 + (sl * 30 + (g < 4 ? 11 + 4 * g : 30)) * (64 / QS);
        return 90 * 64 + (sl * K3N + (g < 4 ? 11 - NHOLE + 4 * g : K3N)) * (64 / QS);
    }
    if (i == 1 && kb < NSL1) return (V_R0 + RC * kb) * 64 + (sl * RC + (g < 4 ? 7 : RC)) * (64 / QS);
    return (V_R0 + 4 * RC * (i - 1) + (i > 1 ? 8 : 0) + RC * kb + (g < 4 ? 7 : RC)) * 64;
}

constexpr unsigned long long DEADLINE_TICKS = 100000000ull;
constexpr int STAGE_BYTES = NSLOT * 7168 > 131072 ? NSLOT * 7168 : 131072;
constexpr int LDS_BYTES = STAGE_BYTES + 4096;

struct Params {
    const u16* x; const u16* w;
    char* ws[NR];
    char* out;
    int rank;
};

__device__ __forceinline__ int rfl(int v) { return __builtin_amdgcn_readfirstlane(v); }

__device__ __forceinline__ __amdgpu_buffer_rsrc_t mk_rsrc(const void* p, unsigned bytes) {
    return __builtin_amdgcn_make_buffer_rsrc(const_cast<void*>(p), 0, (int)bytes, 0x00020000);
}
__device__ __forceinline__ bool timed_out(unsigned long long t0, unsigned* tmo, unsigned call) {
    if (__builtin_amdgcn_s_memrealtime() - t0 > DEADLINE_TICKS) {
        __hip_atomic_store(tmo, call, __ATOMIC_RELAXED, __HIP_MEMORY_SCOPE_AGENT);
        return true;
    }
    return false;
}
__device__ __forceinline__ unsigned lds_rd(unsigned addr) {
    unsigned v; asm volatile("ds_read_b32 %0, %1\n\ts_waitcnt lgkmcnt(0)" : "=v"(v) : "v"(addr) : "memory"); return v;
}
__device__ __forceinline__ u32x4 lds_rd4(unsigned addr) {
    u32x4 v; asm volatile("ds_read_b128 %0, %1\n\ts_waitcnt lgkmcnt(0)" : "=v"(v) : "v"(addr) : "memory"); return v;
}
__device__ __forceinline__ void lds_wr4(unsigned addr, u32x4 v) { asm volatile("ds_write_b128 %0, %1" :: "v"(addr), "v"(v) : "memory"); }
__device__ __forceinline__ void lds_wr(unsigned addr, unsigned v) { asm volatile("ds_write_b32 %0, %1" :: "v"(addr), "v"(v) : "memory"); }

#define WAIT_V(n) asm volatile("s_waitcnt vmcnt(" #n ")" ::: "memory")
#define WAIT_L(n) asm volatile("s_waitcnt lgkmcnt(" #n ")" ::: "memory")
#define SCHED __builtin_amdgcn_sched_barrier(0)
#define BAR __builtin_amdgcn_s_barrier()
#define DRAIN_V() do { SCHED; __builtin_amdgcn_s_waitcnt(0x0F70); WAIT_V(0); SCHED; } while (0)
template <int N> __device__ __forceinline__ void wait_v() { asm volatile("s_waitcnt vmcnt(%0)" :: "n"(N) : "memory"); }


constexpr int AUX_SYS = 17;

__device__ __forceinline__ GAS char* lane_ws(const Params& P, int lane) {
    char* wsl = P.ws[0];
#pragma unroll
    for (int p = 1; p < NR; ++p) {
        char* t = P.ws[p];
        asm volatile("" : "+s"(t));
        wsl = (lane == p) ? t : wsl;
    }
    return (GAS char*)wsl;
}

constexpr unsigned M_OUT = 2;
__device__ __forceinline__ u32x4 unit_entry(int k, int v, int rank) {
    if (v >= NUNIT) return (u32x4){(unsigned)(k + 1) << 16 | 0xffffu, 0u, 0u, 0u};
    int n = v >> 6, u = v & 63;
    if (n >= V_XW) {
        unsigned o = (unsigned)(v - V_XW * 64) + (unsigned)rank * (OWNB / 2u);
        if (o >= OWNB * 4u) o -= OWNB * 4u;
        const unsigned row = o / (OWNB / 1024u);
        return (u32x4){(unsigned)(k + 1) << 16 | (unsigned)v, OFF_OS + o * 1024u + M_OUT, row * 65536u + (o - row * (OWNB / 1024u)) * 1024u, (unsigned)N_XW};
    }
    if (n >= V_R0 + NSL1 * RC) n += NHOLE;
    else if (n >= V_R0) {
        const int v1 = v - V_R0 * 64, sl = v1 / (RC * 64 / QS), rem = v1 - sl * (RC * 64 / QS);
        n = N_R0 + sl / QS * RC + rem / (64 / QS); u = sl % QS * (64 / QS) + rem % (64 / QS);
    } else if (n >= 90) {
        const int v1 = v - 90 * 64, sl = v1 / (K3N * 64 / QS), rem = v1 - sl * (K3N * 64 / QS);
        n = 90 + rem / (64 / QS); u = sl * (64 / QS) + rem % (64 / QS);
        if (n >= HOLE) n += NHOLE;
    } else if (n >= SL0 * 30) {
        const int sl = v / (30 * 64 / QS), rem = v - sl * (30 * 64 / QS);
        n = sl / QS * 30 + rem / (64 / QS); u = sl % QS * (64 / QS) + rem % (64 / QS);
    }
    bool isx; int idx, kb;
    item_decode(n, isx, idx, kb);
    const bool own = !isx && ct_wi(idx) < NOWN;
    const unsigned lb = (isx ? OFF_XA : OFF_WB) + (unsigned)idx * PANEL_BYTES + (unsigned)kb * 524288u + u * 1024u;
    const unsigned rb = own ? OFF_SEND + (unsigned)(rank * NOWN + ct_wi(idx)) * SEND_BYTES + (unsigned)kb * 65536u + u * 1024u : OFF_MINE + (unsigned)n * 65536u + u * 1024u;
    return (u32x4){(unsigned)(k + 1) << 16 | (unsigned)v, rb, lb, (unsigned)n};
}
__device__ __forceinline__ int lead_batches(unsigned cnt) {
    int r = __builtin_amdgcn_readlane((int)cnt, 0) * NS;
#pragma unroll
    for (int s = 1; s < NS; ++s) r = min(r, __builtin_amdgcn_readlane((int)cnt, s) * NS + s);
    return r;
}

__device__ __forceinline__ void comm_role(const Params& P, LAS unsigned char* lds, unsigned call, int c) {
    const int tid = threadIdx.x, wid = rfl(tid >> 6), lane = tid & 63;
    const int rank = P.rank;
    unsigned* tmo = (unsigned*)(P.ws[7] + OFF_TMO);
    const unsigned ctl = (unsigned)(uintptr_t)(lds + STAGE_BYTES);
    constexpr unsigned L_AVAIL = 0, L_FULL = 4, L_USED = 12, L_FREED = 32, L_DONE = 64, L_DQ = 2048;
    constexpr int NDQ = 128;
    static_assert(2 * NS + NSLOT + AHEAD + 64 + NS <= NDQ, "ring of claimed units");
    constexpr unsigned BADV = 0xffffu;
    const unsigned pbase = call << 16;
    auto desc = [&](int k, unsigned& rb, unsigned& lb) -> int {
        const u32x4 e = lds_rd4(ctl + L_DQ + (k & (NDQ - 1)) * 16);
        const unsigned e0 = (unsigned)rfl((int)e.x);
        rb = (unsigned)rfl((int)e.y); lb = (unsigned)rfl((int)e.z);
        return (e0 >> 16) == (unsigned)(k + 1) ? (int)(e0 & 0xffffu) : -1;
    };
    if (tid < 256) lds_wr4(ctl + tid * 16, tid >= 128 && tid < 128 + RUNF ? unit_entry(tid - 128, c * RUNF + tid - 128, rank) : (u32x4){0u, 0u, 0u, 0u});
    WAIT_L(0);
    __syncthreads();
    if (wid == 0) {
        __builtin_amdgcn_s_setprio(3);
        int avail = 0, freed = 0, i = 0;
        __amdgpu_buffer_rsrc_t r[7];
#pragma unroll
        for (int q = 0; q < 7; ++q) r[q] = mk_rsrc(P.ws[q], WS_NEED);
        bool more = true;
#pragma unroll 1
        while (more) {
            const u32x4 e = lds_rd4(ctl + L_DQ + ((i + lane) & (NDQ - 1)) * 16);
            const int nv = (int)__builtin_ctzll(~__ballot((e.x >> 16) == (unsigned)(i + lane + 1)) | 1ull << 63);
            if (!nv) __builtin_amdgcn_s_sleep(1);
#pragma unroll 1
            for (int u = 0; u < nv; ++u, ++i) {
                const int unit0 = __builtin_amdgcn_readlane((int)e.x, u) & 0xffff, n = __builtin_amdgcn_readlane((int)e.w, u);
                const unsigned rb = (unsigned)__builtin_amdgcn_readlane((int)e.y, u);
                if (unit0 == 0xffff) { more = false; break; }
                lds_wr(ctl + L_USED, (unsigned)(i + 1));
                while (n >= avail) {
                    avail = rfl((int)lds_rd(ctl + L_AVAIL));
                    if (n >= avail) { WAIT_V(0); lds_wr(ctl + L_FULL, (unsigned)i); __builtin_amdgcn_s_sleep(1); }
                }
                const unsigned off = (rb & ~3u) + lane * 16u;
                while (i - NSLOT >= freed) { freed = lead_batches(lds_rd(ctl + L_FREED + (lane & 7) * 4)); if (i - NSLOT >= freed) __builtin_amdgcn_s_sleep(1); }
                LAS unsigned char* slot = lds + (unsigned)i % NSLOT * 7168u;
                SCHED;
#pragma unroll
                for (int q = 0; q < 7; ++q) {
                    wait_v<PD - 1>();
                    __builtin_amdgcn_raw_ptr_buffer_load_lds(r[q], (LAS void*)(slot + q * 1024), 16, off, 0, 0, LD_AUX);
                    SCHED;
                    if (q == (PD - 1) % 7 && i >= (PD - 1) / 7) lds_wr(ctl + L_FULL, (unsigned)(i - (PD - 1) / 7));
                }
            }
        }
#pragma unroll
        for (int k = (PD - 1) / 7; k >= 0; --k) {
            if (k == 8) wait_v<56>(); else if (k == 7) wait_v<49>(); else if (k == 6) wait_v<42>(); else if (k == 5) wait_v<35>(); else if (k == 4) wait_v<28>();
            else if (k == 3) wait_v<21>(); else if (k == 2) wait_v<14>(); else if (k == 1) wait_v<7>(); else wait_v<0>();
            if (i >= k) lds_wr(ctl + L_FULL, (unsigned)(i - k));
        }
    } else if (wid == 1) {
        const unsigned* fl = (const unsigned*)(P.ws[7] + OFF_FL) + lane;
        unsigned* prog = (unsigned*)(P.ws[7] + OFF_PROG);
        const unsigned long long t0 = __builtin_amdgcn_s_memrealtime();
        int pos = 0, mine = 0, all = c < 8 ? 0 : V_XW * 64;
        bool pushed = c >= 8;
        while (pos < N_GRP || mine < V_XW * 64 || all < V_XW * 64) {
            unsigned v = call, pv = 0, hv = call;
            if (pos < N_GRP) v = __hip_atomic_load(fl + pos * 8, __ATOMIC_RELAXED, __HIP_MEMORY_SCOPE_SYSTEM);
            if (!pushed) hv = __hip_atomic_load(fl + HOLE * 8, __ATOMIC_RELAXED, __HIP_MEMORY_SCOPE_SYSTEM);
            if (all < V_XW * 64) pv = __hip_atomic_load(prog + min(lane & 31, NCOMM - 1), __ATOMIC_RELAXED, __HIP_MEMORY_SCOPE_AGENT);
            if (mine < V_XW * 64) {
                unsigned rb, lb;
                const int m = desc(lead_batches(lds_rd(ctl + L_DONE + (lane & 7) * 4)), rb, lb);
                if (m > mine) {
                    if (lane == 0) __hip_atomic_store(prog + c, pbase + (unsigned)m, __ATOMIC_RELAXED, __HIP_MEMORY_SCOPE_AGENT);
                    mine = m;
                }
            }
            int adv = 0;
            if (pos < N_GRP) {
                const int it = pos + (lane >> 3);
                const unsigned long long miss = __ballot((int)(v - call) < 0 && it < N_GRP && (it < HOLE || it >= HOLE + NHOLE));
                adv = miss ? (int)(__builtin_ctzll(miss) >> 3) : 8;
                pos += adv;
                if (adv) lds_wr(ctl + L_AVAIL, (unsigned)(pos <= N_XW ? pos : pos < N_GRP ? N_XW : N_XW + 1));
            }
            if (!pushed) pushed = __all((int)(hv - call) >= 0 || lane >= NHOLE * 8);
            if (all < V_XW * 64) {
                int m = (int)(pv - pbase);
#pragma unroll
                for (int o = 16; o >= 1; o >>= 1) m = min(m, __shfl_xor(m, o));
                m = rfl(m);
                if (!pushed) m = min(m, 90 * 64);
                if (m > all) {
                    if (lane == 0) __hip_atomic_store((unsigned*)(P.ws[7] + OFF_GP + c * 128), pbase + (unsigned)m, __ATOMIC_RELAXED, __HIP_MEMORY_SCOPE_AGENT);
                    all = m;
                }
            }
            if (timed_out(t0, tmo, call)) { lds_wr(ctl + L_AVAIL, BADV); break; }
            if (!adv) __builtin_amdgcn_s_sleep(4);
        }
        if (lane < NR - 1)
            __hip_atomic_store((GAS unsigned*)(lane_ws(P, lane) + OFF_FO + c * 128 + rank * 4), call, __ATOMIC_RELAXED, __HIP_MEMORY_SCOPE_SYSTEM);
        const unsigned* fo = (const unsigned*)(P.ws[7] + OFF_FO + c * 128) + (lane & 7);
        for (;;) {
            const unsigned v = __hip_atomic_load(fo, __ATOMIC_RELAXED, __HIP_MEMORY_SCOPE_SYSTEM);
            if (__all((int)(v - call) >= 0 || (lane & 7) == rank)) break;
            if (timed_out(t0, tmo, call)) break;
            __builtin_amdgcn_s_sleep(16);
        }
    } else if (wid == 2) {
        unsigned* wq = (unsigned*)(P.ws[7] + OFF_WQ + (call & 1u) * 128u);
        if (c == 0 && lane == 0)
            __hip_atomic_store((unsigned*)(P.ws[7] + OFF_WQ + ((call + 1u) & 1u) * 128u), 0u, __ATOMIC_RELAXED, __HIP_MEMORY_SCOPE_AGENT);
        int v = 0;
#pragma unroll 1
        for (int k = RUNF; v < NUNIT;) {
            while (rfl((int)lds_rd(ctl + L_USED)) + AHEAD < k) __builtin_amdgcn_s_sleep(1);
            const int m = max(RUNT, min(RUN1, (NUNIT - v) / RUN_DIV));
            if (lane == 0) v = (int)__hip_atomic_fetch_add(wq, (unsigned)m, __ATOMIC_RELAXED, __HIP_MEMORY_SCOPE_AGENT) + NCOMM * RUNF;
            v = rfl(v);
            if (lane < m || (v + m > NUNIT && lane < m + NS)) lds_wr4(ctl + L_DQ + ((k + lane) & (NDQ - 1)) * 16, unit_entry(k + lane, v + lane, rank));
            k += m;
            if (v + m > NUNIT) break;
        }
    } else if (wid < 3 + NS) {
        const int s = wid - 3;
        int full = 0;
        unsigned cnt = 0, done = 0;
        bool bad = false;
#pragma unroll 1
        for (int i = s;; i += NS) {
            int unit0; unsigned rb, lb;
            while ((unit0 = desc(i, rb, lb)) < 0) {
                if (done != cnt) { DRAIN_V(); done = cnt; lds_wr(ctl + L_DONE + s * 4, done); }
                else __builtin_amdgcn_s_sleep(1);
            }
            if (unit0 == 0xffff) break;
            const bool isout = (rb & 3u) == M_OUT;
            const unsigned off = lb + lane * 16u;
            const __amdgpu_buffer_rsrc_t r = mk_rsrc(isout ? P.out : P.ws[7], isout ? OUT_BYTES : WS_NEED);
            while (i >= full) {
                const unsigned cw = lds_rd(ctl + (lane & 1) * 4);
                full = __builtin_amdgcn_readlane((int)cw, 1);
                bad = (unsigned)__builtin_amdgcn_readlane((int)cw, 0) == BADV;
                if (i >= full) {
                    if (done != cnt) { DRAIN_V(); done = cnt; lds_wr(ctl + L_DONE + s * 4, done); }
                    else __builtin_amdgcn_s_sleep(1);
                }
            }
            u32x4 v[7];
#pragma unroll
            for (int q = 0; q < 7; ++q) v[q] = *(const LAS u32x4*)(lds + (unsigned)i % NSLOT * 7168u + q * 1024 + lane * 16);
            SCHED; WAIT_L(0); SCHED;
            ++cnt;
            lds_wr(ctl + L_FREED + s * 4, cnt);
#pragma unroll
            for (int q = 0; q < 7; ++q) {
                if (bad) v[q] = (u32x4){0x7fc07fc0u, 0x7fc07fc0u, 0x7fc07fc0u, 0x7fc07fc0u};
                if (!isout) __builtin_amdgcn_raw_buffer_store_b128(v[q], r, off + q * 65536u, 0, ST_AUX);
                else __builtin_amdgcn_raw_buffer_store_b128(v[q], r, off + (unsigned)((rank + 1 + q) & 7) * OWNB, 0, ST_AUX_OUT);
            }
            SCHED;
            if (done + 1u != cnt) { wait_v<7>(); done = cnt - 1u; lds_wr(ctl + L_DONE + s * 4, done); }
            if (isout) { done = cnt; lds_wr(ctl + L_DONE + s * 4, done); }
        }
        if (done != cnt) { DRAIN_V(); lds_wr(ctl + L_DONE + s * 4, cnt); }
    }
}

#define LDIN(p) __builtin_nontemporal_load(p)
__device__ __forceinline__ unsigned take_ticket(const Params& P, int zv) {
    return __hip_atomic_fetch_add((unsigned*)(P.ws[7] + OFF_TICKET + (blockIdx.x & 15) * 128) + zv, 1u, __ATOMIC_RELAXED, __HIP_MEMORY_SCOPE_AGENT);
}
__device__ __forceinline__ unsigned stage_role(const Params& P, GAS char* wsl, unsigned ctl, int s0) {
    const int tid = threadIdx.x, sub = rfl(tid >> 6), lane = tid & 63;
    const int rank = P.rank;
    const int qi = 4 * (lane & 3) + ((lane & 15) >> 2);
    constexpr int NP8 = 7 * NHOLE, G = 30 + 7 * NOWN, S3 = 3 * G, S4 = S3 + 30, S5 = S4 + 7 * (NOWN - NHOLE), N_STAGE = NP8 + S5 + N_XW - N_R0;
    unsigned call = 0, tk = 0;
    int zv = 0;
    asm volatile("" : "+v"(zv));
    if (s0 >= NP8 + NEARLY) {
        const unsigned long long t0 = __builtin_amdgcn_s_memrealtime();
        while (__builtin_amdgcn_s_memrealtime() - t0 < DLY_L) __builtin_amdgcn_s_sleep(2);
    }
#pragma unroll 1
    for (int s1 = s0; s1 < N_STAGE; s1 += NGEMM) {
        const bool push = s1 < NP8;
        const int s = s1 - NP8;
        int n, q = 7;
        if (push) { const int jo = s1 / 7; q = s1 - jo * 7; n = HOLE + jo; }
        else if (s < S3) {
            const int kb = s / G, r = s - kb * G;
            if (r < 30) n = kb * 30 + r;
            else { const int jo = (r - 30) / 7; q = r - 30 - jo * 7; n = kb * 30 + 5 + jo; }
        } else if (s < S4) n = 90 + s - S3;
        else if (s < S5) { const int jo = (s - S4) / 7; q = s - S4 - jo * 7; n = 95 + jo; }
        else n = N_R0 + s - S5;
        const __amdgpu_buffer_rsrc_t rw = mk_rsrc(P.ws[push ? q : 7], WS_NEED);
        bool isx; int idx, kb;
        item_decode(n, isx, idx, kb);
        const int wi = ct_wi(idx);
        const bool own = !isx && wi < NOWN;
        u32x4 v[8];
        if (isx) {
            const u16* src = P.x + (size_t)(idx * 256 + sub * 32 + (lane & 15)) * KSH + kb * 128 + (lane >> 4) * 8;
#pragma unroll
            for (int t = 0; t < 8; ++t) v[t] = LDIN((const u32x4*)(src + (t >> 2) * 16 * KSH + (t & 3) * 32));
            SCHED;
            if (s1 == s0 && tid == 0) tk = take_ticket(P, zv);
            SCHED;
            const unsigned dst = OFF_MINE + (unsigned)n * 65536u + sub * 2048u + lane * 16u;
#pragma unroll
            for (int t = 0; t < 8; ++t)
                __builtin_amdgcn_raw_buffer_store_b128(v[t], rw, dst + (t >> 2) * 1024 + (t & 3) * 16384, 0, AUX_SYS);
        } else {
            const int wr = sub & 1;
            const int p = (rank + 1 + q) & 7;
            const int col = own ? p * (OWNB / 2) + wi * 256 : (SH0 - NOWN + wi) * 256;
            const u16* src = P.w + (size_t)(kb * 128 + (sub >> 1) * 32 + (lane >> 4) * 8) * NN + col + wr * 128 + qi * 8;
#pragma unroll
            for (int t = 0; t < 8; ++t) v[t] = LDIN((const u32x4*)(src + (size_t)t * NN));
            SCHED;
            if (s1 == s0 && tid == 0) tk = take_ticket(P, zv);
            SCHED;
            const unsigned base = q < 7 && !push ? OFF_SEND + (unsigned)(p * NOWN + wi) * SEND_BYTES + (unsigned)kb * 65536u
                                : push ? OFF_WB + (unsigned)idx * PANEL_BYTES + (unsigned)(kb * 8 + 6 - q) * 65536u : OFF_MINE + (unsigned)n * 65536u;
            const unsigned dst = base + (sub >> 1) * 16384u + wr * 4096u + lane * 16u;
#pragma unroll
            for (int t = 0; t < 8; ++t) {
                u32x4 o;
#pragma unroll
                for (int e = 0; e < 4; ++e) o[e] = __builtin_amdgcn_perm(v[2 * e + 1][t >> 1], v[2 * e][t >> 1], (t & 1) ? 0x07060302u : 0x05040100u);
                __builtin_amdgcn_raw_buffer_store_b128(o, rw, dst + (t >> 2) * 8192 + (t & 3) * 1024, 0, AUX_SYS);
            }
        }
        DRAIN_V();
        if (s1 == s0 && tid == 0) { lds_wr(ctl + 4, tk); WAIT_L(0); }
        BAR;
        if (s1 == s0) call = (unsigned)rfl((int)lds_rd(ctl + 4)) / (GRID / 16) + 1u;
        if (sub == 0 && (own ? lane == q : lane < NR))
            __hip_atomic_store((GAS unsigned*)(wsl + OFF_FL + n * 32 + rank * 4), call, __ATOMIC_RELAXED, __HIP_MEMORY_SCOPE_SYSTEM);
        SCHED;
    }
    return call;
}

__device__ __forceinline__ unsigned cvt_pk_bf16(float lo, float hi) {
    unsigned r; asm volatile("v_cvt_pk_bf16_f32 %0, %1, %2" : "=v"(r) : "v"(lo), "v"(hi)); return r;
}
__device__ __forceinline__ f32x4 silu4(f32x4 y, float one) {
    f32x4 t = y * -1.44269504088896341f;
    asm("" : "+v"(t));
    f32x4 e;
#pragma unroll
    for (int i = 0; i < 4; ++i) e[i] = __builtin_amdgcn_exp2f(__builtin_amdgcn_fmed3f(t[i], -86.5617024533378f, 86.5617024533378f));
    e = e + one;
#pragma unroll
    for (int i = 0; i < 4; ++i) e[i] = __builtin_amdgcn_rcpf(e[i]);
    return e * y;
}

__device__ __forceinline__ void gemm_role(const Params& P, LAS unsigned char* lds, int g, int lw) {
    const int tid = threadIdx.x, wid = rfl(tid >> 6), lane = tid & 63, wr = wid >> 2, wc = wid & 3, fr = lane & 15, fq = lane >> 4;
    const int rank = P.rank;
    char* const ws = P.ws[7];
    unsigned* tmo = (unsigned*)(ws + OFF_TMO);
    const unsigned ldsw = (unsigned)wid * 1024u;
    volatile LAS int* lknown = (volatile LAS int*)(lds + STAGE_BYTES);
    int known = 0;
    GAS char* wsl = lane_ws(P, lane);
    asm volatile("" : "+v"(wsl));
    const unsigned call = stage_role(P, wsl, (unsigned)(uintptr_t)(lds + STAGE_BYTES), lw * 8 + g);

    auto ready = [&](int i, int ks) {
        const int need = lw < 28 ? unit_need(i, ks, g) : (V_R0 + 4 * RC + 4 + (i >> 1) * 4) * 64;
        if ((unsigned)known >= (unsigned)need) return;
        if (wid == 0) {
            const unsigned* f = (const unsigned*)(ws + OFF_GP + g * 128);
            const unsigned long long t0 = __builtin_amdgcn_s_memrealtime();
            int m;
            for (;;) {
                m = rfl((int)(__hip_atomic_load(f, __ATOMIC_RELAXED, __HIP_MEMORY_SCOPE_AGENT) - (call << 16)));
                if (m >= need) break;
                if (timed_out(t0, tmo, call)) { m = -1; break; }
                __builtin_amdgcn_s_sleep(8);
            }
            if (lane == 0) lknown[0] = m;
            if (known == 0) __builtin_amdgcn_fence(__ATOMIC_ACQUIRE, "");
            WAIT_V(0); WAIT_L(0);
        }
        asm volatile("" ::: "memory"); BAR; asm volatile("" ::: "memory");
        known = rfl(lknown[0]);
    };

    const int ntile = lw < 28 ? NROUND : 4;
    int j = lw < 28 ? 2 * (lw >> 2) + (g >> 2) : RC * NROUND;
    int mb = 4 * (g & 3) + (lw < 28 ? lw & 3 : 2 * (g >> 2));
    f32x4 acc[2][2][4][2];
    bf16x8 Ar[2][2][4], Br[2][2][2];
    const __amdgpu_buffer_rsrc_t rw = mk_rsrc(ws, WS_NEED);
    unsigned gv = tid * 16u;
    unsigned la = (unsigned)(uintptr_t)lds + wr * 4096u + lane * 16u, lb = (unsigned)(uintptr_t)lds + 65536u + wc * 2048u + lane * 16u;
    asm volatile("" : "+v"(gv), "+v"(la), "+v"(lb));
    unsigned cA = OFF_WB + (unsigned)j * PANEL_BYTES, cB = OFF_XA + (unsigned)mb * PANEL_BYTES;
#define SSTR 65536u
#define GL(b, i, sa, sb) do { \
        if ((i) & 1) __builtin_amdgcn_raw_ptr_buffer_load_lds(rw, (LAS void*)(lds + 65536 + (b) * 16384 + ((i) >> 1) * 8192 + ldsw), 16, gv, (sb) + (b) * SSTR + ((i) >> 1) * 8192u, 0, GX_AUX); \
        else __builtin_amdgcn_raw_ptr_buffer_load_lds(rw, (LAS void*)(lds + (b) * 16384 + ((i) >> 1) * 8192 + ldsw), 16, gv, (sa) + (b) * SSTR + ((i) >> 1) * 8192u, 0, GW_AUX); } while (0)
#define RDA(r, b, h, m) Ar[r][h][m] = *(const LAS bf16x8*)(uintptr_t)(la + (b) * 16384 + (h) * 8192 + (m) * 1024)
#define RDB(r, b, h, n) Br[r][h][n] = *(const LAS bf16x8*)(uintptr_t)(lb + (b) * 16384 + (h) * 8192 + (n) * 1024)
#define RD(r, b, i) do { if ((i) < 4) RDB(r, b, (i) >> 1, (i) & 1); else RDA(r, b, ((i) - 4) >> 2, ((i) - 4) & 3); } while (0)
#define ACC2(i) acc[(i) >> 4][((i) >> 1) & 1][((i) >> 2) & 3][(i) & 1]
#define MF2(z, r, i) do { \
        if (z) asm volatile("v_mfma_f32_16x16x32_bf16 %0, %1, %2, 0" : "=&v"(ACC2(i)) : "v"(Ar[r][(i) >> 4][((i) >> 2) & 3]), "v"(Br[r][((i) >> 1) & 1][(i) & 1])); \
        else asm volatile("v_mfma_f32_16x16x32_bf16 %0, %1, %2, %0" : "+v"(ACC2(i)) : "v"(Ar[r][(i) >> 4][((i) >> 2) & 3]), "v"(Br[r][((i) >> 1) & 1][(i) & 1])); } while (0)
#define MIDX(i) (((i) & 4) ? (i) ^ 3 : (i))
#define GPOS 1
#define GSTR 8
#define RPOS 0
#define RSTR 2
#define XBAR BAR
#define XGL 4
#define XRD 12
#define STEP(b, z, vw, sa, sb) do { \
        SCHED; wait_v<vw>(); __builtin_amdgcn_s_waitcnt(0xC07F); XBAR; SCHED; \
        _Pragma("unroll") for (int i = 0; i < 32; ++i) { \
            MF2(z, (b) & 1, MIDX(i)); SCHED; \
            if (i >= GPOS && (i - GPOS) % GSTR == 0 && (i - GPOS) / GSTR < XGL) { GL(b, (i - GPOS) / GSTR, sa, sb); SCHED; } \
            if (i >= RPOS && (i - RPOS) % RSTR == 0 && (i - RPOS) / RSTR < XRD) { RD(((b) + 1) & 1, ((b) + 1) & 3, (i - RPOS) / RSTR); SCHED; } \
        } } while (0)
    ready(0, 0);
#pragma unroll
    for (int b = 0; b < 4; ++b)
#pragma unroll
        for (int i = 0; i < 4; ++i) GL(b, i, cA, cB);
    SCHED; WAIT_V(0); BAR; SCHED;
#pragma unroll
    for (int i = 0; i < 12; ++i) RD(0, 0, i);
    const __amdgpu_buffer_rsrc_t ro = mk_rsrc(P.out, OUT_BYTES);
#pragma unroll 1
    for (int ui = 0; ui < ntile; ++ui) {
        const bool has_next = ui + 1 < ntile;
        const int nj = !has_next ? j : lw < 28 ? j + RC : j + (ui & 1), nmb = lw < 28 ? mb : mb ^ 1;
        const unsigned nA = OFF_WB + (unsigned)nj * PANEL_BYTES, nB = OFF_XA + (unsigned)nmb * PANEL_BYTES;
        STEP(0, true, 24, cA + 4 * SSTR, cB + 4 * SSTR); STEP(1, false, 24, cA + 4 * SSTR, cB + 4 * SSTR); STEP(2, false, 24, cA + 4 * SSTR, cB + 4 * SSTR);
        const int jr = j % RC, ja = (jr & 1) * 7 + (jr >> 1);
        const unsigned ia = j < RC ? (j & 1 ? 16 + ja : 10 - ja) : j >= RC * NROUND ? N_R1 + jr * 4 : N_R0 + (j / RC - 1) * (4 * RC) + (j >= 2 * RC ? 8 : 0) + ja;
        const unsigned ka = (j < RC ? 30u : j >= RC * NROUND ? 1u : (unsigned)RC) * 65536u;
        const unsigned mA = OFF_MINE - 3 * SSTR + ia * 65536u, mB = OFF_MINE - 3 * SSTR + (unsigned)(mb < 4 ? mb : mb + 7) * 65536u;
        STEP(3, false, 8, mA, mB);
        const unsigned lim = lw < 28 ? unit_need(ui, 15, g) : 0;
#pragma unroll 1
        for (int t = 2; t <= 32; ++t) {
            const bool last = t == 32;
            const unsigned so = (unsigned)(t >> 3) * 524288u + (unsigned)((t >> 1) & 3) * 16384u + (unsigned)(t & 1) * 262144u;
            const unsigned sa = last ? nA : cA + so, sb = last ? nB : cB + so;
            if (!(t & 1) && ((unsigned)known < lim || last)) {
                if (!last) ready(ui, t >> 1);
                else if (has_next) ready(ui + 1, 0);
            }
            const unsigned qo = (unsigned)((t >> 1) & 3) * 16384u;
            const unsigned ta = t & 1 ? mA + (unsigned)(t >> 3) * ka + qo : sa, tb = t & 1 ? mB + (unsigned)(t >> 3) * (30u * 65536u) + qo : sb;
            STEP(0, false, 8, sa, sb); STEP(1, false, 8, sa, sb); STEP(2, false, 8, sa, sb); STEP(3, false, 8, ta, tb);
        }
        asm volatile("s_nop 15\n\ts_nop 15" ::: "memory");
        SCHED;
        {
            const int wi = ct_wi(j);
            const bool own = wi < NOWN;
            const unsigned row = (unsigned)(mb * 256 + wc * 32 + fr), cb = (unsigned)(wi * 256 + wr * 128 + fq * 8) * 2u;
            const unsigned obase = row * 65536u + cb + (own ? (unsigned)rank * OWNB : (SH0 - NOWN) * 512u);
            const unsigned pbase = OFF_OS + row * OWNB + cb;
            const float one = known < 0 ? __builtin_nanf("") : 1.0f;
            {
#pragma unroll
                for (int bj = 0; bj < 2; ++bj)
#pragma unroll
                    for (int n = 0; n < 2; ++n) {
                        f32x4 sv[8];
#pragma unroll
                        for (int m = 0; m < 8; ++m) sv[m] = silu4(acc[m >> 2][bj][m & 3][n], one);
#pragma unroll
                        for (int r = 0; r < 4; ++r) {
                            u32x4 o;
#pragma unroll
                            for (int e = 0; e < 4; ++e) o[e] = cvt_pk_bf16(sv[2 * e][r], sv[2 * e + 1][r]);
                            __builtin_amdgcn_raw_buffer_store_b128(o, ro, obase + r * 64u + (bj * 128 + n * 16) * 65536u, 0, EPI_AUX);
                            if (own) __builtin_amdgcn_raw_buffer_store_b128(o, rw, pbase + r * 64u + (bj * 128 + n * 16) * (unsigned)OWNB, 0, OS_AUX);
                        }
                    }
            }
            if (own) {
                WAIT_V(0);
                unsigned old = 0;
                if (lane == 0) old = __hip_atomic_fetch_add((unsigned*)(ws + OFF_TD + mb * 128), 1u, __ATOMIC_RELAXED, __HIP_MEMORY_SCOPE_AGENT);
                old = (unsigned)rfl((int)old) + 1u;
                if (old % (8u * NOWN) == 0u && lane < NR)
                    __hip_atomic_store((GAS unsigned*)(wsl + OFF_FL + (N_XW + mb) * 32 + rank * 4), call, __ATOMIC_RELAXED, __HIP_MEMORY_SCOPE_SYSTEM);
            }
        }
        SCHED;
        cA = nA; cB = nB; j = nj; mb = nmb;
    }
    WAIT_V(0);
    BAR;
    if (wid == 0) {
        unsigned old = 0;
        if (lane == 0) old = __hip_atomic_fetch_add((unsigned*)(ws + OFF_GP + g * 128 + 64), 1u, __ATOMIC_RELAXED, __HIP_MEMORY_SCOPE_AGENT);
        if (((unsigned)rfl((int)old) + 1u) % WG_PER_GRP == 0u) __builtin_amdgcn_fence(__ATOMIC_RELEASE, "agent");
    }
}

__global__ __launch_bounds__(512, 2) void fused_kernel(Params P) {
    __shared__ __attribute__((aligned(16))) unsigned char smem[LDS_BYTES];
    LAS unsigned char* lds = (LAS unsigned char*)smem;
    const int b = blockIdx.x;
    if (b < NCOMM) {
        if (threadIdx.x == 0) *(LAS unsigned*)(lds + 1024) = take_ticket(P, 0);
        __syncthreads();
        const unsigned call = (unsigned)rfl((int)*(LAS unsigned*)(lds + 1024)) / (GRID / 16) + 1u;
        comm_role(P, lds, call, b);
    } else {
#define GMAP 0
        const int x = b & 7;
        gemm_role(P, lds, GMAP ? (~x & 1) * 4 + (x >> 1) : x, (b - NCOMM) >> 3);
    }
}

}

extern "C" void kernel_launch(int rank, int nranks, ncclComm_t comm, hipStream_t stream,
                              void* const* d_in, const int* in_sizes, int n_in,
                              void* d_out, int out_size,
                              const int* in_dtypes, int out_dtype,
                              void* const* d_out_all,
                              void* const* d_ws_all, size_t ws_bytes) {
    (void)comm; (void)in_sizes; (void)n_in; (void)d_out_all; (void)out_size; (void)in_dtypes; (void)out_dtype;
    if (nranks != NR || (unsigned)rank >= (unsigned)NR || ws_bytes < WS_NEED) {
        fprintf(stderr, "unsupported configuration\n");
        return;
    }
    Params P;
    P.x = (const u16*)d_in[0]; P.w = (const u16*)d_in[1];
    for (int q = 0; q < NR; ++q) P.ws[q] = (char*)d_ws_all[(rank + 1 + q) % NR];
    P.out = (char*)d_out;
    P.rank = rank;
    hipLaunchKernelGGL(fused_kernel, dim3(GRID), dim3(512), 0, stream, P);
}
```

```cpp
#include <hip/hip_runtime.h>
#include <rccl/rccl.h>
#include <cstdio>

namespace {

typedef unsigned short u16;
typedef short bf16x8 __attribute__((ext_vector_type(8)));
typedef float f32x4 __attribute__((ext_vector_type(4)));
typedef unsigned u32x4 __attribute__((ext_vector_type(4)));
#define LAS __attribute__((address_space(3)))
#define GAS __attribute__((address_space(1)))

constexpr int NR = 8;
constexpr int NN = 32768, KSH = 512;
constexpr int GRID = 256;
constexpr int NCOMM = 24;
constexpr int NGEMM = GRID - NCOMM;
constexpr int WG_PER_GRP = NGEMM / 8;
constexpr int RC = 14, NROUND = 6;
constexpr int NCT = RC * NROUND + 2;
constexpr int NOWN = 6;
constexpr int OWNB = NOWN * 512;
constexpr int SH0 = 8 * NOWN;
static_assert(WG_PER_GRP == 29 && NCT == 128 - 7 * NOWN, "tile map");
#define PD 21
#define NS 5
#define NSLOT 18
#define AHEAD 5
#define NHOLE 6
#define PACE 1
#define QS 4
#define SL0 2
#define NSL1 2
#define DLY_L 300u
#define NEARLY 72
#define ST_AUX_OUT 18
#define ST_AUX 16
#define RUN1 8
#define RUNT 2
#define RUNF 8
#define RUN_DIV 72
#define GX_AUX 0
#define GW_AUX 0
#define LD_AUX 17
#define OS_AUX 17
#define EPI_AUX 18
static_assert(NS >= 1 && NS <= 5 && PD >= 7 && PD <= 63 && RUN1 <= 64, "waves of a communication workgroup");
static_assert(NHOLE >= 0 && NHOLE <= NOWN && (QS == 1 || QS == 2 || QS == 4) && NSL1 >= 0 && NSL1 <= 4, "pushed items");

constexpr int NPW = 56;

constexpr unsigned OFF_TMO = 128;
constexpr unsigned OFF_PROG = 256;
constexpr unsigned OFF_WQ = 512;
constexpr unsigned OFF_GP = 1024;
constexpr unsigned OFF_TICKET = 2048;
constexpr unsigned OFF_TD = 4096;
constexpr unsigned OFF_FO = 8192;
constexpr unsigned OFF_FL = 16384;
constexpr unsigned OFF_XA = 1u << 20;
constexpr unsigned SEND_BYTES = 262144u;
constexpr unsigned PANEL_BYTES = 2u << 20;
constexpr unsigned OFF_WB = OFF_XA + 16u * PANEL_BYTES;
constexpr unsigned OFF_SEND = OFF_WB + (unsigned)NCT * PANEL_BYTES;
constexpr unsigned OFF_OS = OFF_SEND + 8u * NOWN * SEND_BYTES;
constexpr unsigned OFF_MINE = OFF_OS + 4096u * OWNB;
constexpr unsigned WS_NEED = OFF_MINE + (64u + NCT * 4u) * 65536u;
constexpr unsigned OUT_BYTES = 256u << 20;

constexpr int N_R0 = 120, N_R1 = N_R0 + 4 * RC;
constexpr int N_XW = 64 + NCT * 4;
constexpr int HOLE = 101 - NHOLE;
constexpr int K3N = 30 - NHOLE;
constexpr int V_R0 = N_R0 - NHOLE;
constexpr int V_XW = N_XW - NHOLE;
constexpr int V_ALL = V_XW + OWNB / 16;
constexpr int NUNIT = V_ALL * 64;
constexpr int N_GRP = N_XW + 16;
static_assert(NUNIT + 256 < 65535, "progress fits 16 bits");
static_assert(OFF_FL + (N_GRP + 8) * 32 <= 65536, "flag table");
__host__ __device__ constexpr int ct_wi(int j) { return j >= RC ? j : (j & 1) * 7 + (j >> 1); }
__device__ __forceinline__ void item_decode(int n, bool& isx, int& idx, int& kb) {
    if (n < N_R0) {
        kb = n / 30; const int r = n - kb * 30;
        isx = r < 4 || (r >= 11 && r < 23);
        idx = r < 4 ? r : r < 11 ? 20 - 2 * r : r < 23 ? r - 7 : 2 * (r - 23) + 1;
    } else {
        const bool extra = n >= N_R1 && n < N_R1 + 8;
        const int m = n - (n < N_R1 ? N_R0 : N_R0 + 8), i = m / (4 * RC), rem = m - i * (4 * RC);
        kb = rem / RC; const int r = rem - kb * RC;
        isx = false;
        idx = RC * (i + 1) + (r < 7 ? 2 * r : 2 * (r - 7) + 1);
        if (extra) { idx = RC * NROUND + ((n - N_R1) >> 2); kb = (n - N_R1) & 3; }
    }
}
__device__ __forceinline__ int unit_need(int i, int ks, int g) {
    const int kb = ks >> 2, sl = (ks & 3) * QS >> 2;
    if (i == 0) {
        if (kb < SL0) return (30 * kb + (g < 4 ? 11 + 4 * g : 30)) * 64;
        if (kb < 3) return 30 * kb * 64 + (sl * 30 + (g < 4 ? 11 + 4 * g : 30)) * (64 / QS);
        return 90 * 64 + (sl * K3N + (g < 4 ? 11 - NHOLE + 4 * g : K3N)) * (64 / QS);
    }
    if (i == 1 && kb < NSL1) return (V_R0 + RC * kb) * 64 + (sl * RC + (g < 4 ? 7 : RC)) * (64 / QS);
    return (V_R0 + 4 * RC * (i - 1) + (i > 1 ? 8 : 0) + RC * kb + (g < 4 ? 7 : RC)) * 64;
}

constexpr unsigned long long DEADLINE_TICKS = 100000000ull;
constexpr int STAGE_BYTES = NSLOT * 7168 > 131072 ? NSLOT * 7168 : 131072;
constexpr int LDS_BYTES = STAGE_BYTES + 4096;

struct Params {
    const u16* x; const u16* w;
    char* ws[NR];
    char* out;
    int rank;
};

__device__ __forceinline__ int rfl(int v) { return __builtin_amdgcn_readfirstlane(v); }

__device__ __forceinline__ __amdgpu_buffer_rsrc_t mk_rsrc(const void* p, unsigned bytes) {
    return __builtin_amdgcn_make_buffer_rsrc(const_cast<void*>(p), 0, (int)bytes, 0x00020000);
}
__device__ __forceinline__ bool timed_out(unsigned long long t0, unsigned* tmo, unsigned call) {
    if (__builtin_amdgcn_s_memrealtime() - t0 > DEADLINE_TICKS) {
        __hip_atomic_store(tmo, call, __ATOMIC_RELAXED, __HIP_MEMORY_SCOPE_AGENT);
        return true;
    }
    return false;
}
__device__ __forceinline__ unsigned lds_rd(unsigned addr) {
    unsigned v; asm volatile("ds_read_b32 %0, %1\n\ts_waitcnt lgkmcnt(0)" : "=v"(v) : "v"(addr) : "memory"); return v;
}
__device__ __forceinline__ u32x4 lds_rd4(unsigned addr) {
    u32x4 v; asm volatile("ds_read_b128 %0, %1\n\ts_waitcnt lgkmcnt(0)" : "=v"(v) : "v"(addr) : "memory"); return v;
}
__device__ __forceinline__ void lds_wr4(unsigned addr, u32x4 v) { asm volatile("ds_write_b128 %0, %1" :: "v"(addr), "v"(v) : "memory"); }
__device__ __forceinline__ void lds_wr(unsigned addr, unsigned v) { asm volatile("ds_write_b32 %0, %1" :: "v"(addr), "v"(v) : "memory"); }

#define WAIT_V(n) asm volatile("s_waitcnt vmcnt(" #n ")" ::: "memory")
#define WAIT_L(n) asm volatile("s_waitcnt lgkmcnt(" #n ")" ::: "memory")
#define SCHED __builtin_amdgcn_sched_barrier(0)
#define BAR __builtin_amdgcn_s_barrier()
#define DRAIN_V() do { SCHED; __builtin_amdgcn_s_waitcnt(0x0F70); WAIT_V(0); SCHED; } while (0)
template <int N> __device__ __forceinline__ void wait_v() { asm volatile("s_waitcnt vmcnt(%0)" :: "n"(N) : "memory"); }


constexpr int AUX_SYS = 17;

__device__ __forceinline__ GAS char* lane_ws(const Params& P, int lane) {
    char* wsl = P.ws[0];
#pragma unroll
    for (int p = 1; p < NR; ++p) {
        char* t = P.ws[p];
        asm volatile("" : "+s"(t));
        wsl = (lane == p) ? t : wsl;
    }
    return (GAS char*)wsl;
}

constexpr unsigned M_OUT = 2;
__device__ __forceinline__ u32x4 unit_entry(int k, int v, int rank) {
    if (v >= NUNIT) return (u32x4){(unsigned)(k + 1) << 16 | 0xffffu, 0u, 0u, 0u};
    int n = v >> 6, u = v & 63;
    if (n >= V_XW) {
        unsigned o = (unsigned)(v - V_XW * 64) + (unsigned)rank * (OWNB / 2u);
        if (o >= OWNB * 4u) o -= OWNB * 4u;
        const unsigned row = o / (OWNB / 1024u);
        return (u32x4){(unsigned)(k + 1) << 16 | (unsigned)v, OFF_OS + o * 1024u + M_OUT, row * 65536u + (o - row * (OWNB / 1024u)) * 1024u, (unsigned)N_XW};
    }
    if (n >= V_R0 + NSL1 * RC) n += NHOLE;
    else if (n >= V_R0) {
        const int v1 = v - V_R0 * 64, sl = v1 / (RC * 64 / QS), rem = v1 - sl * (RC * 64 / QS);
        n = N_R0 + sl / QS * RC + rem / (64 / QS); u = sl % QS * (64 / QS) + rem % (64 / QS);
    } else if (n >= 90) {
        const int v1 = v - 90 * 64, sl = v1 / (K3N * 64 / QS), rem = v1 - sl * (K3N * 64 / QS);
        n = 90 + rem / (64 / QS); u = sl * (64 / QS) + rem % (64 / QS);
        if (n >= HOLE) n += NHOLE;
    } else if (n >= SL0 * 30) {
        const int sl = v / (30 * 64 / QS), rem = v - sl * (30 * 64 / QS);
        n = sl / QS * 30 + rem / (64 / QS); u = sl % QS * (64 / QS) + rem % (64 / QS);
    }
    bool isx; int idx, kb;
    item_decode(n, isx, idx, kb);
    const bool own = !isx && ct_wi(idx) < NOWN;
    const unsigned lb = (isx ? OFF_XA : OFF_WB) + (unsigned)idx * PANEL_BYTES + (unsigned)kb * 524288u + u * 1024u;
    const unsigned rb = own ? OFF_SEND + (unsigned)(rank * NOWN + ct_wi(idx)) * SEND_BYTES + (unsigned)kb * 65536u + u * 1024u : OFF_MINE + (unsigned)n * 65536u + u * 1024u;
    return (u32x4){(unsigned)(k + 1) << 16 | (unsigned)v, rb, lb, (unsigned)n};
}
__device__ __forceinline__ int lead_batches(unsigned cnt) {
    int r = __builtin_amdgcn_readlane((int)cnt, 0) * NS;
#pragma unroll
    for (int s = 1; s < NS; ++s) r = min(r, __builtin_amdgcn_readlane((int)cnt, s) * NS + s);
    return r;
}

__device__ __forceinline__ void comm_role(const Params& P, LAS unsigned char* lds, unsigned call, int c) {
    const int tid = threadIdx.x, wid = rfl(tid >> 6), lane = tid & 63;
    const int rank = P.rank;
    unsigned* tmo = (unsigned*)(P.ws[7] + OFF_TMO);
    const unsigned ctl = (unsigned)(uintptr_t)(lds + STAGE_BYTES);
    constexpr unsigned L_AVAIL = 0, L_FULL = 4, L_USED = 12, L_FREED = 32, L_DONE = 64, L_DQ = 2048;
    constexpr int NDQ = 128;
    static_assert(2 * NS + NSLOT + AHEAD + 64 + NS <= NDQ, "ring of claimed units");
    constexpr unsigned BADV = 0xffffu;
    const unsigned pbase = call << 16;
    auto desc = [&](int k, unsigned& rb, unsigned& lb) -> int {
        const u32x4 e = lds_rd4(ctl + L_DQ + (k & (NDQ - 1)) * 16);
        const unsigned e0 = (unsigned)rfl((int)e.x);
        rb = (unsigned)rfl((int)e.y); lb = (unsigned)rfl((int)e.z);
        return (e0 >> 16) == (unsigned)(k + 1) ? (int)(e0 & 0xffffu) : -1;
    };
    if (tid < 256) lds_wr4(ctl + tid * 16, tid >= 128 && tid < 128 + RUNF ? unit_entry(tid - 128, c * RUNF + tid - 128, rank) : (u32x4){0u, 0u, 0u, 0u});
    WAIT_L(0);
    __syncthreads();
    if (wid == 0) {
        __builtin_amdgcn_s_setprio(3);
        int avail = 0, freed = 0, i = 0;
        __amdgpu_buffer_rsrc_t r[7];
#pragma unroll
        for (int q = 0; q < 7; ++q) r[q] = mk_rsrc(P.ws[q], WS_NEED);
        bool more = true;
#pragma unroll 1
        while (more) {
            const u32x4 e = lds_rd4(ctl + L_DQ + ((i + lane) & (NDQ - 1)) * 16);
            const int nv = (int)__builtin_ctzll(~__ballot((e.x >> 16) == (unsigned)(i + lane + 1)) | 1ull << 63);
            if (!nv) __builtin_amdgcn_s_sleep(1);
#pragma unroll 1
            for (int u = 0; u < nv; ++u, ++i) {
                const int unit0 = __builtin_amdgcn_readlane((int)e.x, u) & 0xffff, n = __builtin_amdgcn_readlane((int)e.w, u);
                const unsigned rb = (unsigned)__builtin_amdgcn_readlane((int)e.y, u);
                if (unit0 == 0xffff) { more = false; break; }
                lds_wr(ctl + L_USED, (unsigned)(i + 1));
                while (n >= avail) {
                    avail = rfl((int)lds_rd(ctl + L_AVAIL));
                    if (n >= avail) { WAIT_V(0); lds_wr(ctl + L_FULL, (unsigned)i); __builtin_amdgcn_s_sleep(1); }
                }
                const unsigned off = (rb & ~3u) + lane * 16u;
                while (i - NSLOT >= freed) { freed = lead_batches(lds_rd(ctl + L_FREED + (lane & 7) * 4)); if (i - NSLOT >= freed) __builtin_amdgcn_s_sleep(1); }
                LAS unsigned char* slot = lds + (unsigned)i % NSLOT * 7168u;
                SCHED;
#pragma unroll
                for (int q = 0; q < 7; ++q) {
                    wait_v<PD - 1>();
                    __builtin_amdgcn_raw_ptr_buffer_load_lds(r[q], (LAS void*)(slot + q * 1024), 16, off, 0, 0, LD_AUX);
                    SCHED;
                    if (q == (PD - 1) % 7 && i >= (PD - 1) / 7) lds_wr(ctl + L_FULL, (unsigned)(i - (PD - 1) / 7));
                }
            }
        }
#pragma unroll
        for (int k = (PD - 1) / 7; k >= 0; --k) {
            if (k == 8) wait_v<56>(); else if (k == 7) wait_v<49>(); else if (k == 6) wait_v<42>(); else if (k == 5) wait_v<35>(); else if (k == 4) wait_v<28>();
            else if (k == 3) wait_v<21>(); else if (k == 2) wait_v<14>(); else if (k == 1) wait_v<7>(); else wait_v<0>();
            if (i >= k) lds_wr(ctl + L_FULL, (unsigned)(i - k));
        }
    } else if (wid == 1) {
        const unsigned* fl = (const unsigned*)(P.ws[7] + OFF_FL) + lane;
        unsigned* prog = (unsigned*)(P.ws[7] + OFF_PROG);
        const unsigned long long t0 = __builtin_amdgcn_s_memrealtime();
        int pos = 0, mine = 0, all = c < 8 ? 0 : V_XW * 64;
        bool pushed = c >= 8;
        if (c < 8) { __builtin_amdgcn_fence(__ATOMIC_ACQUIRE, ""); WAIT_V(0); }
        while (pos < N_GRP || mine < V_XW * 64 || all < V_XW * 64) {
            unsigned v = call, pv = 0, hv = call;
            if (pos < N_GRP) v = __hip_atomic_load(fl + pos * 8, __ATOMIC_RELAXED, __HIP_MEMORY_SCOPE_SYSTEM);
            if (!pushed) hv = __hip_atomic_load(fl + ((lane & 7) == rank ? HOLE : N_GRP) * 8, __ATOMIC_RELAXED, __HIP_MEMORY_SCOPE_SYSTEM);
            if (all < V_XW * 64) pv = __hip_atomic_load(prog + min(lane & 31, NCOMM - 1), __ATOMIC_RELAXED, __HIP_MEMORY_SCOPE_AGENT);
            if (mine < V_XW * 64) {
                unsigned rb, lb;
                const int m = desc(lead_batches(lds_rd(ctl + L_DONE + (lane & 7) * 4)), rb, lb);
                if (m > mine) {
                    if (lane == 0) __hip_atomic_store(prog + c, pbase + (unsigned)m, __ATOMIC_RELAXED, __HIP_MEMORY_SCOPE_AGENT);
                    mine = m;
                }
            }
            int adv = 0;
            if (pos < N_GRP) {
                const int it = pos + (lane >> 3);
                const unsigned long long miss = __ballot((int)(v - call) < 0 && it < N_GRP && (it < HOLE || it >= HOLE + NHOLE));
                adv = miss ? (int)(__builtin_ctzll(miss) >> 3) : 8;
                pos += adv;
                if (adv) lds_wr(ctl + L_AVAIL, (unsigned)(pos <= N_XW ? pos : pos < N_GRP ? N_XW : N_XW + 1));
            }
            if (!pushed) pushed = __all((int)(hv - call) >= 0 || (lane >= NHOLE * 8 && (lane & 7) == rank));
            if (all < V_XW * 64) {
                int m = (int)(pv - pbase);
#pragma unroll
                for (int o = 16; o >= 1; o >>= 1) m = min(m, __shfl_xor(m, o));
                m = rfl(m);
                if (!pushed) m = min(m, 90 * 64);
                if (m > all) {
                    if (lane == 0) __hip_atomic_store((unsigned*)(P.ws[7] + OFF_GP + c * 128), pbase + (unsigned)m, __ATOMIC_RELAXED, __HIP_MEMORY_SCOPE_AGENT);
                    all = m;
                }
            }
            if (timed_out(t0, tmo, call)) { lds_wr(ctl + L_AVAIL, BADV); break; }
            if (!adv) __builtin_amdgcn_s_sleep(4);
        }
        if (lane < NR - 1)
            __hip_atomic_store((GAS unsigned*)(lane_ws(P, lane) + OFF_FO + c * 128 + rank * 4), call, __ATOMIC_RELAXED, __HIP_MEMORY_SCOPE_SYSTEM);
        const unsigned* fo = (const unsigned*)(P.ws[7] + OFF_FO + c * 128) + (lane & 7);
        for (;;) {
            const unsigned v = __hip_atomic_load(fo, __ATOMIC_RELAXED, __HIP_MEMORY_SCOPE_SYSTEM);
            if (__all((int)(v - call) >= 0 || (lane & 7) == rank)) break;
            if (timed_out(t0, tmo, call)) break;
            __builtin_amdgcn_s_sleep(16);
        }
    } else if (wid == 2) {
        unsigned* wq = (unsigned*)(P.ws[7] + OFF_WQ + (call & 1u) * 128u);
        if (c == 0 && lane == 0)
            __hip_atomic_store((unsigned*)(P.ws[7] + OFF_WQ + ((call + 1u) & 1u) * 128u), 0u, __ATOMIC_RELAXED, __HIP_MEMORY_SCOPE_AGENT);
        int v = 0;
#pragma unroll 1
        for (int k = RUNF; v < NUNIT;) {
            while (rfl((int)lds_rd(ctl + L_USED)) + AHEAD < k) __builtin_amdgcn_s_sleep(1);
            const int m = max(RUNT, min(RUN1, (NUNIT - v) / RUN_DIV));
            if (lane == 0) v = (int)__hip_atomic_fetch_add(wq, (unsigned)m, __ATOMIC_RELAXED, __HIP_MEMORY_SCOPE_AGENT) + NCOMM * RUNF;
            v = rfl(v);
            if (lane < m || (v + m > NUNIT && lane < m + NS)) lds_wr4(ctl + L_DQ + ((k + lane) & (NDQ - 1)) * 16, unit_entry(k + lane, v + lane, rank));
            k += m;
            if (v + m > NUNIT) break;
        }
    } else if (wid < 3 + NS) {
        const int s = wid - 3;
        int full = 0;
        unsigned cnt = 0, done = 0;
        bool bad = false;
#pragma unroll 1
        for (int i = s;; i += NS) {
            int unit0; unsigned rb, lb;
            while ((unit0 = desc(i, rb, lb)) < 0) {
                if (done != cnt) { DRAIN_V(); done = cnt; lds_wr(ctl + L_DONE + s * 4, done); }
                else __builtin_amdgcn_s_sleep(1);
            }
            if (unit0 == 0xffff) break;
            const bool isout = (rb & 3u) == M_OUT;
            const unsigned off = lb + lane * 16u;
            const __amdgpu_buffer_rsrc_t r = mk_rsrc(isout ? P.out : P.ws[7], isout ? OUT_BYTES : WS_NEED);
            while (i >= full) {
                const unsigned cw = lds_rd(ctl + (lane & 1) * 4);
                full = __builtin_amdgcn_readlane((int)cw, 1);
                bad = (unsigned)__builtin_amdgcn_readlane((int)cw, 0) == BADV;
                if (i >= full) {
                    if (done != cnt) { DRAIN_V(); done = cnt; lds_wr(ctl + L_DONE + s * 4, done); }
                    else __builtin_amdgcn_s_sleep(1);
                }
            }
            u32x4 v[7];
#pragma unroll
            for (int q = 0; q < 7; ++q) v[q] = *(const LAS u32x4*)(lds + (unsigned)i % NSLOT * 7168u + q * 1024 + lane * 16);
            SCHED; WAIT_L(0); SCHED;
            ++cnt;
            lds_wr(ctl + L_FREED + s * 4, cnt);
#pragma unroll
            for (int q = 0; q < 7; ++q) {
                if (bad) v[q] = (u32x4){0x7fc07fc0u, 0x7fc07fc0u, 0x7fc07fc0u, 0x7fc07fc0u};
                if (!isout) __builtin_amdgcn_raw_buffer_store_b128(v[q], r, off + q * 65536u, 0, ST_AUX);
                else __builtin_amdgcn_raw_buffer_store_b128(v[q], r, off + (unsigned)((rank + 1 + q) & 7) * OWNB, 0, ST_AUX_OUT);
            }
            SCHED;
            if (done + 1u != cnt) { wait_v<7>(); done = cnt - 1u; lds_wr(ctl + L_DONE + s * 4, done); }
            if (isout) { done = cnt; lds_wr(ctl + L_DONE + s * 4, done); }
        }
        if (done != cnt) { DRAIN_V(); lds_wr(ctl + L_DONE + s * 4, cnt); }
    }
}

#define LDIN(p) __builtin_nontemporal_load(p)
__device__ __forceinline__ unsigned take_ticket(const Params& P, int zv) {
    return __hip_atomic_fetch_add((unsigned*)(P.ws[7] + OFF_TICKET + (blockIdx.x & 15) * 128) + zv, 1u, __ATOMIC_RELAXED, __HIP_MEMORY_SCOPE_AGENT);
}
__device__ __forceinline__ unsigned push_first(const Params& P, unsigned ctl, int s1) {
    const int tid = threadIdx.x, lane = tid & 63;
    const int q = s1 >> 3, wi = rfl(tid >> 6), fi = N_GRP + (s1 & 7);
    const unsigned sub = (unsigned)s1 & 7u;
    const u16* w = P.w; int rank = P.rank; char* wq = P.ws[q]; char* w7 = P.ws[7];
    const unsigned vo = ((sub >> 1) * 32u + ((unsigned)lane >> 4) * 8u) * (NN * 2u) + (sub & 1u) * 256u + (4u * (lane & 3) + ((lane & 15) >> 2)) * 16u;
    unsigned dst = OFF_WB + (unsigned)(2 * wi) * PANEL_BYTES + (unsigned)(30 - q) * 65536u + (sub >> 1) * 16384u + (sub & 1u) * 4096u + lane * 16u;
    int zv = 0;
    asm volatile("" : "+v"(dst), "+v"(zv));
    asm volatile("" : "+s"(w), "+s"(rank), "+s"(wq), "+s"(w7));
    const __amdgpu_buffer_rsrc_t rs = mk_rsrc(w, KSH * NN * 2u), rw = mk_rsrc(wq, WS_NEED);
    const unsigned so = 384u * (NN * 2u) + (unsigned)((rank + 1 + q) & 7) * OWNB + (unsigned)wi * 512u;
    if (wi < NHOLE) {
        u32x4 v[8];
#pragma unroll
        for (int t = 0; t < 8; ++t) v[t] = __builtin_amdgcn_raw_buffer_load_b128(rs, vo, so + t * (NN * 2u), 2);
        SCHED;
#pragma unroll
        for (int t = 0; t < 8; ++t) {
            u32x4 o;
#pragma unroll
            for (int e = 0; e < 4; ++e) o[e] = __builtin_amdgcn_perm(v[2 * e + 1][t >> 1], v[2 * e][t >> 1], (t & 1) ? 0x07060302u : 0x05040100u);
            if (t >= PACE) { SCHED; wait_v<PACE - 1>(); SCHED; }
            __builtin_amdgcn_raw_buffer_store_b128(o, rw, dst + (t >> 2) * 8192 + (t & 3) * 1024, 0, AUX_SYS);
        }
    }
    SCHED;
    unsigned tk = 0;
    if (tid == 448) tk = __hip_atomic_fetch_add((GAS unsigned*)(w7 + OFF_TICKET + (blockIdx.x & 15) * 128) + zv, 1u, __ATOMIC_RELAXED, __HIP_MEMORY_SCOPE_AGENT);
    DRAIN_V();
    if (tid == 448) { lds_wr(ctl + 4, tk); WAIT_L(0); }
    BAR;
    const unsigned call = (unsigned)rfl((int)lds_rd(ctl + 4)) / (GRID / 16) + 1u;
    if (tid == 0) __hip_atomic_store((GAS unsigned*)(wq + OFF_FL + fi * 32 + rank * 4), call, __ATOMIC_RELAXED, __HIP_MEMORY_SCOPE_SYSTEM);
    SCHED;
    return call;
}
__device__ __forceinline__ unsigned stage_role(const Params& P, GAS char* wsl, unsigned ctl, int s0, unsigned call) {
    const int tid = threadIdx.x, sub = rfl(tid >> 6), lane = tid & 63;
    const int rank = P.rank;
    const int qi = 4 * (lane & 3) + ((lane & 15) >> 2);
    constexpr int G = 30 + 7 * NOWN, S3 = 3 * G, S4 = S3 + 30, S5 = S4 + 7 * (NOWN - NHOLE), N_STAGE = NPW + S5 + N_XW - N_R0;
    unsigned tk = 0;
    int zv = 0;
    asm volatile("" : "+v"(zv));
    if (s0 >= NPW + NEARLY) {
        const unsigned long long t0 = __builtin_amdgcn_s_memrealtime();
        while (__builtin_amdgcn_s_memrealtime() - t0 < DLY_L) __builtin_amdgcn_s_sleep(2);
    }
#pragma unroll 1
    for (int s1 = s0 < NPW ? s0 + NGEMM : s0; s1 < N_STAGE; s1 += NGEMM) {
        const int s = s1 - NPW;
        int n, q = 7;
        if (s < S3) {
            const int kb = s / G, r = s - kb * G;
            if (r < 30) n = kb * 30 + r;
            else { const int jo = (r - 30) / 7; q = r - 30 - jo * 7; n = kb * 30 + 5 + jo; }
        } else if (s < S4) n = 90 + s - S3;
        else if (s < S5) { const int jo = (s - S4) / 7; q = s - S4 - jo * 7; n = 95 + jo; }
        else n = N_R0 + s - S5;
        const __amdgpu_buffer_rsrc_t rw = mk_rsrc(P.ws[7], WS_NEED);
        bool isx; int idx, kb;
        item_decode(n, isx, idx, kb);
        const int wi = ct_wi(idx);
        const bool own = !isx && wi < NOWN;
        u32x4 v[8];
        if (isx) {
            const u16* src = P.x + (size_t)(idx * 256 + sub * 32 + (lane & 15)) * KSH + kb * 128 + (lane >> 4) * 8;
#pragma unroll
            for (int t = 0; t < 8; ++t) v[t] = LDIN((const u32x4*)(src + (t >> 2) * 16 * KSH + (t & 3) * 32));
            SCHED;
            if (s1 == s0 && tid == 0) tk = take_ticket(P, zv);
            SCHED;
            const unsigned dst = OFF_MINE + (unsigned)n * 65536u + sub * 2048u + lane * 16u;
#pragma unroll
            for (int t = 0; t < 8; ++t)
                __builtin_amdgcn_raw_buffer_store_b128(v[t], rw, dst + (t >> 2) * 1024 + (t & 3) * 16384, 0, AUX_SYS);
        } else {
            const int wr = sub & 1;
            const int p = (rank + 1 + q) & 7;
            const int col = own ? p * (OWNB / 2) + wi * 256 : (SH0 - NOWN + wi) * 256;
            const u16* src = P.w + (size_t)(kb * 128 + (sub >> 1) * 32 + (lane >> 4) * 8) * NN + col + wr * 128 + qi * 8;
#pragma unroll
            for (int t = 0; t < 8; ++t) v[t] = LDIN((const u32x4*)(src + (size_t)t * NN));
            SCHED;
            if (s1 == s0 && tid == 0) tk = take_ticket(P, zv);
            SCHED;
            const unsigned base = q < 7 ? OFF_SEND + (unsigned)(p * NOWN + wi) * SEND_BYTES + (unsigned)kb * 65536u : OFF_MINE + (unsigned)n * 65536u;
            const unsigned dst = base + (sub >> 1) * 16384u + wr * 4096u + lane * 16u;
#pragma unroll
            for (int t = 0; t < 8; ++t) {
                u32x4 o;
#pragma unroll
                for (int e = 0; e < 4; ++e) o[e] = __builtin_amdgcn_perm(v[2 * e + 1][t >> 1], v[2 * e][t >> 1], (t & 1) ? 0x07060302u : 0x05040100u);
                __builtin_amdgcn_raw_buffer_store_b128(o, rw, dst + (t >> 2) * 8192 + (t & 3) * 1024, 0, AUX_SYS);
            }
        }
        DRAIN_V();
        if (s1 == s0 && tid == 0) { lds_wr(ctl + 4, tk); WAIT_L(0); }
        BAR;
        if (s1 == s0) call = (unsigned)rfl((int)lds_rd(ctl + 4)) / (GRID / 16) + 1u;
        if (sub == 0 && (own ? lane == q : lane < NR))
            __hip_atomic_store((GAS unsigned*)(wsl + OFF_FL + n * 32 + rank * 4), call, __ATOMIC_RELAXED, __HIP_MEMORY_SCOPE_SYSTEM);
        SCHED;
    }
    return call;
}

__device__ __forceinline__ unsigned cvt_pk_bf16(float lo, float hi) {
    unsigned r; asm volatile("v_cvt_pk_bf16_f32 %0, %1, %2" : "=v"(r) : "v"(lo), "v"(hi)); return r;
}
__device__ __forceinline__ f32x4 silu4(f32x4 y, float one) {
    f32x4 t = y * -1.44269504088896341f;
    asm("" : "+v"(t));
    f32x4 e;
#pragma unroll
    for (int i = 0; i < 4; ++i) e[i] = __builtin_amdgcn_exp2f(__builtin_amdgcn_fmed3f(t[i], -86.5617024533378f, 86.5617024533378f));
    e = e + one;
#pragma unroll
    for (int i = 0; i < 4; ++i) e[i] = __builtin_amdgcn_rcpf(e[i]);
    return e * y;
}

__device__ __forceinline__ void gemm_role(const Params& P, LAS unsigned char* lds, int g, int lw, unsigned call0) {
    const int tid = threadIdx.x, wid = rfl(tid >> 6), lane = tid & 63, wr = wid >> 2, wc = wid & 3, fr = lane & 15, fq = lane >> 4;
    const int rank = P.rank;
    char* const ws = P.ws[7];
    unsigned* tmo = (unsigned*)(ws + OFF_TMO);
    const unsigned ldsw = (unsigned)wid * 1024u;
    volatile LAS int* lknown = (volatile LAS int*)(lds + STAGE_BYTES);
    int known = 0;
    GAS char* wsl = lane_ws(P, lane);
    asm volatile("" : "+v"(wsl));
    const unsigned call = stage_role(P, wsl, (unsigned)(uintptr_t)(lds + STAGE_BYTES), lw * 8 + g, call0);

    auto ready = [&](int i, int ks) {
        const int need = lw < 28 ? unit_need(i, ks, g) : (V_R0 + 4 * RC + 4 + (i >> 1) * 4) * 64;
        if ((unsigned)known >= (unsigned)need) return;
        if (wid == 0) {
            const unsigned* f = (const unsigned*)(ws + OFF_GP + g * 128);
            const unsigned long long t0 = __builtin_amdgcn_s_memrealtime();
            int m;
            for (;;) {
                m = rfl((int)(__hip_atomic_load(f, __ATOMIC_RELAXED, __HIP_MEMORY_SCOPE_AGENT) - (call << 16)));
                if (m >= need) break;
                if (timed_out(t0, tmo, call)) { m = -1; break; }
                __builtin_amdgcn_s_sleep(8);
            }
            if (lane == 0) lknown[0] = m;
            WAIT_V(0); WAIT_L(0);
        }
        asm volatile("" ::: "memory"); BAR; asm volatile("" ::: "memory");
        known = rfl(lknown[0]);
    };

    const int ntile = lw < 28 ? NROUND : 4;
    int j = lw < 28 ? 2 * (lw >> 2) + (g >> 2) : RC * NROUND;
    int mb = 4 * (g & 3) + (lw < 28 ? lw & 3 : 2 * (g >> 2));
    f32x4 acc[2][2][4][2];
    bf16x8 Ar[2][2][4], Br[2][2][2];
    const __amdgpu_buffer_rsrc_t rw = mk_rsrc(ws, WS_NEED);
    unsigned gv = tid * 16u;
    unsigned la = (unsigned)(uintptr_t)lds + wr * 4096u + lane * 16u, lb = (unsigned)(uintptr_t)lds + 65536u + wc * 2048u + lane * 16u;
    asm volatile("" : "+v"(gv), "+v"(la), "+v"(lb));
    unsigned cA = OFF_WB + (unsigned)j * PANEL_BYTES, cB = OFF_XA + (unsigned)mb * PANEL_BYTES;
#define SSTR 65536u
#define GL(b, i, sa, sb) do { \
        if ((i) & 1) __builtin_amdgcn_raw_ptr_buffer_load_lds(rw, (LAS void*)(lds + 65536 + (b) * 16384 + ((i) >> 1) * 8192 + ldsw), 16, gv, (sb) + (b) * SSTR + ((i) >> 1) * 8192u, 0, GX_AUX); \
        else __builtin_amdgcn_raw_ptr_buffer_load_lds(rw, (LAS void*)(lds + (b) * 16384 + ((i) >> 1) * 8192 + ldsw), 16, gv, (sa) + (b) * SSTR + ((i) >> 1) * 8192u, 0, GW_AUX); } while (0)
#define RDA(r, b, h, m) Ar[r][h][m] = *(const LAS bf16x8*)(uintptr_t)(la + (b) * 16384 + (h) * 8192 + (m) * 1024)
#define RDB(r, b, h, n) Br[r][h][n] = *(const LAS bf16x8*)(uintptr_t)(lb + (b) * 16384 + (h) * 8192 + (n) * 1024)
#define RD(r, b, i) do { if ((i) < 4) RDB(r, b, (i) >> 1, (i) & 1); else RDA(r, b, ((i) - 4) >> 2, ((i) - 4) & 3); } while (0)
#define ACC2(i) acc[(i) >> 4][((i) >> 1) & 1][((i) >> 2) & 3][(i) & 1]
#define MF2(z, r, i) do { \
        if (z) asm volatile("v_mfma_f32_16x16x32_bf16 %0, %1, %2, 0" : "=&v"(ACC2(i)) : "v"(Ar[r][(i) >> 4][((i) >> 2) & 3]), "v"(Br[r][((i) >> 1) & 1][(i) & 1])); \
        else asm volatile("v_mfma_f32_16x16x32_bf16 %0, %1, %2, %0" : "+v"(ACC2(i)) : "v"(Ar[r][(i) >> 4][((i) >> 2) & 3]), "v"(Br[r][((i) >> 1) & 1][(i) & 1])); } while (0)
#define MIDX(i) (((i) & 4) ? (i) ^ 3 : (i))
#define GPOS 1
#define GSTR 8
#define RPOS 0
#define RSTR 2
#define XBAR BAR
#define XGL 4
#define XRD 12
#define STEP(b, z, vw, sa, sb) do { \
        SCHED; wait_v<vw>(); __builtin_amdgcn_s_waitcnt(0xC07F); XBAR; SCHED; \
        _Pragma("unroll") for (int i = 0; i < 32; ++i) { \
            MF2(z, (b) & 1, MIDX(i)); SCHED; \
            if (i >= GPOS && (i - GPOS) % GSTR == 0 && (i - GPOS) / GSTR < XGL) { GL(b, (i - GPOS) / GSTR, sa, sb); SCHED; } \
            if (i >= RPOS && (i - RPOS) % RSTR == 0 && (i - RPOS) / RSTR < XRD) { RD(((b) + 1) & 1, ((b) + 1) & 3, (i - RPOS) / RSTR); SCHED; } \
        } } while (0)
    ready(0, 0);
#pragma unroll
    for (int b = 0; b < 4; ++b)
#pragma unroll
        for (int i = 0; i < 4; ++i) GL(b, i, cA, cB);
    SCHED; WAIT_V(0); BAR; SCHED;
#pragma unroll
    for (int i = 0; i < 12; ++i) RD(0, 0, i);
    const __amdgpu_buffer_rsrc_t ro = mk_rsrc(P.out, OUT_BYTES);
#pragma unroll 1
    for (int ui = 0; ui < ntile; ++ui) {
        const bool has_next = ui + 1 < ntile;
        const int nj = !has_next ? j : lw < 28 ? j + RC : j + (ui & 1), nmb = lw < 28 ? mb : mb ^ 1;
        const unsigned nA = OFF_WB + (unsigned)nj * PANEL_BYTES, nB = OFF_XA + (unsigned)nmb * PANEL_BYTES;
        STEP(0, true, 24, cA + 4 * SSTR, cB + 4 * SSTR); STEP(1, false, 24, cA + 4 * SSTR, cB + 4 * SSTR); STEP(2, false, 24, cA + 4 * SSTR, cB + 4 * SSTR);
        const int jr = j % RC, ja = (jr & 1) * 7 + (jr >> 1);
        const unsigned ia = j < RC ? (j & 1 ? 16 + ja : 10 - ja) : j >= RC * NROUND ? N_R1 + jr * 4 : N_R0 + (j / RC - 1) * (4 * RC) + (j >= 2 * RC ? 8 : 0) + ja;
        const unsigned ka = (j < RC ? 30u : j >= RC * NROUND ? 1u : (unsigned)RC) * 65536u;
        const unsigned mA = OFF_MINE - 3 * SSTR + ia * 65536u, mB = OFF_MINE - 3 * SSTR + (unsigned)(mb < 4 ? mb : mb + 7) * 65536u;
        STEP(3, false, 8, mA, mB);
        const unsigned lim = lw < 28 ? unit_need(ui, 15, g) : 0;
#pragma unroll 1
        for (int t = 2; t <= 32; ++t) {
            const bool last = t == 32;
            const unsigned so = (unsigned)(t >> 3) * 524288u + (unsigned)((t >> 1) & 3) * 16384u + (unsigned)(t & 1) * 262144u;
            const unsigned sa = last ? nA : cA + so, sb = last ? nB : cB + so;
            if (!(t & 1) && ((unsigned)known < lim || last)) {
                if (!last) ready(ui, t >> 1);
                else if (has_next) ready(ui + 1, 0);
            }
            const unsigned qo = (unsigned)((t >> 1) & 3) * 16384u;
            const unsigned ta = t & 1 ? mA + (unsigned)(t >> 3) * ka + qo : sa, tb = t & 1 ? mB + (unsigned)(t >> 3) * (30u * 65536u) + qo : sb;
            STEP(0, false, 8, sa, sb); STEP(1, false, 8, sa, sb); STEP(2, false, 8, sa, sb); STEP(3, false, 8, ta, tb);
        }
        asm volatile("s_nop 15\n\ts_nop 15" ::: "memory");
        SCHED;
        {
            const int wi = ct_wi(j);
            const bool own = wi < NOWN;
            const unsigned row = (unsigned)(mb * 256 + wc * 32 + fr), cb = (unsigned)(wi * 256 + wr * 128 + fq * 8) * 2u;
            const unsigned obase = row * 65536u + cb + (own ? (unsigned)rank * OWNB : (SH0 - NOWN) * 512u);
            const unsigned pbase = OFF_OS + row * OWNB + cb;
            const float one = known < 0 ? __builtin_nanf("") : 1.0f;
            {
#pragma unroll
                for (int bj = 0; bj < 2; ++bj)
#pragma unroll
                    for (int n = 0; n < 2; ++n) {
                        f32x4 sv[8];
#pragma unroll
                        for (int m = 0; m < 8; ++m) sv[m] = silu4(acc[m >> 2][bj][m & 3][n], one);
#pragma unroll
                        for (int r = 0; r < 4; ++r) {
                            u32x4 o;
#pragma unroll
                            for (int e = 0; e < 4; ++e) o[e] = cvt_pk_bf16(sv[2 * e][r], sv[2 * e + 1][r]);
                            __builtin_amdgcn_raw_buffer_store_b128(o, ro, obase + r * 64u + (bj * 128 + n * 16) * 65536u, 0, EPI_AUX);
                            if (own) __builtin_amdgcn_raw_buffer_store_b128(o, rw, pbase + r * 64u + (bj * 128 + n * 16) * (unsigned)OWNB, 0, OS_AUX);
                        }
                        if (ui == 0 && bj * 2 + n < 3) { SCHED; __builtin_amdgcn_s_sleep(120); SCHED; }
                        if (ui == 1 && bj * 2 + n < 3) { SCHED; __builtin_amdgcn_s_sleep(60); SCHED; }
                    }
            }
            if (own) {
                WAIT_V(0);
                unsigned old = 0;
                if (lane == 0) old = __hip_atomic_fetch_add((unsigned*)(ws + OFF_TD + mb * 128), 1u, __ATOMIC_RELAXED, __HIP_MEMORY_SCOPE_AGENT);
                old = (unsigned)rfl((int)old) + 1u;
                if (old % (8u * NOWN) == 0u && lane < NR)
                    __hip_atomic_store((GAS unsigned*)(wsl + OFF_FL + (N_XW + mb) * 32 + rank * 4), call, __ATOMIC_RELAXED, __HIP_MEMORY_SCOPE_SYSTEM);
            }
        }
        SCHED;
        cA = nA; cB = nB; j = nj; mb = nmb;
    }
    WAIT_V(0);
    BAR;
    if (wid == 0) {
        unsigned old = 0;
        if (lane == 0) old = __hip_atomic_fetch_add((unsigned*)(ws + OFF_GP + g * 128 + 64), 1u, __ATOMIC_RELAXED, __HIP_MEMORY_SCOPE_AGENT);
        if (((unsigned)rfl((int)old) + 1u) % WG_PER_GRP == 0u) __builtin_amdgcn_fence(__ATOMIC_RELEASE, "agent");
    }
}

__global__ __launch_bounds__(512, 2) void fused_kernel(Params P) {
    __shared__ __attribute__((aligned(16))) unsigned char smem[LDS_BYTES];
    LAS unsigned char* lds = (LAS unsigned char*)smem;
    const int b = blockIdx.x;
    unsigned call0 = 0;
    if (__builtin_expect((unsigned)(b - NCOMM) < (unsigned)NPW, 1)) call0 = push_first(P, (unsigned)(uintptr_t)(lds + STAGE_BYTES), b - NCOMM);
    if (b < NCOMM) {
        if (threadIdx.x == 0) *(LAS unsigned*)(lds + 1024) = take_ticket(P, 0);
        __syncthreads();
        const unsigned call = (unsigned)rfl((int)*(LAS unsigned*)(lds + 1024)) / (GRID / 16) + 1u;
        comm_role(P, lds, call, b);
    } else {
        gemm_role(P, lds, b & 7, (b - NCOMM) >> 3, call0);
    }
}

}

extern "C" void kernel_launch(int rank, int nranks, ncclComm_t comm, hipStream_t stream,
                              void* const* d_in, const int* in_sizes, int n_in,
                              void* d_out, int out_size,
                              const int* in_dtypes, int out_dtype,
                              void* const* d_out_all,
                              void* const* d_ws_all, size_t ws_bytes) {
    (void)comm; (void)in_sizes; (void)n_in; (void)d_out_all; (void)out_size; (void)in_dtypes; (void)out_dtype;
    if (nranks != NR || (unsigned)rank >= (unsigned)NR || ws_bytes < WS_NEED) {
        fprintf(stderr, "unsupported configuration\n");
        return;
    }
    Params P;
    P.x = (const u16*)d_in[0]; P.w = (const u16*)d_in[1];
    for (int q = 0; q < NR; ++q) P.ws[q] = (char*)d_ws_all[(rank + 1 + q) % NR];
    P.out = (char*)d_out;
    P.rank = rank;
    hipLaunchKernelGGL(fused_kernel, dim3(GRID), dim3(512), 0, stream, P);
}
```

```cpp
#include <hip/hip_runtime.h>
#include <rccl/rccl.h>
#include <cstdio>

namespace {

typedef unsigned short u16;
typedef short bf16x8 __attribute__((ext_vector_type(8)));
typedef float f32x4 __attribute__((ext_vector_type(4)));
typedef unsigned u32x4 __attribute__((ext_vector_type(4)));
#define LAS __attribute__((address_space(3)))
#define GAS __attribute__((address_space(1)))

constexpr int NR = 8;
constexpr int NN = 32768, KSH = 512;
constexpr int GRID = 256;
constexpr int NCOMM = 32;
constexpr int NGEMM = GRID - NCOMM;
constexpr int WG_PER_GRP = NGEMM / 8;
constexpr int RC = 14, NROUND = 6;
constexpr int NCT = RC * NROUND;
constexpr int NOWN = 6;
#define FR 4
#define CB 56
constexpr int OWNB = NOWN * 512;
constexpr int SH0 = 8 * NOWN;
static_assert(WG_PER_GRP == 28 && NCT + 2 == 128 - 7 * NOWN && FR >= 2 && FR < NROUND, "tile map");
#define PD 14
#define NS 5
#define NSLOT 18
#define AHEAD 6
#define AHT AHEAD
#define AHN 300
#define FX_AUX 17
#define EPI_S0 120
#define EPI_S1 60
#define NHOLE 6
#define PACE 1
#define QS 4
#define SL0 2
#define NSL1 2
#define DLY_L 300u
#define NEARLY 72
#define ST_AUX_OUT 18
#define ST_AUX 16
#define RUN1 8
#define RUNT 2
#define RUNF 8
#define RUN_DIV 64
#define GX_AUX 0
#define GW_AUX 0
#define LD_AUX 17
#define OS_AUX 17
#define EPI_AUX 18
static_assert(NS >= 1 && NS <= 5 && PD >= 7 && PD <= 63 && RUN1 <= 8 && RUNF <= 8, "waves of a communication workgroup");
static_assert(NHOLE >= 0 && NHOLE <= NOWN && (QS == 1 || QS == 2 || QS == 4) && NSL1 >= 0 && NSL1 <= 4, "pushed items");

constexpr int NPW = 56;

constexpr unsigned OFF_TMO = 128;
constexpr unsigned OFF_PROG = 256;
constexpr unsigned OFF_GP = 1024;
constexpr unsigned OFF_TICKET = 2048;
constexpr unsigned OFF_TD = 4096;
constexpr unsigned OFF_FO = 8192;
constexpr unsigned OFF_WQ = 12288;
constexpr unsigned OFF_FL = 16384;
constexpr unsigned OFF_XA = 1u << 20;
constexpr unsigned SEND_BYTES = 262144u;
constexpr unsigned PANEL_BYTES = 2u << 20;
constexpr unsigned OFF_WB = OFF_XA + 16u * PANEL_BYTES;
constexpr unsigned OFF_SEND = OFF_WB + (unsigned)NCT * PANEL_BYTES;
constexpr unsigned OFF_OS = OFF_SEND + 8u * NOWN * SEND_BYTES;
constexpr unsigned OFF_FX = OFF_OS + 4096u * OWNB;
constexpr unsigned OFF_MINE = OFF_FX + 8u * 4096u * 1024u;
constexpr unsigned WS_NEED = OFF_MINE + (64u + (NCT + 2u) * 4u) * 65536u;
constexpr unsigned OUT_BYTES = 256u << 20;

constexpr int N_R0 = 120, N_FR = N_R0 + 4 * RC * (FR - 1);
constexpr int N_XW = 64 + (NCT + 2) * 4;
constexpr int HOLE = 101 - NHOLE;
constexpr int K3N = 30 - NHOLE;
constexpr int V_R0 = N_R0 - NHOLE;
constexpr int V_XW = N_XW - 8 - NHOLE;
constexpr int V_ALL = V_XW + OWNB / 16;
constexpr int NUNIT = V_ALL * 64;
constexpr int NPOOL = 4096;
constexpr int N_GRP = N_XW + 32;
static_assert(NUNIT + NPOOL + 256 < 65535, "progress fits 16 bits");
static_assert(OFF_FL + (N_GRP + 8) * 32 <= 65536, "flag table");
__device__ __forceinline__ int ct_wi(int j, int rank) {
    const int r = j - RC * FR;
    return j < RC ? (j & 1) * 7 + (j >> 1) : r < 0 ? j : r >= RC ? j + 2 : RC * FR + ((2 * (rank + 1) + r) & 15);
}
__device__ __forceinline__ void item_decode(int n, bool& isx, int& idx, int& kb) {
    if (n < N_R0) {
        kb = n / 30; const int r = n - kb * 30;
        isx = r < 4 || (r >= 11 && r < 23);
        idx = r < 4 ? r : r < 11 ? 20 - 2 * r : r < 23 ? r - 7 : 2 * (r - 23) + 1;
    } else {
        const int m = n - N_R0, i = m / (4 * RC), rem = m - i * (4 * RC);
        kb = rem / RC; const int r = rem - kb * RC;
        isx = false;
        idx = RC * (i + 1) + (r < 7 ? 2 * r : 2 * (r - 7) + 1);
    }
}
__device__ __forceinline__ int unit_need(int i, int ks, int g) {
    const int kb = ks >> 2, sl = (ks & 3) * QS >> 2;
    if (i == 0) {
        if (kb < SL0) return (30 * kb + (g < 4 ? 11 + 4 * g : 30)) * 64;
        if (kb < 3) return 30 * kb * 64 + (sl * 30 + (g < 4 ? 11 + 4 * g : 30)) * (64 / QS);
        return 90 * 64 + (sl * K3N + (g < 4 ? 11 - NHOLE + 4 * g : K3N)) * (64 / QS);
    }
    if (i == 1 && kb < NSL1) return (V_R0 + RC * kb) * 64 + (sl * RC + (g < 4 ? 7 : RC)) * (64 / QS);
    return (V_R0 + 4 * RC * (i - 1) + RC * kb + (g < 4 ? 7 : RC)) * 64;
}

constexpr unsigned long long DEADLINE_TICKS = 100000000ull;
constexpr int STAGE_BYTES = NSLOT * 7168 > 131072 ? NSLOT * 7168 : 131072;
constexpr int LDS_BYTES = STAGE_BYTES + 2048 + 8192;

struct Params {
    const u16* x; const u16* w;
    char* ws[NR];
    char* out;
    int rank;
};

__device__ __forceinline__ int rfl(int v) { return __builtin_amdgcn_readfirstlane(v); }

__device__ __forceinline__ __amdgpu_buffer_rsrc_t mk_rsrc(const void* p, unsigned bytes) {
    return __builtin_amdgcn_make_buffer_rsrc(const_cast<void*>(p), 0, (int)bytes, 0x00020000);
}
__device__ __forceinline__ bool timed_out(unsigned long long t0, unsigned* tmo, unsigned call) {
    if (__builtin_amdgcn_s_memrealtime() - t0 > DEADLINE_TICKS) {
        __hip_atomic_store(tmo, call, __ATOMIC_RELAXED, __HIP_MEMORY_SCOPE_AGENT);
        return true;
    }
    return false;
}
__device__ __forceinline__ unsigned lds_rd(unsigned addr) {
    unsigned v; asm volatile("ds_read_b32 %0, %1\n\ts_waitcnt lgkmcnt(0)" : "=v"(v) : "v"(addr) : "memory"); return v;
}
__device__ __forceinline__ u32x4 lds_rd4(unsigned addr) {
    u32x4 v; asm volatile("ds_read_b128 %0, %1\n\ts_waitcnt lgkmcnt(0)" : "=v"(v) : "v"(addr) : "memory"); return v;
}
__device__ __forceinline__ void lds_wr4(unsigned addr, u32x4 v) { asm volatile("ds_write_b128 %0, %1" :: "v"(addr), "v"(v) : "memory"); }
__device__ __forceinline__ void lds_wr(unsigned addr, unsigned v) { asm volatile("ds_write_b32 %0, %1" :: "v"(addr), "v"(v) : "memory"); }

#define WAIT_V(n) asm volatile("s_waitcnt vmcnt(" #n ")" ::: "memory")
#define WAIT_L(n) asm volatile("s_waitcnt lgkmcnt(" #n ")" ::: "memory")
#define SCHED __builtin_amdgcn_sched_barrier(0)
#define BAR __builtin_amdgcn_s_barrier()
#define DRAIN_V() do { SCHED; __builtin_amdgcn_s_waitcnt(0x0F70); WAIT_V(0); SCHED; } while (0)
template <int N> __device__ __forceinline__ void wait_v() { asm volatile("s_waitcnt vmcnt(%0)" :: "n"(N) : "memory"); }

constexpr int AUX_SYS = 17;

__device__ __forceinline__ GAS char* lane_ws(const Params& P, int lane) {
    char* wsl = P.ws[0];
#pragma unroll
    for (int p = 1; p < NR; ++p) {
        char* t = P.ws[p];
        asm volatile("" : "+s"(t));
        wsl = (lane == p) ? t : wsl;
    }
    return (GAS char*)wsl;
}

__device__ __forceinline__ int link_of(int x, int lp) { return (x * 3 + (x % 3 + 1) * lp) % 7; }
constexpr unsigned M_OUT = 2;
__device__ __forceinline__ u32x4 unit_entry(int v, int pv, int l, int rank) {
    if (v >= NUNIT) {
        if (pv >= NPOOL) return (u32x4){0xffffu, 0x80000000u, 0x80000000u + M_OUT, 0u};
        return (u32x4){(unsigned)NUNIT, OFF_FX + (unsigned)(rank * NPOOL + pv) * 1024u, (unsigned)pv * 65536u + (unsigned)(SH0 - NOWN + RC * FR + 2 * rank) * 512u + M_OUT, N_XW + 1u};
    }
    int n = v >> 6, u = v & 63;
    if (n >= V_XW) {
        unsigned o = (unsigned)(v - V_XW * 64) + (unsigned)rank * (OWNB / 2u);
        if (o >= OWNB * 4u) o -= OWNB * 4u;
        const unsigned row = o / (OWNB / 1024u);
        return (u32x4){(unsigned)v, OFF_OS + o * 1024u, row * 65536u + (o - row * (OWNB / 1024u)) * 1024u + (unsigned)((rank + 1 + l) & 7) * OWNB + M_OUT, (unsigned)N_XW};
    }
    if (n >= V_R0 + NSL1 * RC) n += NHOLE;
    else if (n >= V_R0) {
        const int v1 = v - V_R0 * 64, sl = v1 / (RC * 64 / QS), rem = v1 - sl * (RC * 64 / QS);
        n = N_R0 + sl / QS * RC + rem / (64 / QS); u = sl % QS * (64 / QS) + rem % (64 / QS);
    } else if (n >= 90) {
        const int v1 = v - 90 * 64, sl = v1 / (K3N * 64 / QS), rem = v1 - sl * (K3N * 64 / QS);
        n = 90 + rem / (64 / QS); u = sl * (64 / QS) + rem % (64 / QS);
        if (n >= HOLE) n += NHOLE;
    } else if (n >= SL0 * 30) {
        const int sl = v / (30 * 64 / QS), rem = v - sl * (30 * 64 / QS);
        n = sl / QS * 30 + rem / (64 / QS); u = sl % QS * (64 / QS) + rem % (64 / QS);
    }
    bool isx; int idx, kb;
    item_decode(n, isx, idx, kb);
    const int wi = ct_wi(idx, rank);
    const bool own = !isx && wi < NOWN;
    int need = n;
    if (n >= N_FR + 4 * RC) need = n += 8;
    else if (n >= N_FR) { n = N_FR + kb * 16 + wi - RC * FR; need = N_FR + 63; }
    const unsigned lb = (isx ? OFF_XA : OFF_WB) + (unsigned)idx * PANEL_BYTES + (unsigned)(kb * 8 + l) * 65536u + u * 1024u;
    const unsigned rb = own ? OFF_SEND + (unsigned)(rank * NOWN + wi) * SEND_BYTES + (unsigned)kb * 65536u + u * 1024u : OFF_MINE + (unsigned)n * 65536u + u * 1024u;
    return (u32x4){(unsigned)v, rb, lb, (unsigned)need};
}
__device__ __forceinline__ int lead_batches(unsigned cnt) {
    int r = __builtin_amdgcn_readlane((int)cnt, 0) * NS;
#pragma unroll
    for (int s = 1; s < NS; ++s) r = min(r, __builtin_amdgcn_readlane((int)cnt, s) * NS + s);
    return r;
}

__device__ __forceinline__ void comm_role(const Params& P, LAS unsigned char* lds, unsigned call, int c) {
    const int tid = threadIdx.x, wid = rfl(tid >> 6), lane = tid & 63;
    const int rank = P.rank;
    unsigned* tmo = (unsigned*)(P.ws[7] + OFF_TMO);
    const unsigned ctl = (unsigned)(uintptr_t)(lds + STAGE_BYTES);
    constexpr unsigned L_BAD = 0, L_FULL = 4, L_USED = 12, L_FREED = 32, L_DONE = 64, L_AVAIL = 96, L_DQ = 2048;
    constexpr int NDQ = 64;
    static_assert(2 * NS + NSLOT + AHEAD + 16 <= NDQ, "ring of claimed batches");
    const unsigned pbase = call << 16;
    const int s7 = 7 * (c >> 3), q = lane & 7, lb8 = lane >> 3;
    const int lp = (s7 + min(q, 6)) >> 2, lnk = link_of(c & 7, lp);
    const int src = q < 7 ? (rank + 1 + lnk) & 7 : rank;
    if (tid < 128) lds_wr4(ctl + tid * 16, (u32x4){0u, 0u, 0u, 0u});
    lds_wr4(ctl + L_DQ + tid * 16, (u32x4){0u, 0u, 0u, 0u});
    WAIT_L(0);
    __syncthreads();
    if (wid == 0) {
        __builtin_amdgcn_s_setprio(3);
        int freed = 0, i = 0;
        __amdgpu_buffer_rsrc_t r[7];
#pragma unroll
        for (int t = 0; t < 7; ++t) r[t] = mk_rsrc(P.ws[(unsigned)__builtin_amdgcn_readlane(lnk, t) % 7u], WS_NEED);
        bool more = true;
#pragma unroll 1
        while (more) {
            const u32x4 e = lds_rd4(ctl + L_DQ + ((i + lb8) & (NDQ - 1)) * 128 + q * 16);
            const unsigned av = lds_rd(ctl + L_AVAIL + src * 4);
            const bool valid = (e.x >> 16) == ((unsigned)(i + lb8 + 1) & 0xffffu);
            const unsigned long long no = ~__ballot(valid && e.w < av);
            const int nv = no ? (int)__builtin_ctzll(no) >> 3 : 8;
            if (!nv) {
                if ((__ballot(valid) & 255u) == 255u) { WAIT_V(0); lds_wr(ctl + L_FULL, (unsigned)i); }
                __builtin_amdgcn_s_sleep(1);
            }
#pragma unroll 1
            for (int u = 0; u < nv; ++u, ++i) {
                if ((__builtin_amdgcn_readlane((int)e.x, u * 8 + 7) & 0xffff) == 0xffff) { more = false; break; }
                lds_wr(ctl + L_USED, (unsigned)(i + 1));
                unsigned off[7];
#pragma unroll
                for (int t = 0; t < 7; ++t) off[t] = (unsigned)__builtin_amdgcn_readlane((int)e.y, u * 8 + t) + lane * 16u;
                while (i - NSLOT >= freed) { freed = lead_batches(lds_rd(ctl + L_FREED + q * 4)); if (i - NSLOT >= freed) __builtin_amdgcn_s_sleep(1); }
                LAS unsigned char* slot = lds + (unsigned)i % NSLOT * 7168u;
                SCHED;
#pragma unroll
                for (int t = 0; t < 7; ++t) {
                    wait_v<PD - 1>();
                    __builtin_amdgcn_raw_ptr_buffer_load_lds(r[t], (LAS void*)(slot + t * 1024), 16, off[t], 0, 0, LD_AUX);
                    SCHED;
                    if (t == (PD - 1) % 7 && i >= (PD - 1) / 7) lds_wr(ctl + L_FULL, (unsigned)(i - (PD - 1) / 7));
                }
            }
        }
#pragma unroll
        for (int k = (PD - 1) / 7; k >= 0; --k) {
            if (k == 8) wait_v<56>(); else if (k == 7) wait_v<49>(); else if (k == 6) wait_v<42>(); else if (k == 5) wait_v<35>(); else if (k == 4) wait_v<28>();
            else if (k == 3) wait_v<21>(); else if (k == 2) wait_v<14>(); else if (k == 1) wait_v<7>(); else wait_v<0>();
            if (i >= k) lds_wr(ctl + L_FULL, (unsigned)(i - k));
        }
    } else if (wid == 1) {
        const unsigned* fl = (const unsigned*)(P.ws[7] + OFF_FL) + lane;
        unsigned* prog = (unsigned*)(P.ws[7] + OFF_PROG);
        const unsigned long long t0 = __builtin_amdgcn_s_memrealtime();
        unsigned smask = 0;
#pragma unroll
        for (int t = 0; t < 8; ++t) smask |= 1u << __builtin_amdgcn_readlane(src, t);
        int pos = smask >> q & 1u ? 0 : N_GRP;
        int mine = 0, all = c < 8 ? 0 : V_XW * 64;
        bool open = true;
        bool pushed = c >= 8;
        if (c < 8) { __builtin_amdgcn_fence(__ATOMIC_ACQUIRE, ""); WAIT_V(0); }
        while (open || mine < V_XW * 64 || all < V_XW * 64) {
            unsigned v = call, pv = 0, hv = call;
            if (pos < N_GRP) v = __hip_atomic_load(fl + pos * 8, __ATOMIC_RELAXED, __HIP_MEMORY_SCOPE_SYSTEM);
            if (!pushed) hv = __hip_atomic_load(fl + ((lane & 7) == rank ? HOLE : N_GRP) * 8, __ATOMIC_RELAXED, __HIP_MEMORY_SCOPE_SYSTEM);
            if (all < V_XW * 64) pv = __hip_atomic_load(prog + (lane & 31), __ATOMIC_RELAXED, __HIP_MEMORY_SCOPE_AGENT);
            if (mine < V_XW * 64) {
                const int k = lead_batches(lds_rd(ctl + L_DONE + q * 4));
                const unsigned e0 = (unsigned)rfl((int)lds_rd(ctl + L_DQ + (k & (NDQ - 1)) * 128 + 112));
                const int m = (e0 >> 16) == ((unsigned)(k + 1) & 0xffffu) ? (int)(e0 & 0xffffu) : -1;
                if (m > mine) {
                    if (lane == 0) __hip_atomic_store(prog + c, pbase + (unsigned)m, __ATOMIC_RELAXED, __HIP_MEMORY_SCOPE_AGENT);
                    mine = m;
                }
            }
            bool adv = false;
            if (open) {
                const int it = pos + lb8;
                const unsigned long long miss = __ballot((int)(v - call) < 0 && it < N_GRP && (it < HOLE || it >= HOLE + NHOLE)) >> q & 0x0101010101010101ull;
                const int a = pos >= N_GRP ? 0 : miss ? (int)(__builtin_ctzll(miss) >> 3) : 8;
                pos += a;
                adv = __any(a);
                if (a && lane < 8) lds_wr(ctl + L_AVAIL + q * 4, (unsigned)(pos <= N_XW ? pos : N_XW + ((pos - N_XW) >> 4)));
                open = __any(pos < N_GRP);
            }
            if (!pushed) pushed = __all((int)(hv - call) >= 0 || (lane >= NHOLE * 8 && (lane & 7) == rank));
            if (all < V_XW * 64) {
                int m = (int)(pv - pbase);
#pragma unroll
                for (int o = 16; o >= 1; o >>= 1) m = min(m, __shfl_xor(m, o));
                m = rfl(m);
                if (!pushed) m = min(m, 90 * 64);
                if (m > all) {
                    if (lane == 0) __hip_atomic_store((unsigned*)(P.ws[7] + OFF_GP + c * 128), pbase + (unsigned)m, __ATOMIC_RELAXED, __HIP_MEMORY_SCOPE_AGENT);
                    all = m;
                }
            }
            if (timed_out(t0, tmo, call)) { lds_wr(ctl + L_BAD, 1u); WAIT_L(0); lds_wr(ctl + L_AVAIL + q * 4, 0xffffu); break; }
            if (!adv) __builtin_amdgcn_s_sleep(4);
        }
        if (lane < NR - 1)
            __hip_atomic_store((GAS unsigned*)(lane_ws(P, lane) + OFF_FO + c * 128 + rank * 4), call, __ATOMIC_RELAXED, __HIP_MEMORY_SCOPE_SYSTEM);
        const unsigned* fo = (const unsigned*)(P.ws[7] + OFF_FO + c * 128) + (lane & 7);
        for (;;) {
            const unsigned v = __hip_atomic_load(fo, __ATOMIC_RELAXED, __HIP_MEMORY_SCOPE_SYSTEM);
            if (__all((int)(v - call) >= 0 || (lane & 7) == rank)) break;
            if (timed_out(t0, tmo, call)) break;
            __builtin_amdgcn_s_sleep(16);
        }
    } else if (wid == 2) {
        unsigned* wq = (unsigned*)(P.ws[7] + OFF_WQ + (call & 1u) * 1024u + (q < 7 ? lnk : 7) * 128);
        if (c == 0 && lane < 8)
            __hip_atomic_store((unsigned*)(P.ws[7] + OFF_WQ + ((call + 1u) & 1u) * 1024u + lane * 128), 0u, __ATOMIC_RELAXED, __HIP_MEMORY_SCOPE_AGENT);
        const int f0 = max(lp * 4 - s7, 0), mu = min(lp * 4 + 4 - s7, 7) - f0;
#ifndef NVO
#define uo ((q - f0) * m + lb8)
#else
        const int uo = lb8 * mu + q - f0;
#endif
        int m = RUNF, base = ((c & 7) * 4 + ((s7 + f0) & 3)) * RUNF, pv = 0, hp = 0;
        bool ex = false;
        bool shut = false;
#pragma unroll 1
        for (int k = 0;;) {
            u32x4 e = unit_entry(ex || shut ? NUNIT : base + uo, shut ? NPOOL : pv, lnk, rank);
            if (shut) e.x = (unsigned)base;
            unsigned lo = q < 7 ? e.x : 0xffffu, hi = q < 7 ? min(e.w, N_XW - 1u) : 0u;
#pragma unroll
            for (int o = 1; o < 8; o <<= 1) { lo = min(lo, (unsigned)__shfl_xor((int)lo, o)); hi = max(hi, (unsigned)__shfl_xor((int)hi, o)); }
            if (q == 7) { e.x = lo; e.w = hi; }
            e.x |= (unsigned)(k + lb8 + 1) << 16;
            if (lb8 < m) lds_wr4(ctl + L_DQ + ((k + lb8) & (NDQ - 1)) * 128 + q * 16, e);
            k += m;
            if (__any(lb8 < m && lo == 0xffffu)) {
                lds_wr4(ctl + L_DQ + ((k + lb8) & (NDQ - 1)) * 128 + q * 16, (u32x4){(unsigned)(k + lb8 + 1) << 16 | 0xffffu, 0u, 0u, 0u});
                break;
            }
            if (!shut) base = min(base + mu * m, NUNIT);
            ex = base >= NUNIT;
            int left = NUNIT - base;
#pragma unroll
            for (int o = 1; o < 8; o <<= 1) left = min(left, __shfl_xor(left, o));
            left = rfl(left);
            const int rest = left + max(NPOOL - hp, 0) / 7;
            m = max(RUNT, min(RUN1, rest / RUN_DIV));
            while (rfl((int)lds_rd(ctl + L_USED)) + (rest < AHN ? AHT : AHEAD) < k) __builtin_amdgcn_s_sleep(1);
            constexpr int GUARD = 40 * RUN1;
            for (;;) {
                const int av = (int)lds_rd(ctl + L_AVAIL + ((rank + 1 + lnk) & 7) * 4);
                shut = (base >= V_XW * 64 - GUARD && av <= N_XW) || (base >= NUNIT - GUARD && av <= N_XW + 1 && hp < NPOOL);
                if (!__all(shut || (ex && hp >= NPOOL)) || !__any(shut)) break;
                __builtin_amdgcn_s_sleep(8);
            }
            const bool act = lb8 < m && q < 7 && !shut;
            const unsigned long long pm = __ballot(act && ex);
            int r = 0;
            if (lane < 8 && (q < 7 ? q == f0 && !ex && !shut : left < 1024))
                r = (int)__hip_atomic_fetch_add(wq, (unsigned)(q < 7 ? mu * m : __builtin_popcountll(pm)), __ATOMIC_RELAXED, __HIP_MEMORY_SCOPE_AGENT);
            const int got = __shfl(r, f0) + 32 * RUNF;
            if (!ex && !shut) base = min(got, NUNIT);
            hp = __builtin_amdgcn_readlane(r, 7);
            pv = hp + __builtin_popcount((unsigned)pm & ((1u << q) - 1u)) * m + lb8;
            const unsigned long long nm = __ballot(act && !ex && base + uo >= NUNIT);
            if (nm) {
                if (lane == 7) r = (int)__hip_atomic_fetch_add(wq, (unsigned)__builtin_popcountll(nm), __ATOMIC_RELAXED, __HIP_MEMORY_SCOPE_AGENT);
                hp = __builtin_amdgcn_readlane(r, 7);
                if (!ex) pv = hp + (int)__builtin_amdgcn_mbcnt_hi((unsigned)(nm >> 32), __builtin_amdgcn_mbcnt_lo((unsigned)nm, 0u));
            }
        }
    } else if (wid < 3 + NS) {
        const int s = wid - 3;
        int full = 0;
        unsigned cnt = 0, done = 0;
        bool bad = false;
        const __amdgpu_buffer_rsrc_t rw = mk_rsrc(P.ws[7], WS_NEED), ro = mk_rsrc(P.out, OUT_BYTES);
#pragma unroll 1
        for (int i = s;; i += NS) {
            u32x4 e;
            for (;;) {
                e = lds_rd4(ctl + L_DQ + (i & (NDQ - 1)) * 128 + q * 16);
                if (__all((e.x >> 16) == ((unsigned)(i + 1) & 0xffffu))) break;
                if (done != cnt) { DRAIN_V(); done = cnt; lds_wr(ctl + L_DONE + s * 4, done); }
                else __builtin_amdgcn_s_sleep(1);
            }
            const int unit0 = __builtin_amdgcn_readlane((int)e.x, 7) & 0xffff;
            if (unit0 == 0xffff) break;
            while (i >= full) {
                const unsigned cw = lds_rd(ctl + (lane & 1) * 4);
                full = __builtin_amdgcn_readlane((int)cw, 1);
                bad = __builtin_amdgcn_readlane((int)cw, 0) != 0;
                if (i >= full) {
                    if (done != cnt) { DRAIN_V(); done = cnt; lds_wr(ctl + L_DONE + s * 4, done); }
                    else __builtin_amdgcn_s_sleep(1);
                }
            }
            u32x4 v[7];
#pragma unroll
            for (int t = 0; t < 7; ++t) v[t] = *(const LAS u32x4*)(lds + (unsigned)i % NSLOT * 7168u + t * 1024 + lane * 16);
            SCHED; WAIT_L(0); SCHED;
            ++cnt;
            lds_wr(ctl + L_FREED + s * 4, cnt);
#pragma unroll
            for (int t = 0; t < 7; ++t) {
                if (bad) v[t] = (u32x4){0x7fc07fc0u, 0x7fc07fc0u, 0x7fc07fc0u, 0x7fc07fc0u};
                const unsigned lb = (unsigned)__builtin_amdgcn_readlane((int)e.z, t), off = (lb & ~3u) + lane * 16u;
                if (lb & M_OUT) __builtin_amdgcn_raw_buffer_store_b128(v[t], ro, off, 0, ST_AUX_OUT);
                else __builtin_amdgcn_raw_buffer_store_b128(v[t], rw, off, 0, ST_AUX);
            }
            SCHED;
            if (done + 1u != cnt) { wait_v<7>(); done = cnt - 1u; lds_wr(ctl + L_DONE + s * 4, done); }
            if (unit0 >= V_XW * 64) { done = cnt; lds_wr(ctl + L_DONE + s * 4, done); }
        }
        if (done != cnt) { DRAIN_V(); lds_wr(ctl + L_DONE + s * 4, cnt); }
    }
}

#define LDIN(p) __builtin_nontemporal_load(p)
__device__ __forceinline__ unsigned take_ticket(const Params& P, int zv) {
    return __hip_atomic_fetch_add((unsigned*)(P.ws[7] + OFF_TICKET + (blockIdx.x & 15) * 128) + zv, 1u, __ATOMIC_RELAXED, __HIP_MEMORY_SCOPE_AGENT);
}
__device__ __forceinline__ unsigned push_first(const Params& P, unsigned ctl, int s1) {
    const int tid = threadIdx.x, lane = tid & 63;
    const int q = s1 >> 3, wi = rfl(tid >> 6), fi = N_GRP + (s1 & 7);
    const unsigned sub = (unsigned)s1 & 7u;
    const u16* w = P.w; int rank = P.rank; char* wq = P.ws[q]; char* w7 = P.ws[7];
    const unsigned vo = ((sub >> 1) * 32u + ((unsigned)lane >> 4) * 8u) * (NN * 2u) + (sub & 1u) * 256u + (4u * (lane & 3) + ((lane & 15) >> 2)) * 16u;
    unsigned dst = OFF_WB + (unsigned)(2 * wi) * PANEL_BYTES + (unsigned)(30 - q) * 65536u + (sub >> 1) * 16384u + (sub & 1u) * 4096u + lane * 16u;
    int zv = 0;
    asm volatile("" : "+v"(dst), "+v"(zv));
    asm volatile("" : "+s"(w), "+s"(rank), "+s"(wq), "+s"(w7));
    const __amdgpu_buffer_rsrc_t rs = mk_rsrc(w, KSH * NN * 2u), rw = mk_rsrc(wq, WS_NEED);
    const unsigned so = 384u * (NN * 2u) + (unsigned)((rank + 1 + q) & 7) * OWNB + (unsigned)wi * 512u;
    if (wi < NHOLE) {
        u32x4 v[8];
#pragma unroll
        for (int t = 0; t < 8; ++t) v[t] = __builtin_amdgcn_raw_buffer_load_b128(rs, vo, so + t * (NN * 2u), 2);
        SCHED;
#pragma unroll
        for (int t = 0; t < 8; ++t) {
            u32x4 o;
#pragma unroll
            for (int e = 0; e < 4; ++e) o[e] = __builtin_amdgcn_perm(v[2 * e + 1][t >> 1], v[2 * e][t >> 1], (t & 1) ? 0x07060302u : 0x05040100u);
            if (t >= PACE) { SCHED; wait_v<PACE - 1>(); SCHED; }
            __builtin_amdgcn_raw_buffer_store_b128(o, rw, dst + (t >> 2) * 8192 + (t & 3) * 1024, 0, AUX_SYS);
        }
    }
    SCHED;
    unsigned tk = 0;
    if (tid == 448) tk = __hip_atomic_fetch_add((GAS unsigned*)(w7 + OFF_TICKET + (blockIdx.x & 15) * 128) + zv, 1u, __ATOMIC_RELAXED, __HIP_MEMORY_SCOPE_AGENT);
    DRAIN_V();
    if (tid == 448) { lds_wr(ctl + 4, tk); WAIT_L(0); }
    BAR;
    const unsigned call = (unsigned)rfl((int)lds_rd(ctl + 4)) / (GRID / 16) + 1u;
    if (tid == 0) __hip_atomic_store((GAS unsigned*)(wq + OFF_FL + fi * 32 + rank * 4), call, __ATOMIC_RELAXED, __HIP_MEMORY_SCOPE_SYSTEM);
    SCHED;
    return call;
}
__device__ __forceinline__ unsigned stage_role(const Params& P, GAS char* wsl, unsigned ctl, int s0, unsigned call) {
    const int tid = threadIdx.x, sub = rfl(tid >> 6), lane = tid & 63;
    const int rank = P.rank;
    const int qi = 4 * (lane & 3) + ((lane & 15) >> 2);
    constexpr int G = 30 + 7 * NOWN, S3 = 3 * G, S4 = S3 + 30, S5 = S4 + 7 * (NOWN - NHOLE), N_STAGE = NPW + S5 + N_XW - N_R0;
    unsigned tk = 0;
    int zv = 0;
    asm volatile("" : "+v"(zv));
    if (s0 >= NPW + NEARLY) {
        const unsigned long long t0 = __builtin_amdgcn_s_memrealtime();
        while (__builtin_amdgcn_s_memrealtime() - t0 < DLY_L) __builtin_amdgcn_s_sleep(2);
    }
#pragma unroll 1
    for (int s1 = s0 < NPW ? s0 + NGEMM : s0; s1 < N_STAGE; s1 += NGEMM) {
        const int s = s1 - NPW;
        int n, q = 7;
        if (s < S3) {
            const int kb = s / G, r = s - kb * G;
            if (r < 30) n = kb * 30 + r;
            else { const int jo = (r - 30) / 7; q = r - 30 - jo * 7; n = kb * 30 + 5 + jo; }
        } else if (s < S4) n = 90 + s - S3;
        else if (s < S5) { const int jo = (s - S4) / 7; q = s - S4 - jo * 7; n = 95 + jo; }
        else n = N_R0 + s - S5;
        const __amdgpu_buffer_rsrc_t rw = mk_rsrc(P.ws[7], WS_NEED);
        bool isx; int idx, kb;
        item_decode(n < N_FR + 64 ? n : n - 8, isx, idx, kb);
        int wi = ct_wi(idx, rank);
        if (n >= N_FR && n < N_FR + 64) { kb = (n - N_FR) >> 4; wi = RC * FR + ((n - N_FR) & 15); }
        const bool own = !isx && wi < NOWN;
        u32x4 v[8];
        if (isx) {
            const u16* src = P.x + (size_t)(idx * 256 + sub * 32 + (lane & 15)) * KSH + kb * 128 + (lane >> 4) * 8;
#pragma unroll
            for (int t = 0; t < 8; ++t) v[t] = LDIN((const u32x4*)(src + (t >> 2) * 16 * KSH + (t & 3) * 32));
            SCHED;
            if (s1 == s0 && tid == 0) tk = take_ticket(P, zv);
            SCHED;
            const unsigned dst = OFF_MINE + (unsigned)n * 65536u + sub * 2048u + lane * 16u;
#pragma unroll
            for (int t = 0; t < 8; ++t)
                __builtin_amdgcn_raw_buffer_store_b128(v[t], rw, dst + (t >> 2) * 1024 + (t & 3) * 16384, 0, AUX_SYS);
        } else {
            const int wr = sub & 1;
            const int p = (rank + 1 + q) & 7;
            const int col = own ? p * (OWNB / 2) + wi * 256 : (SH0 - NOWN + wi) * 256;
            const u16* src = P.w + (size_t)(kb * 128 + (sub >> 1) * 32 + (lane >> 4) * 8) * NN + col + wr * 128 + qi * 8;
#pragma unroll
            for (int t = 0; t < 8; ++t) v[t] = LDIN((const u32x4*)(src + (size_t)t * NN));
            SCHED;
            if (s1 == s0 && tid == 0) tk = take_ticket(P, zv);
            SCHED;
            const unsigned base = q < 7 ? OFF_SEND + (unsigned)(p * NOWN + wi) * SEND_BYTES + (unsigned)kb * 65536u : OFF_MINE + (unsigned)n * 65536u;
            const unsigned dst = base + (sub >> 1) * 16384u + wr * 4096u + lane * 16u;
#pragma unroll
            for (int t = 0; t < 8; ++t) {
                u32x4 o;
#pragma unroll
                for (int e = 0; e < 4; ++e) o[e] = __builtin_amdgcn_perm(v[2 * e + 1][t >> 1], v[2 * e][t >> 1], (t & 1) ? 0x07060302u : 0x05040100u);
                __builtin_amdgcn_raw_buffer_store_b128(o, rw, dst + (t >> 2) * 8192 + (t & 3) * 1024, 0, AUX_SYS);
            }
        }
        DRAIN_V();
        if (s1 == s0 && tid == 0) { lds_wr(ctl + 4, tk); WAIT_L(0); }
        BAR;
        if (s1 == s0) call = (unsigned)rfl((int)lds_rd(ctl + 4)) / (GRID / 16) + 1u;
        if (sub == 0 && (own ? lane == q : lane < NR))
            __hip_atomic_store((GAS unsigned*)(wsl + OFF_FL + n * 32 + rank * 4), call, __ATOMIC_RELAXED, __HIP_MEMORY_SCOPE_SYSTEM);
        SCHED;
    }
    return call;
}

__device__ __forceinline__ unsigned cvt_pk_bf16(float lo, float hi) {
    unsigned r; asm volatile("v_cvt_pk_bf16_f32 %0, %1, %2" : "=v"(r) : "v"(lo), "v"(hi)); return r;
}
__device__ __forceinline__ f32x4 silu4(f32x4 y, float one) {
    f32x4 t = y * -1.44269504088896341f;
    asm("" : "+v"(t));
    f32x4 e;
#pragma unroll
    for (int i = 0; i < 4; ++i) e[i] = __builtin_amdgcn_exp2f(__builtin_amdgcn_fmed3f(t[i], -86.5617024533378f, 86.5617024533378f));
    e = e + one;
#pragma unroll
    for (int i = 0; i < 4; ++i) e[i] = __builtin_amdgcn_rcpf(e[i]);
    return e * y;
}

__device__ __forceinline__ void gemm_role(const Params& P, LAS unsigned char* lds, int g, int lw, unsigned call0) {
    const int tid = threadIdx.x, wid = rfl(tid >> 6), lane = tid & 63, wr = wid >> 2, wc = wid & 3, fr = lane & 15, fq = lane >> 4;
    const int rank = P.rank;
    char* const ws = P.ws[7];
    unsigned* tmo = (unsigned*)(ws + OFF_TMO);
    const unsigned ldsw = (unsigned)wid * 1024u;
    volatile LAS int* lknown = (volatile LAS int*)(lds + STAGE_BYTES);
    int known = 0;
    GAS char* wsl = lane_ws(P, lane);
    asm volatile("" : "+v"(wsl));
    const unsigned call = stage_role(P, wsl, (unsigned)(uintptr_t)(lds + STAGE_BYTES), lw * 8 + g, call0);

    auto ready = [&](int i, int ks) {
        const int need = unit_need(i, ks, g);
        if ((unsigned)known >= (unsigned)need) return;
        if (wid == 0) {
            const unsigned* f = (const unsigned*)(ws + OFF_GP + g * 128);
            const unsigned long long t0 = __builtin_amdgcn_s_memrealtime();
            int m;
            for (;;) {
                m = rfl((int)(__hip_atomic_load(f, __ATOMIC_RELAXED, __HIP_MEMORY_SCOPE_AGENT) - (call << 16)));
                if (m >= need) break;
                if (timed_out(t0, tmo, call)) { m = -1; break; }
                __builtin_amdgcn_s_sleep(8);
            }
            if (lane == 0) lknown[0] = m;
            WAIT_V(0); WAIT_L(0);
        }
        asm volatile("" ::: "memory"); BAR; asm volatile("" ::: "memory");
        known = rfl(lknown[0]);
    };

    int j = 2 * (lw >> 2) + (g >> 2);
    const int mb = 4 * (g & 3) + (lw & 3), ja = (g >> 2) * 7 + (lw >> 2);
    f32x4 acc[2][2][4][2];
    bf16x8 Ar[2][2][4], Br[2][2][2];
    const __amdgpu_buffer_rsrc_t rw = mk_rsrc(ws, WS_NEED);
    unsigned gv = tid * 16u;
    unsigned la = (unsigned)(uintptr_t)lds + wr * 4096u + lane * 16u, lb = (unsigned)(uintptr_t)lds + 65536u + wc * 2048u + lane * 16u;
    asm volatile("" : "+v"(gv), "+v"(la), "+v"(lb));
    unsigned cA = OFF_WB + (unsigned)j * PANEL_BYTES;
    const unsigned cB = OFF_XA + (unsigned)mb * PANEL_BYTES;
#define SSTR 65536u
#define GL(b, i, sa, sb) do { \
        if ((i) & 1) __builtin_amdgcn_raw_ptr_buffer_load_lds(rw, (LAS void*)(lds + 65536 + (b) * 16384 + ((i) >> 1) * 8192 + ldsw), 16, gv, (sb) + (b) * SSTR + ((i) >> 1) * 8192u, 0, GX_AUX); \
        else __builtin_amdgcn_raw_ptr_buffer_load_lds(rw, (LAS void*)(lds + (b) * 16384 + ((i) >> 1) * 8192 + ldsw), 16, gv, (sa) + (b) * SSTR + ((i) >> 1) * 8192u, 0, GW_AUX); } while (0)
#define RDA(r, b, h, m) Ar[r][h][m] = *(const LAS bf16x8*)(uintptr_t)(la + (b) * 16384 + (h) * 8192 + (m) * 1024)
#define RDB(r, b, h, n) Br[r][h][n] = *(const LAS bf16x8*)(uintptr_t)(lb + (b) * 16384 + (h) * 8192 + (n) * 1024)
#define RD(r, b, i) do { if ((i) < 4) RDB(r, b, (i) >> 1, (i) & 1); else RDA(r, b, ((i) - 4) >> 2, ((i) - 4) & 3); } while (0)
#define ACC2(i) acc[(i) >> 4][((i) >> 1) & 1][((i) >> 2) & 3][(i) & 1]
#define MF2(z, r, i) do { \
        if (z) asm volatile("v_mfma_f32_16x16x32_bf16 %0, %1, %2, 0" : "=&v"(ACC2(i)) : "v"(Ar[r][(i) >> 4][((i) >> 2) & 3]), "v"(Br[r][((i) >> 1) & 1][(i) & 1])); \
        else asm volatile("v_mfma_f32_16x16x32_bf16 %0, %1, %2, %0" : "+v"(ACC2(i)) : "v"(Ar[r][(i) >> 4][((i) >> 2) & 3]), "v"(Br[r][((i) >> 1) & 1][(i) & 1])); } while (0)
#define MIDX(i) (((i) & 4) ? (i) ^ 3 : (i))
#define GPOS 1
#define GSTR 8
#define RPOS 0
#define RSTR 2
#define XBAR BAR
#define XGL 4
#define XRD 12
#define STEP(b, z, vw, sa, sb) do { \
        SCHED; wait_v<vw>(); __builtin_amdgcn_s_waitcnt(0xC07F); XBAR; SCHED; \
        _Pragma("unroll") for (int i = 0; i < 32; ++i) { \
            MF2(z, (b) & 1, MIDX(i)); SCHED; \
            if (i >= GPOS && (i - GPOS) % GSTR == 0 && (i - GPOS) / GSTR < XGL) { GL(b, (i - GPOS) / GSTR, sa, sb); SCHED; } \
            if (i >= RPOS && (i - RPOS) % RSTR == 0 && (i - RPOS) / RSTR < XRD) { RD(((b) + 1) & 1, ((b) + 1) & 3, (i - RPOS) / RSTR); SCHED; } \
        } } while (0)
    ready(0, 0);
#pragma unroll
    for (int b = 0; b < 4; ++b)
#pragma unroll
        for (int i = 0; i < 4; ++i) GL(b, i, cA, cB);
    SCHED; WAIT_V(0); BAR; SCHED;
#pragma unroll
    for (int i = 0; i < 12; ++i) RD(0, 0, i);
    const __amdgpu_buffer_rsrc_t ro = mk_rsrc(P.out, OUT_BYTES);
#pragma unroll 1
    for (int ui = 0; ui < NROUND; ++ui) {
        const bool has_next = ui + 1 < NROUND;
        const int nj = has_next ? j + RC : j, wi = ct_wi(j, rank);
        const unsigned nA = OFF_WB + (unsigned)nj * PANEL_BYTES;
        STEP(0, true, 24, cA + 4 * SSTR, cB + 4 * SSTR); STEP(1, false, 24, cA + 4 * SSTR, cB + 4 * SSTR); STEP(2, false, 24, cA + 4 * SSTR, cB + 4 * SSTR);
        const unsigned ia = ui == 0 ? (j & 1 ? 16 + ja : 10 - ja) : ui == FR ? N_FR + wi - RC * FR : N_R0 + (ui - 1) * (4 * RC) + (ui > FR ? 8 : 0) + ja;
        const unsigned ka = (ui == 0 ? 30u : ui == FR ? 16u : (unsigned)RC) * 65536u;
        const unsigned mA = OFF_MINE - 3 * SSTR + ia * 65536u, mB = OFF_MINE - 3 * SSTR + (unsigned)(mb < 4 ? mb : mb + 7) * 65536u;
        STEP(3, false, 8, mA, mB);
        const unsigned lim = unit_need(ui, 15, g);
#pragma unroll 1
        for (int t = 2; t <= 32; ++t) {
            const bool last = t == 32;
            const unsigned so = (unsigned)(t >> 3) * 524288u + (unsigned)((t >> 1) & 3) * 16384u + (unsigned)(t & 1) * 262144u;
            const unsigned sa = last ? nA : cA + so, sb = last ? cB : cB + so;
            if (!(t & 1) && ((unsigned)known < lim || last)) {
                if (!last) ready(ui, t >> 1);
                else if (has_next) ready(ui + 1, 0);
            }
            const unsigned qo = (unsigned)((t >> 1) & 3) * 16384u;
            const unsigned ta = t & 1 ? mA + (unsigned)(t >> 3) * ka + qo : sa, tb = t & 1 ? mB + (unsigned)(t >> 3) * (30u * 65536u) + qo : sb;
            STEP(0, false, 8, sa, sb); STEP(1, false, 8, sa, sb); STEP(2, false, 8, sa, sb); STEP(3, false, 8, ta, tb);
        }
        asm volatile("s_nop 15\n\ts_nop 15" ::: "memory");
        SCHED;
        {
            const bool own = wi < NOWN, dup = own || ui == FR;
            const unsigned row = (unsigned)(mb * 256 + wc * 32 + fr), cb = (unsigned)(wi * 256 + wr * 128 + fq * 8) * 2u;
            const unsigned obase = row * 65536u + cb + (own ? (unsigned)rank * OWNB : (SH0 - NOWN) * 512u);
            const unsigned pitch = own ? OWNB : 1024u;
            const unsigned pbase = row * pitch + cb + (own ? OFF_OS : OFF_FX + (unsigned)((wi - RC * FR) >> 1) * (NPOOL * 1024u - 1024u) - RC * FR * 512u);
            const float one = known < 0 ? __builtin_nanf("") : 1.0f;
            {
#pragma unroll
                for (int bj = 0; bj < 2; ++bj)
#pragma unroll
                    for (int n = 0; n < 2; ++n) {
                        f32x4 sv[8];
#pragma unroll
                        for (int m = 0; m < 8; ++m) sv[m] = silu4(acc[m >> 2][bj][m & 3][n], one);
#pragma unroll
                        for (int r = 0; r < 4; ++r) {
                            u32x4 o;
#pragma unroll
                            for (int e = 0; e < 4; ++e) o[e] = cvt_pk_bf16(sv[2 * e][r], sv[2 * e + 1][r]);
                            __builtin_amdgcn_raw_buffer_store_b128(o, ro, obase + r * 64u + (bj * 128 + n * 16) * 65536u, 0, EPI_AUX);
                            if (FX_AUX == OS_AUX ? dup : own) __builtin_amdgcn_raw_buffer_store_b128(o, rw, pbase + r * 64u + (bj * 128 + n * 16) * pitch, 0, OS_AUX);
                            if (FX_AUX != OS_AUX && ui == FR) __builtin_amdgcn_raw_buffer_store_b128(o, rw, pbase + r * 64u + (bj * 128 + n * 16) * pitch, 0, FX_AUX);
                        }
                        if (ui == 0 && bj * 2 + n < 3) { SCHED; __builtin_amdgcn_s_sleep(EPI_S0); SCHED; }
                        if (ui == 1 && bj * 2 + n < 3) { SCHED; __builtin_amdgcn_s_sleep(EPI_S1); SCHED; }
                    }
            }
            if (dup) {
                WAIT_V(0);
                unsigned old = 0;
                const int tb = own ? mb : mb + 16;
                if (lane == 0) old = __hip_atomic_fetch_add((unsigned*)(ws + OFF_TD + tb * 128), 1u, __ATOMIC_RELAXED, __HIP_MEMORY_SCOPE_AGENT);
                old = (unsigned)rfl((int)old) + 1u;
                if ((own ? old % (8u * NOWN) : old % (8u * RC)) == 0u && lane < NR)
                    __hip_atomic_store((GAS unsigned*)(wsl + OFF_FL + (N_XW + tb) * 32 + rank * 4), call, __ATOMIC_RELAXED, __HIP_MEMORY_SCOPE_SYSTEM);
            }
        }
        SCHED;
        cA = nA; j = nj;
    }
    WAIT_V(0);
    BAR;
    if (wid == 0) {
        unsigned old = 0;
        if (lane == 0) old = __hip_atomic_fetch_add((unsigned*)(ws + OFF_GP + g * 128 + 64), 1u, __ATOMIC_RELAXED, __HIP_MEMORY_SCOPE_AGENT);
        if (((unsigned)rfl((int)old) + 1u) % WG_PER_GRP == 0u) __builtin_amdgcn_fence(__ATOMIC_RELEASE, "agent");
    }
}

__global__ __launch_bounds__(512, 2) void fused_kernel(Params P) {
    __shared__ __attribute__((aligned(16))) unsigned char smem[LDS_BYTES];
    LAS unsigned char* lds = (LAS unsigned char*)smem;
    const int b = blockIdx.x;
    unsigned call0 = 0;
    constexpr int PB = CB == 0 ? NCOMM : 0;
    static_assert(CB == 0 || CB >= NPW, "roles");
    if (__builtin_expect((unsigned)(b - PB) < (unsigned)NPW, 1)) call0 = push_first(P, (unsigned)(uintptr_t)(lds + STAGE_BYTES), b - PB);
    if ((unsigned)(b - CB) < (unsigned)NCOMM) {
        if (threadIdx.x == 0) *(LAS unsigned*)(lds + 1024) = take_ticket(P, 0);
        __syncthreads();
        const unsigned call = (unsigned)rfl((int)*(LAS unsigned*)(lds + 1024)) / (GRID / 16) + 1u;
        comm_role(P, lds, call, b - CB);
    } else {
        gemm_role(P, lds, b & 7, (b < CB ? b : b - NCOMM) >> 3, call0);
    }
}

}

extern "C" void kernel_launch(int rank, int nranks, ncclComm_t comm, hipStream_t stream,
                              void* const* d_in, const int* in_sizes, int n_in,
                              void* d_out, int out_size,
                              const int* in_dtypes, int out_dtype,
                              void* const* d_out_all,
                              void* const* d_ws_all, size_t ws_bytes) {
    (void)comm; (void)in_sizes; (void)n_in; (void)d_out_all; (void)out_size; (void)in_dtypes; (void)out_dtype;
    if (nranks != NR || (unsigned)rank >= (unsigned)NR || ws_bytes < WS_NEED) {
        fprintf(stderr, "unsupported configuration\n");
        return;
    }
    Params P;
    P.x = (const u16*)d_in[0]; P.w = (const u16*)d_in[1];
    for (int q = 0; q < NR; ++q) P.ws[q] = (char*)d_ws_all[(rank + 1 + q) % NR];
    P.out = (char*)d_out;
    P.rank = rank;
    hipLaunchKernelGGL(fused_kernel, dim3(GRID), dim3(512), 0, stream, P);
}
```

```cpp
#include <hip/hip_runtime.h>
#include <rccl/rccl.h>
#include <cstdio>

namespace {

typedef unsigned short u16;
typedef short bf16x8 __attribute__((ext_vector_type(8)));
typedef unsigned u32x4 __attribute__((ext_vector_type(4)));
#define LAS __attribute__((address_space(3)))
#define GAS __attribute__((address_space(1)))

constexpr int NR = 8;
constexpr int NN = 32768, KSH = 512;
constexpr int GRID = 256;
constexpr int NCOMM = 32;
constexpr int NGEMM = GRID - NCOMM;
constexpr int WG_PER_GRP = NGEMM / 8;
constexpr int RC = 14, NROUND = 7;
constexpr int NCT = RC * NROUND;
constexpr int NOWN = 4;
constexpr int NPRE = 4;
constexpr int PPV = 2, PT0 = 8;
constexpr unsigned STG = 70;
constexpr int LB = 16, JT = 1120;
constexpr int FR = 5;
constexpr int RU = 5, RDI = 4;
constexpr int DR = 3, PDEC = 350, PDEC2 = 397;
constexpr int CB = 56;
constexpr int OWNB = NOWN * 512;
constexpr int SH0 = 8 * NOWN;
static_assert(WG_PER_GRP == 28 && NCT + 2 == 128 - 7 * NOWN, "tile map");
constexpr int MU = 3;
constexpr int PD = 14, PD1 = 6;
constexpr int NS = 5;
constexpr int NSLOT = 64 / MU;
constexpr int AHEAD = 9, AHEADT = 7;
constexpr int RUN1 = 8, RUNT = 2, RUN_DIV = 44;
constexpr int NHOLE = 5;
constexpr int NEARLY = 30;
constexpr unsigned DLY_L = 400;
constexpr int AUX_SYS = 17, AUX_DEV = 16, AUX_OUT = 18;
constexpr int NPW = 56;

constexpr unsigned OFF_TMO = 128;
constexpr unsigned OFF_PROG = 256;
constexpr unsigned OFF_CF = 512;
constexpr unsigned OFF_JOB = 544;
constexpr unsigned OFF_SD = 768;
constexpr unsigned OFF_GP = 1024;
constexpr unsigned OFF_TICKET = 2048;
constexpr unsigned OFF_TD = 4096;
constexpr unsigned OFF_FO = 81920;
constexpr unsigned OFF_WQ = 12288;
constexpr unsigned TD_CF = 80, TD_G = 82;
constexpr unsigned OFF_FL = 16384;
constexpr unsigned OFF_HF = 65536;
constexpr unsigned OFF_XA = 1u << 20;
constexpr unsigned SEND_BYTES = 262144u;
constexpr unsigned PANEL_BYTES = 2u << 20;
constexpr unsigned OFF_WB = OFF_XA + 16u * PANEL_BYTES;
constexpr unsigned OFF_SEND = OFF_WB + RC * RU * PANEL_BYTES;
constexpr unsigned OFF_OS = OFF_SEND + 8u * NOWN * SEND_BYTES;
constexpr unsigned OFF_FX = OFF_OS + 4096u * OWNB;
constexpr unsigned OFF_MINE = OFF_FX + 8u * 4096u * 1024u;
constexpr unsigned OFF_OP = OFF_MINE + (64u + (NCT + 2u) * 4u) * 65536u;
constexpr unsigned OFF_PT = OFF_OP + 8u * PANEL_BYTES;
constexpr unsigned WS_NEED = OFF_PT + NGEMM * 6u * 131072u;
constexpr unsigned OUT_BYTES = 256u << 20;

constexpr int R0N = 30, N_R0 = 4 * R0N, J_FR = RC * FR, W_OP = 28, W_FR = J_FR + RC;
constexpr int N_XW = 64 + (NCT + 2) * 4;
constexpr int HOLE = 3 * R0N + 18 - NHOLE;
constexpr int K3N = R0N - NHOLE;
constexpr int V_R0 = N_R0 - NHOLE;
constexpr int V_A = N_XW - 8 - NHOLE - 4 * RC, NA = 20;
constexpr int V_XW = V_A + NA + 4 * RC;
constexpr int V_ALL = V_XW + OWNB / 16 - NA;
constexpr int NUNIT = V_ALL * 64;
constexpr int NPOOL = 4096;
constexpr int SKIP = 28 * 64, VD = V_XW * 64 - 2 * SKIP, NMAX = NUNIT + 4096;
constexpr int K_ALL = 32768, K_END = K_ALL + 2048;
static_assert(PDEC2 * 64 < VD - 512 && V_XW * 64 < K_ALL, "decision");
constexpr int N_GRP = N_XW + 48;
static_assert(NMAX + NPOOL + 256 < 65535, "progress fits 16 bits");
static_assert(OFF_FL + (N_GRP + 8) * 32 <= 65536, "flag table");
static_assert(W_FR == W_OP + 7 * NR &&  NHOLE >= NOWN && FR == NROUND - 2 && RU == NROUND - 2 && RU - RDI >= 1 && DR == RU - RDI + 2 && DR < FR - 1, "rounds");
__device__ __forceinline__ unsigned w_panel(int j) { return OFF_WB + (unsigned)(j >= RC * RU ? j - RDI * RC : j) * PANEL_BYTES; }
__device__ __forceinline__ int ct_wi(int j, int rank) {
    const int r = j - J_FR, h = (j - W_OP) / 6, e = j - W_OP - 6 * h, p = h + (h >= rank);
    return j < W_OP ? j : r < 0 ? W_OP + 7 * p + e + (e + 1 >= (p ^ rank)) : r < RC ? W_FR + ((2 * (rank + 1) + r) & 15)
         : r < RC + 7 ? W_OP + 7 * (rank ^ (r - RC + 1)) + r - RC : j - (NCT - 7 - W_OP) + 7 * rank;
}
__device__ __forceinline__ void item_decode(int n, bool& isx, int& idx, int& kb) {
    if (n < N_R0) {
        kb = n / R0N; const int r = n - kb * R0N;
        isx = r < 4 || r >= 18;
        idx = r < 4 ? r : r < 18 ? 17 - r : r - RC;
    } else {
        const int m = n - N_R0, i = m / (4 * RC), rem = m - i * (4 * RC);
        kb = rem / RC;
        isx = false;
        idx = RC * (i + 1) + rem - kb * RC;
        if (i == NROUND - 2) { kb = m & 3; idx = RC + (m >> 2); }
    }
}
__device__ __forceinline__ int unit_need(int i, int ks, int g) {
    const int kb = ks >> 2, sl = ks & 3, r = 18 + 4 * (g & 3), h = (g >> 2) + 1;
    if (i == 0) return kb == 0 ? r * 64 : kb < 3 ? (ks * R0N + r) * 16 : 3 * R0N * 64 + (sl * K3N + r - NHOLE) * 16;
    if (i == 1) return V_R0 * 64 + (2 * ks + h) * 112;
    return (V_R0 + 4 * RC * (i - 1) + RC * kb + 7 * h) * 64;
}

constexpr unsigned long long DEADLINE_TICKS = 100000000ull;
constexpr int BS = 2 * MU;
constexpr unsigned SLOTB = BS * 1024u;
constexpr int STAGE_BYTES = 131072;
static_assert(NSLOT * SLOTB <= STAGE_BYTES, "ring");
constexpr int LDS_BYTES = STAGE_BYTES + 2048 + 8192;

struct Params {
    const u16* x; const u16* w;
    char* ws[NR];
    char* out;
    int rank;
};

__device__ __forceinline__ int rfl(int v) { return __builtin_amdgcn_readfirstlane(v); }

__device__ __forceinline__ __amdgpu_buffer_rsrc_t mk_rsrc(const void* p, unsigned bytes) {
    return __builtin_amdgcn_make_buffer_rsrc(const_cast<void*>(p), 0, (int)bytes, 0x00020000);
}
__device__ __forceinline__ bool timed_out(unsigned long long t0, unsigned* tmo, unsigned call) {
    if (__builtin_amdgcn_s_memrealtime() - t0 > DEADLINE_TICKS) {
        __hip_atomic_store(tmo, call, __ATOMIC_RELAXED, __HIP_MEMORY_SCOPE_AGENT);
        return true;
    }
    return false;
}
__device__ __forceinline__ unsigned lds_rd(unsigned addr) {
    unsigned v; asm volatile("ds_read_b32 %0, %1\n\ts_waitcnt lgkmcnt(0)" : "=v"(v) : "v"(addr) : "memory"); return v;
}
__device__ __forceinline__ u32x4 lds_rd4(unsigned addr) {
    u32x4 v; asm volatile("ds_read_b128 %0, %1\n\ts_waitcnt lgkmcnt(0)" : "=v"(v) : "v"(addr) : "memory"); return v;
}
__device__ __forceinline__ void lds_wr4(unsigned addr, u32x4 v) { asm volatile("ds_write_b128 %0, %1" :: "v"(addr), "v"(v) : "memory"); }
__device__ __forceinline__ void lds_wr(unsigned addr, unsigned v) { asm volatile("ds_write_b32 %0, %1" :: "v"(addr), "v"(v) : "memory"); }

#define WAIT_V(n) asm volatile("s_waitcnt vmcnt(" #n ")" ::: "memory")
#define WAIT_L(n) asm volatile("s_waitcnt lgkmcnt(" #n ")" ::: "memory")
#define SCHED __builtin_amdgcn_sched_barrier(0)
#define BAR __builtin_amdgcn_s_barrier()
#define DRAIN_V() do { SCHED; __builtin_amdgcn_s_waitcnt(0x0F70); WAIT_V(0); SCHED; } while (0)
template <int N> __device__ __forceinline__ void wait_v() { asm volatile("s_waitcnt vmcnt(%0)" :: "n"(N) : "memory"); }

__device__ __forceinline__ GAS char* lane_ws(const Params& P, int lane) {
    char* wsl = P.ws[0];
#pragma unroll
    for (int p = 1; p < NR; ++p) {
        char* t = P.ws[p];
        asm volatile("" : "+s"(t));
        wsl = (lane == p) ? t : wsl;
    }
    return (GAS char*)wsl;
}

__device__ __forceinline__ void arrive(char* ws, unsigned call, int lane) {
    asm volatile("" : "+s"(ws));
    unsigned old = 0;
    if (lane == 0) old = __hip_atomic_fetch_add((GAS unsigned*)(ws + OFF_SD), 1u, __ATOMIC_RELAXED, __HIP_MEMORY_SCOPE_AGENT);
    if (((unsigned)rfl((int)old) + 1u) % (NGEMM + 8) == 0u && lane < 8)
        __hip_atomic_store((GAS unsigned long long*)(ws + OFF_GP + 96) + lane * 16, __builtin_amdgcn_s_memrealtime() << 32 | call << 1 | ((__builtin_amdgcn_s_getreg(20 | 3 << 11) ^ blockIdx.x) & 1u), __ATOMIC_RELAXED, __HIP_MEMORY_SCOPE_AGENT);
}

__device__ __forceinline__ int link_of(int x, int lp) { return (x + (lp == 6 ? 0 : lp & 1 ? 6 - (lp >> 1) : (lp >> 1) + 1)) % 7; }
constexpr unsigned M_OUT = 2, M_CUT = 1, M_TWO = 4;
__device__ __forceinline__ u32x4 unit_entry(int v, int pv, int l, int rank) {
    if (v >= NMAX) {
        if (pv >= NPOOL) return (u32x4){0xffffu, 0x80000000u, 0x80000000u + M_OUT, 0u};
        return (u32x4){(unsigned)NMAX, OFF_FX + (unsigned)(rank * NPOOL + pv) * 1024u, (unsigned)pv * 65536u + (unsigned)(SH0 - NOWN + W_FR + 2 * rank) * 512u + M_OUT, N_XW + 2u};
    }
    if (v >= NUNIT) {
        const unsigned o = (unsigned)(v - NUNIT) & 2047u, h = v >= NUNIT + 2048;
        const unsigned d = (h ? 0x63724150u : 0x42351760u) >> 4 * (((rank + 1 + l) ^ rank) & 7) & 7u, p = h ? rank ^ d : rank;
        return (u32x4){(unsigned)v, OFF_OP + (p * 2048u + o) * 1024u, o * 131072u + (SH0 - NOWN + W_OP - 1 + 7 * p + d) * 512u + M_OUT + M_TWO, N_XW + 1u};
    }
    int n = v >> 6, u = v & 63;
    if (n >= V_XW || (unsigned)(n - V_A) < (unsigned)NA) {
        unsigned o = (unsigned)((n - (n >= V_XW ? V_XW - NA : V_A)) * 64 + u) + (unsigned)rank * (OWNB / 2u);
        if (o >= OWNB * 4u) o -= OWNB * 4u;
        const unsigned row = 2u * o / NOWN, cb = 2u * o - row * NOWN;
        return (u32x4){(unsigned)v, OFF_OS + o * 1024u, row * 65536u + cb * 512u + (unsigned)((rank + 1 + l) & 7) * OWNB + M_OUT + (cb == NOWN - 1 ? M_CUT : 0u), (unsigned)N_XW};
    }
    if (n >= V_R0 + 4 * RC) n += n >= V_A ? NHOLE - NA : NHOLE;
    else if (n >= V_R0) {
        const int v1 = v - V_R0 * 64, sl = v1 / (RC * 16), rem = v1 - sl * (RC * 16);
        n = N_R0 + (sl >> 2) * RC + (rem >> 4); u = (sl & 3) * 16 + (rem & 15);
    } else if (n >= 3 * R0N) {
        const int v1 = v - 3 * R0N * 64, sl = v1 / (K3N * 16), rem = v1 - sl * (K3N * 16);
        n = 3 * R0N + rem / 16; u = sl * 16 + rem % 16;
        if (n >= HOLE) n += NHOLE;
    } else if (n >= R0N) {
        const int sl = v / (R0N * 16), rem = v - sl * (R0N * 16);
        n = sl / 4 * R0N + rem / 16; u = sl % 4 * 16 + rem % 16;
    }
    bool isx; int idx, kb;
    item_decode(n, isx, idx, kb);
    const int wi = ct_wi(idx, rank);
    const bool own = !isx && wi < NOWN;
    if (n >= N_R0) n = N_R0 + (wi - RC) * 4 + kb;
    const unsigned lb = (isx ? OFF_XA + (unsigned)idx * PANEL_BYTES : w_panel(idx)) + (unsigned)((kb * 4 + (u >> 4)) * 7 + l) * 16384u + (u & 15) * 1024u;
    const unsigned rb = own ? OFF_SEND + (unsigned)(rank * NOWN + wi) * SEND_BYTES + (unsigned)kb * 65536u + u * 1024u : OFF_MINE + (unsigned)n * 65536u + u * 1024u;
    return (u32x4){(unsigned)v, rb, lb, (unsigned)n};
}
__device__ __forceinline__ int min8(int v) {
    v = min(v, __builtin_amdgcn_update_dpp(v, v, 0xB1, 15, 15, false));
    v = min(v, __builtin_amdgcn_update_dpp(v, v, 0x4E, 15, 15, false));
    return min(v, __builtin_amdgcn_update_dpp(v, v, 0x141, 15, 15, false));
}
__device__ __forceinline__ int lead_batches(unsigned cnt) {
    int r = __builtin_amdgcn_readlane((int)cnt, 0) * NS;
#pragma unroll
    for (int s = 1; s < NS; ++s) r = min(r, __builtin_amdgcn_readlane((int)cnt, s) * NS + s);
    return r;
}

constexpr unsigned L_BAD = 0, L_FULL = 4, L_USED = 12, L_GATE = 16, L_FREED = 32, L_DONE = 64, L_AVAIL = 96, L_DQ = 2048;
constexpr int NDQ = 64;
static_assert(2 * NS + NSLOT + AHEAD + 16 <= NDQ, "ring of claimed batches");
template <int L, int K> __device__ __forceinline__ void drain(unsigned ctl, int i) {
    wait_v<K * L>();
    if (i >= K) lds_wr(ctl + L_FULL, (unsigned)(i - K));
    if constexpr (K > 0) drain<L, K - 1>(ctl, i);
}
template <int L, int W> __device__ __forceinline__ void loader(const Params& P, LAS unsigned char* lds, int lnk, int src) {
    const int lane = threadIdx.x & 63, q = lane & 7, lb8 = lane >> 3;
    const unsigned ctl = (unsigned)(uintptr_t)(lds + STAGE_BYTES);
    int freed = 0, i = 0, dry = 0;
    const __amdgpu_buffer_rsrc_t ra = mk_rsrc(P.ws[(unsigned)__builtin_amdgcn_readlane(lnk, 0) % 7u], WS_NEED);
    const __amdgpu_buffer_rsrc_t rb = mk_rsrc(P.ws[(unsigned)__builtin_amdgcn_readlane(lnk, L - 1) % 7u], WS_NEED);
    bool more = true;
#pragma unroll 1
    while (more) {
        u32x4 e; unsigned av, fr;
        asm volatile("ds_read_b128 %0, %3\n\tds_read_b32 %1, %4\n\tds_read_b32 %2, %5\n\ts_waitcnt lgkmcnt(0)" : "=&v"(e), "=&v"(av), "=&v"(fr)
                     : "v"(ctl + L_DQ + ((i + lb8) & (NDQ - 1)) * 128 + q * 16), "v"(ctl + L_AVAIL + src * 4), "v"(ctl + L_FREED + q * 4) : "memory");
        freed = lead_batches(fr);
        const bool valid = (e.x >> 16) == ((unsigned)(i + lb8 + 1) & 0xffffu);
        const unsigned long long no = ~__ballot(valid && e.w < av);
        const int nv = no ? (int)__builtin_ctzll(no) >> 3 : 8;
        if (!nv) {
            if ((__ballot(valid) & 255u) == 255u || ++dry > 32) { WAIT_V(0); lds_wr(ctl + L_FULL, (unsigned)i); }
            __builtin_amdgcn_s_sleep(1);
        } else dry = 0;
#pragma unroll 1
        for (int u = 0; u < nv; ++u, ++i) {
            if ((__builtin_amdgcn_readlane((int)e.x, u * 8) & 0xffff) == 0xffff) { more = false; break; }
            lds_wr(ctl + L_USED, (unsigned)(i + 1));
            unsigned off[L];
#pragma unroll
            for (int t = 0; t < L; ++t) off[t] = (unsigned)__builtin_amdgcn_readlane((int)e.y, u * 8 + t) + lane * 16u;
            while (i - NSLOT >= freed) { freed = lead_batches(lds_rd(ctl + L_FREED + q * 4)); if (i - NSLOT >= freed) __builtin_amdgcn_s_sleep(1); }
            LAS unsigned char* slot = lds + (unsigned)i % NSLOT * SLOTB;
            SCHED;
#pragma unroll
            for (int t = 0; t < L; ++t) {
                wait_v<W - 1>();
                __builtin_amdgcn_raw_ptr_buffer_load_lds(t >= MU ? rb : ra, (LAS void*)(slot + t * 1024), 16, off[t], 0, 0, AUX_SYS);
                SCHED;
                if (t == (W - 1) % L && i >= (W - 1) / L) lds_wr(ctl + L_FULL, (unsigned)(i - (W - 1) / L));
            }
        }
    }
    drain<L, (W - 1) / L>(ctl, i);
}

__device__ __forceinline__ void comm_role(const Params& P, LAS unsigned char* lds, unsigned call, int c) {
    const int tid = threadIdx.x, wid = rfl(tid >> 6), lane = tid & 63;
    const int rank = P.rank;
    unsigned* tmo = (unsigned*)(P.ws[7] + OFF_TMO);
    const unsigned ctl = (unsigned)(uintptr_t)(lds + STAGE_BYTES);
    const unsigned pbase = call << 16;
    const int q = lane & 7, lb8 = lane >> 3;
    const int nreal = c < 24 ? BS : MU, qq = min(q, nreal - 1);
    const int hf = qq >= MU, f0 = hf * MU, oc = qq - f0;
    const int lnk = link_of(c & 7, 2 * (c >> 3) + hf);
    const int src = (rank + 1 + lnk) & 7;
    if (tid < 128) lds_wr4(ctl + tid * 16, (u32x4){0u, 0u, 0u, 0u});
    lds_wr4(ctl + L_DQ + tid * 16, (u32x4){0u, 0u, 0u, 0u});
    WAIT_L(0);
    __syncthreads();
    if (wid == 0) {
        __builtin_amdgcn_s_setprio(3);
        if (c < 24) loader<BS, PD>(P, lds, lnk, src); else loader<MU, PD1>(P, lds, lnk, src);
    } else if (wid == 1) {
        const unsigned* fl = (const unsigned*)(P.ws[7] + OFF_FL) + lane;
        unsigned* prog = (unsigned*)(P.ws[7] + OFF_PROG);
        const unsigned long long t0 = __builtin_amdgcn_s_memrealtime();
        const unsigned smask = 1u << __builtin_amdgcn_readlane(src, 0) | 1u << __builtin_amdgcn_readlane(src, 7);
        int pos = smask >> q & 1u ? 0 : N_GRP;
        int mine = 0, all = c < 8 ? 0 : K_END;
        bool open = true;
        bool pushed = c >= 8;
        int gate = 0;
        int asked = c ? 2 : 0;
        if (c < 8) { __builtin_amdgcn_fence(__ATOMIC_ACQUIRE, ""); WAIT_V(0); arrive(P.ws[7], call, lane); }
        unsigned tref = 0;
        while (open || mine < V_XW * 64 || all < K_END) {
            unsigned v = call, pv = 0, hv = call, cv = 0;
            if (gate < 3 || all < K_END) cv = __hip_atomic_load((unsigned*)(P.ws[7] + OFF_CF) + (lane & 31), __ATOMIC_RELAXED, __HIP_MEMORY_SCOPE_AGENT);
            if (pos < N_GRP) v = __hip_atomic_load(fl + pos * 8, __ATOMIC_RELAXED, __HIP_MEMORY_SCOPE_SYSTEM);
            if (!pushed) hv = __hip_atomic_load(fl + N_GRP * 8, __ATOMIC_RELAXED, __HIP_MEMORY_SCOPE_SYSTEM);
            if (all < K_END) pv = __hip_atomic_load(prog + (lane & 31), __ATOMIC_RELAXED, __HIP_MEMORY_SCOPE_AGENT);
            if (mine < V_XW * 64) {
                const int k = lead_batches(lds_rd(ctl + L_DONE + q * 4));
                const unsigned e0 = (unsigned)rfl((int)lds_rd(ctl + L_DQ + (k & (NDQ - 1)) * 128));
                const int m = (e0 >> 16) == ((unsigned)(k + 1) & 0xffffu) ? (int)(e0 & 0xffffu) : -1;
                if (m > mine) {
                    if (lane == 0) __hip_atomic_store(prog + c, pbase + (unsigned)m, __ATOMIC_RELAXED, __HIP_MEMORY_SCOPE_AGENT);
                    mine = m;
                }
            }
            bool adv = false;
            if (open) {
                const int it = pos + lb8;
                const unsigned long long miss = __ballot((int)(v - call) < 0 && it < N_GRP && (it < HOLE || it >= HOLE + NHOLE)) >> q & 0x0101010101010101ull;
                const int a = pos >= N_GRP ? 0 : miss ? (int)(__builtin_ctzll(miss) >> 3) : 8;
                pos += a;
                adv = __any(a);
                if (adv && lane < 8) lds_wr(ctl + L_AVAIL + q * 4, (unsigned)(pos <= N_XW ? pos : N_XW + ((pos - N_XW) >> 4)));
                open = __any(pos < N_GRP);
            }
            if (gate == 2) {
                const unsigned dv = (unsigned)__builtin_amdgcn_readlane((int)cv, 3);
                if ((dv ^ call << 2) < 4u) lds_wr(ctl + L_GATE, (unsigned)(gate = 3 + (dv & 3u)));
            }
            if (gate < 2 && (unsigned)__builtin_amdgcn_readlane((int)cv, gate) == call) {
                lds_wr(ctl + L_GATE, (unsigned)++gate);
                if (c < 8) { __builtin_amdgcn_fence(__ATOMIC_ACQUIRE, "agent"); WAIT_V(0); }
            }
            if (asked < 2 && all >= PDEC * 64) {
                if (!asked) tref = (unsigned)__builtin_amdgcn_s_memrealtime();
                const unsigned jt = (unsigned)__shfl((int)cv, (OFF_JOB - OFF_CF) / 4 + 2 * q), jd = (unsigned)__shfl((int)cv, (OFF_JOB - OFF_CF) / 4 + 2 * q + 1);
                const bool far = __all(((jt ^ call) & 255u) == 0u);
                asked = far || all >= PDEC2 * 64 ? 2 : 1;
                if (asked == 2) {
                    const float tc = 24360.f;
                    const float p = fmaxf((float)((int)(jt - (tref << 8)) >> 8) + 1.01f * jd, tc - 14100.f), tau = jd * (1.f / 58.4f);
                    int k = 0, o = 0;
#pragma unroll
                    for (int h = 0; h < 8; ++h) { const float x = __shfl(p, h); k += x < p || (x == p && h < q); }
#pragma unroll
                    for (int h = 0; h < 8; ++h) o = __shfl(k, h) + k == 7 ? h : o;
                    const float po = __shfl(p, o), to = __shfl(tau, o), tf = k < 4 ? tau : to, ts = k < 4 ? to : tau, gap = fabsf(p - po);
                    const int h0 = (int)fminf(fmaxf((gap + 33.6f * ts - 29.6f * tf) / (tf + ts) + 0.5f, 0.f), 20.f), h1 = (int)fminf(fmaxf((gap + 33.2f * ts) / (tf + ts) + 0.5f, 8.f), 28.f);
                    const bool help = h0 >= 7 && fmaxf(p, po) + 28.3f * ts > tc - 800.f;
                    const float e0 = p + tau * (help ? (k < 4 ? h0 + 29.6f : 33.6f - h0) : k < 4 ? 32.6f : 28.3f);
                    const float e1 = p + tau * (k < 4 ? h1 + 3.0f : 35.8f - h1);
                    const unsigned dr = !far ? 2u : __all(e0 <= tc - 300.f) ? 0u : __all(e1 <= tc + 800.f) ? 1u : 2u;
                    const unsigned job = (unsigned)min(k, 7 - k) | (dr == 2 ? 16u : dr ? (k < 4 ? 8u : 12u) | h1 << 6 : !help ? k & 4 : (k < 4 ? 44u : 0u) | (32 - h0) << 6);
                    if (lane < 9) __hip_atomic_store((unsigned*)(P.ws[7] + OFF_CF) + (lane < 8 ? (OFF_JOB - OFF_CF) / 4 + 16 + lane : 3), lane < 8 ? call << 12 | job : call << 2 | dr, __ATOMIC_RELAXED, __HIP_MEMORY_SCOPE_AGENT);
                }
            }
            if (!pushed) pushed = __all((int)(hv - call) >= 0 || (lane & 7) == rank);
            if (all < K_END) {
                int m = (int)(pv - pbase);
#pragma unroll
                for (int o = 16; o >= 1; o >>= 1) m = min(m, __shfl_xor(m, o));
                m = rfl(m);
                if (!pushed) m = min(m, 3 * R0N * 64);
                m = (unsigned)__builtin_amdgcn_readlane((int)cv, 4) == call ? max(m, 1) : 0;
                if (gate < 2) m = min(m, (V_R0 + 4 * RC * (RU - 1 + gate)) * 64);
                if (m >= V_XW * 64 - SKIP) {
                    const unsigned jv = (unsigned)__builtin_amdgcn_readlane((int)cv, (OFF_JOB - OFF_CF) / 4 + 16 + c) ^ call << 12;
                    m = jv < 2048u ? (m >= V_XW * 64 ? K_END : K_ALL) + (int)jv : min(m, V_XW * 64 - 1);
                }
                if (m > all) {
                    if (lane == 0) __hip_atomic_store((unsigned*)(P.ws[7] + OFF_GP + c * 128), pbase + (unsigned)m, __ATOMIC_RELAXED, __HIP_MEMORY_SCOPE_AGENT);
                    all = m;
                }
            }
            if (timed_out(t0, tmo, call)) { lds_wr(ctl + L_BAD, 1u); WAIT_L(0); lds_wr(ctl + L_AVAIL + q * 4, 0xffffu); lds_wr(ctl + L_GATE, (unsigned)max(gate, 3)); break; }
            if (!adv) __builtin_amdgcn_s_sleep(4);
        }
        if (lane < NR - 1)
            __hip_atomic_store((GAS unsigned*)(lane_ws(P, lane) + OFF_FO + c * 128 + rank * 4), call, __ATOMIC_RELAXED, __HIP_MEMORY_SCOPE_SYSTEM);
        const unsigned* fo = (const unsigned*)(P.ws[7] + OFF_FO + c * 128) + (lane & 7);
        for (;;) {
            const unsigned v = __hip_atomic_load(fo, __ATOMIC_RELAXED, __HIP_MEMORY_SCOPE_SYSTEM);
            if (__all((int)(v - call) >= 0 || (lane & 7) == rank)) break;
            if (timed_out(t0, tmo, call)) break;
            __builtin_amdgcn_s_sleep(16);
        }
    } else if (wid == 2) {
        __builtin_amdgcn_s_setprio(2);
        unsigned* wq = (unsigned*)(P.ws[7] + OFF_WQ + (call & 1u) * 1024u + (lane < 63 ? lnk : 7) * 128);
        if (c == 0 && lane < 8)
            __hip_atomic_store((unsigned*)(P.ws[7] + OFF_WQ + ((call + 1u) & 1u) * 1024u + lane * 128), 0u, __ATOMIC_RELAXED, __HIP_MEMORY_SCOPE_AGENT);
#define uo (m == 8 ? oc * 8 + ((3 * lb8 + oc) & 7) : oc * m + lb8)
        int m = RUN1, base = (c & 7) * MU * RUN1, pv = 0, hp = 0;
        bool ex = false;
        bool shut = false;
        int gt = 0, nun = NUNIT, skip = 0;
#define UMAP(v) ((v) + skip >= V_XW * 64 ? (v) + skip : (v))
#pragma unroll 1
        for (int k = 0;;) {
            u32x4 e = unit_entry(shut || q >= nreal || base + uo >= nun ? NMAX : UMAP(base + uo), shut || q >= nreal ? NPOOL : pv, lnk, rank);
            if (shut) e.x = (unsigned)UMAP(base);
            unsigned lo = e.x;
            if (base < nun) lo = (unsigned)UMAP(base);
            lo = (unsigned)min8((int)lo);
            e.x = (unsigned)(k + lb8 + 1) << 16 | lo;
            if (lb8 < m) lds_wr4(ctl + L_DQ + ((k + lb8) & (NDQ - 1)) * 128 + q * 16, e);
            k += m;
            if (__any(lb8 < m && lo == 0xffffu)) {
                lds_wr4(ctl + L_DQ + ((k + lb8) & (NDQ - 1)) * 128 + q * 16, (u32x4){(unsigned)(k + lb8 + 1) << 16 | 0xffffu, 0u, 0u, 0u});
                break;
            }
            if (!shut) base = min(base + MU * m, nun);
            ex = base >= nun;
            const int left = rfl(min8(nun - base));
            const int rest = left + max(NPOOL - hp, 0) / 7;
            m = max(RUNT, min(RUN1, rest / RUN_DIV));
            constexpr int GUARD = 40 * RUN1;
            for (;;) {
                int av;
                for (;;) {
                    av = (int)lds_rd(ctl + (lane == 62 ? L_USED : lane == 63 ? L_GATE : L_AVAIL + src * 4));
                    if (__builtin_amdgcn_readlane(av, 62) + (rest < 256 ? AHEADT : AHEAD) >= k) break;
                    __builtin_amdgcn_s_sleep(1);
                }
                gt = __builtin_amdgcn_readlane(av, 63);
                if (lane >= 62) av = __builtin_amdgcn_readlane(av, 7);
                shut = (gt < 3 && base >= min((V_R0 + 4 * RC * (RU - 1 + gt)) * 64, VD) - GUARD) || (base >= V_A * 64 - GUARD && av <= N_XW) || (base >= NUNIT - skip - GUARD && av <= N_XW + 1 && skip && !ex) || (base >= nun - GUARD && av <= N_XW + 2 && hp < NPOOL);
                if (!__all(shut || (ex && hp >= NPOOL)) || !__any(shut)) break;
                __builtin_amdgcn_s_sleep(8);
            }
            const bool act = lb8 < m && q < nreal && !shut;
            const unsigned long long pm = __ballot(act && ex);
            int r = 0;
            if (lane < 63 ? lane == f0 && !ex && !shut : left < 1024)
                r = (int)__hip_atomic_fetch_add(wq, (unsigned)(lane < 63 ? MU * m : __builtin_popcountll(pm)), __ATOMIC_RELAXED, __HIP_MEMORY_SCOPE_AGENT);
            const int got = (hf ? __builtin_amdgcn_readlane(r, MU) : __builtin_amdgcn_readlane(r, 0)) + 8 * MU * RUN1;
            if (__any(!ex && !shut && got + MU * m > VD)) while (gt < 3) gt = rfl((int)lds_rd(ctl + L_GATE));
            if (gt > 3) { nun = NUNIT + (gt - 3) * (2048 - SKIP); skip = (gt - 3) * SKIP; }
            if (!ex && !shut) base = min(got, nun);
            hp = __builtin_amdgcn_readlane(r, 63);
            pv = hp + __builtin_popcount((unsigned)pm & ((1u << q) - 1u)) * m + lb8;
            const unsigned long long nm = __ballot(act && !ex && base + uo >= nun);
            if (nm) {
                if (lane == 63) r = (int)__hip_atomic_fetch_add(wq, (unsigned)__builtin_popcountll(nm), __ATOMIC_RELAXED, __HIP_MEMORY_SCOPE_AGENT);
                hp = __builtin_amdgcn_readlane(r, 63);
                if (!ex) pv = hp + (int)__builtin_amdgcn_mbcnt_hi((unsigned)(nm >> 32), __builtin_amdgcn_mbcnt_lo((unsigned)nm, 0u));
            }
        }
    } else if (wid < 3 + NS) {
        const int s = wid - 3;
        int full = 0;
        unsigned cnt = 0, done = 0;
        bool bad = false;
        const unsigned cut = lane * 16u + (lane >> 5) * (65536u - OWNB), two = lane * 16u + (lane >> 5) * (65536u - 512u);
        const __amdgpu_buffer_rsrc_t rw = mk_rsrc(P.ws[7], WS_NEED), ro = mk_rsrc(P.out, OUT_BYTES);
#pragma unroll 1
        for (int i = s;; i += NS) {
            u32x4 e;
            for (;;) {
                e = lds_rd4(ctl + L_DQ + (i & (NDQ - 1)) * 128 + q * 16);
                if (__all((e.x >> 16) == ((unsigned)(i + 1) & 0xffffu))) break;
                if (done != cnt) { DRAIN_V(); done = cnt; lds_wr(ctl + L_DONE + s * 4, done); }
                else __builtin_amdgcn_s_sleep(1);
            }
            const int unit0 = rfl((int)e.x) & 0xffff;
            if (unit0 == 0xffff) break;
            while (i >= full) {
                const unsigned cw = lds_rd(ctl + (lane & 1) * 4);
                full = __builtin_amdgcn_readlane((int)cw, 1);
                bad = __builtin_amdgcn_readlane((int)cw, 0) != 0;
                if (i >= full) {
                    if (done != cnt) { DRAIN_V(); done = cnt; lds_wr(ctl + L_DONE + s * 4, done); }
                    else __builtin_amdgcn_s_sleep(1);
                }
            }
            u32x4 v[BS];
#pragma unroll
            for (int t = 0; t < BS; ++t) v[t] = *(const LAS u32x4*)(lds + (unsigned)i % NSLOT * SLOTB + t * 1024 + lane * 16);
            SCHED; WAIT_L(0); SCHED;
            ++cnt;
            lds_wr(ctl + L_FREED + s * 4, cnt);
#pragma unroll
            for (int t = 0; t < BS; ++t) {
                if (bad) v[t] = (u32x4){0x7fc07fc0u, 0x7fc07fc0u, 0x7fc07fc0u, 0x7fc07fc0u};
                const unsigned lb = (unsigned)__builtin_amdgcn_readlane((int)e.z, t), off = (lb & ~7u) + (lb & M_TWO ? two : lb & M_CUT ? cut : lane * 16u);
                if (lb & M_OUT) __builtin_amdgcn_raw_buffer_store_b128(v[t], ro, off, 0, AUX_OUT);
                else __builtin_amdgcn_raw_buffer_store_b128(v[t], rw, off, 0, AUX_DEV);
            }
            SCHED;
            if (done + 1u != cnt) { wait_v<BS>(); done = cnt - 1u; lds_wr(ctl + L_DONE + s * 4, done); }
            if (unit0 >= V_XW * 64) { done = cnt; lds_wr(ctl + L_DONE + s * 4, done); }
        }
        if (done != cnt) { DRAIN_V(); lds_wr(ctl + L_DONE + s * 4, cnt); }
    }
}

#define LDIN(p) __builtin_nontemporal_load(p)
__device__ __forceinline__ unsigned take_ticket(const Params& P, int zv) {
    return __hip_atomic_fetch_add((unsigned*)(P.ws[7] + OFF_TICKET + (blockIdx.x & 15) * 128) + zv, 1u, __ATOMIC_RELAXED, __HIP_MEMORY_SCOPE_AGENT);
}
__device__ __forceinline__ unsigned push_first(const Params& P, unsigned ctl, int s1) {
    const int tid = threadIdx.x, lane = tid & 63;
    const int q = s1 >> 3, wi = rfl(tid >> 6), fi = N_GRP + (s1 & 7);
    const unsigned sub = (unsigned)s1 & 7u;
    const u16* w = P.w; int rank = P.rank; char* wq = P.ws[q]; char* w7 = P.ws[7];
    const unsigned vo = ((sub >> 1) * 32u + ((unsigned)lane >> 4) * 8u) * (NN * 2u) + (sub & 1u) * 256u + (lane & 15) * 16u;
    unsigned dst = OFF_WB + (unsigned)wi * PANEL_BYTES + ((12u + (sub >> 1)) * 7u + 6u - q) * 16384u + (sub & 1u) * 4096u + lane * 16u;
    int zv = 0;
    asm volatile("" : "+v"(dst), "+v"(zv));
    asm volatile("" : "+s"(w), "+s"(rank), "+s"(wq), "+s"(w7));
    const __amdgpu_buffer_rsrc_t rs = mk_rsrc(w, KSH * NN * 2u), rw = mk_rsrc(wq, WS_NEED);
    const unsigned so = 384u * (NN * 2u) + (wi < NOWN ? (unsigned)((rank + 1 + q) & 7) * OWNB : (SH0 - NOWN) * 512u) + (unsigned)wi * 512u;
    if (wi < NHOLE) {
        u32x4 v[8];
#pragma unroll
        for (int t = 0; t < 8; ++t) v[t] = __builtin_amdgcn_raw_buffer_load_b128(rs, vo, so + t * (NN * 2u), 2);
        SCHED;
#pragma unroll
        for (int t = 0; t < 8; ++t) {
            u32x4 o;
#pragma unroll
            for (int e = 0; e < 4; ++e) o[e] = __builtin_amdgcn_perm(v[2 * e + 1][t >> 1], v[2 * e][t >> 1], (t & 1) ? 0x07060302u : 0x05040100u);
            if (t) { SCHED; WAIT_V(0); SCHED; }
            __builtin_amdgcn_raw_buffer_store_b128(o, rw, dst + (t >> 2) * 8192 + (t & 3) * 1024, 0, AUX_SYS);
        }
    }
    SCHED;
    unsigned tk = 0;
    if (tid == 448) tk = __hip_atomic_fetch_add((GAS unsigned*)(w7 + OFF_TICKET + (blockIdx.x & 15) * 128) + zv, 1u, __ATOMIC_RELAXED, __HIP_MEMORY_SCOPE_AGENT);
    DRAIN_V();
    if (tid == 448) { lds_wr(ctl + 4, tk); WAIT_L(0); }
    BAR;
    const unsigned call = (unsigned)rfl((int)lds_rd(ctl + 4)) / (GRID / 16) + 1u;
    if (tid == 0) __hip_atomic_store((GAS unsigned*)(wq + OFF_FL + fi * 32 + rank * 4), call, __ATOMIC_RELAXED, __HIP_MEMORY_SCOPE_SYSTEM);
    SCHED;
    return call;
}
__device__ __forceinline__ unsigned stage_role(const Params& P, GAS char* wsl, unsigned ctl, int s0, unsigned call, bool& fast) {
    const int tid = threadIdx.x, sub = rfl(tid >> 6), lane = tid & 63;
    const int rank = P.rank;
    const int qi = lane & 15;
    constexpr int SA = R0N + 7 * NOWN, SC = N_XW + 7 * NOWN, N_STAGE = NPW + SC + 14 * NOWN;
    static_assert(N_STAGE <= 3 * NGEMM - 64 && 2 * NGEMM - NPW - 7 * NOWN >= N_R0 + 8 * 28, "three passes, the first two cover three rounds");
    const unsigned long long t0 = __builtin_amdgcn_s_memrealtime();
    unsigned tk = 0;
    int zv = 0;
    asm volatile("" : "+v"(zv));
    lds_wr(ctl, 0u);
    if (s0 >= NPW + NEARLY) {
        while (__builtin_amdgcn_s_memrealtime() - t0 < DLY_L) __builtin_amdgcn_s_sleep(2);
    }
#pragma unroll 1
    for (int s1 = s0 < NPW ? s0 + NGEMM : s0; s1 < N_STAGE;) {
        const int s = s1 - NPW;
        int n = s < R0N ? s : s - 7 * NOWN, q = 7;
        if ((s >= R0N && s < SA) || s >= SC) {
            const int r = s < SA ? s - R0N : s - N_XW, jo = r / 7;
            q = r - jo * 7; n = jo / NOWN * R0N + 18 - NOWN + jo % NOWN;
        }
        const __amdgpu_buffer_rsrc_t rw = mk_rsrc(P.ws[7], WS_NEED);
        bool isx; int idx, kb;
        item_decode(n, isx, idx, kb);
        int wi = idx;
        if (n >= N_R0) { kb = (n - N_R0) & 3; wi = RC + ((n - N_R0) >> 2); }
        const bool own = !isx && wi < NOWN;
        u32x4 v[8];
        if (isx) {
            const u16* src = P.x + (size_t)(idx * 256 + sub * 32 + (lane & 15)) * KSH + kb * 128 + (lane >> 4) * 8;
#pragma unroll
            for (int t = 0; t < 8; ++t) v[t] = LDIN((const u32x4*)(src + (t >> 2) * 16 * KSH + (t & 3) * 32));
            SCHED;
            if (s1 == s0 && tid == 0) tk = take_ticket(P, zv);
            SCHED;
            const unsigned dst = OFF_MINE + (unsigned)n * 65536u + sub * 2048u + lane * 16u;
#pragma unroll
            for (int t = 0; t < 8; ++t)
                __builtin_amdgcn_raw_buffer_store_b128(v[t], rw, dst + (t >> 2) * 1024 + (t & 3) * 16384, 0, AUX_SYS);
        } else {
            const int wr = sub & 1;
            const int p = (rank + 1 + q) & 7;
            const int col = own ? p * (OWNB / 2) + wi * 256 : (SH0 - NOWN + wi) * 256;
            const u16* src = P.w + (size_t)(kb * 128 + (sub >> 1) * 32 + (lane >> 4) * 8) * NN + col + wr * 128 + qi * 8;
#pragma unroll
            for (int t = 0; t < 8; ++t) v[t] = LDIN((const u32x4*)(src + (size_t)t * NN));
            SCHED;
            if (s1 == s0 && tid == 0) tk = take_ticket(P, zv);
            SCHED;
            const unsigned base = q < 7 ? OFF_SEND + (unsigned)(p * NOWN + wi) * SEND_BYTES + (unsigned)kb * 65536u : OFF_MINE + (unsigned)n * 65536u;
            const unsigned dst = base + (sub >> 1) * 16384u + wr * 4096u + lane * 16u;
#pragma unroll
            for (int t = 0; t < 8; ++t) {
                u32x4 o;
#pragma unroll
                for (int e = 0; e < 4; ++e) o[e] = __builtin_amdgcn_perm(v[2 * e + 1][t >> 1], v[2 * e][t >> 1], (t & 1) ? 0x07060302u : 0x05040100u);
                __builtin_amdgcn_raw_buffer_store_b128(o, rw, dst + (t >> 2) * 8192 + (t & 3) * 1024, 0, AUX_SYS);
            }
        }
        DRAIN_V();
        if (s1 == s0 && tid == 0) lds_wr(ctl + 4, tk);
        WAIT_L(0);
        BAR;
        if (s1 == s0) call = (unsigned)rfl((int)lds_rd(ctl + 4)) / (GRID / 16) + 1u;
        if (sub == 0 && (own ? lane == q : true) && lane < NR - 1)
            __hip_atomic_store((GAS unsigned*)(wsl + OFF_FL + n * 32 + rank * 4), call, __ATOMIC_RELAXED, __HIP_MEMORY_SCOPE_SYSTEM);
        SCHED;
        if (sub == 0 && s1 >= NGEMM) {
            if (s1 < 2 * NGEMM) arrive(P.ws[7], call, lane);
            else {
                char* w7 = P.ws[7];
                asm volatile("" : "+s"(w7));
                if (lane == 0 && (__hip_atomic_fetch_add((GAS unsigned*)(w7 + OFF_SD + 128), 1u, __ATOMIC_RELAXED, __HIP_MEMORY_SCOPE_AGENT) + 1u) % (N_STAGE - 2 * NGEMM) == 0u)
                    __hip_atomic_store((GAS unsigned*)(w7 + OFF_CF) + 4, call, __ATOMIC_RELAXED, __HIP_MEMORY_SCOPE_AGENT);
            }
        }
        s1 = s1 < NGEMM ? s1 + NGEMM : s1 < 2 * NGEMM && s0 >= 32 && (unsigned)(s0 - 8 * LB) >= 32u ? s1 + NGEMM - (s0 < 8 * LB ? 32 : 64) : N_STAGE;
    }
    if (sub == 0) {
        unsigned long long go;
        while ((unsigned)rfl((int)(go = __hip_atomic_load((unsigned long long*)(P.ws[7] + OFF_GP + 96) + (s0 & 7) * 16, __ATOMIC_RELAXED, __HIP_MEMORY_SCOPE_AGENT))) >> 1 != call) {
            if (timed_out(t0, (unsigned*)(P.ws[7] + OFF_TMO), call)) { lds_wr(ctl, ~0u); break; }
            __builtin_amdgcn_s_sleep(8);
        }
        lds_wr(ctl + 8, (unsigned)go);
        WAIT_L(0);
        const unsigned tg = (unsigned)rfl((int)(go >> 32));
        while ((int)((unsigned)__builtin_amdgcn_s_memrealtime() - tg) < (int)STG * (s0 >> 3) - (s0 >= 8 * LB ? JT : 0)) __builtin_amdgcn_s_sleep(2);
    }
    BAR;
    fast = !((rfl((int)lds_rd(ctl + 8)) ^ s0) & 1);
    return call;
}

#define AR(i) "n"(128 + 4 * (i))
typedef float f32x2 __attribute__((ext_vector_type(2)));
template <int I> struct Int { static constexpr int value = I; };
template <int N, int I = 0, class F> __device__ __forceinline__ void each(F f) { if constexpr (I < N) { f(Int<I>()); each<N, I + 1>(f); } }
#define EACH(N, i) each<N>([&](auto i##_) __attribute__((always_inline)) { constexpr int i = decltype(i##_)::value;
#define END_EACH })
template <int I, bool Z> __device__ __forceinline__ void mfma(bf16x8 a, bf16x8 b) {
    if (Z) asm volatile("v_mfma_f32_16x16x32_bf16 v[%2:%2+3], %1, %0, 0" :: "v"(a), "v"(b), AR(I));
    else asm volatile("v_mfma_f32_16x16x32_bf16 v[%2:%2+3], %1, %0, v[%2:%2+3]" :: "v"(a), "v"(b), AR(I));
}
template <int I> __device__ __forceinline__ void sigmoid(f32x2* e, float one) {
    asm volatile("v_pk_fma_f32 %0, v[%3:%3+1], %2, %2 op_sel:[0,0,1] op_sel_hi:[1,0,1]\n\tv_pk_fma_f32 %1, v[%3+2:%3+3], %2, %2 op_sel:[0,0,1] op_sel_hi:[1,0,1]"
                 : "=v"(e[0]), "=v"(e[1]) : "s"((f32x2){-1.44269504088896341f, -86.5617024533378f}), AR(I));
#pragma unroll
    for (int h = 0; h < 2; ++h) {
#pragma unroll
        for (int i = 0; i < 2; ++i) e[h][i] = __builtin_amdgcn_fmed3f(__builtin_amdgcn_exp2f(e[h][i]), 0.0f, 1.0f);
        e[h] = __builtin_elementwise_fma(e[h], (f32x2)(1.142004588e+26f), (f32x2)(one));
#pragma unroll
        for (int i = 0; i < 2; ++i) e[h][i] = __builtin_amdgcn_rcpf(e[h][i]);
    }
}
template <int L, int H> __device__ __forceinline__ void scale(const f32x2* e) {
    asm volatile("s_nop 0\n\tv_pk_mul_f32 v[%4:%4+1], v[%4:%4+1], %0\n\tv_pk_mul_f32 v[%4+2:%4+3], v[%4+2:%4+3], %1\n\t"
                 "v_pk_mul_f32 v[%5:%5+1], v[%5:%5+1], %2\n\tv_pk_mul_f32 v[%5+2:%5+3], v[%5+2:%5+3], %3" :: "v"(e[0]), "v"(e[1]), "v"(e[2]), "v"(e[3]), AR(L), AR(H));
}
template <int L, int H> __device__ __forceinline__ void pack(unsigned* o) {
    asm volatile("v_cvt_pk_bf16_f32 %0, v[%4], v[%5]\n\tv_cvt_pk_bf16_f32 %1, v[%4+1], v[%5+1]\n\tv_cvt_pk_bf16_f32 %2, v[%4+2], v[%5+2]\n\tv_cvt_pk_bf16_f32 %3, v[%4+3], v[%5+3]"
                 : "=v"(o[0]), "=v"(o[1]), "=v"(o[2]), "=v"(o[3]) : AR(L), AR(H));
}
constexpr int PB = 128 - 4 * PPV, NVIS = 16 / PPV;
constexpr unsigned PMASK = (2u << NVIS) - 1u;
template <int I> __device__ __forceinline__ void add_piece(unsigned idx) {
    unsigned h0[2], h1[2];
    asm volatile("v_and_b32 %0, 0xffff0000, v[%5]\n\tv_and_b32 %1, 0xffff0000, v[%5+1]\n\tv_and_b32 %2, 0xffff0000, v[%5+2]\n\tv_and_b32 %3, 0xffff0000, v[%5+3]\n\t"
                 "v_lshlrev_b32 v[%5], 16, v[%5]\n\tv_lshlrev_b32 v[%5+1], 16, v[%5+1]\n\tv_lshlrev_b32 v[%5+2], 16, v[%5+2]\n\tv_lshlrev_b32 v[%5+3], 16, v[%5+3]\n\t"
                 "s_set_gpr_idx_on %4, gpr_idx(SRC1,DST)\n\t"
                 "v_add_f32 v[%6], v[%5], v[%6]\n\tv_add_f32 v[%6+1], v[%5+1], v[%6+1]\n\tv_add_f32 v[%6+2], v[%5+2], v[%6+2]\n\tv_add_f32 v[%6+3], v[%5+3], v[%6+3]\n\t"
                 "v_add_f32 v[%6+4], %0, v[%6+4]\n\tv_add_f32 v[%6+5], %1, v[%6+5]\n\tv_add_f32 v[%6+6], %2, v[%6+6]\n\tv_add_f32 v[%6+7], %3, v[%6+7]\n\t"
                 "s_set_gpr_idx_off" : "=&v"(h0[0]), "=&v"(h0[1]), "=&v"(h1[0]), "=&v"(h1[1]) : "s"(idx), "n"(PB + 4 * I), "n"(128 + 8 * I) : "m0");
}
template <int I> __device__ __forceinline__ void get_piece(unsigned vo, __amdgpu_buffer_rsrc_t r, unsigned so) {
    asm volatile("buffer_load_dwordx4 v[%3:%3+3], %0, %1, %2 offen offset:%4 sc1 nt" :: "v"(vo), "s"(r), "s"(so), "n"(PB + 4 * I), "n"(I * 1024) : "memory");
}
template <int I> __device__ __forceinline__ void add_regs(u32x4 w) {
    const u32x4 l = w << 16, h = w & 0xffff0000u;
    asm volatile("v_add_f32 v[%8], %0, v[%8]\n\tv_add_f32 v[%8+1], %1, v[%8+1]\n\tv_add_f32 v[%8+2], %2, v[%8+2]\n\tv_add_f32 v[%8+3], %3, v[%8+3]\n\t"
                 "v_add_f32 v[%8+4], %4, v[%8+4]\n\tv_add_f32 v[%8+5], %5, v[%8+5]\n\tv_add_f32 v[%8+6], %6, v[%8+6]\n\tv_add_f32 v[%8+7], %7, v[%8+7]"
                 :: "v"(l[0]), "v"(l[1]), "v"(l[2]), "v"(l[3]), "v"(h[0]), "v"(h[1]), "v"(h[2]), "v"(h[3]), "n"(128 + 8 * I));
}
template <bool W> __device__ __forceinline__ void visit(unsigned k, unsigned vo, __amdgpu_buffer_rsrc_t r, unsigned so) {
    if (k) {
        if (W) WAIT_V(0);
        EACH(PPV, p) add_piece<p>((k - 1u) * (8u * PPV)); END_EACH;
    }
    if (k < NVIS) { EACH(PPV, p) get_piece<p>(vo, r, so + k * (1024u * PPV)); END_EACH; }
    asm volatile("s_nop 1");
}

template <bool LD> __device__ __forceinline__ void gemm_role(const Params& P, LAS unsigned char* lds, int xg, int lw, const unsigned call, bool fast) {
    const int tid = threadIdx.x, wid = rfl(tid >> 6), lane = tid & 63, wr = wid >> 2, wc = wid & 3, fr = lane & 15, fq = lane >> 4;
    const int rank = P.rank;
    char* const ws = P.ws[7];
    unsigned* tmo = (unsigned*)(ws + OFF_TMO);
    const unsigned lknown = (unsigned)(uintptr_t)(lds + STAGE_BYTES);
    auto tell = [&](unsigned f) {
        const unsigned l = __builtin_amdgcn_mbcnt_lo(~0u, 0u);
        GAS char* w;
        asm volatile("ds_read_b64 %0, %1\n\ts_waitcnt lgkmcnt(0)" : "=v"(w) : "v"(lknown + 1024u + l * 8u) : "memory");
        if (l < NR) __hip_atomic_store((GAS unsigned*)(w + OFF_FL + (N_XW + f) * 32 + rank * 4), call, __ATOMIC_RELAXED, __HIP_MEMORY_SCOPE_SYSTEM);
    };
    int known = rfl((int)lds_rd(lknown));

    auto ready = [&](int need) {
        if ((unsigned)known >= (unsigned)need) return;
        if (wc == 0) {
            const unsigned* f = (const unsigned*)(ws + OFF_GP + xg * 128);
            const unsigned long long t0 = __builtin_amdgcn_s_memrealtime();
            for (;;) {
                known = rfl((int)(__hip_atomic_load(f, __ATOMIC_RELAXED, __HIP_MEMORY_SCOPE_AGENT) - (call << 16)));
                if (known >= need) break;
                if (timed_out(t0, tmo, call)) { known = -1; break; }
                __builtin_amdgcn_s_sleep(8);
            }
            lds_wr(lknown, (unsigned)known);
        } else {
            while ((unsigned)(known = rfl((int)lds_rd(lknown))) < (unsigned)need) __builtin_amdgcn_s_sleep(1);
        }
    };

    const int g = xg, j0 = 7 * (g >> 2) + (lw >> 2);
    int j = RC + j0;
    int mb = 4 * (g & 3) + (lw & 3);
    const int np = NPRE + (lw >= LB), nj = np + fast;
    bool pt = false;
    int job = 0;
    auto helped = [&]() {
        const unsigned long long t0 = __builtin_amdgcn_s_memrealtime();
        while ((unsigned)rfl((int)__hip_atomic_load((unsigned*)(ws + OFF_HF) + ((job & 3) * WG_PER_GRP + lw) * 32, __ATOMIC_RELAXED, __HIP_MEMORY_SCOPE_AGENT)) != call) {
            if (timed_out(t0, tmo, call)) { lds_wr(lknown, ~0u); break; }
            __builtin_amdgcn_s_sleep(8);
        }
    };
    bf16x8 Ar[8][2], Br[2][2][4];
    const __amdgpu_buffer_rsrc_t rw = mk_rsrc(ws, WS_NEED);
    const bool isx = wc & 1;
    unsigned lm0 = isx ? 65536u + (wc >> 1) * 8192u : 32768u + (wc >> 1) * 16384u, lm1 = lm0 + (isx ? 16384u : 8192u), flip = isx ? 0u : 32768u;
    const unsigned sub = (wc & 1) * 65536u + (wc >> 1) * 8192u;
    unsigned gv = lane * 16u + (wc >> 1) * 8192u;
    unsigned la = (unsigned)(uintptr_t)lds + wr * 4096u + lane * 16u, lb = (unsigned)(uintptr_t)lds + 65536u + wc * 2048u + lane * 16u;
    asm volatile("" : "+v"(gv), "+v"(la), "+v"(lb));
    auto mine = [&](int rr, int j, int mb, unsigned& ka) {
        const int wi = ct_wi(j, rank);
        ka = (isx || rr == 0 ? (unsigned)R0N : 1u) * 65536u;
        return OFF_MINE + (unsigned)(isx ? (mb < 4 ? mb : mb + RC) : rr == 0 ? 17 - j : N_R0 + (wi - RC) * 4) * 65536u;
    };
    unsigned nS, nA = mine(1, j, mb, nS);
#define GL(j, p) do { \
        constexpr int g_ = !((j) & 1) ? (p) >> 2 : (p) < 6 ? (p) / 3 : (p) - 6, o_ = !((j) & 1) ? (p) & 3 : (p) < 6 ? (p) % 3 : 3; \
        if ((p) == 0 || ((p) == 6 && ((j) & 1))) { lx = ((j) & 1 ? lm1 : lm0) ^ ((j) & 2 ? 32768u : 0u); if (p) { asm volatile("" : "+s"(flip)); lx ^= flip; } ls = ((j) < 2 && !(p) ? bp : sn) + lx; } \
        if (g_ == 1 && (o_ == 0 || (p) == 7) && !(o_ && !((j) & 1))) asm volatile("s_add_u32 m0, m0, 0x1000\n\ts_addk_i32 %0, 0x1000\n\tbuffer_load_dwordx4 %1, %2, %0 offen offset:%3 sc1 lds" : "+s"(ls) : "v"(gv), "s"(rw), "n"(o_ * 1024) : "memory", "scc"); \
        else if ((p) == 0 || ((p) == 6 && ((j) & 1))) asm volatile("s_mov_b32 m0, %4\n\ts_nop 0\n\tbuffer_load_dwordx4 %1, %2, %0 offen offset:%3 sc1 lds" :: "s"(ls), "v"(gv), "s"(rw), "n"(o_ * 1024), "s"(lx) : "memory"); \
        else asm volatile("buffer_load_dwordx4 %1, %2, %0 offen offset:%3 sc1 lds" :: "s"(ls), "v"(gv), "s"(rw), "n"(o_ * 1024) : "memory"); } while (0)
#define RDA(f, k, b) Ar[f][k] = *(const LAS bf16x8*)(uintptr_t)(la + (((b) & 2) + (k)) * 16384 + ((f) >> 2) * 8192 + ((f) & 3) * 1024)
#define RDB(b, q) Br[(b) >> 1 & 1][(b) & 1][q] = *(const LAS bf16x8*)(uintptr_t)(lb + ((b) & 3) * 16384 + ((q) >> 1) * 8192 + ((q) & 1) * 1024)
#define RD(b, n) do { if ((n) < 6) RDA(((b) & 1) * 4 + ((n) >> 1 & 3), (~(n) ^ ((b) & (n) >> 2)) & 1, b); else if (((n) & 14) == 8) RDA(((b) & 1) * 4 + 3, (n) & 1, (b) + 1); else RDB((b) + 2, ((n) >> 2 & 2) + ((n) & 1)); } while (0)
#define MROW(b, i) (((b) & 1) && (i) >= 16 ? 2 + (((i) >> 1 ^ (i) >> 2) & 1) : (i) >> 3)
#define MCOL(b, i) (((b) & 1) && (i) >= 16 ? (i) >> 2 & 3 : (i) & 8 ? 3 - ((i) >> 1 & 3) : (i) >> 1 & 3)
#define MFR(b, i) ((((b) & 1) * 4 + 7 + MROW(b, i)) & 7)
#define MF2(z, b, i) mfma<MFR(b, i) * 4 + MCOL(b, i), z && !((i) & 1)>(Ar[MFR(b, i)][((i) ^ (i) >> 1) & 1], Br[(b) >> 1][((i) ^ (i) >> 1) & 1][MCOL(b, i)])
#define HEAD(vw, vn) do { SCHED; if ((LD ? vw : vn) < 63) wait_v<LD ? vw : vn>(); __builtin_amdgcn_s_waitcnt(0xC07F); BAR; SCHED; } while (0)
#define BODY(b, z) do { \
        EACH(32, i) \
            MF2(z, b, i); SCHED; \
            if (LD && i % 4 == 1) { GL(((b) + 1) & 3, i / 4); SCHED; } \
            if (i < 3 || i == 4 || (i >= 8 && i < 24 && i % 2 == 0)) { RD(b, i < 3 ? i : i == 4 ? 3 : i / 2 % 12); SCHED; } \
        END_EACH; } while (0)
    unsigned lx = 0, ls = 0, sn = nA - sub, bp = sn;
    __builtin_amdgcn_s_setprio(LD ? 2 : 1);
    if (LD) { EACH(32, i) GL(i >> 3, i & 7); END_EACH; }
    SCHED; WAIT_V(0); BAR; SCHED;
    EACH(8, i) RDB(i >> 2, i & 3); END_EACH;
    RDA(7, 0, 0); RDA(7, 1, 0);
    HEAD(63, 63);
    sn += nS;
    if (isx) bp = sn;
    if (LD) { EACH(8, i) GL(0, i); END_EACH; }
    asm volatile("" ::: "v255");
    unsigned kv = call << 16;
    unsigned ask = wc ? 0u : OFF_GP + (unsigned)xg * 128u;
    unsigned am = 0x44444444u;
    const __amdgpu_buffer_rsrc_t ro = mk_rsrc(P.out, OUT_BYTES);
#pragma unroll 1
    for (int ui = -nj; ui < NROUND; ++ui) {
        const bool has_next = ui + 1 < NROUND, npt = (unsigned)ui < (unsigned)np;
        const int wi = ct_wi(j, rank), rr = ui >= 0 ? ui : ui + nj < np ? ui + nj + 1 : NROUND - 1;
        const unsigned a0 = nA, st = nS;
        BODY(0, true);
        const bool xn = ui < -1 && rr == np;
        const int t1 = ui < 0 ? 4 : !(job & 4) && job >> 6 ? job >> 6 : 32, jn = ui == -1 ? j0 : xn ? NCT - 7 + (lw >> 2) : j + RC, mn = 4 * (xn ? xg >> 1 : g & 3) + (lw & 3);
        const unsigned pa = OFF_PT + (unsigned)((((rr - 1) * WG_PER_GRP + lw) * 8 + (rr < NROUND - 1 ? xg : mb >> 2 | (job >> 6 ? 4 : 0))) * 16 + wid * 2 - (wc >> 1)) * 8192u;
        const unsigned pm = pt ? PMASK << PT0 : !(job & 12) && job >> 6 ? (2u * PMASK + !LD) << (t1 - NVIS - 2) : 0u;
        const int ts = pt ? 6 : (job & 12) == 12 ? (job >> 6) + 2 : 2;
        am &= ~0u << ts;
        unsigned cm = am << 1 | pm;
        unsigned vs = 0, vn = 0;
        if (LD) {
            const unsigned pan = isx ? OFF_XA + (unsigned)mb * PANEL_BYTES : w_panel(j);
            if (npt || (job & 32)) { nA = isx ? pan : w_panel(npt ? jn : j + 7); nS = 65536u; }
            else if (has_next) nA = mine(ui == -1 ? 0 : rr + 1, jn, mn, nS);
            int ln = lane;
            asm volatile("" : "+v"(ln));
            vs = (ln == t1 ? nA : ln > t1 ? nA + nS : ln < 4 ? a0 + (unsigned)ln * st : pan + (unsigned)(ln - 4) * 65536u) - sub;
            vn = ui < 0 ? ln == 4 && rr >= 3 : !has_next ? (ln == 32 && (job & 32) ? K_END : 0) : ln < 4 ? 0 : ln < 32 ? unit_need(ui, (4 * ln - 13) / 7, g)
               : npt ? unit_need(ui + 1, ln & 1, g) : ui == NROUND - 2 && ln == 32 ? 0x7fffffff : 0;
            asm volatile("" : "+v"(vs), "+v"(vn));
        }
        auto turn = [&](int t) __attribute__((always_inline)) {
            const int need = __builtin_amdgcn_readlane((int)vn, t);
            if (__builtin_expect((unsigned)known < (unsigned)need, 0)) {
                ready(need > K_END ? K_ALL : need);
                if (need > K_END) {
                    if ((known & 12) == 4) ready(K_END);
                    const int ml = 4 * (known & 3) + (lw & 3);
                    if (known & 4) { nA = (isx ? OFF_XA + (unsigned)ml * PANEL_BYTES : w_panel(NCT - (known & 8 ? RC : 7) + (lw >> 2))) + (known & 8 ? (unsigned)((known >> 6 & 31) - 4) * 65536u : 0u); nS = 65536u; }
                    else nA = mine(NROUND - 1, NCT - RC + (lw >> 2), ml, nS);
                    vs = nA - sub;
                    asm volatile("v_writelane_b32 %0, %1, 33\n\ts_nop 1" : "+v"(vs) : "s"(nA + nS - sub));
                }
            }
            bp = sn;
            sn = (unsigned)__builtin_amdgcn_readlane((int)vs, t);
            if (isx) bp = sn;
        };
        HEAD(63, 63); BODY(1, true);
        if (LD && ui == -nj) WAIT_V(16);
        HEAD(63, 63); BODY(2, false);
        HEAD(16, 0);
        if (!LD && (job & 16) && tid == 256) __hip_atomic_store((unsigned*)(ws + OFF_HF) + ((job & 3) * WG_PER_GRP + lw) * 32, call, __ATOMIC_RELAXED, __HIP_MEMORY_SCOPE_AGENT);
        if (LD) turn(ts);
        SCHED;
        BODY(3, false);
#pragma unroll 1
        for (int t = ts; t <= t1; ++t) {
            HEAD(16, 63);
            BODY(0, false);
            HEAD(16, 63);
            if (!LD && __builtin_expect(pm >> (t & 31) & 1u, 0)) {
                const int k = t + __builtin_clz(pm) - (31 - NVIS);
                if (k < 0) helped(); else visit<true>((unsigned)k, gv, rw, pa);
            }
            SCHED;
            BODY(1, false);
            HEAD(16, 63);
            if (LD) asm volatile("s_bitcmp1_b32 %3, %4\n\ts_cbranch_scc0 3f\n\ts_cmp_eq_u32 %2, 0\n\ts_cbranch_scc1 2f\n\ts_cmp_lt_u32 %2, %5\n\ts_cbranch_scc1 1f\n\t"
                                 "s_mov_b64 exec, 1\n\tv_mov_b32 %0, 1\n\tbuffer_atomic_add %0, off, %1, %2 sc0\n\ts_mov_b64 exec, -1\n\ts_branch 3f\n"
                                 "1:\n\tbuffer_load_dword %0, off, %1, %2 sc1\n\ts_branch 3f\n"
                                 "2:\n\tv_mov_b32 %0, %6\n\tds_read_b32 %0, %0\n3:"
                                 : "+v"(kv) : "s"(rw), "s"(ask), "s"(am), "s"(t), "n"(OFF_TD), "s"(lknown) : "scc", "memory");
            SCHED;
            BODY(2, false);
            HEAD(16, 63);
            if (LD && __builtin_expect(cm >> (t & 31) & 1u, 0)) {
                if (pm >> (t & 31) & 1u) { __builtin_amdgcn_s_setprio(0); visit<false>((unsigned)(t + __builtin_clz(pm) - (31 - NVIS)), gv, rw, pa); __builtin_amdgcn_s_setprio(2); }
                if (am << 1 >> (t & 31) & 1u) {
                    unsigned kr;
                    asm volatile("v_readfirstlane_b32 %0, %1" : "=s"(kr) : "v"(kv));
                    if (ask >= OFF_TD) {
                        const unsigned pd = (ask - OFF_TD) >> 7;
                        ask = OFF_GP + (unsigned)xg * 128u;
                        if (pd >= TD_G) {
                            const unsigned now = (unsigned)__builtin_amdgcn_s_memrealtime();
                            if ((kr + 1u) % LB == 0u && lane == 0)
                                __hip_atomic_store((unsigned long long*)(ws + OFF_JOB) + xg, (unsigned long long)(now - lds_rd(lknown + 16)) << 32 | (now << 8 | (call & 255u)), __ATOMIC_RELAXED, __HIP_MEMORY_SCOPE_AGENT);
                        } else if (pd >= TD_CF) {
                            if (pd == TD_CF) lds_wr(lknown + 16, (unsigned)__builtin_amdgcn_s_memrealtime());
                            if ((kr + 1u) % NGEMM == 0u && lane == 0) __hip_atomic_store((unsigned*)(ws + OFF_CF) + (pd - TD_CF), call, __ATOMIC_RELAXED, __HIP_MEMORY_SCOPE_AGENT);
                        } else if ((pd < 16 ? (kr + 1u) % NOWN : (kr + 1u) % RC) == 0u) tell(pd);
                    } else if ((unsigned)known < (unsigned)K_ALL) {
                        known = max(known, (int)(kr - (ask ? call << 16 : 0u)));
                        if (ask) lds_wr(lknown, (unsigned)known);
                    } else { am = 0u; cm = pm; }
                }
            }
            if (LD) turn(t + 1);
            SCHED;
            BODY(3, false);
        }
        if ((job & 12) == 8) {
            if (wid == 4) helped();
            HEAD(63, 63);
            EACH(2, h)
                u32x4 w[8];
#pragma unroll
                for (int k = 0; k < 8; ++k) w[k] = __builtin_amdgcn_raw_buffer_load_b128(rw, gv + (k & 3) * 1024u, pa + (2 * h + (k >> 2)) * 4096u, AUX_OUT);
                EACH(8, k) add_regs<8 * h + k>(w[k]); END_EACH;
                SCHED;
            END_EACH;
        }
        const int bad = (int)lds_rd(lknown);
        if (!LD) HEAD(63, 63);
        asm volatile("s_nop 15\n\ts_nop 15" ::: "memory");
        SCHED;
#define AI(q, m) ((m) * 4 + (q))
        if (ui < 0 || (job & 12) == 12) {
            if (LD) HEAD(0, 0);
            EACH(16, k)
                unsigned o[4];
                pack<2 * k, 2 * k + 1>(o);
                __builtin_amdgcn_raw_buffer_store_b128((u32x4){o[0], o[1], o[2], o[3]}, rw, gv + (k & 3) * 1024u, pa + (k >> 2) * 4096u, AUX_OUT);
            END_EACH;
        } else {
            const int rop = wi / 7 - W_OP / 7;
            const bool own = wi < NOWN, opt = (0x2147563 >> 4 * (wi % 7) & 7) == (rop ^ rank), dup = own || opt || ui == FR;
            const unsigned row = (unsigned)(mb * 256 + wc * 32 + fq * 4), cb = (unsigned)(wi * 256 + wr * 128 + fr * 8) * 2u;
            const unsigned obase = row * 65536u + cb + (own ? (unsigned)rank * OWNB : (SH0 - NOWN) * 512u);
            const unsigned pitch = own ? OWNB : opt ? 512u : 1024u;
            const unsigned pbase = row * pitch + cb + (own ? OFF_OS : opt ? OFF_OP + (unsigned)rop * PANEL_BYTES - (unsigned)wi * 512u : OFF_FX + (unsigned)((wi - W_FR) >> 1) * (NPOOL * 1024u - 1024u) - W_FR * 512u);
            const float one = bad < 0 ? __builtin_nanf("") : 1.0f;
            EACH(4, qd)
                    constexpr int bj = qd >> 1, n = qd & 1;
                    f32x2 sg[16];
                    unsigned o[4][4];
                    EACH(8, m) sigmoid<AI(qd, m)>(sg + 2 * m, one); END_EACH;
                    EACH(4, e) scale<AI(qd, 2 * e), AI(qd, 2 * e + 1)>(sg + 4 * e); pack<AI(qd, 2 * e), AI(qd, 2 * e + 1)>(o[e]); END_EACH;
                    if (LD && bj + n == 0) HEAD(0, 0);
#pragma unroll
                    for (int r = 0; r < 4; ++r) {
                        const u32x4 ov = {o[0][r], o[1][r], o[2][r], o[3][r]};
                        __builtin_amdgcn_raw_buffer_store_b128(ov, ro, obase + (bj * 128 + n * 16 + r) * 65536u, 0, AUX_OUT);
                        if (dup) __builtin_amdgcn_raw_buffer_store_b128(ov, rw, pbase + (bj * 128 + n * 16 + r) * pitch, 0, AUX_SYS);
                    }
            END_EACH;
            if (wc == 0 && (own || (unsigned)(ui - (FR - 1)) < 2u)) ask = OFF_TD + (unsigned)(own ? mb : mb + 16 * (ui - FR + 2)) * 128u;
            if (wc == 0 && (unsigned)(ui - (RU - RDI)) < 2u) ask = OFF_TD + (TD_CF + (unsigned)(ui - (RU - RDI))) * 128u;
            if (wc == 0 && ui == DR && lw < LB) ask = OFF_TD + (TD_G + (unsigned)xg) * 128u;
        }
        am = 0x44444444u;
        SCHED;
        j = jn; mb = mn; pt = npt;
        if (ui == NROUND - 2 || (job & 32)) {
            if (rfl(bad) & 16) break;
            ui = NROUND - 2;
            job = job ? (job & 3) | 20 : rfl(bad) & 2047;
            mb = 4 * (job & 3) + (lw & 3); pt = (job & 12) == 4; j = NCT - (pt ? 7 : RC) + (lw >> 2);
        }
    }
    WAIT_V(0);
    BAR;
    if (wid == 0 && ask >= OFF_TD) {
        unsigned old = 0;
        if (lane == 0) old = __hip_atomic_fetch_add((unsigned*)(ws + ask), 1u, __ATOMIC_RELAXED, __HIP_MEMORY_SCOPE_AGENT);
        if (((unsigned)rfl((int)old) + 1u) % RC == 0u) tell((ask - OFF_TD) >> 7);
    }
    if (tid == 0 && (job & 12) == 12) __hip_atomic_store((unsigned*)(ws + OFF_HF) + ((job & 3) * WG_PER_GRP + lw) * 32, call, __ATOMIC_RELAXED, __HIP_MEMORY_SCOPE_AGENT);
    if (wid == 0) {
        unsigned old = 0;
        if (lane == 0) old = __hip_atomic_fetch_add((unsigned*)(ws + OFF_GP + xg * 128 + 64), 1u, __ATOMIC_RELAXED, __HIP_MEMORY_SCOPE_AGENT);
        if (((unsigned)rfl((int)old) + 1u) % WG_PER_GRP == 0u) __builtin_amdgcn_fence(__ATOMIC_RELEASE, "agent");
    }
}

__global__ __launch_bounds__(512, 2) __attribute__((amdgpu_num_vgpr(64 - 2 * PPV))) void fused_kernel(Params P) {
    extern __shared__ __attribute__((aligned(16))) unsigned char smem[];
    LAS unsigned char* lds = (LAS unsigned char*)smem;
    const int b = blockIdx.x;
    unsigned call0 = 0;
    static_assert(CB >= NPW && CB % 8 == 0, "roles");
    if (__builtin_expect((unsigned)b < (unsigned)NPW, 1)) call0 = push_first(P, (unsigned)(uintptr_t)(lds + STAGE_BYTES), b);
    if ((unsigned)(b - CB) < (unsigned)NCOMM) {
        if (threadIdx.x == 0) *(LAS unsigned*)(lds + 1024) = take_ticket(P, 0);
        __syncthreads();
        const unsigned call = (unsigned)rfl((int)*(LAS unsigned*)(lds + 1024)) / (GRID / 16) + 1u;
        comm_role(P, lds, call, b - CB);
    } else {
        const int xg = b & 7, lw = (b < CB ? b : b - NCOMM) >> 3;
        GAS char* wsl = lane_ws(P, threadIdx.x & 63);
        asm volatile("ds_write_b64 %1, %0" : "+v"(wsl) : "v"((unsigned)(uintptr_t)(lds + STAGE_BYTES + 1024) + threadIdx.x * 8u) : "memory");
        bool fast;
        const unsigned call = stage_role(P, wsl, (unsigned)(uintptr_t)(lds + STAGE_BYTES), lw * 8 + xg, call0, fast);
        if (rfl(threadIdx.x >> 8) == 0) gemm_role<true>(P, lds, xg, lw, call, fast);
        else gemm_role<false>(P, lds, xg, lw, call, fast);
    }
}

}

extern "C" void kernel_launch(int rank, int nranks, ncclComm_t comm, hipStream_t stream,
                              void* const* d_in, const int* in_sizes, int n_in,
                              void* d_out, int out_size,
                              const int* in_dtypes, int out_dtype,
                              void* const* d_out_all,
                              void* const* d_ws_all, size_t ws_bytes) {
    (void)comm; (void)in_sizes; (void)n_in; (void)d_out_all; (void)out_size; (void)in_dtypes; (void)out_dtype;
    if (nranks != NR || (unsigned)rank >= (unsigned)NR || ws_bytes < WS_NEED) {
        fprintf(stderr, "unsupported configuration\n");
        return;
    }
    Params P;
    P.x = (const u16*)d_in[0]; P.w = (const u16*)d_in[1];
    for (int q = 0; q < NR; ++q) P.ws[q] = (char*)d_ws_all[(rank + 1 + q) % NR];
    P.out = (char*)d_out;
    P.rank = rank;
    (void)hipFuncSetAttribute((const void*)fused_kernel, hipFuncAttributeMaxDynamicSharedMemorySize, LDS_BYTES);
    hipLaunchKernelGGL(fused_kernel, dim3(GRID), dim3(512), LDS_BYTES, stream, P);
}
```
